```python
import numpy as np

kernel_name = "gpu_a2a_gemm_m4096_k4096_n16384_f32_gelu"


ABSMAX_RTOL = 0.001

COMPUTE_DTYPE = "float32"
EPILOGUE = "gelu"
IN_QUANT = None
OUT_QUANT = None
IN_DTYPES = {"x": "float32", "w_mat": "float32"}
WORLD_SIZE = 8

def setup_inputs():

    x = np.random.randn(4096, 4096).astype(np.float32)
    w_mat = (np.random.randn(4096, 16384) / np.sqrt(4096)).astype(np.float32)
    return {"x": x, "w_mat": w_mat}

def _epilogue(y):
    c = 0.7978845608028654
    return (0.5 * y * (1.0 + np.tanh(c * (y + 0.044715 * y ** 3)))).astype(np.float32)

def reference(x, w_mat):
    y = (x @ w_mat).astype(np.float32)
    return _epilogue(y)
```

```cpp
#include <hip/hip_runtime.h>
#include <rccl/rccl.h>
#include <cstdio>

#define HIP_CHECK(expr) do { hipError_t _e = (expr); if (_e != hipSuccess) { \
    fprintf(stderr, "HIP error %s at %s:%d\n", hipGetErrorString(_e), __FILE__, __LINE__); \
    return; } } while (0)

namespace {

typedef unsigned u32x4 __attribute__((ext_vector_type(4)));
typedef float f32x4 __attribute__((ext_vector_type(4)));
typedef float f32x2 __attribute__((ext_vector_type(2)));
typedef _Float16 f16x2 __attribute__((ext_vector_type(2)));
typedef _Float16 f16x8 __attribute__((ext_vector_type(8)));
#define LAS __attribute__((address_space(3)))
#define GAS __attribute__((address_space(1)))

constexpr int NR = 8;
constexpr int KTOT = 4096, NTOT = 16384, MLOC = 512, KSH = 512;
constexpr int NWG = 256, NTHR = 512;
constexpr int NSTEP = KTOT / 32;
constexpr int RING = 16;
constexpr int SLOT_B = 8192;
constexpr int CTRL_OFF = RING * SLOT_B;
constexpr int LDS_B = CTRL_OFF + 64;

constexpr size_t WS_SEQ = 0;
constexpr size_t WS_TMO = 32768;
constexpr size_t WS_CNT = 36864;
constexpr size_t WS_A = 1u << 20;
constexpr size_t A_BYTES = (size_t)MLOC * KTOT * 2;
constexpr size_t WS_NEED = WS_A + 2 * A_BYTES;
constexpr unsigned long long DEADLINE_TICKS = 50000000ull;

#define W_AUX 0
#define A_AUX 17
#define O_AUX 0
#define WGRP 6

struct Params {
    const float* x;
    const float* w;
    float* out;
    char* ws[NR];
    int rank;
    int pad;
};

__device__ __forceinline__ unsigned pk16(float a, float b) {
    f32x2 v = {a, b};
    return __builtin_bit_cast(unsigned, __builtin_convertvector(v, f16x2));
}

__device__ __forceinline__ int unit_kk(int u, int rank) {
    u &= 31;
    if (u < 4) return rank * 16 + u * 4;
    const int v = u - 4, q = v / 7, s = (rank + 1 + (v - q * 7)) & 7;
    return s * 16 + q * 4;
}

__device__ __forceinline__ u32x4 lds_read_ctrl(unsigned addr) {
    u32x4 r;
    asm volatile("ds_read_b128 %0, %1\n\ts_waitcnt lgkmcnt(0)" : "=v"(r) : "v"(addr) : "memory");
    return r;
}
__device__ __forceinline__ void lds_write_word(unsigned addr, unsigned v) {
    asm volatile("ds_write_b32 %0, %1" :: "v"(addr), "v"(v) : "memory");
}

__device__ __forceinline__ float gelu(float y) {
    const float c2 = -2.0f * 0.7978845608028654f * 1.4426950408889634f;
    const float t = y * y;
    const float p = __builtin_fmaf(t, 0.044715f * c2, c2);
    const float e = __builtin_amdgcn_exp2f(y * p);
    return y * __builtin_amdgcn_rcpf(1.0f + e);
}

__global__ void __launch_bounds__(NTHR, 2) fused_a2a_gemm_gelu(Params p) {
    __shared__ __attribute__((aligned(16))) char smem[LDS_B];
    const unsigned lds0 = (unsigned)(size_t)(LAS char*)smem;
    const int tid = threadIdx.x, lane = tid & 63;
    const int wave = __builtin_amdgcn_readfirstlane(tid >> 6);
    const int b = blockIdx.x;
    const int xcd = b & 7, li = b >> 3;
    const int mh = li & 1;
    const int ncol0 = (xcd * 16 + (li >> 1)) * 128;
    const int rank = p.rank;
    char* const myws = p.ws[rank];
    const unsigned ctrl = lds0 + CTRL_OFF;

    if (wave < 4) {
        __builtin_amdgcn_s_barrier();
        asm volatile("" ::: "memory");
        unsigned call;
        {
            u32x4 c = lds_read_ctrl(ctrl + 32);
            call = __builtin_amdgcn_readfirstlane(c.x);
        }
        const __amdgpu_buffer_rsrc_t rsA = __builtin_amdgcn_make_buffer_rsrc(
            myws + WS_A + (size_t)(call & 1) * A_BYTES, 0, (int)A_BYTES, 0x00020000);
        const int avoff = (mh * 16 + wave * 4) * 1024 + lane * 16;
        const unsigned bbase = lds0 + (lane >> 4) * 2048 + (lane & 15) * 16;
        unsigned c_full0 = 0, c_full1 = 0, c_ardy = 0;

        auto refresh = [&]() {
            u32x4 c = lds_read_ctrl(ctrl);
            c_full0 = __builtin_amdgcn_readfirstlane(c.x);
            c_full1 = __builtin_amdgcn_readfirstlane(c.y);
            c_ardy = __builtin_amdgcn_readfirstlane(c.z);
        };
        auto need_a = [&](unsigned units) {
            if (c_ardy < units) {
                refresh();
                while (c_ardy < units) { __builtin_amdgcn_s_sleep(2); refresh(); }
            }
            asm volatile("" ::: "memory");
        };
        auto need_b = [&](int t) {
            const unsigned want = (unsigned)(t >> 1) + 1;
            if (t & 1) {
                if (c_full1 < want) {
                    refresh();
                    while (c_full1 < want) { __builtin_amdgcn_s_sleep(1); refresh(); }
                }
            } else {
                if (c_full0 < want) {
                    refresh();
                    while (c_full0 < want) { __builtin_amdgcn_s_sleep(1); refresh(); }
                }
            }
            asm volatile("" ::: "memory");
        };

        f32x4 acc[4][8];
#pragma unroll
        for (int m = 0; m < 4; ++m)
#pragma unroll
            for (int n = 0; n < 8; ++n) acc[m][n] = f32x4{0.f, 0.f, 0.f, 0.f};
        u32x4 a[4][4];
        u32x4 bq[4];

        need_a(1);
        int kk_cur = unit_kk(0, rank);
#pragma unroll
        for (int s = 0; s < 3; ++s)
#pragma unroll
            for (int m = 0; m < 4; ++m)
                a[s][m] = __builtin_amdgcn_raw_buffer_load_b128(rsA, avoff + m * 1024, (kk_cur + s) * 32768, A_AUX);
        need_b(0);
#pragma unroll
        for (int f = 0; f < 3; ++f)
            bq[f] = *(const LAS u32x4*)(size_t)(bbase + (f & 3) * 512 + (f >> 2) * 256);
        __builtin_amdgcn_sched_barrier(0);

        for (int u = 0; u < 32; ++u) {
            const int kk_nxt = unit_kk(u + 1, rank);
#pragma unroll
            for (int tt = 0; tt < 4; ++tt) {
                const int t = u * 4 + tt;
                const unsigned sb = bbase + (t & (RING - 1)) * SLOT_B;
                const unsigned sbn = bbase + ((t + 1) & (RING - 1)) * SLOT_B;
                if (tt == 1) need_a(u + 2 < 32 ? u + 2 : 32);
                const int asoff = (tt == 0 ? kk_cur + 3 : kk_nxt + tt - 1) * 32768;
#pragma unroll
                for (int f = 0; f < 8; ++f) {
                    if (f == 5 && t + 1 < NSTEP) need_b(t + 1);
                    const int fn = (f + 3) & 7;
                    bq[(f + 3) & 3] = *(const LAS u32x4*)(size_t)((f + 3 < 8 ? sb : sbn) + (fn & 3) * 512 + (fn >> 2) * 256);
                    if (f == 4) lds_write_word(ctrl + 16 + wave * 4, t + 1);
                    if (f < 4)
                        a[(tt + 3) & 3][f] = __builtin_amdgcn_raw_buffer_load_b128(rsA, avoff + f * 1024, asoff, A_AUX);
#pragma unroll
                    for (int m = 0; m < 4; ++m)
                        acc[m][f] = __builtin_amdgcn_mfma_f32_16x16x32_f16(
                            __builtin_bit_cast(f16x8, a[tt][m]), __builtin_bit_cast(f16x8, bq[f & 3]), acc[m][f], 0, 0, 0);
                    __builtin_amdgcn_sched_barrier(0);
                }
            }
            kk_cur = kk_nxt;
        }

        unsigned bad;
        {
            u32x4 c = lds_read_ctrl(ctrl);
            bad = __builtin_amdgcn_readfirstlane(c.w);
        }
        const __amdgpu_buffer_rsrc_t rsO = __builtin_amdgcn_make_buffer_rsrc(
            p.out, 0, MLOC * NTOT * 4, 0x00020000);
        const int ovoff = ((mh * 256 + wave * 64 + (lane >> 4) * 4) * NTOT + ncol0 + (lane & 15) * 4) * 4;
#pragma unroll
        for (int m = 0; m < 4; ++m)
#pragma unroll
            for (int r = 0; r < 4; ++r)
#pragma unroll
                for (int T = 0; T < 2; ++T) {
                    f32x4 o;
#pragma unroll
                    for (int i = 0; i < 4; ++i) o[i] = gelu(acc[m][T * 4 + i][r]);
                    if (bad) o = f32x4{__builtin_nanf(""), __builtin_nanf(""), __builtin_nanf(""), __builtin_nanf("")};
                    __builtin_amdgcn_raw_buffer_store_b128(__builtin_bit_cast(u32x4, o), rsO, ovoff + T * 256,
                                                           (m * 16 + r) * NTOT * 4, O_AUX);
                }
    } else if (wave < 6) {
        const int ld = wave - 4;
        const __amdgpu_buffer_rsrc_t rsW = __builtin_amdgcn_make_buffer_rsrc(
            (void*)p.w, 0, KTOT * NTOT * 4, 0x00020000);
        const int h = lane >> 5, c = lane & 31;
        const int wvoff = (h * 8 * NTOT + ncol0 + c * 4) * 4;
        const unsigned wbase = lds0 + h * 2048 + c * 16;
        constexpr int NGRP = NSTEP;
        u32x4 w[WGRP][8];
        auto grp_soff = [&](int g) -> int {
            const int t = 2 * (g >> 1) + ld;
            const int row = (unit_kk(t >> 2, rank) + (t & 3)) * 32 + (g & 1) * 16;
            return g < NGRP ? row * (NTOT * 4) : KTOT * NTOT * 4;
        };
#pragma unroll
        for (int s = 0; s < WGRP; ++s) {
            const int so = grp_soff(s);
#pragma unroll
            for (int j = 0; j < 8; ++j)
                w[s][j] = __builtin_amdgcn_raw_buffer_load_b128(rsW, wvoff, so + j * (NTOT * 4), W_AUX);
        }
        __builtin_amdgcn_sched_barrier(0);
        __builtin_amdgcn_s_barrier();
        asm volatile("" ::: "memory");
        unsigned c_done = 0;
        for (int g0 = 0; g0 < NGRP; g0 += WGRP) {
#pragma unroll
            for (int s = 0; s < WGRP; ++s) {
                const int g = g0 + s;
                if (g < NGRP) {
                    const int t = 2 * (g >> 1) + ld;
                    if (t >= RING && c_done < (unsigned)(t - RING + 1)) {
                        for (;;) {
                            u32x4 d = lds_read_ctrl(ctrl + 16);
                            const unsigned m0 = d.x < d.y ? d.x : d.y, m1 = d.z < d.w ? d.z : d.w;
                            c_done = __builtin_amdgcn_readfirstlane(m0 < m1 ? m0 : m1);
                            if (c_done >= (unsigned)(t - RING + 1)) break;
                            __builtin_amdgcn_s_sleep(2);
                        }
                    }
                    const unsigned dst = wbase + (t & (RING - 1)) * SLOT_B + (g & 1) * 4096;
#pragma unroll
                    for (int i = 0; i < 4; ++i) {
                        u32x4 o;
#pragma unroll
                        for (int j = 0; j < 4; ++j)
                            o[j] = pk16(__builtin_bit_cast(float, w[s][2 * j][i]), __builtin_bit_cast(float, w[s][2 * j + 1][i]));
                        *(LAS u32x4*)(size_t)(dst + i * 512) = o;
                    }
                    if (g & 1) lds_write_word(ctrl + ld * 4, (g >> 1) + 1);
                }
                const int so = grp_soff(g + WGRP);
#pragma unroll
                for (int j = 0; j < 8; ++j)
                    w[s][j] = __builtin_amdgcn_raw_buffer_load_b128(rsW, wvoff, so + j * (NTOT * 4), W_AUX);
                __builtin_amdgcn_sched_barrier(0);
            }
        }
    } else if (wave == 6) {
        const unsigned d = (unsigned)(xcd + li) & 7u;
        const GAS f32x4* src = (const GAS f32x4*)(p.x + (size_t)(d * MLOC + li * 16 + (lane & 15)) * KSH + (lane >> 4) * 8);
        f32x4 v[16][2];
#pragma unroll
        for (int s = 0; s < 16; ++s) {
            v[s][0] = __builtin_nontemporal_load(src + s * 8);
            v[s][1] = __builtin_nontemporal_load(src + s * 8 + 1);
        }
        __builtin_amdgcn_sched_barrier(0);
        __builtin_amdgcn_s_barrier();
        asm volatile("" ::: "memory");
        unsigned call;
        {
            u32x4 c = lds_read_ctrl(ctrl + 32);
            call = __builtin_amdgcn_readfirstlane(c.x);
        }
        char* const dws = p.ws[d];
        char* dst = dws + WS_A + (size_t)(call & 1) * A_BYTES + (size_t)((rank * 16) * 32 + li) * 1024 + lane * 16;
        unsigned* cnt = (unsigned*)(dws + WS_CNT + (size_t)((li >> 4) * 8 + rank) * 128);
        u32x4 o[16];
#pragma unroll
        for (int s = 0; s < 16; ++s) {
            o[s][0] = pk16(v[s][0][0], v[s][0][1]);
            o[s][1] = pk16(v[s][0][2], v[s][0][3]);
            o[s][2] = pk16(v[s][1][0], v[s][1][1]);
            o[s][3] = pk16(v[s][1][2], v[s][1][3]);
        }
#pragma unroll
        for (int q = 0; q < 4; ++q) {
#pragma unroll
            for (int s = 0; s < 4; ++s)
                asm volatile("global_store_dwordx4 %0, %1, off sc0 sc1\n\ts_nop 1"
                             :: "v"(dst + (size_t)(q * 4 + s) * 32768), "v"(o[q * 4 + s]) : "memory");
            asm volatile("s_waitcnt vmcnt(0)" ::: "memory");
            if (lane == 0)
                __hip_atomic_fetch_add(cnt + q, 1u, __ATOMIC_RELAXED, __HIP_MEMORY_SCOPE_SYSTEM);
        }
    } else {
        unsigned call = 0;
        if (lane == 0)
            call = __hip_atomic_fetch_add((unsigned*)(myws + WS_SEQ + (size_t)b * 128), 1u,
                                          __ATOMIC_RELAXED, __HIP_MEMORY_SCOPE_AGENT) + 1u;
        call = __builtin_amdgcn_readfirstlane(call);
        if (lane < 16) lds_write_word(ctrl + lane * 4, lane == 8 ? call : 0u);
        asm volatile("s_waitcnt lgkmcnt(0)" ::: "memory");
        __builtin_amdgcn_s_barrier();
        asm volatile("" ::: "memory");
        const int kk = unit_kk(lane & 31, rank);
        unsigned* cnt = (unsigned*)(myws + WS_CNT + (size_t)(mh * 8 + (kk >> 4)) * 128) + ((kk >> 2) & 3);
        unsigned* tmo = (unsigned*)(myws + WS_TMO);
        const unsigned target = call * 16u;
        const unsigned long long t0 = __builtin_amdgcn_s_memrealtime();
        unsigned published = 0;
        bool ok = false;
        bool bad = false;
        for (;;) {
            if (!ok) ok = __hip_atomic_load(cnt, __ATOMIC_RELAXED, __HIP_MEMORY_SCOPE_SYSTEM) >= target;
            const unsigned mask = (unsigned)__ballot(ok);
            const unsigned n = mask == 0xffffffffu ? 32u : (unsigned)__builtin_ctz(~mask);
            if (n == 32u) break;
            if (n > published) {
                published = n;
                lds_write_word(ctrl + 8, n);
            }
            if (__hip_atomic_load(tmo, __ATOMIC_RELAXED, __HIP_MEMORY_SCOPE_AGENT) == call ||
                __builtin_amdgcn_s_memrealtime() - t0 > DEADLINE_TICKS) {
                __hip_atomic_store(tmo, call, __ATOMIC_RELAXED, __HIP_MEMORY_SCOPE_AGENT);
                bad = true;
                break;
            }
            __builtin_amdgcn_s_sleep(4);
        }
        if (__hip_atomic_load(tmo, __ATOMIC_RELAXED, __HIP_MEMORY_SCOPE_AGENT) == call) bad = true;
        if (bad) lds_write_word(ctrl + 12, 1u);
        lds_write_word(ctrl + 8, 32u);
    }
}

}

extern "C" void kernel_launch(int rank, int nranks, ncclComm_t comm,
                              hipStream_t stream,
                              void* const* d_in, const int* in_sizes, int n_in,
                              void* d_out, int out_size,
                              const int* in_dtypes, int out_dtype,
                              void* const* d_out_all,
                              void* const* d_ws_all, size_t ws_bytes) {
    if (nranks != NR || (unsigned)rank >= (unsigned)NR || ws_bytes < WS_NEED || n_in < 2 ||
        in_sizes[0] != KTOT * KSH || in_sizes[1] != KTOT * NTOT || out_size != MLOC * NTOT) {
        fprintf(stderr, "kernel_launch: unexpected problem shape\n");
        return;
    }
    Params p;
    p.x = (const float*)d_in[0];
    p.w = (const float*)d_in[1];
    p.out = (float*)d_out;
    for (int r = 0; r < NR; ++r) p.ws[r] = (char*)d_ws_all[r];
    p.rank = rank;
    p.pad = 0;
    hipLaunchKernelGGL(fused_a2a_gemm_gelu, dim3(NWG), dim3(NTHR), 0, stream, p);
    HIP_CHECK(hipGetLastError());
}
```

```cpp
#include <hip/hip_runtime.h>
#include <rccl/rccl.h>
#include <cstdio>


#define HIP_CHECK(expr) do { hipError_t _e = (expr); if (_e != hipSuccess) { \
    fprintf(stderr, "HIP error %s at %s:%d\n", hipGetErrorString(_e), __FILE__, __LINE__); \
    return; } } while (0)

namespace {

typedef unsigned u32x32 __attribute__((ext_vector_type(32)));
#define LAS __attribute__((address_space(3)))

constexpr int NR = 8;
constexpr int KTOT = 4096, NTOT = 16384, MLOC = 512, KSH = 512;
constexpr int NWG = 256, NTHR = 256;
constexpr int RING = 20;
constexpr int SLOT_B = 8192;
constexpr int LDS_B = RING * SLOT_B;
constexpr int GS = 2;
constexpr int LA = 2;
constexpr int DELTA = 20;
constexpr int NM = 64 - DELTA;
constexpr int ROWB = NTOT * 4;

constexpr int WS_SEQ = 0;
constexpr int WS_TMO = 0x20000;
constexpr int WS_CNT = 0x21000;
constexpr int WS_XF = 0x30000;
constexpr int WS_A = 1 << 20;
constexpr int A_BYTES = MLOC * KTOT * 2;
constexpr int WS_X = 16 << 20;
constexpr int WS_REC = WS_X + NWG * 4 * 32768;
constexpr int OOR = 0x6000000;
static_assert(WS_TMO == 0x20000 && A_BYTES == 1 << 22 && OOR >= WS_REC, "constants of the assembly");

struct Params {
    const float* x;
    const float* w;
    float* out;
    char* my;
    char* ws[NR];
    int rank;
    int pad;
};

__device__ __forceinline__ int uni(int v) { return __builtin_amdgcn_readfirstlane(v); }

__device__ __forceinline__ int step_kk(int t, int rank, int kh) {
    const unsigned v = (unsigned)(t - 8) >> 1, j = (v * 37u) >> 8;
    const int i = (int)(v - j * 7u);
    const int peer = ((rank + 1 + i) & 7) * 16 + (((int)j + 1 * i) & 3) * GS + (t & 1);
    return (t < 8 ? rank * 16 + t : peer) + kh * 8;
}

__global__ void __launch_bounds__(NTHR, 1) fused_a2a_gemm_gelu(Params p) {
    __shared__ __attribute__((aligned(16))) char smem[LDS_B];
    const unsigned lds0 = (unsigned)(size_t)(LAS char*)smem;
    const int tid = threadIdx.x, lane = tid & 63;
    const int wave = __builtin_amdgcn_readfirstlane(tid >> 6);
    const int b = blockIdx.x;
    const int xcd = b & 7, li = b >> 3;
    const int kh = xcd >> 2;
    const int ntile = (xcd & 3) * 32 + li;
    const int partner = b ^ 4;
    constexpr int PAT = 0;
    constexpr int PIECE = PAT == 1 ? NTOT / 2 : 64;
    const int lcol = (PAT == 1 ? ntile * 64 : ntile * 128) + (lane & 15) * 4;
    const unsigned d = (unsigned)(xcd + li) & 7u;
    int rank = p.rank;
    char* myws = p.my;
    char* dws = p.ws[d];
    const float* px = p.x;
    const float* pw = p.w;
    float* pout = p.out;
    asm volatile("" : "+s"(rank), "+s"(myws), "+s"(dws), "+s"(px), "+s"(pw), "+s"(pout));

    const int sj = li >> 3, sh = (li >> 2) & 1, sq = (sj + 1 * ((rank - (int)d - 1) & 7)) & 3;
    const int ss = sh * 8 + sq * 2, sm = (li & 3) * 8 + wave * 2;
    const bool early = d == (unsigned)rank || sj == 0;
    const unsigned role = __builtin_amdgcn_readfirstlane((early ? 2u : 4u) | (wave == 0 ? 1u : 0u) |
                                                         (b == NWG - 1 ? 8u : 0u) | (wave == 0 && li >= 30 ? 16u : 0u) |
                                                         (li >= 8 && (li & 7) == 7 ? 32u : 0u));
    const int xv = ((lane >> 4) * KSH + (lane & 15) * 4) * 4;
    const int xl = uni(lds0 + LDS_B - 16384 + wave * 4096);
    const int xrem = uni(((d * MLOC + sm * 16) * KSH + ss * 32) * 4);
    const unsigned soff = uni(WS_A + ((rank * 16 + ss) * 32 + sm) * 1024);
    const int seq = uni(WS_SEQ + (b * 4 + wave) * 128);
    const __amdgpu_buffer_rsrc_t rsX = __builtin_amdgcn_make_buffer_rsrc((void*)px, 0, KTOT * KSH * 4, 0x00020000);
    const __amdgpu_buffer_rsrc_t rsA = __builtin_amdgcn_make_buffer_rsrc(myws, 0, WS_REC, 0x00020000);
    u32x32 remx;
    unsigned long long slot;
    asm volatile(
        "s_nop 4\n"
        "s_bitcmp1_b32 %[role], 1\n"
        "s_cbranch_scc0 LWb_%=\n"
        "s_add_u32 s45, %[xrem], 0\n"
        "buffer_load_dwordx4 v[28:31], %[xv], %[rsX], s45 offen nt\n"
        "s_add_u32 s45, %[xrem], 8192\n"
        "buffer_load_dwordx4 v[32:35], %[xv], %[rsX], s45 offen nt\n"
        "s_add_u32 s45, %[xrem], 16384\n"
        "buffer_load_dwordx4 v[36:39], %[xv], %[rsX], s45 offen nt\n"
        "s_add_u32 s45, %[xrem], 24576\n"
        "buffer_load_dwordx4 v[40:43], %[xv], %[rsX], s45 offen nt\n"
        "s_add_u32 s45, %[xrem], 32768\n"
        "buffer_load_dwordx4 v[44:47], %[xv], %[rsX], s45 offen nt\n"
        "s_add_u32 s45, %[xrem], 40960\n"
        "buffer_load_dwordx4 v[48:51], %[xv], %[rsX], s45 offen nt\n"
        "s_add_u32 s45, %[xrem], 49152\n"
        "buffer_load_dwordx4 v[52:55], %[xv], %[rsX], s45 offen nt\n"
        "s_add_u32 s45, %[xrem], 57344\n"
        "buffer_load_dwordx4 v[56:59], %[xv], %[rsX], s45 offen nt\n"
        "LWb_%=:\n"
        "v_mov_b32 v67, %[seq]\n"
        "s_mov_b64 exec, 1\n"
        "buffer_load_dwordx2 v[68:69], v67, %[rsA], 0 offen sc0 sc1\n"
        "s_mov_b64 exec, -1\n"
        : "=&{v[28:59]}"(remx), "=&{v[68:69]}"(slot)
        : [xv] "v"(xv), [rsX] "s"(rsX), [rsA] "s"(rsA), [role] "s"(role), [xrem] "s"(xrem), [seq] "s"(seq)
        : "memory", "v67", "s45", "scc");
    __builtin_amdgcn_sched_barrier(0);

    const unsigned scnt = uni(WS_CNT + (rank * 2 + sh) * 512 + sq * 128);
    const unsigned slim = uni(role & 4u ? 1 + (unsigned)(li - 8) * 9 / 24 + (li >= 16 ? 0 : 0) : ~0u);

    const int tab = step_kk(lane, rank, kh);
    unsigned tabw = (unsigned)tab << 21;
    int cntoff = WS_CNT + ((tab >> 4) * 2 + kh) * 512 + ((tab & 7) / GS) * 128;
    const int keepofs = WS_A + wave * 8192 + kh * 4096, sendofs = keepofs ^ 4096;
    const int a0 = uni(((rank * 16 + kh * 8) << 15) + keepofs);
    const int cpo = uni((li / (7 + 1) * 4 + wave) % 6 * 8192);
    const int pf0 = uni(((li * 13) & 31) * 240 / 32 - 240 / 2);
    const int hs = 2 * (lane + LA - NM);
    int tablo = (__shfl(tab, hs < 0 ? lane + LA : (NM + hs) & 63) << 15) + (hs < 0 ? keepofs : sendofs);
    int tabhi = (__shfl(tab, hs < 0 ? lane + LA : (NM + hs + 1) & 63) << 15) + sendofs;
    const int tabh = (__shfl(tab, (NM + lane) & 63) << 15) + keepofs;
    const int ps = lane < 20 ? lane + 8 + (wave & 1) : 2 * lane - 20 + 8 + wave;
    const int tp = __shfl(tab, ps & 63) << 15;
    int tabpf = (lane >= 20 || ((lane >> 1) & 1) == (wave >> 1)) && ps < 64 ? tp + WS_A + li * 1024 + ps : OOR;

    const __amdgpu_buffer_rsrc_t rsW = __builtin_amdgcn_make_buffer_rsrc((void*)pw, 0, KTOT * NTOT * 4, 0x00020000);
    const __amdgpu_buffer_rsrc_t rsO = __builtin_amdgcn_make_buffer_rsrc(pout, 0, MLOC * NTOT * 4, 0x00020000);
    int l16 = lane * 16;
    const unsigned bbase = lds0 + (lane >> 4) * 2048 + (lane & 15) * 16;
    const unsigned wdst = lds0 + ((wave >> 1) * 2 + (lane >> 5)) * 2048 + ((wave & 1) * 16 + (lane & 15)) * 16 + ((lane >> 4) & 1) * 8;
    const int wv0 = ((wave >> 1) * 16 + (lane >> 4) * 4) * ROWB + (lcol + (wave & 1) * PIECE) * 4;
    const int okeep = ((lane >> 4) * 4 + (wave * 8 + kh * 4) * 16) * ROWB + lcol * 4;
    const int xsend = uni(WS_X + (partner * 4 + wave) * 32768), xrecv = uni(WS_X + (b * 4 + wave) * 32768);
    const int mflag = uni(WS_XF + (b * 4 + wave) * 128), pflag = uni(WS_XF + (partner * 4 + wave) * 128);
    asm volatile(
        "s_memrealtime s[48:49]\n"
        "v_add_u32 v62, 65536, %[wv0]\n"
        "v_add_u32 v63, 131072, %[wv0]\n"
        "v_add_u32 v66, 196608, %[wv0]\n"
        "s_bitcmp1_b32 %[role], 2\n"
        "s_cbranch_scc0 LWx_%=\n"
        "s_add_u32 s45, %[xrem], 0\n"
        "buffer_load_dwordx4 v[28:31], %[xv], %[rsX], s45 offen nt\n"
        "s_add_u32 s45, %[xrem], 8192\n"
        "buffer_load_dwordx4 v[32:35], %[xv], %[rsX], s45 offen nt\n"
        "s_add_u32 s45, %[xrem], 16384\n"
        "buffer_load_dwordx4 v[36:39], %[xv], %[rsX], s45 offen nt\n"
        "s_add_u32 s45, %[xrem], 24576\n"
        "buffer_load_dwordx4 v[40:43], %[xv], %[rsX], s45 offen nt\n"
        "s_add_u32 s45, %[xrem], 32768\n"
        "buffer_load_dwordx4 v[44:47], %[xv], %[rsX], s45 offen nt\n"
        "s_add_u32 s45, %[xrem], 40960\n"
        "buffer_load_dwordx4 v[48:51], %[xv], %[rsX], s45 offen nt\n"
        "s_add_u32 s45, %[xrem], 49152\n"
        "buffer_load_dwordx4 v[52:55], %[xv], %[rsX], s45 offen nt\n"
        "s_add_u32 s45, %[xrem], 57344\n"
        "buffer_load_dwordx4 v[56:59], %[xv], %[rsX], s45 offen nt\n"
        "LWx_%=:\n"
        "v_readlane_b32 s42, %[tabw], 0\n"
        "s_nop 4\n"
        "buffer_load_dwordx4 v[192:195], %[wv0], %[rsW], s42 offen nt\n"
        "buffer_load_dwordx4 v[196:199], v62, %[rsW], s42 offen nt\n"
        "buffer_load_dwordx4 v[200:203], v63, %[rsW], s42 offen nt\n"
        "buffer_load_dwordx4 v[204:207], v66, %[rsW], s42 offen nt\n"
        "s_mov_b32 s53, 0\n"
        "s_mov_b32 s80, 0\n"
        "s_mov_b32 s81, 12\n"
        "s_mov_b32 s82, 16\n"
        "s_mov_b32 s40, 0\n"
        "s_mov_b64 s[54:55], 15\n"
        "s_mov_b32 s60, 0\n"
        "s_bfm_b64 s[88:89], 22, 0\n"
        "s_mov_b32 s87, 12\n"
        "s_mov_b32 s58, 0x6000000\n"
        "s_mov_b32 s61, %[slim]\n"
        "s_mov_b32 s66, 6\n"
        "s_mov_b32 s67, 16384\n"
        "s_mov_b32 s69, 0x8000\n"
        "s_waitcnt vmcnt(12) lgkmcnt(0)\n"
        "s_bitcmp1_b32 %[role], 1\n"
        "s_cbranch_scc0 LWt_%=\n"
        "s_waitcnt vmcnt(4)\n"
        "LWt_%=:\n"
        "v_readfirstlane_b32 s45, v69\n"
        "s_nop 1\n"
        "s_sub_u32 s45, s48, s45\n"
        "s_cmp_lt_u32 s45, 4000\n"
        "s_cselect_b32 s45, 1, 0\n"
        "s_mov_b32 s85, s45\n"
        "s_mul_i32 s72, s45, -1305\n"
        "s_add_u32 s72, s72, 1305\n"
        "s_mul_i32 s74, s45, -284\n"
        "s_add_u32 s74, s74, 284\n"
        "s_mul_i32 s75, s45, -4\n"
        "s_add_u32 s75, s75, 4\n"
        "s_mul_i32 s79, s45, 0\n"
        "s_add_u32 s79, s79, 0\n"
        "s_mul_i32 s76, s45, -2\n"
        "s_add_u32 s76, s76, 10\n"
        "s_mul_i32 s92, s45, -248\n"
        "s_add_u32 s92, s92, 248\n"
        "s_add_u32 s72, s72, s48\n"
        "s_add_u32 s72, s72, %[pf0]\n"
        "s_mov_b32 s73, s48\n"
        "v_readfirstlane_b32 s70, v68\n"
        "s_nop 1\n"
        "s_add_u32 s70, s70, 1\n"
        "s_lshl_b32 s71, s70, 5\n"
        "s_and_b32 s43, s70, 1\n"
        "s_lshl_b32 s43, s43, 22\n"
        "v_add_u32 v64, s43, v64\n"
        "s_bitcmp1_b32 %[role], 1\n"
        "s_cbranch_scc0 LWd_%=\n"
        "v_bfe_u32 v68, v64, 7, 1\n"
        "v_lshlrev_b32 v68, 11, v68\n"
        "v_bfe_u32 v60, v64, 5, 2\n"
        "v_lshl_add_u32 v68, v60, 8, v68\n"
        "v_bfe_u32 v60, v64, 4, 1\n"
        "v_lshl_add_u32 v68, v60, 3, v68\n"
        "v_bfe_u32 v60, v64, 8, 2\n"
        "v_lshl_add_u32 v68, v60, 4, v68\n"
        "v_add_u32 v68, %[xl], v68\n"
        "v_and_b32 v60, 0x3ff, v64\n"
        "v_add_u32 v60, %[xl], v60\n"
        "v_cvt_pk_f16_f32 v28, v28, v29\n"
        "v_cvt_pk_f16_f32 v29, v30, v31\n"
        "ds_write_b64 v68, v[28:29] offset:0\n"
        "v_cvt_pk_f16_f32 v32, v32, v33\n"
        "v_cvt_pk_f16_f32 v33, v34, v35\n"
        "ds_write_b64 v68, v[32:33] offset:64\n"
        "v_cvt_pk_f16_f32 v36, v36, v37\n"
        "v_cvt_pk_f16_f32 v37, v38, v39\n"
        "ds_write_b64 v68, v[36:37] offset:128\n"
        "v_cvt_pk_f16_f32 v40, v40, v41\n"
        "v_cvt_pk_f16_f32 v41, v42, v43\n"
        "ds_write_b64 v68, v[40:41] offset:192\n"
        "v_cvt_pk_f16_f32 v44, v44, v45\n"
        "v_cvt_pk_f16_f32 v45, v46, v47\n"
        "ds_write_b64 v68, v[44:45] offset:1024\n"
        "v_cvt_pk_f16_f32 v48, v48, v49\n"
        "v_cvt_pk_f16_f32 v49, v50, v51\n"
        "ds_write_b64 v68, v[48:49] offset:1088\n"
        "v_cvt_pk_f16_f32 v52, v52, v53\n"
        "v_cvt_pk_f16_f32 v53, v54, v55\n"
        "ds_write_b64 v68, v[52:53] offset:1152\n"
        "v_cvt_pk_f16_f32 v56, v56, v57\n"
        "v_cvt_pk_f16_f32 v57, v58, v59\n"
        "ds_write_b64 v68, v[56:57] offset:1216\n"
        "ds_read_b128 v[28:31], v60 offset:0\n"
        "ds_read_b128 v[32:35], v60 offset:1024\n"
        "ds_read_b128 v[36:39], v60 offset:2048\n"
        "ds_read_b128 v[40:43], v60 offset:3072\n"
        "s_waitcnt lgkmcnt(0)\n"
        "v_add_u32 v60, %[soff], v64\n"
        "global_store_dwordx4 v60, v[28:31], %[dws] offset:0 sc0 sc1\n"
        "global_store_dwordx4 v60, v[32:35], %[dws] offset:1024 sc0 sc1\n"
        "v_add_u32 v60, 0x8000, v60\n"
        "global_store_dwordx4 v60, v[36:39], %[dws] offset:0 sc0 sc1\n"
        "global_store_dwordx4 v60, v[40:43], %[dws] offset:1024 sc0 sc1\n"
        "s_nop 1\n"
        "LWd_%=:\n"
        "v_mov_b32 v128, 0\n"
        "v_mov_b32 v129, 0\n"
        "v_mov_b32 v130, 0\n"
        "v_mov_b32 v131, 0\n"
        "s_nop 1\n"
        "v_mfma_f32_32x32x16_f16 a[0:15], v[128:131], v[128:131], 0\n"
        "v_mfma_f32_32x32x16_f16 a[16:31], v[128:131], v[128:131], 0\n"
        "v_mfma_f32_32x32x16_f16 a[32:47], v[128:131], v[128:131], 0\n"
        "v_mfma_f32_32x32x16_f16 a[48:63], v[128:131], v[128:131], 0\n"
        "v_mfma_f32_32x32x16_f16 a[64:79], v[128:131], v[128:131], 0\n"
        "v_mfma_f32_32x32x16_f16 a[80:95], v[128:131], v[128:131], 0\n"
        "v_mfma_f32_32x32x16_f16 a[96:111], v[128:131], v[128:131], 0\n"
        "v_mfma_f32_32x32x16_f16 a[112:127], v[128:131], v[128:131], 0\n"
        "v_mfma_f32_32x32x16_f16 a[128:143], v[128:131], v[128:131], 0\n"
        "v_mfma_f32_32x32x16_f16 a[144:159], v[128:131], v[128:131], 0\n"
        "v_mfma_f32_32x32x16_f16 a[160:175], v[128:131], v[128:131], 0\n"
        "v_mfma_f32_32x32x16_f16 a[176:191], v[128:131], v[128:131], 0\n"
        "v_mfma_f32_32x32x16_f16 a[192:207], v[128:131], v[128:131], 0\n"
        "v_mfma_f32_32x32x16_f16 a[208:223], v[128:131], v[128:131], 0\n"
        "v_mfma_f32_32x32x16_f16 a[224:239], v[128:131], v[128:131], 0\n"
        "v_mfma_f32_32x32x16_f16 a[240:255], v[128:131], v[128:131], 0\n"
        "s_bitcmp1_b32 %[role], 1\n"
        "s_cbranch_scc0 LWe_%=\n"
        "s_waitcnt vmcnt(0)\n"
        "s_barrier\n"
        "s_bitcmp1_b32 %[role], 0\n"
        "s_cbranch_scc0 LW1_%=\n"
        "v_mov_b32 v68, %[scnt]\n"
        "v_mov_b32 v67, 8\n"
        "s_mov_b64 exec, 1\n"
        "global_atomic_add v68, v67, %[dws] sc1\n"
        "s_mov_b64 exec, -1\n"
        "LW1_%=:\n"
        "LWe_%=:\n"
        "s_getpc_b64 s[90:91]\n"
        "LCP0_%=:\n"
        "s_min_u32 s45, %[cpo], LXFIN_%=-LCP0_%=-8192\n"
        "s_add_u32 s90, s90, s45\n"
        "s_addc_u32 s91, s91, 0\n"
        "s_bitcmp1_b32 %[role], 5\n"
        "s_cbranch_scc0 LCPSa_%=\n"
        "v_and_b32 v68, 0x3ff, v64\n"
        "v_lshlrev_b32 v68, 3, v68\n"
        "global_load_dword v65, v68, s[90:91]\n"
        "LCPSa_%=:\n"
        "LCPF_%=:\n"
        "LHOLD_%=:\n"
        "s_memrealtime s[50:51]\n"
        "s_waitcnt lgkmcnt(0)\n"
        "s_sub_u32 s50, s50, s48\n"
        "s_cmp_lt_u32 s50, 160\n"
        "s_cbranch_scc1 LHOLD_%=\n"
        "v_readlane_b32 s42, %[tabw], 1\n"
        "s_nop 4\n"
        "buffer_load_dwordx4 v[208:211], %[wv0], %[rsW], s42 offen nt\n"
        "buffer_load_dwordx4 v[212:215], v62, %[rsW], s42 offen nt\n"
        "buffer_load_dwordx4 v[216:219], v63, %[rsW], s42 offen nt\n"
        "buffer_load_dwordx4 v[220:223], v66, %[rsW], s42 offen nt\n"
        "s_waitcnt vmcnt(4)\n"
        "v_cvt_pk_f16_f32 v72, v192, v196\n"
        "v_cvt_pk_f16_f32 v73, v200, v204\n"
        "ds_write_b64 %[wdst], v[72:73] offset:0\n"
        "v_cvt_pk_f16_f32 v74, v193, v197\n"
        "v_cvt_pk_f16_f32 v75, v201, v205\n"
        "ds_write_b64 %[wdst], v[74:75] offset:512\n"
        "v_cvt_pk_f16_f32 v76, v194, v198\n"
        "v_cvt_pk_f16_f32 v77, v202, v206\n"
        "ds_write_b64 %[wdst], v[76:77] offset:1024\n"
        "v_cvt_pk_f16_f32 v78, v195, v199\n"
        "v_cvt_pk_f16_f32 v79, v203, v207\n"
        "ds_write_b64 %[wdst], v[78:79] offset:1536\n"
        "s_bitcmp1_b32 %[role], 2\n"
        "s_cbranch_scc0 LWs_%=\n"
        "v_bfe_u32 v68, v64, 7, 1\n"
        "v_lshlrev_b32 v68, 11, v68\n"
        "v_bfe_u32 v60, v64, 5, 2\n"
        "v_lshl_add_u32 v68, v60, 8, v68\n"
        "v_bfe_u32 v60, v64, 4, 1\n"
        "v_lshl_add_u32 v68, v60, 3, v68\n"
        "v_bfe_u32 v60, v64, 8, 2\n"
        "v_lshl_add_u32 v68, v60, 4, v68\n"
        "v_add_u32 v68, %[xl], v68\n"
        "v_and_b32 v60, 0x3ff, v64\n"
        "v_add_u32 v60, %[xl], v60\n"
        "v_cvt_pk_f16_f32 v28, v28, v29\n"
        "v_cvt_pk_f16_f32 v29, v30, v31\n"
        "ds_write_b64 v68, v[28:29] offset:0\n"
        "v_cvt_pk_f16_f32 v32, v32, v33\n"
        "v_cvt_pk_f16_f32 v33, v34, v35\n"
        "ds_write_b64 v68, v[32:33] offset:64\n"
        "v_cvt_pk_f16_f32 v36, v36, v37\n"
        "v_cvt_pk_f16_f32 v37, v38, v39\n"
        "ds_write_b64 v68, v[36:37] offset:128\n"
        "v_cvt_pk_f16_f32 v40, v40, v41\n"
        "v_cvt_pk_f16_f32 v41, v42, v43\n"
        "ds_write_b64 v68, v[40:41] offset:192\n"
        "v_cvt_pk_f16_f32 v44, v44, v45\n"
        "v_cvt_pk_f16_f32 v45, v46, v47\n"
        "ds_write_b64 v68, v[44:45] offset:1024\n"
        "v_cvt_pk_f16_f32 v48, v48, v49\n"
        "v_cvt_pk_f16_f32 v49, v50, v51\n"
        "ds_write_b64 v68, v[48:49] offset:1088\n"
        "v_cvt_pk_f16_f32 v52, v52, v53\n"
        "v_cvt_pk_f16_f32 v53, v54, v55\n"
        "ds_write_b64 v68, v[52:53] offset:1152\n"
        "v_cvt_pk_f16_f32 v56, v56, v57\n"
        "v_cvt_pk_f16_f32 v57, v58, v59\n"
        "ds_write_b64 v68, v[56:57] offset:1216\n"
        "ds_read_b128 v[28:31], v60 offset:0\n"
        "ds_read_b128 v[32:35], v60 offset:1024\n"
        "ds_read_b128 v[36:39], v60 offset:2048\n"
        "ds_read_b128 v[40:43], v60 offset:3072\n"
        "s_waitcnt lgkmcnt(0)\n"
        "LWs_%=:\n"
        "s_waitcnt vmcnt(0)\n"
        "v_cvt_pk_f16_f32 v72, v208, v212\n"
        "v_cvt_pk_f16_f32 v73, v216, v220\n"
        "ds_write_b64 %[wdst], v[72:73] offset:8192\n"
        "v_cvt_pk_f16_f32 v74, v209, v213\n"
        "v_cvt_pk_f16_f32 v75, v217, v221\n"
        "ds_write_b64 %[wdst], v[74:75] offset:8704\n"
        "v_cvt_pk_f16_f32 v76, v210, v214\n"
        "v_cvt_pk_f16_f32 v77, v218, v222\n"
        "ds_write_b64 %[wdst], v[76:77] offset:9216\n"
        "v_cvt_pk_f16_f32 v78, v211, v215\n"
        "v_cvt_pk_f16_f32 v79, v219, v223\n"
        "ds_write_b64 %[wdst], v[78:79] offset:9728\n"
        "v_readlane_b32 s42, %[tabw], 2\n"
        "s_nop 4\n"
        "buffer_load_dwordx4 v[12:15], %[wv0], %[rsW], s42 offen nt\n"
        "buffer_load_dwordx4 v[16:19], v62, %[rsW], s42 offen nt\n"
        "buffer_load_dwordx4 v[20:23], v63, %[rsW], s42 offen nt\n"
        "buffer_load_dwordx4 v[24:27], v66, %[rsW], s42 offen nt\n"
        "s_waitcnt vmcnt(0)\n"
        "v_mov_b32 v61, 0\n"
        "s_mov_b64 exec, 15\n"
        "buffer_load_dword v61, %[cnt], %[rsA], 0 offen sc0 sc1\n"
        "s_mov_b64 exec, -1\n"
        "s_waitcnt vmcnt(0)\n"
        "v_cmp_le_u32 s[46:47], s71, v61\n"
        "s_andn2_b64 s[56:57], s[54:55], s[46:47]\n"
        "s_cbranch_scc1 LS_p_%=\n"
        "LR_p_%=:\n"
        "s_mov_b64 s[54:55], 15\n"
        "s_and_b32 s86, s70, 3\n"
        "s_lshl_b32 s86, s86, 7\n"
        "s_add_u32 s86, s86, 0x24000\n"
        "v_readlane_b32 s42, %[tabw], 3\n"
        "s_add_u32 s41, %[a0], 0\n"
        "s_xor_b32 s44, s41, 0x1000\n"
        "buffer_load_dwordx4 v[128:131], v64, %[rsA], s41 offen offset:0 sc0 sc1\n"
        "buffer_load_dwordx4 v[132:135], v64, %[rsA], s41 offen offset:1024 sc0 sc1\n"
        "buffer_load_dwordx4 v[136:139], v64, %[rsA], s41 offen offset:2048 sc0 sc1\n"
        "buffer_load_dwordx4 v[140:143], v64, %[rsA], s41 offen offset:3072 sc0 sc1\n"
        "buffer_load_dwordx4 v[144:147], v64, %[rsA], s44 offen offset:0 sc0 sc1\n"
        "buffer_load_dwordx4 v[148:151], v64, %[rsA], s44 offen offset:1024 sc0 sc1\n"
        "buffer_load_dwordx4 v[152:155], v64, %[rsA], s44 offen offset:2048 sc0 sc1\n"
        "buffer_load_dwordx4 v[156:159], v64, %[rsA], s44 offen offset:3072 sc0 sc1\n"
        "s_add_u32 s41, %[a0], 32768\n"
        "s_xor_b32 s44, s41, 0x1000\n"
        "buffer_load_dwordx4 v[44:47], %[wv0], %[rsW], s42 offen nt\n"
        "buffer_load_dwordx4 v[48:51], v62, %[rsW], s42 offen nt\n"
        "buffer_load_dwordx4 v[52:55], v63, %[rsW], s42 offen nt\n"
        "buffer_load_dwordx4 v[56:59], v66, %[rsW], s42 offen nt\n"
        "buffer_load_dwordx4 v[160:163], v64, %[rsA], s41 offen offset:0 sc0 sc1\n"
        "buffer_load_dwordx4 v[164:167], v64, %[rsA], s41 offen offset:1024 sc0 sc1\n"
        "buffer_load_dwordx4 v[168:171], v64, %[rsA], s41 offen offset:2048 sc0 sc1\n"
        "buffer_load_dwordx4 v[172:175], v64, %[rsA], s41 offen offset:3072 sc0 sc1\n"
        "buffer_load_dwordx4 v[176:179], v64, %[rsA], s44 offen offset:0 sc0 sc1\n"
        "buffer_load_dwordx4 v[180:183], v64, %[rsA], s44 offen offset:1024 sc0 sc1\n"
        "buffer_load_dwordx4 v[184:187], v64, %[rsA], s44 offen offset:2048 sc0 sc1\n"
        "buffer_load_dwordx4 v[188:191], v64, %[rsA], s44 offen offset:3072 sc0 sc1\n"
        "v_mov_b32 v71, %[bbase]\n"
        "v_add_u32 v70, 0x8000, %[bbase]\n"
        "v_readlane_b32 s42, %[tabw], 4\n"
        "v_readlane_b32 s68, %[tabw], 5\n"
        "s_memtime s[56:57]\n"
        "s_memrealtime s[50:51]\n"
        "s_waitcnt lgkmcnt(0)\n"
        "s_mov_b32 s83, s56\n"
        "s_mov_b32 s84, s50\n"
        "s_barrier\n"
        "ds_read_b128 v[112:115], v71 offset:0\n"
        "ds_read_b128 v[116:119], v71 offset:512\n"
        "ds_read_b128 v[120:123], v71 offset:1024\n"
        "s_waitcnt vmcnt(12) lgkmcnt(2)\n"
        "s_branch LENTER_%=\n"
        "LOOPN_%=:\n"
        "s_sub_u32 s45, s40, 8\n"
        "s_cmp_lt_u32 s45, 40\n"
        "s_cbranch_scc1 LTPN0_%=\n"
        "LTRN0_%=:\n"
        "s_barrier\n"
        "s_waitcnt vmcnt(8) lgkmcnt(2)\n"
        "LENTER_%=:\n"
        "v_mfma_f32_16x16x32_f16 a[0:3], v[128:131], v[112:115], a[0:3]\n"
        "buffer_load_dwordx4 v[80:83], %[wv0], %[rsW], s42 offen nt\n"
        "v_readlane_b32 s41, %[tablo], s40\n"
        "v_mfma_f32_16x16x32_f16 a[32:35], v[132:135], v[112:115], a[32:35]\n"
        "ds_read_b128 v[124:127], v71 offset:1536\n"
        "v_readlane_b32 s44, %[tabhi], s40\n"
        "s_add_u32 s40, s40, 1\n"
        "v_mfma_f32_16x16x32_f16 a[64:67], v[136:139], v[112:115], a[64:67]\n"
        "buffer_load_dwordx4 v[84:87], v62, %[rsW], s42 offen nt\n"
        "v_mfma_f32_16x16x32_f16 a[96:99], v[140:143], v[112:115], a[96:99]\n"
        "buffer_load_dwordx4 v[88:91], v63, %[rsW], s42 offen nt\n"
        "v_mfma_f32_16x16x32_f16 a[128:131], v[144:147], v[112:115], a[128:131]\n"
        "buffer_load_dwordx4 v[92:95], v66, %[rsW], s42 offen nt\n"
        "v_mfma_f32_16x16x32_f16 a[160:163], v[148:151], v[112:115], a[160:163]\n"
        "buffer_load_dwordx4 v[96:99], %[wv0], %[rsW], s68 offen nt\n"
        "v_mfma_f32_16x16x32_f16 a[192:195], v[152:155], v[112:115], a[192:195]\n"
        "buffer_load_dwordx4 v[100:103], v62, %[rsW], s68 offen nt\n"
        "v_mfma_f32_16x16x32_f16 a[224:227], v[156:159], v[112:115], a[224:227]\n"
        "s_waitcnt lgkmcnt(2)\n"
        "v_mfma_f32_16x16x32_f16 a[4:7], v[128:131], v[116:119], a[4:7]\n"
        "buffer_load_dwordx4 v[104:107], v63, %[rsW], s68 offen nt\n"
        "v_mfma_f32_16x16x32_f16 a[36:39], v[132:135], v[116:119], a[36:39]\n"
        "ds_read_b128 v[112:115], v71 offset:256\n"
        "v_mfma_f32_16x16x32_f16 a[68:71], v[136:139], v[116:119], a[68:71]\n"
        "buffer_load_dwordx4 v[108:111], v66, %[rsW], s68 offen nt\n"
        "s_cmp_lt_u32 s40, s87\n"
        "s_cbranch_scc1 LSR0_%=\n"
        "LSB0_%=:\n"
        "s_mov_b64 exec, s[88:89]\n"
        "buffer_load_dword v61, %[cnt], %[rsA], s60 offen sc0 sc1\n"
        "s_mov_b64 exec, -1\n"
        "buffer_load_dword v65, v64, %[rsA], s58 offen sc0 sc1\n"
        "v_mfma_f32_16x16x32_f16 a[100:103], v[140:143], v[116:119], a[100:103]\n"
        "v_readlane_b32 s42, %[tabw], s66\n"
        "s_cmp_lt_u32 s66, 64\n"
        "s_cselect_b32 s42, s42, 0x80000000\n"
        "s_add_u32 s66, s66, 1\n"
        "v_mfma_f32_16x16x32_f16 a[132:135], v[144:147], v[116:119], a[132:135]\n"
        "v_readlane_b32 s68, %[tabw], s66\n"
        "s_cmp_lt_u32 s66, 64\n"
        "s_cselect_b32 s68, s68, 0x80000000\n"
        "s_add_u32 s66, s66, 1\n"
        "v_mfma_f32_16x16x32_f16 a[164:167], v[148:151], v[116:119], a[164:167]\n"
        "s_cmp_eq_u32 s40, s61\n"
        "s_cbranch_scc1 LSENDN0_%=\n"
        "LSENTN0_%=:\n"
        "v_mfma_f32_16x16x32_f16 a[196:199], v[152:155], v[116:119], a[196:199]\n"
        "v_mfma_f32_16x16x32_f16 a[228:231], v[156:159], v[116:119], a[228:231]\n"
        "s_waitcnt lgkmcnt(2)\n"
        "v_mfma_f32_16x16x32_f16 a[8:11], v[128:131], v[120:123], a[8:11]\n"
        "v_add_u32 v69, s67, %[wdst]\n"
        "s_add_u32 s67, s67, 0x2000\n"
        "v_mfma_f32_16x16x32_f16 a[40:43], v[132:135], v[120:123], a[40:43]\n"
        "ds_read_b128 v[116:119], v71 offset:768\n"
        "v_mfma_f32_16x16x32_f16 a[72:75], v[136:139], v[120:123], a[72:75]\n"
        "s_waitcnt vmcnt(22)\n"
        "s_cmp_lt_u32 s67, 163840\n"
        "s_cselect_b32 s67, s67, 0\n"
        "v_mfma_f32_16x16x32_f16 a[104:107], v[140:143], v[120:123], a[104:107]\n"
        "v_cvt_pk_f16_f32 v72, v12, v16\n"
        "v_cvt_pk_f16_f32 v73, v20, v24\n"
        "v_mfma_f32_16x16x32_f16 a[136:139], v[144:147], v[120:123], a[136:139]\n"
        "ds_write_b64 v69, v[72:73] offset:0\n"
        "v_mfma_f32_16x16x32_f16 a[168:171], v[148:151], v[120:123], a[168:171]\n"
        "v_cvt_pk_f16_f32 v74, v13, v17\n"
        "v_cvt_pk_f16_f32 v75, v21, v25\n"
        "v_mfma_f32_16x16x32_f16 a[200:203], v[152:155], v[120:123], a[200:203]\n"
        "ds_write_b64 v69, v[74:75] offset:512\n"
        "v_mfma_f32_16x16x32_f16 a[232:235], v[156:159], v[120:123], a[232:235]\n"
        "s_waitcnt lgkmcnt(4)\n"
        "v_mfma_f32_16x16x32_f16 a[12:15], v[128:131], v[124:127], a[12:15]\n"
        "v_cvt_pk_f16_f32 v76, v14, v18\n"
        "v_cvt_pk_f16_f32 v77, v22, v26\n"
        "v_mfma_f32_16x16x32_f16 a[44:47], v[132:135], v[124:127], a[44:47]\n"
        "ds_read_b128 v[120:123], v71 offset:1280\n"
        "v_mfma_f32_16x16x32_f16 a[76:79], v[136:139], v[124:127], a[76:79]\n"
        "ds_write_b64 v69, v[76:77] offset:1024\n"
        "v_mfma_f32_16x16x32_f16 a[108:111], v[140:143], v[124:127], a[108:111]\n"
        "v_cvt_pk_f16_f32 v78, v15, v19\n"
        "v_cvt_pk_f16_f32 v79, v23, v27\n"
        "v_mfma_f32_16x16x32_f16 a[140:143], v[144:147], v[124:127], a[140:143]\n"
        "ds_write_b64 v69, v[78:79] offset:1536\n"
        "v_mfma_f32_16x16x32_f16 a[172:175], v[148:151], v[124:127], a[172:175]\n"
        "v_mfma_f32_16x16x32_f16 a[204:207], v[152:155], v[124:127], a[204:207]\n"
        "v_mfma_f32_16x16x32_f16 a[236:239], v[156:159], v[124:127], a[236:239]\n"
        "s_waitcnt lgkmcnt(6)\n"
        "v_mfma_f32_16x16x32_f16 a[16:19], v[128:131], v[112:115], a[16:19]\n"
        "v_mfma_f32_16x16x32_f16 a[48:51], v[132:135], v[112:115], a[48:51]\n"
        "ds_read_b128 v[124:127], v71 offset:1792\n"
        "v_mfma_f32_16x16x32_f16 a[80:83], v[136:139], v[112:115], a[80:83]\n"
        "v_mfma_f32_16x16x32_f16 a[112:115], v[140:143], v[112:115], a[112:115]\n"
        "v_mfma_f32_16x16x32_f16 a[144:147], v[144:147], v[112:115], a[144:147]\n"
        "v_mfma_f32_16x16x32_f16 a[176:179], v[148:151], v[112:115], a[176:179]\n"
        "v_mfma_f32_16x16x32_f16 a[208:211], v[152:155], v[112:115], a[208:211]\n"
        "v_mfma_f32_16x16x32_f16 a[240:243], v[156:159], v[112:115], a[240:243]\n"
        "s_waitcnt lgkmcnt(6)\n"
        "v_mfma_f32_16x16x32_f16 a[20:23], v[128:131], v[116:119], a[20:23]\n"
        "v_mfma_f32_16x16x32_f16 a[52:55], v[132:135], v[116:119], a[52:55]\n"
        "ds_read_b128 v[112:115], v71 offset:8192\n"
        "v_mfma_f32_16x16x32_f16 a[84:87], v[136:139], v[116:119], a[84:87]\n"
        "v_mfma_f32_16x16x32_f16 a[116:119], v[140:143], v[116:119], a[116:119]\n"
        "v_mfma_f32_16x16x32_f16 a[148:151], v[144:147], v[116:119], a[148:151]\n"
        "v_mfma_f32_16x16x32_f16 a[180:183], v[148:151], v[116:119], a[180:183]\n"
        "v_mfma_f32_16x16x32_f16 a[212:215], v[152:155], v[116:119], a[212:215]\n"
        "v_mfma_f32_16x16x32_f16 a[244:247], v[156:159], v[116:119], a[244:247]\n"
        "s_waitcnt lgkmcnt(4)\n"
        "v_mfma_f32_16x16x32_f16 a[24:27], v[128:131], v[120:123], a[24:27]\n"
        "v_mfma_f32_16x16x32_f16 a[56:59], v[132:135], v[120:123], a[56:59]\n"
        "ds_read_b128 v[116:119], v71 offset:8704\n"
        "v_mfma_f32_16x16x32_f16 a[88:91], v[136:139], v[120:123], a[88:91]\n"
        "v_mfma_f32_16x16x32_f16 a[120:123], v[140:143], v[120:123], a[120:123]\n"
        "v_mfma_f32_16x16x32_f16 a[152:155], v[144:147], v[120:123], a[152:155]\n"
        "v_mfma_f32_16x16x32_f16 a[184:187], v[148:151], v[120:123], a[184:187]\n"
        "v_mfma_f32_16x16x32_f16 a[216:219], v[152:155], v[120:123], a[216:219]\n"
        "v_mfma_f32_16x16x32_f16 a[248:251], v[156:159], v[120:123], a[248:251]\n"
        "s_waitcnt lgkmcnt(2)\n"
        "v_mfma_f32_16x16x32_f16 a[28:31], v[128:131], v[124:127], a[28:31]\n"
        "v_mfma_f32_16x16x32_f16 a[60:63], v[132:135], v[124:127], a[60:63]\n"
        "ds_read_b128 v[120:123], v71 offset:9216\n"
        "v_mfma_f32_16x16x32_f16 a[92:95], v[136:139], v[124:127], a[92:95]\n"
        "v_mfma_f32_16x16x32_f16 a[124:127], v[140:143], v[124:127], a[124:127]\n"
        "v_mfma_f32_16x16x32_f16 a[156:159], v[144:147], v[124:127], a[156:159]\n"
        "v_mfma_f32_16x16x32_f16 a[188:191], v[148:151], v[124:127], a[188:191]\n"
        "v_mfma_f32_16x16x32_f16 a[220:223], v[152:155], v[124:127], a[220:223]\n"
        "v_mfma_f32_16x16x32_f16 a[252:255], v[156:159], v[124:127], a[252:255]\n"
        "s_barrier\n"
        "s_waitcnt vmcnt(10) lgkmcnt(2)\n"
        "v_mfma_f32_16x16x32_f16 a[0:3], v[160:163], v[112:115], a[0:3]\n"
        "v_readlane_b32 s77, %[tablo], s40\n"
        "v_mfma_f32_16x16x32_f16 a[32:35], v[164:167], v[112:115], a[32:35]\n"
        "ds_read_b128 v[124:127], v71 offset:9728\n"
        "v_readlane_b32 s78, %[tabhi], s40\n"
        "s_add_u32 s40, s40, 1\n"
        "v_mfma_f32_16x16x32_f16 a[64:67], v[168:171], v[112:115], a[64:67]\n"
        "s_cmp_eq_u32 s40, s61\n"
        "s_cbranch_scc1 LSENDN1_%=\n"
        "LSENTN1_%=:\n"
        "v_mfma_f32_16x16x32_f16 a[96:99], v[172:175], v[112:115], a[96:99]\n"
        "v_add_u32 v69, s67, %[wdst]\n"
        "s_add_u32 s67, s67, 0x2000\n"
        "v_mfma_f32_16x16x32_f16 a[128:131], v[176:179], v[112:115], a[128:131]\n"
        "s_waitcnt vmcnt(18)\n"
        "s_cmp_lt_u32 s67, 163840\n"
        "s_cselect_b32 s67, s67, 0\n"
        "v_mfma_f32_16x16x32_f16 a[160:163], v[180:183], v[112:115], a[160:163]\n"
        "v_cvt_pk_f16_f32 v72, v44, v48\n"
        "v_cvt_pk_f16_f32 v73, v52, v56\n"
        "v_mfma_f32_16x16x32_f16 a[192:195], v[184:187], v[112:115], a[192:195]\n"
        "buffer_load_dwordx4 v[192:195], v64, %[rsA], s41 offen offset:0 sc0 sc1\n"
        "s_lshl_b64 s[54:55], s[54:55], 2\n"
        "v_mfma_f32_16x16x32_f16 a[224:227], v[188:191], v[112:115], a[224:227]\n"
        "s_waitcnt lgkmcnt(2)\n"
        "v_mfma_f32_16x16x32_f16 a[4:7], v[160:163], v[116:119], a[4:7]\n"
        "ds_write_b64 v69, v[72:73] offset:0\n"
        "s_or_b32 s54, s54, 3\n"
        "v_mfma_f32_16x16x32_f16 a[36:39], v[164:167], v[116:119], a[36:39]\n"
        "buffer_load_dwordx4 v[196:199], v64, %[rsA], s41 offen offset:1024 sc0 sc1\n"
        "v_mfma_f32_16x16x32_f16 a[68:71], v[168:171], v[116:119], a[68:71]\n"
        "ds_read_b128 v[112:115], v71 offset:8448\n"
        "v_mfma_f32_16x16x32_f16 a[100:103], v[172:175], v[116:119], a[100:103]\n"
        "v_cvt_pk_f16_f32 v74, v45, v49\n"
        "v_cvt_pk_f16_f32 v75, v53, v57\n"
        "v_mfma_f32_16x16x32_f16 a[132:135], v[176:179], v[116:119], a[132:135]\n"
        "buffer_load_dwordx4 v[200:203], v64, %[rsA], s41 offen offset:2048 sc0 sc1\n"
        "v_mfma_f32_16x16x32_f16 a[164:167], v[180:183], v[116:119], a[164:167]\n"
        "ds_write_b64 v69, v[74:75] offset:512\n"
        "v_mfma_f32_16x16x32_f16 a[196:199], v[184:187], v[116:119], a[196:199]\n"
        "v_cvt_pk_f16_f32 v76, v46, v50\n"
        "v_cvt_pk_f16_f32 v77, v54, v58\n"
        "v_mfma_f32_16x16x32_f16 a[228:231], v[188:191], v[116:119], a[228:231]\n"
        "s_waitcnt lgkmcnt(4)\n"
        "v_mfma_f32_16x16x32_f16 a[8:11], v[160:163], v[120:123], a[8:11]\n"
        "buffer_load_dwordx4 v[204:207], v64, %[rsA], s41 offen offset:3072 sc0 sc1\n"
        "v_mfma_f32_16x16x32_f16 a[40:43], v[164:167], v[120:123], a[40:43]\n"
        "ds_read_b128 v[116:119], v71 offset:8960\n"
        "v_mfma_f32_16x16x32_f16 a[72:75], v[168:171], v[120:123], a[72:75]\n"
        "ds_write_b64 v69, v[76:77] offset:1024\n"
        "v_mfma_f32_16x16x32_f16 a[104:107], v[172:175], v[120:123], a[104:107]\n"
        "buffer_load_dwordx4 v[208:211], v64, %[rsA], s44 offen offset:0 sc0 sc1\n"
        "v_mfma_f32_16x16x32_f16 a[136:139], v[176:179], v[120:123], a[136:139]\n"
        "v_cvt_pk_f16_f32 v78, v47, v51\n"
        "v_cvt_pk_f16_f32 v79, v55, v59\n"
        "v_mfma_f32_16x16x32_f16 a[168:171], v[180:183], v[120:123], a[168:171]\n"
        "ds_write_b64 v69, v[78:79] offset:1536\n"
        "v_mfma_f32_16x16x32_f16 a[200:203], v[184:187], v[120:123], a[200:203]\n"
        "buffer_load_dwordx4 v[212:215], v64, %[rsA], s44 offen offset:1024 sc0 sc1\n"
        "v_mfma_f32_16x16x32_f16 a[232:235], v[188:191], v[120:123], a[232:235]\n"
        "s_waitcnt lgkmcnt(6)\n"
        "v_mfma_f32_16x16x32_f16 a[12:15], v[160:163], v[124:127], a[12:15]\n"
        "s_cmp_ge_u32 s40, 41\n"
        "s_cselect_b64 s[54:55], -1, s[54:55]\n"
        "v_mfma_f32_16x16x32_f16 a[44:47], v[164:167], v[124:127], a[44:47]\n"
        "ds_read_b128 v[120:123], v71 offset:9472\n"
        "v_mfma_f32_16x16x32_f16 a[76:79], v[168:171], v[124:127], a[76:79]\n"
        "buffer_load_dwordx4 v[216:219], v64, %[rsA], s44 offen offset:2048 sc0 sc1\n"
        "v_mfma_f32_16x16x32_f16 a[108:111], v[172:175], v[124:127], a[108:111]\n"
        "v_readlane_b32 s58, %[tabpf], s40\n"
        "v_mfma_f32_16x16x32_f16 a[140:143], v[176:179], v[124:127], a[140:143]\n"
        "v_mfma_f32_16x16x32_f16 a[172:175], v[180:183], v[124:127], a[172:175]\n"
        "buffer_load_dwordx4 v[220:223], v64, %[rsA], s44 offen offset:3072 sc0 sc1\n"
        "v_mfma_f32_16x16x32_f16 a[204:207], v[184:187], v[124:127], a[204:207]\n"
        "v_mfma_f32_16x16x32_f16 a[236:239], v[188:191], v[124:127], a[236:239]\n"
        "s_waitcnt lgkmcnt(5)\n"
        "v_mfma_f32_16x16x32_f16 a[16:19], v[160:163], v[112:115], a[16:19]\n"
        "buffer_load_dwordx4 v[224:227], v64, %[rsA], s77 offen offset:0 sc0 sc1\n"
        "v_mfma_f32_16x16x32_f16 a[48:51], v[164:167], v[112:115], a[48:51]\n"
        "ds_read_b128 v[124:127], v71 offset:9984\n"
        "v_mfma_f32_16x16x32_f16 a[80:83], v[168:171], v[112:115], a[80:83]\n"
        "v_mfma_f32_16x16x32_f16 a[112:115], v[172:175], v[112:115], a[112:115]\n"
        "buffer_load_dwordx4 v[228:231], v64, %[rsA], s77 offen offset:1024 sc0 sc1\n"
        "v_mfma_f32_16x16x32_f16 a[144:147], v[176:179], v[112:115], a[144:147]\n"
        "v_mfma_f32_16x16x32_f16 a[176:179], v[180:183], v[112:115], a[176:179]\n"
        "v_mfma_f32_16x16x32_f16 a[208:211], v[184:187], v[112:115], a[208:211]\n"
        "buffer_load_dwordx4 v[232:235], v64, %[rsA], s77 offen offset:2048 sc0 sc1\n"
        "v_mfma_f32_16x16x32_f16 a[240:243], v[188:191], v[112:115], a[240:243]\n"
        "s_waitcnt lgkmcnt(4)\n"
        "v_mfma_f32_16x16x32_f16 a[20:23], v[160:163], v[116:119], a[20:23]\n"
        "v_mfma_f32_16x16x32_f16 a[52:55], v[164:167], v[116:119], a[52:55]\n"
        "ds_read_b128 v[112:115], v71 offset:16384\n"
        "v_mfma_f32_16x16x32_f16 a[84:87], v[168:171], v[116:119], a[84:87]\n"
        "buffer_load_dwordx4 v[236:239], v64, %[rsA], s77 offen offset:3072 sc0 sc1\n"
        "v_mfma_f32_16x16x32_f16 a[116:119], v[172:175], v[116:119], a[116:119]\n"
        "v_mfma_f32_16x16x32_f16 a[148:151], v[176:179], v[116:119], a[148:151]\n"
        "v_mfma_f32_16x16x32_f16 a[180:183], v[180:183], v[116:119], a[180:183]\n"
        "buffer_load_dwordx4 v[240:243], v64, %[rsA], s78 offen offset:0 sc0 sc1\n"
        "v_mfma_f32_16x16x32_f16 a[212:215], v[184:187], v[116:119], a[212:215]\n"
        "s_waitcnt vmcnt(14)\n"
        "v_cmp_le_u32 vcc, s71, v61\n"
        "v_mfma_f32_16x16x32_f16 a[244:247], v[188:191], v[116:119], a[244:247]\n"
        "s_waitcnt lgkmcnt(2)\n"
        "v_mfma_f32_16x16x32_f16 a[24:27], v[160:163], v[120:123], a[24:27]\n"
        "buffer_load_dwordx4 v[244:247], v64, %[rsA], s78 offen offset:1024 sc0 sc1\n"
        "s_or_b64 s[46:47], s[46:47], vcc\n"
        "v_mfma_f32_16x16x32_f16 a[56:59], v[164:167], v[120:123], a[56:59]\n"
        "ds_read_b128 v[116:119], v71 offset:16896\n"
        "s_lshl_b64 s[62:63], s[54:55], 8\n"
        "v_mfma_f32_16x16x32_f16 a[88:91], v[168:171], v[120:123], a[88:91]\n"
        "s_andn2_b64 s[62:63], s[62:63], s[46:47]\n"
        "s_cselect_b32 s60, 0, 0x6000000\n"
        "v_mfma_f32_16x16x32_f16 a[120:123], v[172:175], v[120:123], a[120:123]\n"
        "s_ff0_i32_b64 s52, s[46:47]\n"
        "v_mfma_f32_16x16x32_f16 a[152:155], v[176:179], v[120:123], a[152:155]\n"
        "buffer_load_dwordx4 v[248:251], v64, %[rsA], s78 offen offset:2048 sc0 sc1\n"
        "s_bfm_b64 s[88:89], 44, s52\n"
        "v_mfma_f32_16x16x32_f16 a[184:187], v[180:183], v[120:123], a[184:187]\n"
        "s_andn2_b64 s[88:89], s[88:89], s[46:47]\n"
        "v_mfma_f32_16x16x32_f16 a[216:219], v[184:187], v[120:123], a[216:219]\n"
        "s_cmp_lt_u32 s40, 14\n"
        "s_cselect_b64 s[62:63], 0x3fffff, -1\n"
        "v_mfma_f32_16x16x32_f16 a[248:251], v[188:191], v[120:123], a[248:251]\n"
        "s_waitcnt lgkmcnt(2)\n"
        "v_mfma_f32_16x16x32_f16 a[28:31], v[160:163], v[124:127], a[28:31]\n"
        "buffer_load_dwordx4 v[252:255], v64, %[rsA], s78 offen offset:3072 sc0 sc1\n"
        "s_and_b64 s[88:89], s[88:89], s[62:63]\n"
        "v_mfma_f32_16x16x32_f16 a[60:63], v[164:167], v[124:127], a[60:63]\n"
        "ds_read_b128 v[120:123], v71 offset:17408\n"
        "s_or_b32 s88, s88, 1\n"
        "v_mfma_f32_16x16x32_f16 a[92:95], v[168:171], v[124:127], a[92:95]\n"
        "s_cmp_lt_u32 s40, s87\n"
        "s_cbranch_scc1 LSW1_%=\n"
        "LSX1_%=:\n"
        "v_mfma_f32_16x16x32_f16 a[124:127], v[172:175], v[124:127], a[124:127]\n"
        "s_cmp_eq_u32 s40, 10\n"
        "s_cbranch_scc1 LCWN_%=\n"
        "LCXN_%=:\n"
        "v_mfma_f32_16x16x32_f16 a[156:159], v[176:179], v[124:127], a[156:159]\n"
        "s_andn2_b64 s[56:57], s[54:55], s[46:47]\n"
        "s_cbranch_scc1 LS_N1_%=\n"
        "LR_N1_%=:\n"
        "v_mfma_f32_16x16x32_f16 a[188:191], v[180:183], v[124:127], a[188:191]\n"
        "s_bitcmp1_b64 s[46:47], s58\n"
        "s_cselect_b32 s58, s58, 0x6000000\n"
        "v_mfma_f32_16x16x32_f16 a[220:223], v[184:187], v[124:127], a[220:223]\n"
        "s_andn2_b32 s58, s58, 63\n"
        "v_mfma_f32_16x16x32_f16 a[252:255], v[188:191], v[124:127], a[252:255]\n"
        "s_sub_u32 s45, s40, 8\n"
        "s_cmp_lt_u32 s45, 40\n"
        "s_cbranch_scc1 LTPN2_%=\n"
        "LTRN2_%=:\n"
        "s_barrier\n"
        "s_waitcnt vmcnt(8) lgkmcnt(2)\n"
        "v_mfma_f32_16x16x32_f16 a[0:3], v[192:195], v[112:115], a[0:3]\n"
        "buffer_load_dwordx4 v[12:15], %[wv0], %[rsW], s42 offen nt\n"
        "v_readlane_b32 s41, %[tablo], s40\n"
        "v_mfma_f32_16x16x32_f16 a[32:35], v[196:199], v[112:115], a[32:35]\n"
        "ds_read_b128 v[124:127], v71 offset:17920\n"
        "v_readlane_b32 s44, %[tabhi], s40\n"
        "s_add_u32 s40, s40, 1\n"
        "v_mfma_f32_16x16x32_f16 a[64:67], v[200:203], v[112:115], a[64:67]\n"
        "buffer_load_dwordx4 v[16:19], v62, %[rsW], s42 offen nt\n"
        "v_mfma_f32_16x16x32_f16 a[96:99], v[204:207], v[112:115], a[96:99]\n"
        "buffer_load_dwordx4 v[20:23], v63, %[rsW], s42 offen nt\n"
        "v_mfma_f32_16x16x32_f16 a[128:131], v[208:211], v[112:115], a[128:131]\n"
        "buffer_load_dwordx4 v[24:27], v66, %[rsW], s42 offen nt\n"
        "v_mfma_f32_16x16x32_f16 a[160:163], v[212:215], v[112:115], a[160:163]\n"
        "buffer_load_dwordx4 v[44:47], %[wv0], %[rsW], s68 offen nt\n"
        "v_mfma_f32_16x16x32_f16 a[192:195], v[216:219], v[112:115], a[192:195]\n"
        "buffer_load_dwordx4 v[48:51], v62, %[rsW], s68 offen nt\n"
        "v_mfma_f32_16x16x32_f16 a[224:227], v[220:223], v[112:115], a[224:227]\n"
        "s_waitcnt lgkmcnt(2)\n"
        "v_mfma_f32_16x16x32_f16 a[4:7], v[192:195], v[116:119], a[4:7]\n"
        "buffer_load_dwordx4 v[52:55], v63, %[rsW], s68 offen nt\n"
        "v_mfma_f32_16x16x32_f16 a[36:39], v[196:199], v[116:119], a[36:39]\n"
        "ds_read_b128 v[112:115], v71 offset:16640\n"
        "v_mfma_f32_16x16x32_f16 a[68:71], v[200:203], v[116:119], a[68:71]\n"
        "buffer_load_dwordx4 v[56:59], v66, %[rsW], s68 offen nt\n"
        "s_cmp_lt_u32 s40, s87\n"
        "s_cbranch_scc1 LSR2_%=\n"
        "LSB2_%=:\n"
        "s_mov_b64 exec, s[88:89]\n"
        "buffer_load_dword v61, %[cnt], %[rsA], s60 offen sc0 sc1\n"
        "s_mov_b64 exec, -1\n"
        "buffer_load_dword v65, v64, %[rsA], s58 offen sc0 sc1\n"
        "v_mfma_f32_16x16x32_f16 a[100:103], v[204:207], v[116:119], a[100:103]\n"
        "v_readlane_b32 s42, %[tabw], s66\n"
        "s_cmp_lt_u32 s66, 64\n"
        "s_cselect_b32 s42, s42, 0x80000000\n"
        "s_add_u32 s66, s66, 1\n"
        "v_mfma_f32_16x16x32_f16 a[132:135], v[208:211], v[116:119], a[132:135]\n"
        "v_readlane_b32 s68, %[tabw], s66\n"
        "s_cmp_lt_u32 s66, 64\n"
        "s_cselect_b32 s68, s68, 0x80000000\n"
        "s_add_u32 s66, s66, 1\n"
        "v_mfma_f32_16x16x32_f16 a[164:167], v[212:215], v[116:119], a[164:167]\n"
        "s_cmp_eq_u32 s40, s61\n"
        "s_cbranch_scc1 LSENDN2_%=\n"
        "LSENTN2_%=:\n"
        "v_mfma_f32_16x16x32_f16 a[196:199], v[216:219], v[116:119], a[196:199]\n"
        "v_mfma_f32_16x16x32_f16 a[228:231], v[220:223], v[116:119], a[228:231]\n"
        "s_waitcnt lgkmcnt(2)\n"
        "v_mfma_f32_16x16x32_f16 a[8:11], v[192:195], v[120:123], a[8:11]\n"
        "v_add_u32 v69, s67, %[wdst]\n"
        "s_add_u32 s67, s67, 0x2000\n"
        "v_mfma_f32_16x16x32_f16 a[40:43], v[196:199], v[120:123], a[40:43]\n"
        "ds_read_b128 v[116:119], v71 offset:17152\n"
        "v_mfma_f32_16x16x32_f16 a[72:75], v[200:203], v[120:123], a[72:75]\n"
        "s_waitcnt vmcnt(32)\n"
        "s_cmp_lt_u32 s67, 163840\n"
        "s_cselect_b32 s67, s67, 0\n"
        "v_mfma_f32_16x16x32_f16 a[104:107], v[204:207], v[120:123], a[104:107]\n"
        "v_cvt_pk_f16_f32 v72, v80, v84\n"
        "v_cvt_pk_f16_f32 v73, v88, v92\n"
        "v_mfma_f32_16x16x32_f16 a[136:139], v[208:211], v[120:123], a[136:139]\n"
        "ds_write_b64 v69, v[72:73] offset:0\n"
        "v_mfma_f32_16x16x32_f16 a[168:171], v[212:215], v[120:123], a[168:171]\n"
        "v_cvt_pk_f16_f32 v74, v81, v85\n"
        "v_cvt_pk_f16_f32 v75, v89, v93\n"
        "v_mfma_f32_16x16x32_f16 a[200:203], v[216:219], v[120:123], a[200:203]\n"
        "ds_write_b64 v69, v[74:75] offset:512\n"
        "v_mfma_f32_16x16x32_f16 a[232:235], v[220:223], v[120:123], a[232:235]\n"
        "s_waitcnt lgkmcnt(4)\n"
        "v_mfma_f32_16x16x32_f16 a[12:15], v[192:195], v[124:127], a[12:15]\n"
        "v_cvt_pk_f16_f32 v76, v82, v86\n"
        "v_cvt_pk_f16_f32 v77, v90, v94\n"
        "v_mfma_f32_16x16x32_f16 a[44:47], v[196:199], v[124:127], a[44:47]\n"
        "ds_read_b128 v[120:123], v71 offset:17664\n"
        "v_mfma_f32_16x16x32_f16 a[76:79], v[200:203], v[124:127], a[76:79]\n"
        "ds_write_b64 v69, v[76:77] offset:1024\n"
        "v_mfma_f32_16x16x32_f16 a[108:111], v[204:207], v[124:127], a[108:111]\n"
        "v_cvt_pk_f16_f32 v78, v83, v87\n"
        "v_cvt_pk_f16_f32 v79, v91, v95\n"
        "v_mfma_f32_16x16x32_f16 a[140:143], v[208:211], v[124:127], a[140:143]\n"
        "ds_write_b64 v69, v[78:79] offset:1536\n"
        "v_mfma_f32_16x16x32_f16 a[172:175], v[212:215], v[124:127], a[172:175]\n"
        "v_mfma_f32_16x16x32_f16 a[204:207], v[216:219], v[124:127], a[204:207]\n"
        "v_mfma_f32_16x16x32_f16 a[236:239], v[220:223], v[124:127], a[236:239]\n"
        "s_waitcnt lgkmcnt(6)\n"
        "v_mfma_f32_16x16x32_f16 a[16:19], v[192:195], v[112:115], a[16:19]\n"
        "v_mfma_f32_16x16x32_f16 a[48:51], v[196:199], v[112:115], a[48:51]\n"
        "ds_read_b128 v[124:127], v71 offset:18176\n"
        "v_mfma_f32_16x16x32_f16 a[80:83], v[200:203], v[112:115], a[80:83]\n"
        "v_mfma_f32_16x16x32_f16 a[112:115], v[204:207], v[112:115], a[112:115]\n"
        "v_mfma_f32_16x16x32_f16 a[144:147], v[208:211], v[112:115], a[144:147]\n"
        "v_mfma_f32_16x16x32_f16 a[176:179], v[212:215], v[112:115], a[176:179]\n"
        "v_mfma_f32_16x16x32_f16 a[208:211], v[216:219], v[112:115], a[208:211]\n"
        "v_mfma_f32_16x16x32_f16 a[240:243], v[220:223], v[112:115], a[240:243]\n"
        "s_waitcnt lgkmcnt(6)\n"
        "v_mfma_f32_16x16x32_f16 a[20:23], v[192:195], v[116:119], a[20:23]\n"
        "v_mfma_f32_16x16x32_f16 a[52:55], v[196:199], v[116:119], a[52:55]\n"
        "ds_read_b128 v[112:115], v71 offset:24576\n"
        "v_mfma_f32_16x16x32_f16 a[84:87], v[200:203], v[116:119], a[84:87]\n"
        "v_mfma_f32_16x16x32_f16 a[116:119], v[204:207], v[116:119], a[116:119]\n"
        "v_mfma_f32_16x16x32_f16 a[148:151], v[208:211], v[116:119], a[148:151]\n"
        "v_mfma_f32_16x16x32_f16 a[180:183], v[212:215], v[116:119], a[180:183]\n"
        "v_mfma_f32_16x16x32_f16 a[212:215], v[216:219], v[116:119], a[212:215]\n"
        "v_mfma_f32_16x16x32_f16 a[244:247], v[220:223], v[116:119], a[244:247]\n"
        "s_waitcnt lgkmcnt(4)\n"
        "v_mfma_f32_16x16x32_f16 a[24:27], v[192:195], v[120:123], a[24:27]\n"
        "v_mfma_f32_16x16x32_f16 a[56:59], v[196:199], v[120:123], a[56:59]\n"
        "ds_read_b128 v[116:119], v71 offset:25088\n"
        "v_mfma_f32_16x16x32_f16 a[88:91], v[200:203], v[120:123], a[88:91]\n"
        "v_mfma_f32_16x16x32_f16 a[120:123], v[204:207], v[120:123], a[120:123]\n"
        "v_mfma_f32_16x16x32_f16 a[152:155], v[208:211], v[120:123], a[152:155]\n"
        "v_mfma_f32_16x16x32_f16 a[184:187], v[212:215], v[120:123], a[184:187]\n"
        "v_mfma_f32_16x16x32_f16 a[216:219], v[216:219], v[120:123], a[216:219]\n"
        "v_mfma_f32_16x16x32_f16 a[248:251], v[220:223], v[120:123], a[248:251]\n"
        "s_waitcnt lgkmcnt(2)\n"
        "v_mfma_f32_16x16x32_f16 a[28:31], v[192:195], v[124:127], a[28:31]\n"
        "v_mfma_f32_16x16x32_f16 a[60:63], v[196:199], v[124:127], a[60:63]\n"
        "ds_read_b128 v[120:123], v71 offset:25600\n"
        "v_mfma_f32_16x16x32_f16 a[92:95], v[200:203], v[124:127], a[92:95]\n"
        "v_mfma_f32_16x16x32_f16 a[124:127], v[204:207], v[124:127], a[124:127]\n"
        "v_mfma_f32_16x16x32_f16 a[156:159], v[208:211], v[124:127], a[156:159]\n"
        "v_mfma_f32_16x16x32_f16 a[188:191], v[212:215], v[124:127], a[188:191]\n"
        "v_mfma_f32_16x16x32_f16 a[220:223], v[216:219], v[124:127], a[220:223]\n"
        "v_mfma_f32_16x16x32_f16 a[252:255], v[220:223], v[124:127], a[252:255]\n"
        "s_barrier\n"
        "s_waitcnt vmcnt(10) lgkmcnt(2)\n"
        "v_mfma_f32_16x16x32_f16 a[0:3], v[224:227], v[112:115], a[0:3]\n"
        "v_readlane_b32 s77, %[tablo], s40\n"
        "v_mfma_f32_16x16x32_f16 a[32:35], v[228:231], v[112:115], a[32:35]\n"
        "ds_read_b128 v[124:127], v71 offset:26112\n"
        "v_readlane_b32 s78, %[tabhi], s40\n"
        "s_add_u32 s40, s40, 1\n"
        "v_mfma_f32_16x16x32_f16 a[64:67], v[232:235], v[112:115], a[64:67]\n"
        "s_cmp_eq_u32 s40, s61\n"
        "s_cbranch_scc1 LSENDN3_%=\n"
        "LSENTN3_%=:\n"
        "v_mfma_f32_16x16x32_f16 a[96:99], v[236:239], v[112:115], a[96:99]\n"
        "v_add_u32 v69, s67, %[wdst]\n"
        "s_add_u32 s67, s67, 0x2000\n"
        "v_mfma_f32_16x16x32_f16 a[128:131], v[240:243], v[112:115], a[128:131]\n"
        "s_waitcnt vmcnt(28)\n"
        "s_cmp_lt_u32 s67, 163840\n"
        "s_cselect_b32 s67, s67, 0\n"
        "v_mfma_f32_16x16x32_f16 a[160:163], v[244:247], v[112:115], a[160:163]\n"
        "v_cvt_pk_f16_f32 v72, v96, v100\n"
        "v_cvt_pk_f16_f32 v73, v104, v108\n"
        "v_mfma_f32_16x16x32_f16 a[192:195], v[248:251], v[112:115], a[192:195]\n"
        "buffer_load_dwordx4 v[128:131], v64, %[rsA], s41 offen offset:0 sc0 sc1\n"
        "s_lshl_b64 s[54:55], s[54:55], 2\n"
        "v_mfma_f32_16x16x32_f16 a[224:227], v[252:255], v[112:115], a[224:227]\n"
        "s_waitcnt lgkmcnt(2)\n"
        "v_mfma_f32_16x16x32_f16 a[4:7], v[224:227], v[116:119], a[4:7]\n"
        "ds_write_b64 v69, v[72:73] offset:0\n"
        "s_or_b32 s54, s54, 3\n"
        "v_mfma_f32_16x16x32_f16 a[36:39], v[228:231], v[116:119], a[36:39]\n"
        "buffer_load_dwordx4 v[132:135], v64, %[rsA], s41 offen offset:1024 sc0 sc1\n"
        "s_add_u32 s69, s69, 0x8000\n"
        "v_mfma_f32_16x16x32_f16 a[68:71], v[232:235], v[116:119], a[68:71]\n"
        "ds_read_b128 v[112:115], v71 offset:24832\n"
        "v_mfma_f32_16x16x32_f16 a[100:103], v[236:239], v[116:119], a[100:103]\n"
        "v_cvt_pk_f16_f32 v74, v97, v101\n"
        "v_cvt_pk_f16_f32 v75, v105, v109\n"
        "v_mfma_f32_16x16x32_f16 a[132:135], v[240:243], v[116:119], a[132:135]\n"
        "buffer_load_dwordx4 v[136:139], v64, %[rsA], s41 offen offset:2048 sc0 sc1\n"
        "v_mfma_f32_16x16x32_f16 a[164:167], v[244:247], v[116:119], a[164:167]\n"
        "ds_write_b64 v69, v[74:75] offset:512\n"
        "v_mfma_f32_16x16x32_f16 a[196:199], v[248:251], v[116:119], a[196:199]\n"
        "v_cvt_pk_f16_f32 v76, v98, v102\n"
        "v_cvt_pk_f16_f32 v77, v106, v110\n"
        "v_mfma_f32_16x16x32_f16 a[228:231], v[252:255], v[116:119], a[228:231]\n"
        "s_waitcnt lgkmcnt(4)\n"
        "v_mfma_f32_16x16x32_f16 a[8:11], v[224:227], v[120:123], a[8:11]\n"
        "buffer_load_dwordx4 v[140:143], v64, %[rsA], s41 offen offset:3072 sc0 sc1\n"
        "v_mfma_f32_16x16x32_f16 a[40:43], v[228:231], v[120:123], a[40:43]\n"
        "ds_read_b128 v[116:119], v71 offset:25344\n"
        "v_mfma_f32_16x16x32_f16 a[72:75], v[232:235], v[120:123], a[72:75]\n"
        "ds_write_b64 v69, v[76:77] offset:1024\n"
        "v_mfma_f32_16x16x32_f16 a[104:107], v[236:239], v[120:123], a[104:107]\n"
        "buffer_load_dwordx4 v[144:147], v64, %[rsA], s44 offen offset:0 sc0 sc1\n"
        "v_mfma_f32_16x16x32_f16 a[136:139], v[240:243], v[120:123], a[136:139]\n"
        "v_cvt_pk_f16_f32 v78, v99, v103\n"
        "v_cvt_pk_f16_f32 v79, v107, v111\n"
        "v_mfma_f32_16x16x32_f16 a[168:171], v[244:247], v[120:123], a[168:171]\n"
        "ds_write_b64 v69, v[78:79] offset:1536\n"
        "v_mfma_f32_16x16x32_f16 a[200:203], v[248:251], v[120:123], a[200:203]\n"
        "buffer_load_dwordx4 v[148:151], v64, %[rsA], s44 offen offset:1024 sc0 sc1\n"
        "v_mfma_f32_16x16x32_f16 a[232:235], v[252:255], v[120:123], a[232:235]\n"
        "s_waitcnt lgkmcnt(6)\n"
        "v_mfma_f32_16x16x32_f16 a[12:15], v[224:227], v[124:127], a[12:15]\n"
        "s_cmp_ge_u32 s40, 41\n"
        "s_cselect_b64 s[54:55], -1, s[54:55]\n"
        "v_mfma_f32_16x16x32_f16 a[44:47], v[228:231], v[124:127], a[44:47]\n"
        "ds_read_b128 v[120:123], v71 offset:25856\n"
        "v_mfma_f32_16x16x32_f16 a[76:79], v[232:235], v[124:127], a[76:79]\n"
        "buffer_load_dwordx4 v[152:155], v64, %[rsA], s44 offen offset:2048 sc0 sc1\n"
        "v_mfma_f32_16x16x32_f16 a[108:111], v[236:239], v[124:127], a[108:111]\n"
        "v_readlane_b32 s58, %[tabpf], s40\n"
        "v_mfma_f32_16x16x32_f16 a[140:143], v[240:243], v[124:127], a[140:143]\n"
        "s_cmp_lt_u32 s69, 163840\n"
        "s_cselect_b32 s69, s69, 0\n"
        "v_mfma_f32_16x16x32_f16 a[172:175], v[244:247], v[124:127], a[172:175]\n"
        "buffer_load_dwordx4 v[156:159], v64, %[rsA], s44 offen offset:3072 sc0 sc1\n"
        "v_mfma_f32_16x16x32_f16 a[204:207], v[248:251], v[124:127], a[204:207]\n"
        "v_mfma_f32_16x16x32_f16 a[236:239], v[252:255], v[124:127], a[236:239]\n"
        "s_waitcnt lgkmcnt(5)\n"
        "v_mfma_f32_16x16x32_f16 a[16:19], v[224:227], v[112:115], a[16:19]\n"
        "buffer_load_dwordx4 v[160:163], v64, %[rsA], s77 offen offset:0 sc0 sc1\n"
        "v_mfma_f32_16x16x32_f16 a[48:51], v[228:231], v[112:115], a[48:51]\n"
        "ds_read_b128 v[124:127], v71 offset:26368\n"
        "v_mfma_f32_16x16x32_f16 a[80:83], v[232:235], v[112:115], a[80:83]\n"
        "v_mov_b32 v71, v70\n"
        "v_mfma_f32_16x16x32_f16 a[112:115], v[236:239], v[112:115], a[112:115]\n"
        "buffer_load_dwordx4 v[164:167], v64, %[rsA], s77 offen offset:1024 sc0 sc1\n"
        "v_mfma_f32_16x16x32_f16 a[144:147], v[240:243], v[112:115], a[144:147]\n"
        "v_mfma_f32_16x16x32_f16 a[176:179], v[244:247], v[112:115], a[176:179]\n"
        "v_mfma_f32_16x16x32_f16 a[208:211], v[248:251], v[112:115], a[208:211]\n"
        "buffer_load_dwordx4 v[168:171], v64, %[rsA], s77 offen offset:2048 sc0 sc1\n"
        "v_mfma_f32_16x16x32_f16 a[240:243], v[252:255], v[112:115], a[240:243]\n"
        "s_waitcnt lgkmcnt(4)\n"
        "v_mfma_f32_16x16x32_f16 a[20:23], v[224:227], v[116:119], a[20:23]\n"
        "v_mfma_f32_16x16x32_f16 a[52:55], v[228:231], v[116:119], a[52:55]\n"
        "ds_read_b128 v[112:115], v70 offset:0\n"
        "v_mfma_f32_16x16x32_f16 a[84:87], v[232:235], v[116:119], a[84:87]\n"
        "buffer_load_dwordx4 v[172:175], v64, %[rsA], s77 offen offset:3072 sc0 sc1\n"
        "v_mfma_f32_16x16x32_f16 a[116:119], v[236:239], v[116:119], a[116:119]\n"
        "v_mfma_f32_16x16x32_f16 a[148:151], v[240:243], v[116:119], a[148:151]\n"
        "v_mfma_f32_16x16x32_f16 a[180:183], v[244:247], v[116:119], a[180:183]\n"
        "buffer_load_dwordx4 v[176:179], v64, %[rsA], s78 offen offset:0 sc0 sc1\n"
        "v_mfma_f32_16x16x32_f16 a[212:215], v[248:251], v[116:119], a[212:215]\n"
        "s_waitcnt vmcnt(14)\n"
        "v_cmp_le_u32 vcc, s71, v61\n"
        "v_mfma_f32_16x16x32_f16 a[244:247], v[252:255], v[116:119], a[244:247]\n"
        "s_waitcnt lgkmcnt(2)\n"
        "v_mfma_f32_16x16x32_f16 a[24:27], v[224:227], v[120:123], a[24:27]\n"
        "buffer_load_dwordx4 v[180:183], v64, %[rsA], s78 offen offset:1024 sc0 sc1\n"
        "s_or_b64 s[46:47], s[46:47], vcc\n"
        "v_mfma_f32_16x16x32_f16 a[56:59], v[228:231], v[120:123], a[56:59]\n"
        "ds_read_b128 v[116:119], v70 offset:512\n"
        "s_lshl_b64 s[62:63], s[54:55], 8\n"
        "v_mfma_f32_16x16x32_f16 a[88:91], v[232:235], v[120:123], a[88:91]\n"
        "s_andn2_b64 s[62:63], s[62:63], s[46:47]\n"
        "s_cselect_b32 s60, 0, 0x6000000\n"
        "v_mfma_f32_16x16x32_f16 a[120:123], v[236:239], v[120:123], a[120:123]\n"
        "s_ff0_i32_b64 s52, s[46:47]\n"
        "v_mfma_f32_16x16x32_f16 a[152:155], v[240:243], v[120:123], a[152:155]\n"
        "buffer_load_dwordx4 v[184:187], v64, %[rsA], s78 offen offset:2048 sc0 sc1\n"
        "s_bfm_b64 s[88:89], 44, s52\n"
        "v_mfma_f32_16x16x32_f16 a[184:187], v[244:247], v[120:123], a[184:187]\n"
        "s_andn2_b64 s[88:89], s[88:89], s[46:47]\n"
        "v_mfma_f32_16x16x32_f16 a[216:219], v[248:251], v[120:123], a[216:219]\n"
        "s_cmp_lt_u32 s40, 14\n"
        "s_cselect_b64 s[62:63], 0x3fffff, -1\n"
        "v_mfma_f32_16x16x32_f16 a[248:251], v[252:255], v[120:123], a[248:251]\n"
        "s_waitcnt lgkmcnt(2)\n"
        "v_mfma_f32_16x16x32_f16 a[28:31], v[224:227], v[124:127], a[28:31]\n"
        "buffer_load_dwordx4 v[188:191], v64, %[rsA], s78 offen offset:3072 sc0 sc1\n"
        "s_and_b64 s[88:89], s[88:89], s[62:63]\n"
        "v_mfma_f32_16x16x32_f16 a[60:63], v[228:231], v[124:127], a[60:63]\n"
        "ds_read_b128 v[120:123], v70 offset:1024\n"
        "s_or_b32 s88, s88, 1\n"
        "v_mfma_f32_16x16x32_f16 a[92:95], v[232:235], v[124:127], a[92:95]\n"
        "s_cmp_lt_u32 s40, s87\n"
        "s_cbranch_scc1 LSW3_%=\n"
        "LSX3_%=:\n"
        "v_mfma_f32_16x16x32_f16 a[124:127], v[236:239], v[124:127], a[124:127]\n"
        "s_andn2_b64 s[56:57], s[54:55], s[46:47]\n"
        "s_cbranch_scc1 LS_N3_%=\n"
        "LR_N3_%=:\n"
        "v_mfma_f32_16x16x32_f16 a[156:159], v[240:243], v[124:127], a[156:159]\n"
        "s_bitcmp1_b64 s[46:47], s58\n"
        "s_cselect_b32 s58, s58, 0x6000000\n"
        "v_mfma_f32_16x16x32_f16 a[188:191], v[244:247], v[124:127], a[188:191]\n"
        "s_andn2_b32 s58, s58, 63\n"
        "v_mfma_f32_16x16x32_f16 a[220:223], v[248:251], v[124:127], a[220:223]\n"
        "v_add_u32 v70, s69, %[bbase]\n"
        "v_mfma_f32_16x16x32_f16 a[252:255], v[252:255], v[124:127], a[252:255]\n"
        "s_cmp_eq_u32 s40, 8\n"
        "s_cbranch_scc1 LCLP_%=\n"
        "s_cmp_eq_u32 s40, 12\n"
        "s_cbranch_scc1 LCLS_%=\n"
        "LBACK_%=:\n"
        "s_cmp_eq_u32 s82, 0\n"
        "s_cbranch_scc1 LNOH_%=\n"
        "s_cmp_ge_u32 s40, s81\n"
        "s_cbranch_scc1 LOOPH_%=\n"
        "s_add_u32 s45, s40, s82\n"
        "s_cmp_ge_u32 s45, 44\n"
        "s_cbranch_scc1 LOOPH_%=\n"
        "LNOH_%=:\n"
        "s_cmp_lt_u32 s40, 44\n"
        "s_cbranch_scc1 LOOPN_%=\n"
        "s_waitcnt vmcnt(16) lgkmcnt(2)\n"
        "s_waitcnt vmcnt(8)\n"
        "s_barrier\n"
        "v_add_u32 v69, 16384, %[wdst]\n"
        "v_cvt_pk_f16_f32 v72, v12, v16\n"
        "v_cvt_pk_f16_f32 v73, v20, v24\n"
        "ds_write_b64 v69, v[72:73] offset:0\n"
        "v_cvt_pk_f16_f32 v74, v13, v17\n"
        "v_cvt_pk_f16_f32 v75, v21, v25\n"
        "ds_write_b64 v69, v[74:75] offset:512\n"
        "v_cvt_pk_f16_f32 v76, v14, v18\n"
        "v_cvt_pk_f16_f32 v77, v22, v26\n"
        "ds_write_b64 v69, v[76:77] offset:1024\n"
        "v_cvt_pk_f16_f32 v78, v15, v19\n"
        "v_cvt_pk_f16_f32 v79, v23, v27\n"
        "ds_write_b64 v69, v[78:79] offset:1536\n"
        "v_add_u32 v69, 24576, %[wdst]\n"
        "v_cvt_pk_f16_f32 v72, v44, v48\n"
        "v_cvt_pk_f16_f32 v73, v52, v56\n"
        "ds_write_b64 v69, v[72:73] offset:0\n"
        "v_cvt_pk_f16_f32 v74, v45, v49\n"
        "v_cvt_pk_f16_f32 v75, v53, v57\n"
        "ds_write_b64 v69, v[74:75] offset:512\n"
        "v_cvt_pk_f16_f32 v76, v46, v50\n"
        "v_cvt_pk_f16_f32 v77, v54, v58\n"
        "ds_write_b64 v69, v[76:77] offset:1024\n"
        "v_cvt_pk_f16_f32 v78, v47, v51\n"
        "v_cvt_pk_f16_f32 v79, v55, v59\n"
        "ds_write_b64 v69, v[78:79] offset:1536\n"
        "v_add_u32 v70, 0x10000, %[bbase]\n"
        "v_add_u32 v71, 0x20000, %[bbase]\n"
        "v_and_b32 v61, 0x3ff, v64\n"
        "s_lshl_b32 s66, s70, 2\n"
        "s_sub_u32 s66, s66, 4\n"
        "s_mov_b32 s67, 0\n"
        "s_waitcnt vmcnt(12) lgkmcnt(10)\n"
        "v_mfma_f32_16x16x32_f16 a[128:131], v[128:131], v[112:115], a[128:131]\n"
        "ds_read_b128 v[124:127], %[bbase] offset:34304\n"
        "v_mfma_f32_16x16x32_f16 a[160:163], v[132:135], v[112:115], a[160:163]\n"
        "ds_read_b128 v[72:75], %[bbase] offset:33024\n"
        "s_waitcnt lgkmcnt(11)\n"
        "v_mfma_f32_16x16x32_f16 a[132:135], v[128:131], v[116:119], a[132:135]\n"
        "ds_read_b128 v[76:79], %[bbase] offset:33536\n"
        "v_mfma_f32_16x16x32_f16 a[164:167], v[132:135], v[116:119], a[164:167]\n"
        "ds_read_b128 v[104:107], %[bbase] offset:34048\n"
        "s_waitcnt lgkmcnt(12)\n"
        "v_mfma_f32_16x16x32_f16 a[136:139], v[128:131], v[120:123], a[136:139]\n"
        "ds_read_b128 v[108:111], %[bbase] offset:34560\n"
        "v_mfma_f32_16x16x32_f16 a[168:171], v[132:135], v[120:123], a[168:171]\n"
        "ds_read_b128 v[112:115], %[bbase] offset:40960\n"
        "s_waitcnt lgkmcnt(5)\n"
        "v_mfma_f32_16x16x32_f16 a[140:143], v[128:131], v[124:127], a[140:143]\n"
        "ds_read_b128 v[116:119], %[bbase] offset:41472\n"
        "v_mfma_f32_16x16x32_f16 a[172:175], v[132:135], v[124:127], a[172:175]\n"
        "ds_read_b128 v[120:123], %[bbase] offset:41984\n"
        "s_waitcnt lgkmcnt(6)\n"
        "v_mfma_f32_16x16x32_f16 a[144:147], v[128:131], v[72:75], a[144:147]\n"
        "ds_read_b128 v[124:127], %[bbase] offset:42496\n"
        "v_mfma_f32_16x16x32_f16 a[176:179], v[132:135], v[72:75], a[176:179]\n"
        "v_readlane_b32 s44, %[tabh], 4\n"
        "s_xor_b32 s44, s44, 0x1000\n"
        "s_waitcnt lgkmcnt(6)\n"
        "v_mfma_f32_16x16x32_f16 a[148:151], v[128:131], v[76:79], a[148:151]\n"
        "ds_read_b128 v[72:75], %[bbase] offset:41216\n"
        "v_mfma_f32_16x16x32_f16 a[180:183], v[132:135], v[76:79], a[180:183]\n"
        "buffer_load_dwordx4 v[192:195], v64, %[rsA], s44 offen offset:0 sc0 sc1\n"
        "s_waitcnt lgkmcnt(6)\n"
        "v_mfma_f32_16x16x32_f16 a[152:155], v[128:131], v[104:107], a[152:155]\n"
        "ds_read_b128 v[76:79], %[bbase] offset:41728\n"
        "v_mfma_f32_16x16x32_f16 a[184:187], v[132:135], v[104:107], a[184:187]\n"
        "s_waitcnt lgkmcnt(6)\n"
        "v_mfma_f32_16x16x32_f16 a[156:159], v[128:131], v[108:111], a[156:159]\n"
        "ds_read_b128 v[104:107], %[bbase] offset:42240\n"
        "v_mfma_f32_16x16x32_f16 a[188:191], v[132:135], v[108:111], a[188:191]\n"
        "buffer_load_dwordx4 v[196:199], v64, %[rsA], s44 offen offset:1024 sc0 sc1\n"
        "s_waitcnt vmcnt(10) lgkmcnt(6)\n"
        "v_mfma_f32_16x16x32_f16 a[128:131], v[144:147], v[112:115], a[128:131]\n"
        "ds_read_b128 v[108:111], %[bbase] offset:42752\n"
        "v_mfma_f32_16x16x32_f16 a[160:163], v[148:151], v[112:115], a[160:163]\n"
        "s_waitcnt lgkmcnt(6)\n"
        "v_mfma_f32_16x16x32_f16 a[132:135], v[144:147], v[116:119], a[132:135]\n"
        "ds_read_b128 v[112:115], %[bbase] offset:49152\n"
        "v_mfma_f32_16x16x32_f16 a[164:167], v[148:151], v[116:119], a[164:167]\n"
        "v_readlane_b32 s78, %[tabh], 5\n"
        "s_xor_b32 s78, s78, 0x1000\n"
        "s_waitcnt lgkmcnt(6)\n"
        "v_mfma_f32_16x16x32_f16 a[136:139], v[144:147], v[120:123], a[136:139]\n"
        "ds_read_b128 v[116:119], %[bbase] offset:49664\n"
        "v_mfma_f32_16x16x32_f16 a[168:171], v[148:151], v[120:123], a[168:171]\n"
        "buffer_load_dwordx4 v[200:203], v64, %[rsA], s78 offen offset:0 sc0 sc1\n"
        "s_waitcnt lgkmcnt(6)\n"
        "v_mfma_f32_16x16x32_f16 a[140:143], v[144:147], v[124:127], a[140:143]\n"
        "ds_read_b128 v[120:123], %[bbase] offset:50176\n"
        "v_mfma_f32_16x16x32_f16 a[172:175], v[148:151], v[124:127], a[172:175]\n"
        "s_waitcnt lgkmcnt(6)\n"
        "v_mfma_f32_16x16x32_f16 a[144:147], v[144:147], v[72:75], a[144:147]\n"
        "ds_read_b128 v[124:127], %[bbase] offset:50688\n"
        "v_mfma_f32_16x16x32_f16 a[176:179], v[148:151], v[72:75], a[176:179]\n"
        "buffer_load_dwordx4 v[204:207], v64, %[rsA], s78 offen offset:1024 sc0 sc1\n"
        "s_waitcnt lgkmcnt(6)\n"
        "v_mfma_f32_16x16x32_f16 a[148:151], v[144:147], v[76:79], a[148:151]\n"
        "ds_read_b128 v[72:75], %[bbase] offset:49408\n"
        "v_mfma_f32_16x16x32_f16 a[180:183], v[148:151], v[76:79], a[180:183]\n"
        "s_waitcnt lgkmcnt(6)\n"
        "v_mfma_f32_16x16x32_f16 a[152:155], v[144:147], v[104:107], a[152:155]\n"
        "ds_read_b128 v[76:79], %[bbase] offset:49920\n"
        "v_mfma_f32_16x16x32_f16 a[184:187], v[148:151], v[104:107], a[184:187]\n"
        "v_readlane_b32 s44, %[tabh], 6\n"
        "s_xor_b32 s44, s44, 0x1000\n"
        "s_waitcnt lgkmcnt(6)\n"
        "v_mfma_f32_16x16x32_f16 a[156:159], v[144:147], v[108:111], a[156:159]\n"
        "ds_read_b128 v[104:107], %[bbase] offset:50432\n"
        "v_mfma_f32_16x16x32_f16 a[188:191], v[148:151], v[108:111], a[188:191]\n"
        "buffer_load_dwordx4 v[208:211], v64, %[rsA], s44 offen offset:0 sc0 sc1\n"
        "s_waitcnt vmcnt(9) lgkmcnt(6)\n"
        "v_mfma_f32_16x16x32_f16 a[128:131], v[160:163], v[112:115], a[128:131]\n"
        "ds_read_b128 v[108:111], %[bbase] offset:50944\n"
        "v_mfma_f32_16x16x32_f16 a[160:163], v[164:167], v[112:115], a[160:163]\n"
        "s_waitcnt lgkmcnt(6)\n"
        "v_mfma_f32_16x16x32_f16 a[132:135], v[160:163], v[116:119], a[132:135]\n"
        "ds_read_b128 v[112:115], %[bbase] offset:57344\n"
        "v_mfma_f32_16x16x32_f16 a[164:167], v[164:167], v[116:119], a[164:167]\n"
        "buffer_load_dwordx4 v[212:215], v64, %[rsA], s44 offen offset:1024 sc0 sc1\n"
        "s_waitcnt lgkmcnt(6)\n"
        "v_mfma_f32_16x16x32_f16 a[136:139], v[160:163], v[120:123], a[136:139]\n"
        "ds_read_b128 v[116:119], %[bbase] offset:57856\n"
        "v_mfma_f32_16x16x32_f16 a[168:171], v[164:167], v[120:123], a[168:171]\n"
        "s_waitcnt lgkmcnt(6)\n"
        "v_mfma_f32_16x16x32_f16 a[140:143], v[160:163], v[124:127], a[140:143]\n"
        "ds_read_b128 v[120:123], %[bbase] offset:58368\n"
        "v_mfma_f32_16x16x32_f16 a[172:175], v[164:167], v[124:127], a[172:175]\n"
        "v_readlane_b32 s78, %[tabh], 7\n"
        "s_xor_b32 s78, s78, 0x1000\n"
        "s_waitcnt lgkmcnt(6)\n"
        "v_mfma_f32_16x16x32_f16 a[144:147], v[160:163], v[72:75], a[144:147]\n"
        "ds_read_b128 v[124:127], %[bbase] offset:58880\n"
        "v_mfma_f32_16x16x32_f16 a[176:179], v[164:167], v[72:75], a[176:179]\n"
        "buffer_load_dwordx4 v[216:219], v64, %[rsA], s78 offen offset:0 sc0 sc1\n"
        "s_waitcnt lgkmcnt(6)\n"
        "v_mfma_f32_16x16x32_f16 a[148:151], v[160:163], v[76:79], a[148:151]\n"
        "ds_read_b128 v[72:75], %[bbase] offset:57600\n"
        "v_mfma_f32_16x16x32_f16 a[180:183], v[164:167], v[76:79], a[180:183]\n"
        "s_waitcnt lgkmcnt(6)\n"
        "v_mfma_f32_16x16x32_f16 a[152:155], v[160:163], v[104:107], a[152:155]\n"
        "ds_read_b128 v[76:79], %[bbase] offset:58112\n"
        "v_mfma_f32_16x16x32_f16 a[184:187], v[164:167], v[104:107], a[184:187]\n"
        "buffer_load_dwordx4 v[220:223], v64, %[rsA], s78 offen offset:1024 sc0 sc1\n"
        "s_waitcnt lgkmcnt(6)\n"
        "v_mfma_f32_16x16x32_f16 a[156:159], v[160:163], v[108:111], a[156:159]\n"
        "ds_read_b128 v[104:107], %[bbase] offset:58624\n"
        "v_mfma_f32_16x16x32_f16 a[188:191], v[164:167], v[108:111], a[188:191]\n"
        "s_waitcnt vmcnt(8) lgkmcnt(6)\n"
        "v_mfma_f32_16x16x32_f16 a[128:131], v[176:179], v[112:115], a[128:131]\n"
        "ds_read_b128 v[108:111], %[bbase] offset:59136\n"
        "v_mfma_f32_16x16x32_f16 a[160:163], v[180:183], v[112:115], a[160:163]\n"
        "v_readlane_b32 s44, %[tabh], 8\n"
        "s_xor_b32 s44, s44, 0x1000\n"
        "s_waitcnt lgkmcnt(6)\n"
        "v_mfma_f32_16x16x32_f16 a[132:135], v[176:179], v[116:119], a[132:135]\n"
        "ds_read_b128 v[112:115], v70 offset:0\n"
        "v_mfma_f32_16x16x32_f16 a[164:167], v[180:183], v[116:119], a[164:167]\n"
        "buffer_load_dwordx4 v[224:227], v64, %[rsA], s44 offen offset:0 sc0 sc1\n"
        "s_waitcnt lgkmcnt(6)\n"
        "v_mfma_f32_16x16x32_f16 a[136:139], v[176:179], v[120:123], a[136:139]\n"
        "ds_read_b128 v[116:119], v70 offset:512\n"
        "v_mfma_f32_16x16x32_f16 a[168:171], v[180:183], v[120:123], a[168:171]\n"
        "s_waitcnt lgkmcnt(6)\n"
        "v_mfma_f32_16x16x32_f16 a[140:143], v[176:179], v[124:127], a[140:143]\n"
        "ds_read_b128 v[120:123], v70 offset:1024\n"
        "v_mfma_f32_16x16x32_f16 a[172:175], v[180:183], v[124:127], a[172:175]\n"
        "buffer_load_dwordx4 v[228:231], v64, %[rsA], s44 offen offset:1024 sc0 sc1\n"
        "s_waitcnt lgkmcnt(6)\n"
        "v_mfma_f32_16x16x32_f16 a[144:147], v[176:179], v[72:75], a[144:147]\n"
        "ds_read_b128 v[124:127], v70 offset:1536\n"
        "v_mfma_f32_16x16x32_f16 a[176:179], v[180:183], v[72:75], a[176:179]\n"
        "s_waitcnt lgkmcnt(6)\n"
        "v_mfma_f32_16x16x32_f16 a[148:151], v[176:179], v[76:79], a[148:151]\n"
        "ds_read_b128 v[72:75], v70 offset:256\n"
        "v_mfma_f32_16x16x32_f16 a[180:183], v[180:183], v[76:79], a[180:183]\n"
        "v_readlane_b32 s78, %[tabh], 9\n"
        "s_xor_b32 s78, s78, 0x1000\n"
        "s_waitcnt lgkmcnt(6)\n"
        "v_mfma_f32_16x16x32_f16 a[152:155], v[176:179], v[104:107], a[152:155]\n"
        "ds_read_b128 v[76:79], v70 offset:768\n"
        "v_mfma_f32_16x16x32_f16 a[184:187], v[180:183], v[104:107], a[184:187]\n"
        "buffer_load_dwordx4 v[232:235], v64, %[rsA], s78 offen offset:0 sc0 sc1\n"
        "s_waitcnt lgkmcnt(6)\n"
        "v_mfma_f32_16x16x32_f16 a[156:159], v[176:179], v[108:111], a[156:159]\n"
        "ds_read_b128 v[104:107], v70 offset:1280\n"
        "v_mfma_f32_16x16x32_f16 a[188:191], v[180:183], v[108:111], a[188:191]\n"
        "s_waitcnt lgkmcnt(15)\n"
        "s_barrier\n"
        "s_waitcnt vmcnt(9) lgkmcnt(6)\n"
        "v_mfma_f32_16x16x32_f16 a[128:131], v[192:195], v[112:115], a[128:131]\n"
        "ds_read_b128 v[108:111], v70 offset:1792\n"
        "v_mfma_f32_16x16x32_f16 a[160:163], v[196:199], v[112:115], a[160:163]\n"
        "buffer_load_dwordx4 v[236:239], v64, %[rsA], s78 offen offset:1024 sc0 sc1\n"
        "s_waitcnt lgkmcnt(6)\n"
        "v_mfma_f32_16x16x32_f16 a[132:135], v[192:195], v[116:119], a[132:135]\n"
        "ds_read_b128 v[112:115], v70 offset:8192\n"
        "v_mfma_f32_16x16x32_f16 a[164:167], v[196:199], v[116:119], a[164:167]\n"
        "s_waitcnt lgkmcnt(6)\n"
        "v_mfma_f32_16x16x32_f16 a[136:139], v[192:195], v[120:123], a[136:139]\n"
        "ds_read_b128 v[116:119], v70 offset:8704\n"
        "v_mfma_f32_16x16x32_f16 a[168:171], v[196:199], v[120:123], a[168:171]\n"
        "v_readlane_b32 s44, %[tabh], 10\n"
        "s_xor_b32 s44, s44, 0x1000\n"
        "s_waitcnt lgkmcnt(6)\n"
        "v_mfma_f32_16x16x32_f16 a[140:143], v[192:195], v[124:127], a[140:143]\n"
        "ds_read_b128 v[120:123], v70 offset:9216\n"
        "v_mfma_f32_16x16x32_f16 a[172:175], v[196:199], v[124:127], a[172:175]\n"
        "buffer_load_dwordx4 v[240:243], v64, %[rsA], s44 offen offset:0 sc0 sc1\n"
        "s_waitcnt lgkmcnt(6)\n"
        "v_mfma_f32_16x16x32_f16 a[144:147], v[192:195], v[72:75], a[144:147]\n"
        "ds_read_b128 v[124:127], v70 offset:9728\n"
        "v_mfma_f32_16x16x32_f16 a[176:179], v[196:199], v[72:75], a[176:179]\n"
        "s_waitcnt lgkmcnt(6)\n"
        "v_mfma_f32_16x16x32_f16 a[148:151], v[192:195], v[76:79], a[148:151]\n"
        "ds_read_b128 v[72:75], v70 offset:8448\n"
        "v_mfma_f32_16x16x32_f16 a[180:183], v[196:199], v[76:79], a[180:183]\n"
        "buffer_load_dwordx4 v[244:247], v64, %[rsA], s44 offen offset:1024 sc0 sc1\n"
        "s_waitcnt lgkmcnt(6)\n"
        "v_mfma_f32_16x16x32_f16 a[152:155], v[192:195], v[104:107], a[152:155]\n"
        "ds_read_b128 v[76:79], v70 offset:8960\n"
        "v_mfma_f32_16x16x32_f16 a[184:187], v[196:199], v[104:107], a[184:187]\n"
        "s_waitcnt lgkmcnt(6)\n"
        "v_mfma_f32_16x16x32_f16 a[156:159], v[192:195], v[108:111], a[156:159]\n"
        "ds_read_b128 v[104:107], v70 offset:9472\n"
        "v_mfma_f32_16x16x32_f16 a[188:191], v[196:199], v[108:111], a[188:191]\n"
        "v_readlane_b32 s78, %[tabh], 11\n"
        "s_xor_b32 s78, s78, 0x1000\n"
        "s_waitcnt vmcnt(10) lgkmcnt(6)\n"
        "v_mfma_f32_16x16x32_f16 a[128:131], v[200:203], v[112:115], a[128:131]\n"
        "ds_read_b128 v[108:111], v70 offset:9984\n"
        "v_mfma_f32_16x16x32_f16 a[160:163], v[204:207], v[112:115], a[160:163]\n"
        "buffer_load_dwordx4 v[248:251], v64, %[rsA], s78 offen offset:0 sc0 sc1\n"
        "s_waitcnt lgkmcnt(6)\n"
        "v_mfma_f32_16x16x32_f16 a[132:135], v[200:203], v[116:119], a[132:135]\n"
        "ds_read_b128 v[112:115], v70 offset:16384\n"
        "v_mfma_f32_16x16x32_f16 a[164:167], v[204:207], v[116:119], a[164:167]\n"
        "s_waitcnt lgkmcnt(6)\n"
        "v_mfma_f32_16x16x32_f16 a[136:139], v[200:203], v[120:123], a[136:139]\n"
        "ds_read_b128 v[116:119], v70 offset:16896\n"
        "v_mfma_f32_16x16x32_f16 a[168:171], v[204:207], v[120:123], a[168:171]\n"
        "buffer_load_dwordx4 v[252:255], v64, %[rsA], s78 offen offset:1024 sc0 sc1\n"
        "s_waitcnt lgkmcnt(6)\n"
        "v_mfma_f32_16x16x32_f16 a[140:143], v[200:203], v[124:127], a[140:143]\n"
        "ds_read_b128 v[120:123], v70 offset:17408\n"
        "v_mfma_f32_16x16x32_f16 a[172:175], v[204:207], v[124:127], a[172:175]\n"
        "s_waitcnt lgkmcnt(6)\n"
        "v_mfma_f32_16x16x32_f16 a[144:147], v[200:203], v[72:75], a[144:147]\n"
        "ds_read_b128 v[124:127], v70 offset:17920\n"
        "v_mfma_f32_16x16x32_f16 a[176:179], v[204:207], v[72:75], a[176:179]\n"
        "v_readlane_b32 s44, %[tabh], 12\n"
        "s_xor_b32 s44, s44, 0x1000\n"
        "s_waitcnt lgkmcnt(6)\n"
        "v_mfma_f32_16x16x32_f16 a[148:151], v[200:203], v[76:79], a[148:151]\n"
        "ds_read_b128 v[72:75], v70 offset:16640\n"
        "v_mfma_f32_16x16x32_f16 a[180:183], v[204:207], v[76:79], a[180:183]\n"
        "buffer_load_dwordx4 v[12:15], v64, %[rsA], s44 offen offset:0 sc0 sc1\n"
        "s_waitcnt lgkmcnt(6)\n"
        "v_mfma_f32_16x16x32_f16 a[152:155], v[200:203], v[104:107], a[152:155]\n"
        "ds_read_b128 v[76:79], v70 offset:17152\n"
        "v_mfma_f32_16x16x32_f16 a[184:187], v[204:207], v[104:107], a[184:187]\n"
        "s_waitcnt lgkmcnt(6)\n"
        "v_mfma_f32_16x16x32_f16 a[156:159], v[200:203], v[108:111], a[156:159]\n"
        "ds_read_b128 v[104:107], v70 offset:17664\n"
        "v_mfma_f32_16x16x32_f16 a[188:191], v[204:207], v[108:111], a[188:191]\n"
        "buffer_load_dwordx4 v[16:19], v64, %[rsA], s44 offen offset:1024 sc0 sc1\n"
        "s_waitcnt vmcnt(12) lgkmcnt(6)\n"
        "v_mfma_f32_16x16x32_f16 a[128:131], v[208:211], v[112:115], a[128:131]\n"
        "ds_read_b128 v[108:111], v70 offset:18176\n"
        "v_mfma_f32_16x16x32_f16 a[160:163], v[212:215], v[112:115], a[160:163]\n"
        "s_waitcnt lgkmcnt(6)\n"
        "v_mfma_f32_16x16x32_f16 a[132:135], v[208:211], v[116:119], a[132:135]\n"
        "ds_read_b128 v[112:115], v70 offset:24576\n"
        "v_mfma_f32_16x16x32_f16 a[164:167], v[212:215], v[116:119], a[164:167]\n"
        "v_readlane_b32 s78, %[tabh], 13\n"
        "s_xor_b32 s78, s78, 0x1000\n"
        "s_waitcnt lgkmcnt(6)\n"
        "v_mfma_f32_16x16x32_f16 a[136:139], v[208:211], v[120:123], a[136:139]\n"
        "ds_read_b128 v[116:119], v70 offset:25088\n"
        "v_mfma_f32_16x16x32_f16 a[168:171], v[212:215], v[120:123], a[168:171]\n"
        "buffer_load_dwordx4 v[20:23], v64, %[rsA], s78 offen offset:0 sc0 sc1\n"
        "s_waitcnt lgkmcnt(6)\n"
        "v_mfma_f32_16x16x32_f16 a[140:143], v[208:211], v[124:127], a[140:143]\n"
        "ds_read_b128 v[120:123], v70 offset:25600\n"
        "v_mfma_f32_16x16x32_f16 a[172:175], v[212:215], v[124:127], a[172:175]\n"
        "s_waitcnt lgkmcnt(6)\n"
        "v_mfma_f32_16x16x32_f16 a[144:147], v[208:211], v[72:75], a[144:147]\n"
        "ds_read_b128 v[124:127], v70 offset:26112\n"
        "v_mfma_f32_16x16x32_f16 a[176:179], v[212:215], v[72:75], a[176:179]\n"
        "buffer_load_dwordx4 v[24:27], v64, %[rsA], s78 offen offset:1024 sc0 sc1\n"
        "s_waitcnt lgkmcnt(6)\n"
        "v_mfma_f32_16x16x32_f16 a[148:151], v[208:211], v[76:79], a[148:151]\n"
        "ds_read_b128 v[72:75], v70 offset:24832\n"
        "v_mfma_f32_16x16x32_f16 a[180:183], v[212:215], v[76:79], a[180:183]\n"
        "s_waitcnt lgkmcnt(6)\n"
        "v_mfma_f32_16x16x32_f16 a[152:155], v[208:211], v[104:107], a[152:155]\n"
        "ds_read_b128 v[76:79], v70 offset:25344\n"
        "v_mfma_f32_16x16x32_f16 a[184:187], v[212:215], v[104:107], a[184:187]\n"
        "v_readlane_b32 s44, %[tabh], 14\n"
        "s_xor_b32 s44, s44, 0x1000\n"
        "s_waitcnt lgkmcnt(6)\n"
        "v_mfma_f32_16x16x32_f16 a[156:159], v[208:211], v[108:111], a[156:159]\n"
        "ds_read_b128 v[104:107], v70 offset:25856\n"
        "v_mfma_f32_16x16x32_f16 a[188:191], v[212:215], v[108:111], a[188:191]\n"
        "buffer_load_dwordx4 v[28:31], v64, %[rsA], s44 offen offset:0 sc0 sc1\n"
        "s_waitcnt vmcnt(13) lgkmcnt(6)\n"
        "v_mfma_f32_16x16x32_f16 a[128:131], v[216:219], v[112:115], a[128:131]\n"
        "ds_read_b128 v[108:111], v70 offset:26368\n"
        "v_mfma_f32_16x16x32_f16 a[160:163], v[220:223], v[112:115], a[160:163]\n"
        "s_waitcnt lgkmcnt(6)\n"
        "v_mfma_f32_16x16x32_f16 a[132:135], v[216:219], v[116:119], a[132:135]\n"
        "ds_read_b128 v[112:115], v70 offset:32768\n"
        "v_mfma_f32_16x16x32_f16 a[164:167], v[220:223], v[116:119], a[164:167]\n"
        "buffer_load_dwordx4 v[32:35], v64, %[rsA], s44 offen offset:1024 sc0 sc1\n"
        "s_waitcnt lgkmcnt(6)\n"
        "v_mfma_f32_16x16x32_f16 a[136:139], v[216:219], v[120:123], a[136:139]\n"
        "ds_read_b128 v[116:119], v70 offset:33280\n"
        "v_mfma_f32_16x16x32_f16 a[168:171], v[220:223], v[120:123], a[168:171]\n"
        "s_waitcnt lgkmcnt(6)\n"
        "v_mfma_f32_16x16x32_f16 a[140:143], v[216:219], v[124:127], a[140:143]\n"
        "ds_read_b128 v[120:123], v70 offset:33792\n"
        "v_mfma_f32_16x16x32_f16 a[172:175], v[220:223], v[124:127], a[172:175]\n"
        "v_readlane_b32 s78, %[tabh], 15\n"
        "s_xor_b32 s78, s78, 0x1000\n"
        "s_waitcnt lgkmcnt(6)\n"
        "v_mfma_f32_16x16x32_f16 a[144:147], v[216:219], v[72:75], a[144:147]\n"
        "ds_read_b128 v[124:127], v70 offset:34304\n"
        "v_mfma_f32_16x16x32_f16 a[176:179], v[220:223], v[72:75], a[176:179]\n"
        "buffer_load_dwordx4 v[36:39], v64, %[rsA], s78 offen offset:0 sc0 sc1\n"
        "s_waitcnt lgkmcnt(6)\n"
        "v_mfma_f32_16x16x32_f16 a[148:151], v[216:219], v[76:79], a[148:151]\n"
        "ds_read_b128 v[72:75], v70 offset:33024\n"
        "v_mfma_f32_16x16x32_f16 a[180:183], v[220:223], v[76:79], a[180:183]\n"
        "s_waitcnt lgkmcnt(6)\n"
        "v_mfma_f32_16x16x32_f16 a[152:155], v[216:219], v[104:107], a[152:155]\n"
        "ds_read_b128 v[76:79], v70 offset:33536\n"
        "v_mfma_f32_16x16x32_f16 a[184:187], v[220:223], v[104:107], a[184:187]\n"
        "buffer_load_dwordx4 v[40:43], v64, %[rsA], s78 offen offset:1024 sc0 sc1\n"
        "s_waitcnt lgkmcnt(6)\n"
        "v_mfma_f32_16x16x32_f16 a[156:159], v[216:219], v[108:111], a[156:159]\n"
        "ds_read_b128 v[104:107], v70 offset:34048\n"
        "v_mfma_f32_16x16x32_f16 a[188:191], v[220:223], v[108:111], a[188:191]\n"
        "s_waitcnt vmcnt(14) lgkmcnt(6)\n"
        "v_mfma_f32_16x16x32_f16 a[128:131], v[224:227], v[112:115], a[128:131]\n"
        "ds_read_b128 v[108:111], v70 offset:34560\n"
        "v_mfma_f32_16x16x32_f16 a[160:163], v[228:231], v[112:115], a[160:163]\n"
        "v_readlane_b32 s44, %[tabh], 16\n"
        "s_xor_b32 s44, s44, 0x1000\n"
        "s_waitcnt lgkmcnt(6)\n"
        "v_mfma_f32_16x16x32_f16 a[132:135], v[224:227], v[116:119], a[132:135]\n"
        "ds_read_b128 v[112:115], v70 offset:40960\n"
        "v_mfma_f32_16x16x32_f16 a[164:167], v[228:231], v[116:119], a[164:167]\n"
        "buffer_load_dwordx4 v[44:47], v64, %[rsA], s44 offen offset:0 sc0 sc1\n"
        "s_waitcnt lgkmcnt(6)\n"
        "v_mfma_f32_16x16x32_f16 a[136:139], v[224:227], v[120:123], a[136:139]\n"
        "ds_read_b128 v[116:119], v70 offset:41472\n"
        "v_mfma_f32_16x16x32_f16 a[168:171], v[228:231], v[120:123], a[168:171]\n"
        "s_waitcnt lgkmcnt(6)\n"
        "v_mfma_f32_16x16x32_f16 a[140:143], v[224:227], v[124:127], a[140:143]\n"
        "ds_read_b128 v[120:123], v70 offset:41984\n"
        "v_mfma_f32_16x16x32_f16 a[172:175], v[228:231], v[124:127], a[172:175]\n"
        "buffer_load_dwordx4 v[48:51], v64, %[rsA], s44 offen offset:1024 sc0 sc1\n"
        "s_waitcnt lgkmcnt(6)\n"
        "v_mfma_f32_16x16x32_f16 a[144:147], v[224:227], v[72:75], a[144:147]\n"
        "ds_read_b128 v[124:127], v70 offset:42496\n"
        "v_mfma_f32_16x16x32_f16 a[176:179], v[228:231], v[72:75], a[176:179]\n"
        "s_waitcnt lgkmcnt(6)\n"
        "v_mfma_f32_16x16x32_f16 a[148:151], v[224:227], v[76:79], a[148:151]\n"
        "ds_read_b128 v[72:75], v70 offset:41216\n"
        "v_mfma_f32_16x16x32_f16 a[180:183], v[228:231], v[76:79], a[180:183]\n"
        "v_readlane_b32 s78, %[tabh], 17\n"
        "s_xor_b32 s78, s78, 0x1000\n"
        "s_waitcnt lgkmcnt(6)\n"
        "v_mfma_f32_16x16x32_f16 a[152:155], v[224:227], v[104:107], a[152:155]\n"
        "ds_read_b128 v[76:79], v70 offset:41728\n"
        "v_mfma_f32_16x16x32_f16 a[184:187], v[228:231], v[104:107], a[184:187]\n"
        "buffer_load_dwordx4 v[52:55], v64, %[rsA], s78 offen offset:0 sc0 sc1\n"
        "s_waitcnt lgkmcnt(6)\n"
        "v_mfma_f32_16x16x32_f16 a[156:159], v[224:227], v[108:111], a[156:159]\n"
        "ds_read_b128 v[104:107], v70 offset:42240\n"
        "v_mfma_f32_16x16x32_f16 a[188:191], v[228:231], v[108:111], a[188:191]\n"
        "s_waitcnt vmcnt(15) lgkmcnt(6)\n"
        "v_mfma_f32_16x16x32_f16 a[128:131], v[232:235], v[112:115], a[128:131]\n"
        "ds_read_b128 v[108:111], v70 offset:42752\n"
        "v_mfma_f32_16x16x32_f16 a[160:163], v[236:239], v[112:115], a[160:163]\n"
        "buffer_load_dwordx4 v[56:59], v64, %[rsA], s78 offen offset:1024 sc0 sc1\n"
        "s_waitcnt lgkmcnt(6)\n"
        "v_mfma_f32_16x16x32_f16 a[132:135], v[232:235], v[116:119], a[132:135]\n"
        "ds_read_b128 v[112:115], v70 offset:49152\n"
        "v_mfma_f32_16x16x32_f16 a[164:167], v[236:239], v[116:119], a[164:167]\n"
        "s_waitcnt lgkmcnt(6)\n"
        "v_mfma_f32_16x16x32_f16 a[136:139], v[232:235], v[120:123], a[136:139]\n"
        "ds_read_b128 v[116:119], v70 offset:49664\n"
        "v_mfma_f32_16x16x32_f16 a[168:171], v[236:239], v[120:123], a[168:171]\n"
        "v_readlane_b32 s44, %[tabh], 18\n"
        "s_xor_b32 s44, s44, 0x1000\n"
        "s_waitcnt lgkmcnt(6)\n"
        "v_mfma_f32_16x16x32_f16 a[140:143], v[232:235], v[124:127], a[140:143]\n"
        "ds_read_b128 v[120:123], v70 offset:50176\n"
        "v_mfma_f32_16x16x32_f16 a[172:175], v[236:239], v[124:127], a[172:175]\n"
        "buffer_load_dwordx4 v[80:83], v64, %[rsA], s44 offen offset:0 sc0 sc1\n"
        "s_waitcnt lgkmcnt(6)\n"
        "v_mfma_f32_16x16x32_f16 a[144:147], v[232:235], v[72:75], a[144:147]\n"
        "ds_read_b128 v[124:127], v70 offset:50688\n"
        "v_mfma_f32_16x16x32_f16 a[176:179], v[236:239], v[72:75], a[176:179]\n"
        "s_waitcnt lgkmcnt(6)\n"
        "v_mfma_f32_16x16x32_f16 a[148:151], v[232:235], v[76:79], a[148:151]\n"
        "ds_read_b128 v[72:75], v70 offset:49408\n"
        "v_mfma_f32_16x16x32_f16 a[180:183], v[236:239], v[76:79], a[180:183]\n"
        "buffer_load_dwordx4 v[84:87], v64, %[rsA], s44 offen offset:1024 sc0 sc1\n"
        "s_waitcnt lgkmcnt(6)\n"
        "v_mfma_f32_16x16x32_f16 a[152:155], v[232:235], v[104:107], a[152:155]\n"
        "ds_read_b128 v[76:79], v70 offset:49920\n"
        "v_mfma_f32_16x16x32_f16 a[184:187], v[236:239], v[104:107], a[184:187]\n"
        "s_waitcnt lgkmcnt(6)\n"
        "v_mfma_f32_16x16x32_f16 a[156:159], v[232:235], v[108:111], a[156:159]\n"
        "ds_read_b128 v[104:107], v70 offset:50432\n"
        "v_mfma_f32_16x16x32_f16 a[188:191], v[236:239], v[108:111], a[188:191]\n"
        "v_readlane_b32 s78, %[tabh], 19\n"
        "s_xor_b32 s78, s78, 0x1000\n"
        "s_waitcnt vmcnt(16) lgkmcnt(6)\n"
        "v_mfma_f32_16x16x32_f16 a[128:131], v[240:243], v[112:115], a[128:131]\n"
        "ds_read_b128 v[108:111], v70 offset:50944\n"
        "v_mfma_f32_16x16x32_f16 a[160:163], v[244:247], v[112:115], a[160:163]\n"
        "buffer_load_dwordx4 v[88:91], v64, %[rsA], s78 offen offset:0 sc0 sc1\n"
        "s_waitcnt lgkmcnt(6)\n"
        "v_mfma_f32_16x16x32_f16 a[132:135], v[240:243], v[116:119], a[132:135]\n"
        "ds_read_b128 v[112:115], v70 offset:57344\n"
        "v_mfma_f32_16x16x32_f16 a[164:167], v[244:247], v[116:119], a[164:167]\n"
        "s_waitcnt lgkmcnt(6)\n"
        "v_mfma_f32_16x16x32_f16 a[136:139], v[240:243], v[120:123], a[136:139]\n"
        "ds_read_b128 v[116:119], v70 offset:57856\n"
        "v_mfma_f32_16x16x32_f16 a[168:171], v[244:247], v[120:123], a[168:171]\n"
        "buffer_load_dwordx4 v[92:95], v64, %[rsA], s78 offen offset:1024 sc0 sc1\n"
        "s_waitcnt lgkmcnt(6)\n"
        "v_mfma_f32_16x16x32_f16 a[140:143], v[240:243], v[124:127], a[140:143]\n"
        "ds_read_b128 v[120:123], v70 offset:58368\n"
        "v_mfma_f32_16x16x32_f16 a[172:175], v[244:247], v[124:127], a[172:175]\n"
        "s_waitcnt lgkmcnt(6)\n"
        "v_mfma_f32_16x16x32_f16 a[144:147], v[240:243], v[72:75], a[144:147]\n"
        "ds_read_b128 v[124:127], v70 offset:58880\n"
        "v_mfma_f32_16x16x32_f16 a[176:179], v[244:247], v[72:75], a[176:179]\n"
        "v_readlane_b32 s44, %[tabh], 4\n"
        "s_xor_b32 s44, s44, 0x1000\n"
        "s_waitcnt lgkmcnt(6)\n"
        "v_mfma_f32_16x16x32_f16 a[148:151], v[240:243], v[76:79], a[148:151]\n"
        "ds_read_b128 v[72:75], v70 offset:57600\n"
        "v_mfma_f32_16x16x32_f16 a[180:183], v[244:247], v[76:79], a[180:183]\n"
        "buffer_load_dwordx4 v[96:99], v64, %[rsA], s44 offen offset:2048 sc0 sc1\n"
        "s_waitcnt lgkmcnt(6)\n"
        "v_mfma_f32_16x16x32_f16 a[152:155], v[240:243], v[104:107], a[152:155]\n"
        "ds_read_b128 v[76:79], v70 offset:58112\n"
        "v_mfma_f32_16x16x32_f16 a[184:187], v[244:247], v[104:107], a[184:187]\n"
        "s_waitcnt lgkmcnt(6)\n"
        "v_mfma_f32_16x16x32_f16 a[156:159], v[240:243], v[108:111], a[156:159]\n"
        "ds_read_b128 v[104:107], v70 offset:58624\n"
        "v_mfma_f32_16x16x32_f16 a[188:191], v[244:247], v[108:111], a[188:191]\n"
        "buffer_load_dwordx4 v[100:103], v64, %[rsA], s44 offen offset:3072 sc0 sc1\n"
        "s_waitcnt vmcnt(18) lgkmcnt(6)\n"
        "v_mfma_f32_16x16x32_f16 a[128:131], v[248:251], v[112:115], a[128:131]\n"
        "ds_read_b128 v[108:111], v70 offset:59136\n"
        "v_mfma_f32_16x16x32_f16 a[160:163], v[252:255], v[112:115], a[160:163]\n"
        "s_waitcnt lgkmcnt(6)\n"
        "v_mfma_f32_16x16x32_f16 a[132:135], v[248:251], v[116:119], a[132:135]\n"
        "ds_read_b128 v[112:115], v71 offset:0\n"
        "v_mfma_f32_16x16x32_f16 a[164:167], v[252:255], v[116:119], a[164:167]\n"
        "v_readlane_b32 s78, %[tabh], 5\n"
        "s_xor_b32 s78, s78, 0x1000\n"
        "s_waitcnt lgkmcnt(6)\n"
        "v_mfma_f32_16x16x32_f16 a[136:139], v[248:251], v[120:123], a[136:139]\n"
        "ds_read_b128 v[116:119], v71 offset:512\n"
        "v_mfma_f32_16x16x32_f16 a[168:171], v[252:255], v[120:123], a[168:171]\n"
        "buffer_load_dwordx4 v[128:131], v64, %[rsA], s78 offen offset:2048 sc0 sc1\n"
        "s_waitcnt lgkmcnt(6)\n"
        "v_mfma_f32_16x16x32_f16 a[140:143], v[248:251], v[124:127], a[140:143]\n"
        "ds_read_b128 v[120:123], v71 offset:1024\n"
        "v_mfma_f32_16x16x32_f16 a[172:175], v[252:255], v[124:127], a[172:175]\n"
        "s_waitcnt lgkmcnt(6)\n"
        "v_mfma_f32_16x16x32_f16 a[144:147], v[248:251], v[72:75], a[144:147]\n"
        "ds_read_b128 v[124:127], v71 offset:1536\n"
        "v_mfma_f32_16x16x32_f16 a[176:179], v[252:255], v[72:75], a[176:179]\n"
        "buffer_load_dwordx4 v[132:135], v64, %[rsA], s78 offen offset:3072 sc0 sc1\n"
        "s_waitcnt lgkmcnt(6)\n"
        "v_mfma_f32_16x16x32_f16 a[148:151], v[248:251], v[76:79], a[148:151]\n"
        "ds_read_b128 v[72:75], v71 offset:256\n"
        "v_mfma_f32_16x16x32_f16 a[180:183], v[252:255], v[76:79], a[180:183]\n"
        "s_waitcnt lgkmcnt(6)\n"
        "v_mfma_f32_16x16x32_f16 a[152:155], v[248:251], v[104:107], a[152:155]\n"
        "ds_read_b128 v[76:79], v71 offset:768\n"
        "v_mfma_f32_16x16x32_f16 a[184:187], v[252:255], v[104:107], a[184:187]\n"
        "v_readlane_b32 s44, %[tabh], 6\n"
        "s_xor_b32 s44, s44, 0x1000\n"
        "s_waitcnt lgkmcnt(6)\n"
        "v_mfma_f32_16x16x32_f16 a[156:159], v[248:251], v[108:111], a[156:159]\n"
        "ds_read_b128 v[104:107], v71 offset:1280\n"
        "v_mfma_f32_16x16x32_f16 a[188:191], v[252:255], v[108:111], a[188:191]\n"
        "buffer_load_dwordx4 v[144:147], v64, %[rsA], s44 offen offset:2048 sc0 sc1\n"
        "s_waitcnt vmcnt(19) lgkmcnt(6)\n"
        "v_mfma_f32_16x16x32_f16 a[128:131], v[12:15], v[112:115], a[128:131]\n"
        "ds_read_b128 v[108:111], v71 offset:1792\n"
        "v_mfma_f32_16x16x32_f16 a[160:163], v[16:19], v[112:115], a[160:163]\n"
        "s_waitcnt lgkmcnt(6)\n"
        "v_mfma_f32_16x16x32_f16 a[132:135], v[12:15], v[116:119], a[132:135]\n"
        "ds_read_b128 v[112:115], v71 offset:8192\n"
        "v_mfma_f32_16x16x32_f16 a[164:167], v[16:19], v[116:119], a[164:167]\n"
        "buffer_load_dwordx4 v[148:151], v64, %[rsA], s44 offen offset:3072 sc0 sc1\n"
        "s_waitcnt lgkmcnt(6)\n"
        "v_mfma_f32_16x16x32_f16 a[136:139], v[12:15], v[120:123], a[136:139]\n"
        "ds_read_b128 v[116:119], v71 offset:8704\n"
        "v_mfma_f32_16x16x32_f16 a[168:171], v[16:19], v[120:123], a[168:171]\n"
        "s_waitcnt lgkmcnt(6)\n"
        "v_mfma_f32_16x16x32_f16 a[140:143], v[12:15], v[124:127], a[140:143]\n"
        "ds_read_b128 v[120:123], v71 offset:9216\n"
        "v_mfma_f32_16x16x32_f16 a[172:175], v[16:19], v[124:127], a[172:175]\n"
        "v_readlane_b32 s78, %[tabh], 7\n"
        "s_xor_b32 s78, s78, 0x1000\n"
        "s_waitcnt lgkmcnt(6)\n"
        "v_mfma_f32_16x16x32_f16 a[144:147], v[12:15], v[72:75], a[144:147]\n"
        "ds_read_b128 v[124:127], v71 offset:9728\n"
        "v_mfma_f32_16x16x32_f16 a[176:179], v[16:19], v[72:75], a[176:179]\n"
        "buffer_load_dwordx4 v[160:163], v64, %[rsA], s78 offen offset:2048 sc0 sc1\n"
        "s_waitcnt lgkmcnt(6)\n"
        "v_mfma_f32_16x16x32_f16 a[148:151], v[12:15], v[76:79], a[148:151]\n"
        "ds_read_b128 v[72:75], v71 offset:8448\n"
        "v_mfma_f32_16x16x32_f16 a[180:183], v[16:19], v[76:79], a[180:183]\n"
        "s_waitcnt lgkmcnt(6)\n"
        "v_mfma_f32_16x16x32_f16 a[152:155], v[12:15], v[104:107], a[152:155]\n"
        "ds_read_b128 v[76:79], v71 offset:8960\n"
        "v_mfma_f32_16x16x32_f16 a[184:187], v[16:19], v[104:107], a[184:187]\n"
        "buffer_load_dwordx4 v[164:167], v64, %[rsA], s78 offen offset:3072 sc0 sc1\n"
        "s_waitcnt lgkmcnt(6)\n"
        "v_mfma_f32_16x16x32_f16 a[156:159], v[12:15], v[108:111], a[156:159]\n"
        "ds_read_b128 v[104:107], v71 offset:9472\n"
        "v_mfma_f32_16x16x32_f16 a[188:191], v[16:19], v[108:111], a[188:191]\n"
        "s_waitcnt vmcnt(20) lgkmcnt(6)\n"
        "v_mfma_f32_16x16x32_f16 a[128:131], v[20:23], v[112:115], a[128:131]\n"
        "ds_read_b128 v[108:111], v71 offset:9984\n"
        "v_mfma_f32_16x16x32_f16 a[160:163], v[24:27], v[112:115], a[160:163]\n"
        "v_readlane_b32 s44, %[tabh], 8\n"
        "s_xor_b32 s44, s44, 0x1000\n"
        "s_waitcnt lgkmcnt(6)\n"
        "v_mfma_f32_16x16x32_f16 a[132:135], v[20:23], v[116:119], a[132:135]\n"
        "ds_read_b128 v[112:115], v71 offset:16384\n"
        "v_mfma_f32_16x16x32_f16 a[164:167], v[24:27], v[116:119], a[164:167]\n"
        "buffer_load_dwordx4 v[176:179], v64, %[rsA], s44 offen offset:2048 sc0 sc1\n"
        "s_waitcnt lgkmcnt(6)\n"
        "v_mfma_f32_16x16x32_f16 a[136:139], v[20:23], v[120:123], a[136:139]\n"
        "ds_read_b128 v[116:119], v71 offset:16896\n"
        "v_mfma_f32_16x16x32_f16 a[168:171], v[24:27], v[120:123], a[168:171]\n"
        "s_waitcnt lgkmcnt(6)\n"
        "v_mfma_f32_16x16x32_f16 a[140:143], v[20:23], v[124:127], a[140:143]\n"
        "ds_read_b128 v[120:123], v71 offset:17408\n"
        "v_mfma_f32_16x16x32_f16 a[172:175], v[24:27], v[124:127], a[172:175]\n"
        "buffer_load_dwordx4 v[180:183], v64, %[rsA], s44 offen offset:3072 sc0 sc1\n"
        "s_waitcnt lgkmcnt(6)\n"
        "v_mfma_f32_16x16x32_f16 a[144:147], v[20:23], v[72:75], a[144:147]\n"
        "ds_read_b128 v[124:127], v71 offset:17920\n"
        "v_mfma_f32_16x16x32_f16 a[176:179], v[24:27], v[72:75], a[176:179]\n"
        "s_waitcnt lgkmcnt(6)\n"
        "v_mfma_f32_16x16x32_f16 a[148:151], v[20:23], v[76:79], a[148:151]\n"
        "ds_read_b128 v[72:75], v71 offset:16640\n"
        "v_mfma_f32_16x16x32_f16 a[180:183], v[24:27], v[76:79], a[180:183]\n"
        "v_readlane_b32 s78, %[tabh], 9\n"
        "s_xor_b32 s78, s78, 0x1000\n"
        "s_waitcnt lgkmcnt(6)\n"
        "v_mfma_f32_16x16x32_f16 a[152:155], v[20:23], v[104:107], a[152:155]\n"
        "ds_read_b128 v[76:79], v71 offset:17152\n"
        "v_mfma_f32_16x16x32_f16 a[184:187], v[24:27], v[104:107], a[184:187]\n"
        "buffer_load_dwordx4 v[192:195], v64, %[rsA], s78 offen offset:2048 sc0 sc1\n"
        "s_waitcnt lgkmcnt(6)\n"
        "v_mfma_f32_16x16x32_f16 a[156:159], v[20:23], v[108:111], a[156:159]\n"
        "ds_read_b128 v[104:107], v71 offset:17664\n"
        "v_mfma_f32_16x16x32_f16 a[188:191], v[24:27], v[108:111], a[188:191]\n"
        "s_waitcnt vmcnt(21) lgkmcnt(6)\n"
        "v_mfma_f32_16x16x32_f16 a[128:131], v[28:31], v[112:115], a[128:131]\n"
        "ds_read_b128 v[108:111], v71 offset:18176\n"
        "v_mfma_f32_16x16x32_f16 a[160:163], v[32:35], v[112:115], a[160:163]\n"
        "buffer_load_dwordx4 v[196:199], v64, %[rsA], s78 offen offset:3072 sc0 sc1\n"
        "s_waitcnt lgkmcnt(6)\n"
        "v_mfma_f32_16x16x32_f16 a[132:135], v[28:31], v[116:119], a[132:135]\n"
        "ds_read_b128 v[112:115], v71 offset:24576\n"
        "v_mfma_f32_16x16x32_f16 a[164:167], v[32:35], v[116:119], a[164:167]\n"
        "s_waitcnt lgkmcnt(6)\n"
        "v_mfma_f32_16x16x32_f16 a[136:139], v[28:31], v[120:123], a[136:139]\n"
        "ds_read_b128 v[116:119], v71 offset:25088\n"
        "v_mfma_f32_16x16x32_f16 a[168:171], v[32:35], v[120:123], a[168:171]\n"
        "v_readlane_b32 s44, %[tabh], 10\n"
        "s_xor_b32 s44, s44, 0x1000\n"
        "s_waitcnt lgkmcnt(6)\n"
        "v_mfma_f32_16x16x32_f16 a[140:143], v[28:31], v[124:127], a[140:143]\n"
        "ds_read_b128 v[120:123], v71 offset:25600\n"
        "v_mfma_f32_16x16x32_f16 a[172:175], v[32:35], v[124:127], a[172:175]\n"
        "buffer_load_dwordx4 v[200:203], v64, %[rsA], s44 offen offset:2048 sc0 sc1\n"
        "s_waitcnt lgkmcnt(6)\n"
        "v_mfma_f32_16x16x32_f16 a[144:147], v[28:31], v[72:75], a[144:147]\n"
        "ds_read_b128 v[124:127], v71 offset:26112\n"
        "v_mfma_f32_16x16x32_f16 a[176:179], v[32:35], v[72:75], a[176:179]\n"
        "s_waitcnt lgkmcnt(6)\n"
        "v_mfma_f32_16x16x32_f16 a[148:151], v[28:31], v[76:79], a[148:151]\n"
        "ds_read_b128 v[72:75], v71 offset:24832\n"
        "v_mfma_f32_16x16x32_f16 a[180:183], v[32:35], v[76:79], a[180:183]\n"
        "buffer_load_dwordx4 v[204:207], v64, %[rsA], s44 offen offset:3072 sc0 sc1\n"
        "s_waitcnt lgkmcnt(6)\n"
        "v_mfma_f32_16x16x32_f16 a[152:155], v[28:31], v[104:107], a[152:155]\n"
        "ds_read_b128 v[76:79], v71 offset:25344\n"
        "v_mfma_f32_16x16x32_f16 a[184:187], v[32:35], v[104:107], a[184:187]\n"
        "s_waitcnt lgkmcnt(6)\n"
        "v_mfma_f32_16x16x32_f16 a[156:159], v[28:31], v[108:111], a[156:159]\n"
        "ds_read_b128 v[104:107], v71 offset:25856\n"
        "v_mfma_f32_16x16x32_f16 a[188:191], v[32:35], v[108:111], a[188:191]\n"
        "v_readlane_b32 s78, %[tabh], 11\n"
        "s_xor_b32 s78, s78, 0x1000\n"
        "s_waitcnt vmcnt(22) lgkmcnt(6)\n"
        "v_mfma_f32_16x16x32_f16 a[128:131], v[36:39], v[112:115], a[128:131]\n"
        "ds_read_b128 v[108:111], v71 offset:26368\n"
        "v_mfma_f32_16x16x32_f16 a[160:163], v[40:43], v[112:115], a[160:163]\n"
        "buffer_load_dwordx4 v[208:211], v64, %[rsA], s78 offen offset:2048 sc0 sc1\n"
        "s_waitcnt lgkmcnt(6)\n"
        "v_mfma_f32_16x16x32_f16 a[132:135], v[36:39], v[116:119], a[132:135]\n"
        "ds_read_b128 v[112:115], %[bbase] offset:0\n"
        "v_mfma_f32_16x16x32_f16 a[164:167], v[40:43], v[116:119], a[164:167]\n"
        "s_waitcnt lgkmcnt(6)\n"
        "v_mfma_f32_16x16x32_f16 a[136:139], v[36:39], v[120:123], a[136:139]\n"
        "ds_read_b128 v[116:119], %[bbase] offset:512\n"
        "v_mfma_f32_16x16x32_f16 a[168:171], v[40:43], v[120:123], a[168:171]\n"
        "buffer_load_dwordx4 v[212:215], v64, %[rsA], s78 offen offset:3072 sc0 sc1\n"
        "s_waitcnt lgkmcnt(6)\n"
        "v_mfma_f32_16x16x32_f16 a[140:143], v[36:39], v[124:127], a[140:143]\n"
        "ds_read_b128 v[120:123], %[bbase] offset:1024\n"
        "v_mfma_f32_16x16x32_f16 a[172:175], v[40:43], v[124:127], a[172:175]\n"
        "s_waitcnt lgkmcnt(6)\n"
        "v_mfma_f32_16x16x32_f16 a[144:147], v[36:39], v[72:75], a[144:147]\n"
        "ds_read_b128 v[124:127], %[bbase] offset:1536\n"
        "v_mfma_f32_16x16x32_f16 a[176:179], v[40:43], v[72:75], a[176:179]\n"
        "v_readlane_b32 s44, %[tabh], 12\n"
        "s_xor_b32 s44, s44, 0x1000\n"
        "s_waitcnt lgkmcnt(6)\n"
        "v_mfma_f32_16x16x32_f16 a[148:151], v[36:39], v[76:79], a[148:151]\n"
        "ds_read_b128 v[72:75], %[bbase] offset:256\n"
        "v_mfma_f32_16x16x32_f16 a[180:183], v[40:43], v[76:79], a[180:183]\n"
        "buffer_load_dwordx4 v[216:219], v64, %[rsA], s44 offen offset:2048 sc0 sc1\n"
        "s_waitcnt lgkmcnt(6)\n"
        "v_mfma_f32_16x16x32_f16 a[152:155], v[36:39], v[104:107], a[152:155]\n"
        "ds_read_b128 v[76:79], %[bbase] offset:768\n"
        "v_mfma_f32_16x16x32_f16 a[184:187], v[40:43], v[104:107], a[184:187]\n"
        "s_waitcnt lgkmcnt(6)\n"
        "v_mfma_f32_16x16x32_f16 a[156:159], v[36:39], v[108:111], a[156:159]\n"
        "ds_read_b128 v[104:107], %[bbase] offset:1280\n"
        "v_mfma_f32_16x16x32_f16 a[188:191], v[40:43], v[108:111], a[188:191]\n"
        "buffer_load_dwordx4 v[220:223], v64, %[rsA], s44 offen offset:3072 sc0 sc1\n"
        "s_waitcnt vmcnt(24) lgkmcnt(6)\n"
        "v_mfma_f32_16x16x32_f16 a[128:131], v[44:47], v[112:115], a[128:131]\n"
        "ds_read_b128 v[108:111], %[bbase] offset:1792\n"
        "v_mfma_f32_16x16x32_f16 a[160:163], v[48:51], v[112:115], a[160:163]\n"
        "s_waitcnt lgkmcnt(6)\n"
        "v_mfma_f32_16x16x32_f16 a[132:135], v[44:47], v[116:119], a[132:135]\n"
        "ds_read_b128 v[112:115], %[bbase] offset:8192\n"
        "v_mfma_f32_16x16x32_f16 a[164:167], v[48:51], v[116:119], a[164:167]\n"
        "v_readlane_b32 s78, %[tabh], 13\n"
        "s_xor_b32 s78, s78, 0x1000\n"
        "s_waitcnt lgkmcnt(6)\n"
        "v_mfma_f32_16x16x32_f16 a[136:139], v[44:47], v[120:123], a[136:139]\n"
        "ds_read_b128 v[116:119], %[bbase] offset:8704\n"
        "v_mfma_f32_16x16x32_f16 a[168:171], v[48:51], v[120:123], a[168:171]\n"
        "buffer_load_dwordx4 v[224:227], v64, %[rsA], s78 offen offset:2048 sc0 sc1\n"
        "s_waitcnt lgkmcnt(6)\n"
        "v_mfma_f32_16x16x32_f16 a[140:143], v[44:47], v[124:127], a[140:143]\n"
        "ds_read_b128 v[120:123], %[bbase] offset:9216\n"
        "v_mfma_f32_16x16x32_f16 a[172:175], v[48:51], v[124:127], a[172:175]\n"
        "s_waitcnt lgkmcnt(6)\n"
        "v_mfma_f32_16x16x32_f16 a[144:147], v[44:47], v[72:75], a[144:147]\n"
        "ds_read_b128 v[124:127], %[bbase] offset:9728\n"
        "v_mfma_f32_16x16x32_f16 a[176:179], v[48:51], v[72:75], a[176:179]\n"
        "buffer_load_dwordx4 v[228:231], v64, %[rsA], s78 offen offset:3072 sc0 sc1\n"
        "s_waitcnt lgkmcnt(6)\n"
        "v_mfma_f32_16x16x32_f16 a[148:151], v[44:47], v[76:79], a[148:151]\n"
        "ds_read_b128 v[72:75], %[bbase] offset:8448\n"
        "v_mfma_f32_16x16x32_f16 a[180:183], v[48:51], v[76:79], a[180:183]\n"
        "s_waitcnt lgkmcnt(6)\n"
        "v_mfma_f32_16x16x32_f16 a[152:155], v[44:47], v[104:107], a[152:155]\n"
        "ds_read_b128 v[76:79], %[bbase] offset:8960\n"
        "v_mfma_f32_16x16x32_f16 a[184:187], v[48:51], v[104:107], a[184:187]\n"
        "v_readlane_b32 s44, %[tabh], 14\n"
        "s_xor_b32 s44, s44, 0x1000\n"
        "s_waitcnt lgkmcnt(6)\n"
        "v_mfma_f32_16x16x32_f16 a[156:159], v[44:47], v[108:111], a[156:159]\n"
        "ds_read_b128 v[104:107], %[bbase] offset:9472\n"
        "v_mfma_f32_16x16x32_f16 a[188:191], v[48:51], v[108:111], a[188:191]\n"
        "buffer_load_dwordx4 v[232:235], v64, %[rsA], s44 offen offset:2048 sc0 sc1\n"
        "s_waitcnt vmcnt(25) lgkmcnt(6)\n"
        "v_mfma_f32_16x16x32_f16 a[128:131], v[52:55], v[112:115], a[128:131]\n"
        "ds_read_b128 v[108:111], %[bbase] offset:9984\n"
        "v_mfma_f32_16x16x32_f16 a[160:163], v[56:59], v[112:115], a[160:163]\n"
        "s_waitcnt lgkmcnt(6)\n"
        "v_mfma_f32_16x16x32_f16 a[132:135], v[52:55], v[116:119], a[132:135]\n"
        "ds_read_b128 v[112:115], %[bbase] offset:16384\n"
        "v_mfma_f32_16x16x32_f16 a[164:167], v[56:59], v[116:119], a[164:167]\n"
        "buffer_load_dwordx4 v[236:239], v64, %[rsA], s44 offen offset:3072 sc0 sc1\n"
        "s_waitcnt lgkmcnt(6)\n"
        "v_mfma_f32_16x16x32_f16 a[136:139], v[52:55], v[120:123], a[136:139]\n"
        "ds_read_b128 v[116:119], %[bbase] offset:16896\n"
        "v_mfma_f32_16x16x32_f16 a[168:171], v[56:59], v[120:123], a[168:171]\n"
        "s_waitcnt lgkmcnt(6)\n"
        "v_mfma_f32_16x16x32_f16 a[140:143], v[52:55], v[124:127], a[140:143]\n"
        "ds_read_b128 v[120:123], %[bbase] offset:17408\n"
        "v_mfma_f32_16x16x32_f16 a[172:175], v[56:59], v[124:127], a[172:175]\n"
        "v_readlane_b32 s78, %[tabh], 15\n"
        "s_xor_b32 s78, s78, 0x1000\n"
        "s_waitcnt lgkmcnt(6)\n"
        "v_mfma_f32_16x16x32_f16 a[144:147], v[52:55], v[72:75], a[144:147]\n"
        "ds_read_b128 v[124:127], %[bbase] offset:17920\n"
        "v_mfma_f32_16x16x32_f16 a[176:179], v[56:59], v[72:75], a[176:179]\n"
        "buffer_load_dwordx4 v[240:243], v64, %[rsA], s78 offen offset:2048 sc0 sc1\n"
        "s_waitcnt lgkmcnt(6)\n"
        "v_mfma_f32_16x16x32_f16 a[148:151], v[52:55], v[76:79], a[148:151]\n"
        "ds_read_b128 v[72:75], %[bbase] offset:16640\n"
        "v_mfma_f32_16x16x32_f16 a[180:183], v[56:59], v[76:79], a[180:183]\n"
        "s_waitcnt lgkmcnt(6)\n"
        "v_mfma_f32_16x16x32_f16 a[152:155], v[52:55], v[104:107], a[152:155]\n"
        "ds_read_b128 v[76:79], %[bbase] offset:17152\n"
        "v_mfma_f32_16x16x32_f16 a[184:187], v[56:59], v[104:107], a[184:187]\n"
        "buffer_load_dwordx4 v[244:247], v64, %[rsA], s78 offen offset:3072 sc0 sc1\n"
        "s_waitcnt lgkmcnt(6)\n"
        "v_mfma_f32_16x16x32_f16 a[156:159], v[52:55], v[108:111], a[156:159]\n"
        "ds_read_b128 v[104:107], %[bbase] offset:17664\n"
        "v_mfma_f32_16x16x32_f16 a[188:191], v[56:59], v[108:111], a[188:191]\n"
        "s_waitcnt vmcnt(26) lgkmcnt(6)\n"
        "v_mfma_f32_16x16x32_f16 a[128:131], v[80:83], v[112:115], a[128:131]\n"
        "ds_read_b128 v[108:111], %[bbase] offset:18176\n"
        "v_mfma_f32_16x16x32_f16 a[160:163], v[84:87], v[112:115], a[160:163]\n"
        "v_readlane_b32 s44, %[tabh], 16\n"
        "s_xor_b32 s44, s44, 0x1000\n"
        "s_waitcnt lgkmcnt(6)\n"
        "v_mfma_f32_16x16x32_f16 a[132:135], v[80:83], v[116:119], a[132:135]\n"
        "ds_read_b128 v[112:115], %[bbase] offset:24576\n"
        "v_mfma_f32_16x16x32_f16 a[164:167], v[84:87], v[116:119], a[164:167]\n"
        "buffer_load_dwordx4 v[248:251], v64, %[rsA], s44 offen offset:2048 sc0 sc1\n"
        "s_waitcnt lgkmcnt(6)\n"
        "v_mfma_f32_16x16x32_f16 a[136:139], v[80:83], v[120:123], a[136:139]\n"
        "ds_read_b128 v[116:119], %[bbase] offset:25088\n"
        "v_mfma_f32_16x16x32_f16 a[168:171], v[84:87], v[120:123], a[168:171]\n"
        "s_waitcnt lgkmcnt(6)\n"
        "v_mfma_f32_16x16x32_f16 a[140:143], v[80:83], v[124:127], a[140:143]\n"
        "ds_read_b128 v[120:123], %[bbase] offset:25600\n"
        "v_mfma_f32_16x16x32_f16 a[172:175], v[84:87], v[124:127], a[172:175]\n"
        "buffer_load_dwordx4 v[252:255], v64, %[rsA], s44 offen offset:3072 sc0 sc1\n"
        "s_waitcnt lgkmcnt(6)\n"
        "v_mfma_f32_16x16x32_f16 a[144:147], v[80:83], v[72:75], a[144:147]\n"
        "ds_read_b128 v[124:127], %[bbase] offset:26112\n"
        "v_mfma_f32_16x16x32_f16 a[176:179], v[84:87], v[72:75], a[176:179]\n"
        "s_waitcnt lgkmcnt(6)\n"
        "v_mfma_f32_16x16x32_f16 a[148:151], v[80:83], v[76:79], a[148:151]\n"
        "ds_read_b128 v[72:75], %[bbase] offset:24832\n"
        "v_mfma_f32_16x16x32_f16 a[180:183], v[84:87], v[76:79], a[180:183]\n"
        "v_readlane_b32 s78, %[tabh], 17\n"
        "s_xor_b32 s78, s78, 0x1000\n"
        "s_waitcnt lgkmcnt(6)\n"
        "v_mfma_f32_16x16x32_f16 a[152:155], v[80:83], v[104:107], a[152:155]\n"
        "ds_read_b128 v[76:79], %[bbase] offset:25344\n"
        "v_mfma_f32_16x16x32_f16 a[184:187], v[84:87], v[104:107], a[184:187]\n"
        "buffer_load_dwordx4 v[12:15], v64, %[rsA], s78 offen offset:2048 sc0 sc1\n"
        "s_waitcnt lgkmcnt(6)\n"
        "v_mfma_f32_16x16x32_f16 a[156:159], v[80:83], v[108:111], a[156:159]\n"
        "ds_read_b128 v[104:107], %[bbase] offset:25856\n"
        "v_mfma_f32_16x16x32_f16 a[188:191], v[84:87], v[108:111], a[188:191]\n"
        "s_waitcnt vmcnt(27) lgkmcnt(6)\n"
        "v_mfma_f32_16x16x32_f16 a[128:131], v[88:91], v[112:115], a[128:131]\n"
        "ds_read_b128 v[108:111], %[bbase] offset:26368\n"
        "v_mfma_f32_16x16x32_f16 a[160:163], v[92:95], v[112:115], a[160:163]\n"
        "buffer_load_dwordx4 v[16:19], v64, %[rsA], s78 offen offset:3072 sc0 sc1\n"
        "s_waitcnt lgkmcnt(6)\n"
        "v_mfma_f32_16x16x32_f16 a[132:135], v[88:91], v[116:119], a[132:135]\n"
        "ds_read_b128 v[112:115], %[bbase] offset:32768\n"
        "v_mfma_f32_16x16x32_f16 a[164:167], v[92:95], v[116:119], a[164:167]\n"
        "s_waitcnt lgkmcnt(6)\n"
        "v_mfma_f32_16x16x32_f16 a[136:139], v[88:91], v[120:123], a[136:139]\n"
        "ds_read_b128 v[116:119], %[bbase] offset:33280\n"
        "v_mfma_f32_16x16x32_f16 a[168:171], v[92:95], v[120:123], a[168:171]\n"
        "v_readlane_b32 s44, %[tabh], 18\n"
        "s_xor_b32 s44, s44, 0x1000\n"
        "s_waitcnt lgkmcnt(6)\n"
        "v_mfma_f32_16x16x32_f16 a[140:143], v[88:91], v[124:127], a[140:143]\n"
        "ds_read_b128 v[120:123], %[bbase] offset:33792\n"
        "v_mfma_f32_16x16x32_f16 a[172:175], v[92:95], v[124:127], a[172:175]\n"
        "buffer_load_dwordx4 v[20:23], v64, %[rsA], s44 offen offset:2048 sc0 sc1\n"
        "s_waitcnt lgkmcnt(6)\n"
        "v_mfma_f32_16x16x32_f16 a[144:147], v[88:91], v[72:75], a[144:147]\n"
        "ds_read_b128 v[124:127], %[bbase] offset:34304\n"
        "v_mfma_f32_16x16x32_f16 a[176:179], v[92:95], v[72:75], a[176:179]\n"
        "s_waitcnt lgkmcnt(6)\n"
        "v_mfma_f32_16x16x32_f16 a[148:151], v[88:91], v[76:79], a[148:151]\n"
        "ds_read_b128 v[72:75], %[bbase] offset:33024\n"
        "v_mfma_f32_16x16x32_f16 a[180:183], v[92:95], v[76:79], a[180:183]\n"
        "buffer_load_dwordx4 v[24:27], v64, %[rsA], s44 offen offset:3072 sc0 sc1\n"
        "s_waitcnt lgkmcnt(6)\n"
        "v_mfma_f32_16x16x32_f16 a[152:155], v[88:91], v[104:107], a[152:155]\n"
        "ds_read_b128 v[76:79], %[bbase] offset:33536\n"
        "v_mfma_f32_16x16x32_f16 a[184:187], v[92:95], v[104:107], a[184:187]\n"
        "s_waitcnt lgkmcnt(6)\n"
        "v_mfma_f32_16x16x32_f16 a[156:159], v[88:91], v[108:111], a[156:159]\n"
        "ds_read_b128 v[104:107], %[bbase] offset:34048\n"
        "v_mfma_f32_16x16x32_f16 a[188:191], v[92:95], v[108:111], a[188:191]\n"
        "v_readlane_b32 s78, %[tabh], 19\n"
        "s_xor_b32 s78, s78, 0x1000\n"
        "s_waitcnt vmcnt(63) lgkmcnt(6)\n"
        "v_mfma_f32_16x16x32_f16 a[192:195], v[136:139], v[112:115], a[192:195]\n"
        "ds_read_b128 v[108:111], %[bbase] offset:34560\n"
        "v_mfma_f32_16x16x32_f16 a[224:227], v[140:143], v[112:115], a[224:227]\n"
        "buffer_load_dwordx4 v[28:31], v64, %[rsA], s78 offen offset:2048 sc0 sc1\n"
        "s_waitcnt lgkmcnt(6)\n"
        "v_mfma_f32_16x16x32_f16 a[196:199], v[136:139], v[116:119], a[196:199]\n"
        "ds_read_b128 v[112:115], %[bbase] offset:40960\n"
        "v_mfma_f32_16x16x32_f16 a[228:231], v[140:143], v[116:119], a[228:231]\n"
        "s_waitcnt lgkmcnt(6)\n"
        "v_mfma_f32_16x16x32_f16 a[200:203], v[136:139], v[120:123], a[200:203]\n"
        "ds_read_b128 v[116:119], %[bbase] offset:41472\n"
        "v_mfma_f32_16x16x32_f16 a[232:235], v[140:143], v[120:123], a[232:235]\n"
        "buffer_load_dwordx4 v[32:35], v64, %[rsA], s78 offen offset:3072 sc0 sc1\n"
        "s_waitcnt lgkmcnt(6)\n"
        "v_mfma_f32_16x16x32_f16 a[204:207], v[136:139], v[124:127], a[204:207]\n"
        "ds_read_b128 v[120:123], %[bbase] offset:41984\n"
        "v_mfma_f32_16x16x32_f16 a[236:239], v[140:143], v[124:127], a[236:239]\n"
        "s_waitcnt lgkmcnt(6)\n"
        "v_mfma_f32_16x16x32_f16 a[208:211], v[136:139], v[72:75], a[208:211]\n"
        "ds_read_b128 v[124:127], %[bbase] offset:42496\n"
        "v_mfma_f32_16x16x32_f16 a[240:243], v[140:143], v[72:75], a[240:243]\n"
        "v_readlane_b32 s41, %[tabh], 0\n"
        "s_xor_b32 s41, s41, 0\n"
        "s_waitcnt lgkmcnt(6)\n"
        "v_mfma_f32_16x16x32_f16 a[212:215], v[136:139], v[76:79], a[212:215]\n"
        "ds_read_b128 v[72:75], %[bbase] offset:41216\n"
        "v_mfma_f32_16x16x32_f16 a[244:247], v[140:143], v[76:79], a[244:247]\n"
        "buffer_load_dwordx4 v[36:39], v64, %[rsA], s41 offen offset:0 sc0 sc1\n"
        "s_waitcnt lgkmcnt(6)\n"
        "v_mfma_f32_16x16x32_f16 a[216:219], v[136:139], v[104:107], a[216:219]\n"
        "ds_read_b128 v[76:79], %[bbase] offset:41728\n"
        "v_mfma_f32_16x16x32_f16 a[248:251], v[140:143], v[104:107], a[248:251]\n"
        "s_add_u32 s43, %[xsend], 0\n"
        "v_add_u32 v60, s43, v61\n"
        "global_store_dwordx4 v60, a[128:131], %[my] offset:0 sc0 sc1\n"
        "s_waitcnt lgkmcnt(6)\n"
        "v_mfma_f32_16x16x32_f16 a[220:223], v[136:139], v[108:111], a[220:223]\n"
        "ds_read_b128 v[104:107], %[bbase] offset:42240\n"
        "v_mfma_f32_16x16x32_f16 a[252:255], v[140:143], v[108:111], a[252:255]\n"
        "buffer_load_dwordx4 v[40:43], v64, %[rsA], s41 offen offset:1024 sc0 sc1\n"
        "s_waitcnt vmcnt(63) lgkmcnt(6)\n"
        "v_mfma_f32_16x16x32_f16 a[192:195], v[152:155], v[112:115], a[192:195]\n"
        "ds_read_b128 v[108:111], %[bbase] offset:42752\n"
        "v_mfma_f32_16x16x32_f16 a[224:227], v[156:159], v[112:115], a[224:227]\n"
        "s_waitcnt lgkmcnt(6)\n"
        "v_mfma_f32_16x16x32_f16 a[196:199], v[152:155], v[116:119], a[196:199]\n"
        "ds_read_b128 v[112:115], %[bbase] offset:49152\n"
        "v_mfma_f32_16x16x32_f16 a[228:231], v[156:159], v[116:119], a[228:231]\n"
        "buffer_load_dwordx4 v[44:47], v64, %[rsA], s41 offen offset:2048 sc0 sc1\n"
        "s_waitcnt lgkmcnt(6)\n"
        "v_mfma_f32_16x16x32_f16 a[200:203], v[152:155], v[120:123], a[200:203]\n"
        "ds_read_b128 v[116:119], %[bbase] offset:49664\n"
        "v_mfma_f32_16x16x32_f16 a[232:235], v[156:159], v[120:123], a[232:235]\n"
        "global_store_dwordx4 v60, a[132:135], %[my] offset:1024 sc0 sc1\n"
        "s_waitcnt lgkmcnt(6)\n"
        "v_mfma_f32_16x16x32_f16 a[204:207], v[152:155], v[124:127], a[204:207]\n"
        "ds_read_b128 v[120:123], %[bbase] offset:50176\n"
        "v_mfma_f32_16x16x32_f16 a[236:239], v[156:159], v[124:127], a[236:239]\n"
        "buffer_load_dwordx4 v[48:51], v64, %[rsA], s41 offen offset:3072 sc0 sc1\n"
        "s_waitcnt lgkmcnt(6)\n"
        "v_mfma_f32_16x16x32_f16 a[208:211], v[152:155], v[72:75], a[208:211]\n"
        "ds_read_b128 v[124:127], %[bbase] offset:50688\n"
        "v_mfma_f32_16x16x32_f16 a[240:243], v[156:159], v[72:75], a[240:243]\n"
        "s_waitcnt lgkmcnt(6)\n"
        "v_mfma_f32_16x16x32_f16 a[212:215], v[152:155], v[76:79], a[212:215]\n"
        "ds_read_b128 v[72:75], %[bbase] offset:49408\n"
        "v_mfma_f32_16x16x32_f16 a[244:247], v[156:159], v[76:79], a[244:247]\n"
        "v_readlane_b32 s77, %[tabh], 1\n"
        "s_xor_b32 s77, s77, 0\n"
        "s_waitcnt lgkmcnt(6)\n"
        "v_mfma_f32_16x16x32_f16 a[216:219], v[152:155], v[104:107], a[216:219]\n"
        "ds_read_b128 v[76:79], %[bbase] offset:49920\n"
        "v_mfma_f32_16x16x32_f16 a[248:251], v[156:159], v[104:107], a[248:251]\n"
        "buffer_load_dwordx4 v[52:55], v64, %[rsA], s77 offen offset:0 sc0 sc1\n"
        "s_waitcnt lgkmcnt(6)\n"
        "v_mfma_f32_16x16x32_f16 a[220:223], v[152:155], v[108:111], a[220:223]\n"
        "ds_read_b128 v[104:107], %[bbase] offset:50432\n"
        "v_mfma_f32_16x16x32_f16 a[252:255], v[156:159], v[108:111], a[252:255]\n"
        "global_store_dwordx4 v60, a[136:139], %[my] offset:2048 sc0 sc1\n"
        "s_waitcnt vmcnt(63) lgkmcnt(6)\n"
        "v_mfma_f32_16x16x32_f16 a[192:195], v[168:171], v[112:115], a[192:195]\n"
        "ds_read_b128 v[108:111], %[bbase] offset:50944\n"
        "v_mfma_f32_16x16x32_f16 a[224:227], v[172:175], v[112:115], a[224:227]\n"
        "buffer_load_dwordx4 v[56:59], v64, %[rsA], s77 offen offset:1024 sc0 sc1\n"
        "s_waitcnt lgkmcnt(6)\n"
        "v_mfma_f32_16x16x32_f16 a[196:199], v[168:171], v[116:119], a[196:199]\n"
        "ds_read_b128 v[112:115], %[bbase] offset:57344\n"
        "v_mfma_f32_16x16x32_f16 a[228:231], v[172:175], v[116:119], a[228:231]\n"
        "s_waitcnt lgkmcnt(6)\n"
        "v_mfma_f32_16x16x32_f16 a[200:203], v[168:171], v[120:123], a[200:203]\n"
        "ds_read_b128 v[116:119], %[bbase] offset:57856\n"
        "v_mfma_f32_16x16x32_f16 a[232:235], v[172:175], v[120:123], a[232:235]\n"
        "buffer_load_dwordx4 v[80:83], v64, %[rsA], s77 offen offset:2048 sc0 sc1\n"
        "s_waitcnt lgkmcnt(6)\n"
        "v_mfma_f32_16x16x32_f16 a[204:207], v[168:171], v[124:127], a[204:207]\n"
        "ds_read_b128 v[120:123], %[bbase] offset:58368\n"
        "v_mfma_f32_16x16x32_f16 a[236:239], v[172:175], v[124:127], a[236:239]\n"
        "global_store_dwordx4 v60, a[140:143], %[my] offset:3072 sc0 sc1\n"
        "s_waitcnt lgkmcnt(6)\n"
        "v_mfma_f32_16x16x32_f16 a[208:211], v[168:171], v[72:75], a[208:211]\n"
        "ds_read_b128 v[124:127], %[bbase] offset:58880\n"
        "v_mfma_f32_16x16x32_f16 a[240:243], v[172:175], v[72:75], a[240:243]\n"
        "buffer_load_dwordx4 v[84:87], v64, %[rsA], s77 offen offset:3072 sc0 sc1\n"
        "s_waitcnt lgkmcnt(6)\n"
        "v_mfma_f32_16x16x32_f16 a[212:215], v[168:171], v[76:79], a[212:215]\n"
        "ds_read_b128 v[72:75], %[bbase] offset:57600\n"
        "v_mfma_f32_16x16x32_f16 a[244:247], v[172:175], v[76:79], a[244:247]\n"
        "s_waitcnt lgkmcnt(6)\n"
        "v_mfma_f32_16x16x32_f16 a[216:219], v[168:171], v[104:107], a[216:219]\n"
        "ds_read_b128 v[76:79], %[bbase] offset:58112\n"
        "v_mfma_f32_16x16x32_f16 a[248:251], v[172:175], v[104:107], a[248:251]\n"
        "v_readlane_b32 s41, %[tabh], 2\n"
        "s_xor_b32 s41, s41, 0\n"
        "s_waitcnt lgkmcnt(6)\n"
        "v_mfma_f32_16x16x32_f16 a[220:223], v[168:171], v[108:111], a[220:223]\n"
        "ds_read_b128 v[104:107], %[bbase] offset:58624\n"
        "v_mfma_f32_16x16x32_f16 a[252:255], v[172:175], v[108:111], a[252:255]\n"
        "buffer_load_dwordx4 v[88:91], v64, %[rsA], s41 offen offset:0 sc0 sc1\n"
        "s_waitcnt vmcnt(63) lgkmcnt(6)\n"
        "v_mfma_f32_16x16x32_f16 a[192:195], v[184:187], v[112:115], a[192:195]\n"
        "ds_read_b128 v[108:111], %[bbase] offset:59136\n"
        "v_mfma_f32_16x16x32_f16 a[224:227], v[188:191], v[112:115], a[224:227]\n"
        "s_add_u32 s43, %[xsend], 4096\n"
        "v_add_u32 v60, s43, v61\n"
        "global_store_dwordx4 v60, a[144:147], %[my] offset:0 sc0 sc1\n"
        "s_waitcnt lgkmcnt(6)\n"
        "v_mfma_f32_16x16x32_f16 a[196:199], v[184:187], v[116:119], a[196:199]\n"
        "ds_read_b128 v[112:115], v70 offset:0\n"
        "v_mfma_f32_16x16x32_f16 a[228:231], v[188:191], v[116:119], a[228:231]\n"
        "buffer_load_dwordx4 v[92:95], v64, %[rsA], s41 offen offset:1024 sc0 sc1\n"
        "s_waitcnt lgkmcnt(6)\n"
        "v_mfma_f32_16x16x32_f16 a[200:203], v[184:187], v[120:123], a[200:203]\n"
        "ds_read_b128 v[116:119], v70 offset:512\n"
        "v_mfma_f32_16x16x32_f16 a[232:235], v[188:191], v[120:123], a[232:235]\n"
        "s_waitcnt lgkmcnt(6)\n"
        "v_mfma_f32_16x16x32_f16 a[204:207], v[184:187], v[124:127], a[204:207]\n"
        "ds_read_b128 v[120:123], v70 offset:1024\n"
        "v_mfma_f32_16x16x32_f16 a[236:239], v[188:191], v[124:127], a[236:239]\n"
        "buffer_load_dwordx4 v[136:139], v64, %[rsA], s41 offen offset:2048 sc0 sc1\n"
        "s_waitcnt lgkmcnt(6)\n"
        "v_mfma_f32_16x16x32_f16 a[208:211], v[184:187], v[72:75], a[208:211]\n"
        "ds_read_b128 v[124:127], v70 offset:1536\n"
        "v_mfma_f32_16x16x32_f16 a[240:243], v[188:191], v[72:75], a[240:243]\n"
        "global_store_dwordx4 v60, a[148:151], %[my] offset:1024 sc0 sc1\n"
        "s_waitcnt lgkmcnt(6)\n"
        "v_mfma_f32_16x16x32_f16 a[212:215], v[184:187], v[76:79], a[212:215]\n"
        "ds_read_b128 v[72:75], v70 offset:256\n"
        "v_mfma_f32_16x16x32_f16 a[244:247], v[188:191], v[76:79], a[244:247]\n"
        "buffer_load_dwordx4 v[140:143], v64, %[rsA], s41 offen offset:3072 sc0 sc1\n"
        "s_waitcnt lgkmcnt(6)\n"
        "v_mfma_f32_16x16x32_f16 a[216:219], v[184:187], v[104:107], a[216:219]\n"
        "ds_read_b128 v[76:79], v70 offset:768\n"
        "v_mfma_f32_16x16x32_f16 a[248:251], v[188:191], v[104:107], a[248:251]\n"
        "s_waitcnt lgkmcnt(6)\n"
        "v_mfma_f32_16x16x32_f16 a[220:223], v[184:187], v[108:111], a[220:223]\n"
        "ds_read_b128 v[104:107], v70 offset:1280\n"
        "v_mfma_f32_16x16x32_f16 a[252:255], v[188:191], v[108:111], a[252:255]\n"
        "v_readlane_b32 s77, %[tabh], 3\n"
        "s_xor_b32 s77, s77, 0\n"
        "s_waitcnt vmcnt(48) lgkmcnt(6)\n"
        "v_mfma_f32_16x16x32_f16 a[192:195], v[96:99], v[112:115], a[192:195]\n"
        "ds_read_b128 v[108:111], v70 offset:1792\n"
        "v_mfma_f32_16x16x32_f16 a[224:227], v[100:103], v[112:115], a[224:227]\n"
        "buffer_load_dwordx4 v[152:155], v64, %[rsA], s77 offen offset:0 sc0 sc1\n"
        "s_waitcnt lgkmcnt(6)\n"
        "v_mfma_f32_16x16x32_f16 a[196:199], v[96:99], v[116:119], a[196:199]\n"
        "ds_read_b128 v[112:115], v70 offset:8192\n"
        "v_mfma_f32_16x16x32_f16 a[228:231], v[100:103], v[116:119], a[228:231]\n"
        "global_store_dwordx4 v60, a[152:155], %[my] offset:2048 sc0 sc1\n"
        "s_waitcnt lgkmcnt(6)\n"
        "v_mfma_f32_16x16x32_f16 a[200:203], v[96:99], v[120:123], a[200:203]\n"
        "ds_read_b128 v[116:119], v70 offset:8704\n"
        "v_mfma_f32_16x16x32_f16 a[232:235], v[100:103], v[120:123], a[232:235]\n"
        "buffer_load_dwordx4 v[156:159], v64, %[rsA], s77 offen offset:1024 sc0 sc1\n"
        "s_waitcnt lgkmcnt(6)\n"
        "v_mfma_f32_16x16x32_f16 a[204:207], v[96:99], v[124:127], a[204:207]\n"
        "ds_read_b128 v[120:123], v70 offset:9216\n"
        "v_mfma_f32_16x16x32_f16 a[236:239], v[100:103], v[124:127], a[236:239]\n"
        "s_waitcnt lgkmcnt(6)\n"
        "v_mfma_f32_16x16x32_f16 a[208:211], v[96:99], v[72:75], a[208:211]\n"
        "ds_read_b128 v[124:127], v70 offset:9728\n"
        "v_mfma_f32_16x16x32_f16 a[240:243], v[100:103], v[72:75], a[240:243]\n"
        "buffer_load_dwordx4 v[168:171], v64, %[rsA], s77 offen offset:2048 sc0 sc1\n"
        "s_waitcnt lgkmcnt(6)\n"
        "v_mfma_f32_16x16x32_f16 a[212:215], v[96:99], v[76:79], a[212:215]\n"
        "ds_read_b128 v[72:75], v70 offset:8448\n"
        "v_mfma_f32_16x16x32_f16 a[244:247], v[100:103], v[76:79], a[244:247]\n"
        "global_store_dwordx4 v60, a[156:159], %[my] offset:3072 sc0 sc1\n"
        "s_waitcnt lgkmcnt(6)\n"
        "v_mfma_f32_16x16x32_f16 a[216:219], v[96:99], v[104:107], a[216:219]\n"
        "ds_read_b128 v[76:79], v70 offset:8960\n"
        "v_mfma_f32_16x16x32_f16 a[248:251], v[100:103], v[104:107], a[248:251]\n"
        "buffer_load_dwordx4 v[172:175], v64, %[rsA], s77 offen offset:3072 sc0 sc1\n"
        "s_waitcnt lgkmcnt(6)\n"
        "v_mfma_f32_16x16x32_f16 a[220:223], v[96:99], v[108:111], a[220:223]\n"
        "ds_read_b128 v[104:107], v70 offset:9472\n"
        "v_mfma_f32_16x16x32_f16 a[252:255], v[100:103], v[108:111], a[252:255]\n"
        "s_waitcnt vmcnt(52) lgkmcnt(6)\n"
        "v_mfma_f32_16x16x32_f16 a[192:195], v[128:131], v[112:115], a[192:195]\n"
        "ds_read_b128 v[108:111], v70 offset:9984\n"
        "v_mfma_f32_16x16x32_f16 a[224:227], v[132:135], v[112:115], a[224:227]\n"
        "v_readlane_b32 s41, %[tabh], 4\n"
        "s_xor_b32 s41, s41, 0\n"
        "s_waitcnt lgkmcnt(6)\n"
        "v_mfma_f32_16x16x32_f16 a[196:199], v[128:131], v[116:119], a[196:199]\n"
        "ds_read_b128 v[112:115], v70 offset:16384\n"
        "v_mfma_f32_16x16x32_f16 a[228:231], v[132:135], v[116:119], a[228:231]\n"
        "buffer_load_dwordx4 v[184:187], v64, %[rsA], s41 offen offset:0 sc0 sc1\n"
        "s_waitcnt lgkmcnt(6)\n"
        "v_mfma_f32_16x16x32_f16 a[200:203], v[128:131], v[120:123], a[200:203]\n"
        "ds_read_b128 v[116:119], v70 offset:16896\n"
        "v_mfma_f32_16x16x32_f16 a[232:235], v[132:135], v[120:123], a[232:235]\n"
        "s_add_u32 s43, %[xsend], 8192\n"
        "v_add_u32 v60, s43, v61\n"
        "global_store_dwordx4 v60, a[160:163], %[my] offset:0 sc0 sc1\n"
        "s_waitcnt lgkmcnt(6)\n"
        "v_mfma_f32_16x16x32_f16 a[204:207], v[128:131], v[124:127], a[204:207]\n"
        "ds_read_b128 v[120:123], v70 offset:17408\n"
        "v_mfma_f32_16x16x32_f16 a[236:239], v[132:135], v[124:127], a[236:239]\n"
        "buffer_load_dwordx4 v[188:191], v64, %[rsA], s41 offen offset:1024 sc0 sc1\n"
        "s_waitcnt lgkmcnt(6)\n"
        "v_mfma_f32_16x16x32_f16 a[208:211], v[128:131], v[72:75], a[208:211]\n"
        "ds_read_b128 v[124:127], v70 offset:17920\n"
        "v_mfma_f32_16x16x32_f16 a[240:243], v[132:135], v[72:75], a[240:243]\n"
        "s_waitcnt lgkmcnt(6)\n"
        "v_mfma_f32_16x16x32_f16 a[212:215], v[128:131], v[76:79], a[212:215]\n"
        "ds_read_b128 v[72:75], v70 offset:16640\n"
        "v_mfma_f32_16x16x32_f16 a[244:247], v[132:135], v[76:79], a[244:247]\n"
        "buffer_load_dwordx4 v[96:99], v64, %[rsA], s41 offen offset:2048 sc0 sc1\n"
        "s_waitcnt lgkmcnt(6)\n"
        "v_mfma_f32_16x16x32_f16 a[216:219], v[128:131], v[104:107], a[216:219]\n"
        "ds_read_b128 v[76:79], v70 offset:17152\n"
        "v_mfma_f32_16x16x32_f16 a[248:251], v[132:135], v[104:107], a[248:251]\n"
        "global_store_dwordx4 v60, a[164:167], %[my] offset:1024 sc0 sc1\n"
        "s_waitcnt lgkmcnt(6)\n"
        "v_mfma_f32_16x16x32_f16 a[220:223], v[128:131], v[108:111], a[220:223]\n"
        "ds_read_b128 v[104:107], v70 offset:17664\n"
        "v_mfma_f32_16x16x32_f16 a[252:255], v[132:135], v[108:111], a[252:255]\n"
        "buffer_load_dwordx4 v[100:103], v64, %[rsA], s41 offen offset:3072 sc0 sc1\n"
        "s_waitcnt vmcnt(56) lgkmcnt(6)\n"
        "v_mfma_f32_16x16x32_f16 a[192:195], v[144:147], v[112:115], a[192:195]\n"
        "ds_read_b128 v[108:111], v70 offset:18176\n"
        "v_mfma_f32_16x16x32_f16 a[224:227], v[148:151], v[112:115], a[224:227]\n"
        "s_waitcnt lgkmcnt(6)\n"
        "v_mfma_f32_16x16x32_f16 a[196:199], v[144:147], v[116:119], a[196:199]\n"
        "ds_read_b128 v[112:115], v70 offset:24576\n"
        "v_mfma_f32_16x16x32_f16 a[228:231], v[148:151], v[116:119], a[228:231]\n"
        "s_waitcnt lgkmcnt(6)\n"
        "v_mfma_f32_16x16x32_f16 a[200:203], v[144:147], v[120:123], a[200:203]\n"
        "ds_read_b128 v[116:119], v70 offset:25088\n"
        "v_mfma_f32_16x16x32_f16 a[232:235], v[148:151], v[120:123], a[232:235]\n"
        "global_store_dwordx4 v60, a[168:171], %[my] offset:2048 sc0 sc1\n"
        "s_waitcnt lgkmcnt(6)\n"
        "v_mfma_f32_16x16x32_f16 a[204:207], v[144:147], v[124:127], a[204:207]\n"
        "ds_read_b128 v[120:123], v70 offset:25600\n"
        "v_mfma_f32_16x16x32_f16 a[236:239], v[148:151], v[124:127], a[236:239]\n"
        "s_waitcnt lgkmcnt(6)\n"
        "v_mfma_f32_16x16x32_f16 a[208:211], v[144:147], v[72:75], a[208:211]\n"
        "ds_read_b128 v[124:127], v70 offset:26112\n"
        "v_mfma_f32_16x16x32_f16 a[240:243], v[148:151], v[72:75], a[240:243]\n"
        "s_waitcnt lgkmcnt(6)\n"
        "v_mfma_f32_16x16x32_f16 a[212:215], v[144:147], v[76:79], a[212:215]\n"
        "ds_read_b128 v[72:75], v70 offset:24832\n"
        "v_mfma_f32_16x16x32_f16 a[244:247], v[148:151], v[76:79], a[244:247]\n"
        "s_waitcnt lgkmcnt(6)\n"
        "v_mfma_f32_16x16x32_f16 a[216:219], v[144:147], v[104:107], a[216:219]\n"
        "ds_read_b128 v[76:79], v70 offset:25344\n"
        "v_mfma_f32_16x16x32_f16 a[248:251], v[148:151], v[104:107], a[248:251]\n"
        "global_store_dwordx4 v60, a[172:175], %[my] offset:3072 sc0 sc1\n"
        "s_waitcnt lgkmcnt(6)\n"
        "v_mfma_f32_16x16x32_f16 a[220:223], v[144:147], v[108:111], a[220:223]\n"
        "ds_read_b128 v[104:107], v70 offset:25856\n"
        "v_mfma_f32_16x16x32_f16 a[252:255], v[148:151], v[108:111], a[252:255]\n"
        "s_waitcnt vmcnt(56) lgkmcnt(6)\n"
        "v_mfma_f32_16x16x32_f16 a[192:195], v[160:163], v[112:115], a[192:195]\n"
        "ds_read_b128 v[108:111], v70 offset:26368\n"
        "v_mfma_f32_16x16x32_f16 a[224:227], v[164:167], v[112:115], a[224:227]\n"
        "v_readlane_b32 s77, %[tabh], 5\n"
        "s_xor_b32 s77, s77, 0\n"
        "s_waitcnt lgkmcnt(6)\n"
        "v_mfma_f32_16x16x32_f16 a[196:199], v[160:163], v[116:119], a[196:199]\n"
        "ds_read_b128 v[112:115], v70 offset:32768\n"
        "v_mfma_f32_16x16x32_f16 a[228:231], v[164:167], v[116:119], a[228:231]\n"
        "buffer_load_dwordx4 v[128:131], v64, %[rsA], s77 offen offset:0 sc0 sc1\n"
        "s_waitcnt lgkmcnt(6)\n"
        "v_mfma_f32_16x16x32_f16 a[200:203], v[160:163], v[120:123], a[200:203]\n"
        "ds_read_b128 v[116:119], v70 offset:33280\n"
        "v_mfma_f32_16x16x32_f16 a[232:235], v[164:167], v[120:123], a[232:235]\n"
        "s_add_u32 s43, %[xsend], 12288\n"
        "v_add_u32 v60, s43, v61\n"
        "global_store_dwordx4 v60, a[176:179], %[my] offset:0 sc0 sc1\n"
        "s_waitcnt lgkmcnt(6)\n"
        "v_mfma_f32_16x16x32_f16 a[204:207], v[160:163], v[124:127], a[204:207]\n"
        "ds_read_b128 v[120:123], v70 offset:33792\n"
        "v_mfma_f32_16x16x32_f16 a[236:239], v[164:167], v[124:127], a[236:239]\n"
        "buffer_load_dwordx4 v[132:135], v64, %[rsA], s77 offen offset:1024 sc0 sc1\n"
        "s_waitcnt lgkmcnt(6)\n"
        "v_mfma_f32_16x16x32_f16 a[208:211], v[160:163], v[72:75], a[208:211]\n"
        "ds_read_b128 v[124:127], v70 offset:34304\n"
        "v_mfma_f32_16x16x32_f16 a[240:243], v[164:167], v[72:75], a[240:243]\n"
        "s_waitcnt lgkmcnt(6)\n"
        "v_mfma_f32_16x16x32_f16 a[212:215], v[160:163], v[76:79], a[212:215]\n"
        "ds_read_b128 v[72:75], v70 offset:33024\n"
        "v_mfma_f32_16x16x32_f16 a[244:247], v[164:167], v[76:79], a[244:247]\n"
        "buffer_load_dwordx4 v[144:147], v64, %[rsA], s77 offen offset:2048 sc0 sc1\n"
        "s_waitcnt lgkmcnt(6)\n"
        "v_mfma_f32_16x16x32_f16 a[216:219], v[160:163], v[104:107], a[216:219]\n"
        "ds_read_b128 v[76:79], v70 offset:33536\n"
        "v_mfma_f32_16x16x32_f16 a[248:251], v[164:167], v[104:107], a[248:251]\n"
        "global_store_dwordx4 v60, a[180:183], %[my] offset:1024 sc0 sc1\n"
        "s_waitcnt lgkmcnt(6)\n"
        "v_mfma_f32_16x16x32_f16 a[220:223], v[160:163], v[108:111], a[220:223]\n"
        "ds_read_b128 v[104:107], v70 offset:34048\n"
        "v_mfma_f32_16x16x32_f16 a[252:255], v[164:167], v[108:111], a[252:255]\n"
        "buffer_load_dwordx4 v[148:151], v64, %[rsA], s77 offen offset:3072 sc0 sc1\n"
        "s_waitcnt vmcnt(60) lgkmcnt(6)\n"
        "v_mfma_f32_16x16x32_f16 a[192:195], v[176:179], v[112:115], a[192:195]\n"
        "ds_read_b128 v[108:111], v70 offset:34560\n"
        "v_mfma_f32_16x16x32_f16 a[224:227], v[180:183], v[112:115], a[224:227]\n"
        "s_waitcnt lgkmcnt(6)\n"
        "v_mfma_f32_16x16x32_f16 a[196:199], v[176:179], v[116:119], a[196:199]\n"
        "ds_read_b128 v[112:115], v70 offset:40960\n"
        "v_mfma_f32_16x16x32_f16 a[228:231], v[180:183], v[116:119], a[228:231]\n"
        "s_waitcnt lgkmcnt(6)\n"
        "v_mfma_f32_16x16x32_f16 a[200:203], v[176:179], v[120:123], a[200:203]\n"
        "ds_read_b128 v[116:119], v70 offset:41472\n"
        "v_mfma_f32_16x16x32_f16 a[232:235], v[180:183], v[120:123], a[232:235]\n"
        "global_store_dwordx4 v60, a[184:187], %[my] offset:2048 sc0 sc1\n"
        "s_waitcnt lgkmcnt(6)\n"
        "v_mfma_f32_16x16x32_f16 a[204:207], v[176:179], v[124:127], a[204:207]\n"
        "ds_read_b128 v[120:123], v70 offset:41984\n"
        "v_mfma_f32_16x16x32_f16 a[236:239], v[180:183], v[124:127], a[236:239]\n"
        "s_waitcnt lgkmcnt(6)\n"
        "v_mfma_f32_16x16x32_f16 a[208:211], v[176:179], v[72:75], a[208:211]\n"
        "ds_read_b128 v[124:127], v70 offset:42496\n"
        "v_mfma_f32_16x16x32_f16 a[240:243], v[180:183], v[72:75], a[240:243]\n"
        "s_waitcnt lgkmcnt(6)\n"
        "v_mfma_f32_16x16x32_f16 a[212:215], v[176:179], v[76:79], a[212:215]\n"
        "ds_read_b128 v[72:75], v70 offset:41216\n"
        "v_mfma_f32_16x16x32_f16 a[244:247], v[180:183], v[76:79], a[244:247]\n"
        "s_waitcnt lgkmcnt(6)\n"
        "v_mfma_f32_16x16x32_f16 a[216:219], v[176:179], v[104:107], a[216:219]\n"
        "ds_read_b128 v[76:79], v70 offset:41728\n"
        "v_mfma_f32_16x16x32_f16 a[248:251], v[180:183], v[104:107], a[248:251]\n"
        "global_store_dwordx4 v60, a[188:191], %[my] offset:3072 sc0 sc1\n"
        "s_waitcnt lgkmcnt(6)\n"
        "v_mfma_f32_16x16x32_f16 a[220:223], v[176:179], v[108:111], a[220:223]\n"
        "ds_read_b128 v[104:107], v70 offset:42240\n"
        "v_mfma_f32_16x16x32_f16 a[252:255], v[180:183], v[108:111], a[252:255]\n"
        "s_waitcnt vmcnt(60) lgkmcnt(6)\n"
        "v_mfma_f32_16x16x32_f16 a[192:195], v[192:195], v[112:115], a[192:195]\n"
        "ds_read_b128 v[108:111], v70 offset:42752\n"
        "v_mfma_f32_16x16x32_f16 a[224:227], v[196:199], v[112:115], a[224:227]\n"
        "v_readlane_b32 s41, %[tabh], 6\n"
        "s_xor_b32 s41, s41, 0\n"
        "s_waitcnt lgkmcnt(6)\n"
        "v_mfma_f32_16x16x32_f16 a[196:199], v[192:195], v[116:119], a[196:199]\n"
        "ds_read_b128 v[112:115], v70 offset:49152\n"
        "v_mfma_f32_16x16x32_f16 a[228:231], v[196:199], v[116:119], a[228:231]\n"
        "buffer_load_dwordx4 v[160:163], v64, %[rsA], s41 offen offset:0 sc0 sc1\n"
        "s_waitcnt lgkmcnt(6)\n"
        "v_mfma_f32_16x16x32_f16 a[200:203], v[192:195], v[120:123], a[200:203]\n"
        "ds_read_b128 v[116:119], v70 offset:49664\n"
        "v_mfma_f32_16x16x32_f16 a[232:235], v[196:199], v[120:123], a[232:235]\n"
        "s_waitcnt lgkmcnt(6)\n"
        "v_mfma_f32_16x16x32_f16 a[204:207], v[192:195], v[124:127], a[204:207]\n"
        "ds_read_b128 v[120:123], v70 offset:50176\n"
        "v_mfma_f32_16x16x32_f16 a[236:239], v[196:199], v[124:127], a[236:239]\n"
        "buffer_load_dwordx4 v[164:167], v64, %[rsA], s41 offen offset:1024 sc0 sc1\n"
        "s_waitcnt lgkmcnt(6)\n"
        "v_mfma_f32_16x16x32_f16 a[208:211], v[192:195], v[72:75], a[208:211]\n"
        "ds_read_b128 v[124:127], v70 offset:50688\n"
        "v_mfma_f32_16x16x32_f16 a[240:243], v[196:199], v[72:75], a[240:243]\n"
        "s_waitcnt lgkmcnt(6)\n"
        "v_mfma_f32_16x16x32_f16 a[212:215], v[192:195], v[76:79], a[212:215]\n"
        "ds_read_b128 v[72:75], v70 offset:49408\n"
        "v_mfma_f32_16x16x32_f16 a[244:247], v[196:199], v[76:79], a[244:247]\n"
        "buffer_load_dwordx4 v[176:179], v64, %[rsA], s41 offen offset:2048 sc0 sc1\n"
        "s_waitcnt lgkmcnt(6)\n"
        "v_mfma_f32_16x16x32_f16 a[216:219], v[192:195], v[104:107], a[216:219]\n"
        "ds_read_b128 v[76:79], v70 offset:49920\n"
        "v_mfma_f32_16x16x32_f16 a[248:251], v[196:199], v[104:107], a[248:251]\n"
        "s_waitcnt lgkmcnt(6)\n"
        "v_mfma_f32_16x16x32_f16 a[220:223], v[192:195], v[108:111], a[220:223]\n"
        "ds_read_b128 v[104:107], v70 offset:50432\n"
        "v_mfma_f32_16x16x32_f16 a[252:255], v[196:199], v[108:111], a[252:255]\n"
        "buffer_load_dwordx4 v[180:183], v64, %[rsA], s41 offen offset:3072 sc0 sc1\n"
        "s_add_u32 s43, s66, 1\n"
        "v_mov_b32 v68, %[pflag]\n"
        "v_mov_b32 v67, s43\n"
        "s_waitcnt vmcnt(21)\n"
        "s_mov_b64 exec, 1\n"
        "buffer_store_dword v67, v68, %[rsA], 0 offen sc0 sc1\n"
        "s_mov_b64 exec, -1\n"
        "s_waitcnt vmcnt(63) lgkmcnt(6)\n"
        "v_mfma_f32_16x16x32_f16 a[192:195], v[200:203], v[112:115], a[192:195]\n"
        "ds_read_b128 v[108:111], v70 offset:50944\n"
        "v_mfma_f32_16x16x32_f16 a[224:227], v[204:207], v[112:115], a[224:227]\n"
        "s_waitcnt lgkmcnt(6)\n"
        "v_mfma_f32_16x16x32_f16 a[196:199], v[200:203], v[116:119], a[196:199]\n"
        "ds_read_b128 v[112:115], v70 offset:57344\n"
        "v_mfma_f32_16x16x32_f16 a[228:231], v[204:207], v[116:119], a[228:231]\n"
        "s_waitcnt lgkmcnt(6)\n"
        "v_mfma_f32_16x16x32_f16 a[200:203], v[200:203], v[120:123], a[200:203]\n"
        "ds_read_b128 v[116:119], v70 offset:57856\n"
        "v_mfma_f32_16x16x32_f16 a[232:235], v[204:207], v[120:123], a[232:235]\n"
        "s_waitcnt lgkmcnt(6)\n"
        "v_mfma_f32_16x16x32_f16 a[204:207], v[200:203], v[124:127], a[204:207]\n"
        "ds_read_b128 v[120:123], v70 offset:58368\n"
        "v_mfma_f32_16x16x32_f16 a[236:239], v[204:207], v[124:127], a[236:239]\n"
        "s_waitcnt lgkmcnt(6)\n"
        "v_mfma_f32_16x16x32_f16 a[208:211], v[200:203], v[72:75], a[208:211]\n"
        "ds_read_b128 v[124:127], v70 offset:58880\n"
        "v_mfma_f32_16x16x32_f16 a[240:243], v[204:207], v[72:75], a[240:243]\n"
        "s_waitcnt lgkmcnt(6)\n"
        "v_mfma_f32_16x16x32_f16 a[212:215], v[200:203], v[76:79], a[212:215]\n"
        "ds_read_b128 v[72:75], v70 offset:57600\n"
        "v_mfma_f32_16x16x32_f16 a[244:247], v[204:207], v[76:79], a[244:247]\n"
        "s_waitcnt lgkmcnt(6)\n"
        "v_mfma_f32_16x16x32_f16 a[216:219], v[200:203], v[104:107], a[216:219]\n"
        "ds_read_b128 v[76:79], v70 offset:58112\n"
        "v_mfma_f32_16x16x32_f16 a[248:251], v[204:207], v[104:107], a[248:251]\n"
        "s_waitcnt lgkmcnt(6)\n"
        "v_mfma_f32_16x16x32_f16 a[220:223], v[200:203], v[108:111], a[220:223]\n"
        "ds_read_b128 v[104:107], v70 offset:58624\n"
        "v_mfma_f32_16x16x32_f16 a[252:255], v[204:207], v[108:111], a[252:255]\n"
        "s_waitcnt vmcnt(61) lgkmcnt(6)\n"
        "v_mfma_f32_16x16x32_f16 a[192:195], v[208:211], v[112:115], a[192:195]\n"
        "ds_read_b128 v[108:111], v70 offset:59136\n"
        "v_mfma_f32_16x16x32_f16 a[224:227], v[212:215], v[112:115], a[224:227]\n"
        "v_readlane_b32 s77, %[tabh], 7\n"
        "s_xor_b32 s77, s77, 0\n"
        "s_waitcnt lgkmcnt(6)\n"
        "v_mfma_f32_16x16x32_f16 a[196:199], v[208:211], v[116:119], a[196:199]\n"
        "ds_read_b128 v[112:115], v71 offset:0\n"
        "v_mfma_f32_16x16x32_f16 a[228:231], v[212:215], v[116:119], a[228:231]\n"
        "buffer_load_dwordx4 v[192:195], v64, %[rsA], s77 offen offset:0 sc0 sc1\n"
        "s_waitcnt lgkmcnt(6)\n"
        "v_mfma_f32_16x16x32_f16 a[200:203], v[208:211], v[120:123], a[200:203]\n"
        "ds_read_b128 v[116:119], v71 offset:512\n"
        "v_mfma_f32_16x16x32_f16 a[232:235], v[212:215], v[120:123], a[232:235]\n"
        "s_waitcnt lgkmcnt(6)\n"
        "v_mfma_f32_16x16x32_f16 a[204:207], v[208:211], v[124:127], a[204:207]\n"
        "ds_read_b128 v[120:123], v71 offset:1024\n"
        "v_mfma_f32_16x16x32_f16 a[236:239], v[212:215], v[124:127], a[236:239]\n"
        "buffer_load_dwordx4 v[196:199], v64, %[rsA], s77 offen offset:1024 sc0 sc1\n"
        "s_waitcnt lgkmcnt(6)\n"
        "v_mfma_f32_16x16x32_f16 a[208:211], v[208:211], v[72:75], a[208:211]\n"
        "ds_read_b128 v[124:127], v71 offset:1536\n"
        "v_mfma_f32_16x16x32_f16 a[240:243], v[212:215], v[72:75], a[240:243]\n"
        "s_waitcnt lgkmcnt(6)\n"
        "v_mfma_f32_16x16x32_f16 a[212:215], v[208:211], v[76:79], a[212:215]\n"
        "ds_read_b128 v[72:75], v71 offset:256\n"
        "v_mfma_f32_16x16x32_f16 a[244:247], v[212:215], v[76:79], a[244:247]\n"
        "buffer_load_dwordx4 v[200:203], v64, %[rsA], s77 offen offset:2048 sc0 sc1\n"
        "s_waitcnt lgkmcnt(6)\n"
        "v_mfma_f32_16x16x32_f16 a[216:219], v[208:211], v[104:107], a[216:219]\n"
        "ds_read_b128 v[76:79], v71 offset:768\n"
        "v_mfma_f32_16x16x32_f16 a[248:251], v[212:215], v[104:107], a[248:251]\n"
        "s_waitcnt lgkmcnt(6)\n"
        "v_mfma_f32_16x16x32_f16 a[220:223], v[208:211], v[108:111], a[220:223]\n"
        "ds_read_b128 v[104:107], v71 offset:1280\n"
        "v_mfma_f32_16x16x32_f16 a[252:255], v[212:215], v[108:111], a[252:255]\n"
        "buffer_load_dwordx4 v[204:207], v64, %[rsA], s77 offen offset:3072 sc0 sc1\n"
        "s_waitcnt vmcnt(63) lgkmcnt(6)\n"
        "v_mfma_f32_16x16x32_f16 a[192:195], v[216:219], v[112:115], a[192:195]\n"
        "ds_read_b128 v[108:111], v71 offset:1792\n"
        "v_mfma_f32_16x16x32_f16 a[224:227], v[220:223], v[112:115], a[224:227]\n"
        "s_waitcnt lgkmcnt(6)\n"
        "v_mfma_f32_16x16x32_f16 a[196:199], v[216:219], v[116:119], a[196:199]\n"
        "ds_read_b128 v[112:115], v71 offset:8192\n"
        "v_mfma_f32_16x16x32_f16 a[228:231], v[220:223], v[116:119], a[228:231]\n"
        "s_waitcnt lgkmcnt(6)\n"
        "v_mfma_f32_16x16x32_f16 a[200:203], v[216:219], v[120:123], a[200:203]\n"
        "ds_read_b128 v[116:119], v71 offset:8704\n"
        "v_mfma_f32_16x16x32_f16 a[232:235], v[220:223], v[120:123], a[232:235]\n"
        "s_waitcnt lgkmcnt(6)\n"
        "v_mfma_f32_16x16x32_f16 a[204:207], v[216:219], v[124:127], a[204:207]\n"
        "ds_read_b128 v[120:123], v71 offset:9216\n"
        "v_mfma_f32_16x16x32_f16 a[236:239], v[220:223], v[124:127], a[236:239]\n"
        "s_waitcnt lgkmcnt(6)\n"
        "v_mfma_f32_16x16x32_f16 a[208:211], v[216:219], v[72:75], a[208:211]\n"
        "ds_read_b128 v[124:127], v71 offset:9728\n"
        "v_mfma_f32_16x16x32_f16 a[240:243], v[220:223], v[72:75], a[240:243]\n"
        "s_waitcnt lgkmcnt(6)\n"
        "v_mfma_f32_16x16x32_f16 a[212:215], v[216:219], v[76:79], a[212:215]\n"
        "ds_read_b128 v[72:75], v71 offset:8448\n"
        "v_mfma_f32_16x16x32_f16 a[244:247], v[220:223], v[76:79], a[244:247]\n"
        "s_waitcnt lgkmcnt(6)\n"
        "v_mfma_f32_16x16x32_f16 a[216:219], v[216:219], v[104:107], a[216:219]\n"
        "ds_read_b128 v[76:79], v71 offset:8960\n"
        "v_mfma_f32_16x16x32_f16 a[248:251], v[220:223], v[104:107], a[248:251]\n"
        "s_waitcnt lgkmcnt(6)\n"
        "v_mfma_f32_16x16x32_f16 a[220:223], v[216:219], v[108:111], a[220:223]\n"
        "ds_read_b128 v[104:107], v71 offset:9472\n"
        "v_mfma_f32_16x16x32_f16 a[252:255], v[220:223], v[108:111], a[252:255]\n"
        "s_waitcnt vmcnt(61) lgkmcnt(6)\n"
        "v_mfma_f32_16x16x32_f16 a[192:195], v[224:227], v[112:115], a[192:195]\n"
        "ds_read_b128 v[108:111], v71 offset:9984\n"
        "v_mfma_f32_16x16x32_f16 a[224:227], v[228:231], v[112:115], a[224:227]\n"
        "v_readlane_b32 s41, %[tabh], 8\n"
        "s_xor_b32 s41, s41, 0\n"
        "s_waitcnt lgkmcnt(6)\n"
        "v_mfma_f32_16x16x32_f16 a[196:199], v[224:227], v[116:119], a[196:199]\n"
        "ds_read_b128 v[112:115], v71 offset:16384\n"
        "v_mfma_f32_16x16x32_f16 a[228:231], v[228:231], v[116:119], a[228:231]\n"
        "buffer_load_dwordx4 v[208:211], v64, %[rsA], s41 offen offset:0 sc0 sc1\n"
        "s_waitcnt lgkmcnt(6)\n"
        "v_mfma_f32_16x16x32_f16 a[200:203], v[224:227], v[120:123], a[200:203]\n"
        "ds_read_b128 v[116:119], v71 offset:16896\n"
        "v_mfma_f32_16x16x32_f16 a[232:235], v[228:231], v[120:123], a[232:235]\n"
        "s_waitcnt lgkmcnt(6)\n"
        "v_mfma_f32_16x16x32_f16 a[204:207], v[224:227], v[124:127], a[204:207]\n"
        "ds_read_b128 v[120:123], v71 offset:17408\n"
        "v_mfma_f32_16x16x32_f16 a[236:239], v[228:231], v[124:127], a[236:239]\n"
        "buffer_load_dwordx4 v[212:215], v64, %[rsA], s41 offen offset:1024 sc0 sc1\n"
        "s_waitcnt lgkmcnt(6)\n"
        "v_mfma_f32_16x16x32_f16 a[208:211], v[224:227], v[72:75], a[208:211]\n"
        "ds_read_b128 v[124:127], v71 offset:17920\n"
        "v_mfma_f32_16x16x32_f16 a[240:243], v[228:231], v[72:75], a[240:243]\n"
        "s_waitcnt lgkmcnt(6)\n"
        "v_mfma_f32_16x16x32_f16 a[212:215], v[224:227], v[76:79], a[212:215]\n"
        "ds_read_b128 v[72:75], v71 offset:16640\n"
        "v_mfma_f32_16x16x32_f16 a[244:247], v[228:231], v[76:79], a[244:247]\n"
        "buffer_load_dwordx4 v[216:219], v64, %[rsA], s41 offen offset:2048 sc0 sc1\n"
        "s_waitcnt lgkmcnt(6)\n"
        "v_mfma_f32_16x16x32_f16 a[216:219], v[224:227], v[104:107], a[216:219]\n"
        "ds_read_b128 v[76:79], v71 offset:17152\n"
        "v_mfma_f32_16x16x32_f16 a[248:251], v[228:231], v[104:107], a[248:251]\n"
        "s_waitcnt lgkmcnt(6)\n"
        "v_mfma_f32_16x16x32_f16 a[220:223], v[224:227], v[108:111], a[220:223]\n"
        "ds_read_b128 v[104:107], v71 offset:17664\n"
        "v_mfma_f32_16x16x32_f16 a[252:255], v[228:231], v[108:111], a[252:255]\n"
        "buffer_load_dwordx4 v[220:223], v64, %[rsA], s41 offen offset:3072 sc0 sc1\n"
        "s_add_u32 s43, s66, 2\n"
        "v_mov_b32 v68, %[pflag]\n"
        "v_mov_b32 v67, s43\n"
        "s_waitcnt vmcnt(13)\n"
        "s_mov_b64 exec, 1\n"
        "buffer_store_dword v67, v68, %[rsA], 0 offen sc0 sc1\n"
        "s_mov_b64 exec, -1\n"
        "s_waitcnt vmcnt(63) lgkmcnt(6)\n"
        "v_mfma_f32_16x16x32_f16 a[192:195], v[232:235], v[112:115], a[192:195]\n"
        "ds_read_b128 v[108:111], v71 offset:18176\n"
        "v_mfma_f32_16x16x32_f16 a[224:227], v[236:239], v[112:115], a[224:227]\n"
        "s_waitcnt lgkmcnt(6)\n"
        "v_mfma_f32_16x16x32_f16 a[196:199], v[232:235], v[116:119], a[196:199]\n"
        "ds_read_b128 v[112:115], v71 offset:24576\n"
        "v_mfma_f32_16x16x32_f16 a[228:231], v[236:239], v[116:119], a[228:231]\n"
        "s_waitcnt lgkmcnt(6)\n"
        "v_mfma_f32_16x16x32_f16 a[200:203], v[232:235], v[120:123], a[200:203]\n"
        "ds_read_b128 v[116:119], v71 offset:25088\n"
        "v_mfma_f32_16x16x32_f16 a[232:235], v[236:239], v[120:123], a[232:235]\n"
        "s_waitcnt lgkmcnt(6)\n"
        "v_mfma_f32_16x16x32_f16 a[204:207], v[232:235], v[124:127], a[204:207]\n"
        "ds_read_b128 v[120:123], v71 offset:25600\n"
        "v_mfma_f32_16x16x32_f16 a[236:239], v[236:239], v[124:127], a[236:239]\n"
        "s_waitcnt lgkmcnt(6)\n"
        "v_mfma_f32_16x16x32_f16 a[208:211], v[232:235], v[72:75], a[208:211]\n"
        "ds_read_b128 v[124:127], v71 offset:26112\n"
        "v_mfma_f32_16x16x32_f16 a[240:243], v[236:239], v[72:75], a[240:243]\n"
        "s_waitcnt lgkmcnt(6)\n"
        "v_mfma_f32_16x16x32_f16 a[212:215], v[232:235], v[76:79], a[212:215]\n"
        "ds_read_b128 v[72:75], v71 offset:24832\n"
        "v_mfma_f32_16x16x32_f16 a[244:247], v[236:239], v[76:79], a[244:247]\n"
        "s_waitcnt lgkmcnt(6)\n"
        "v_mfma_f32_16x16x32_f16 a[216:219], v[232:235], v[104:107], a[216:219]\n"
        "ds_read_b128 v[76:79], v71 offset:25344\n"
        "v_mfma_f32_16x16x32_f16 a[248:251], v[236:239], v[104:107], a[248:251]\n"
        "s_waitcnt lgkmcnt(6)\n"
        "v_mfma_f32_16x16x32_f16 a[220:223], v[232:235], v[108:111], a[220:223]\n"
        "ds_read_b128 v[104:107], v71 offset:25856\n"
        "v_mfma_f32_16x16x32_f16 a[252:255], v[236:239], v[108:111], a[252:255]\n"
        "s_waitcnt vmcnt(62) lgkmcnt(6)\n"
        "v_mfma_f32_16x16x32_f16 a[192:195], v[240:243], v[112:115], a[192:195]\n"
        "ds_read_b128 v[108:111], v71 offset:26368\n"
        "v_mfma_f32_16x16x32_f16 a[224:227], v[244:247], v[112:115], a[224:227]\n"
        "v_readlane_b32 s77, %[tabh], 9\n"
        "s_xor_b32 s77, s77, 0\n"
        "s_waitcnt lgkmcnt(6)\n"
        "v_mfma_f32_16x16x32_f16 a[196:199], v[240:243], v[116:119], a[196:199]\n"
        "ds_read_b128 v[112:115], %[bbase] offset:0\n"
        "v_mfma_f32_16x16x32_f16 a[228:231], v[244:247], v[116:119], a[228:231]\n"
        "buffer_load_dwordx4 v[224:227], v64, %[rsA], s77 offen offset:0 sc0 sc1\n"
        "s_waitcnt lgkmcnt(6)\n"
        "v_mfma_f32_16x16x32_f16 a[200:203], v[240:243], v[120:123], a[200:203]\n"
        "ds_read_b128 v[116:119], %[bbase] offset:512\n"
        "v_mfma_f32_16x16x32_f16 a[232:235], v[244:247], v[120:123], a[232:235]\n"
        "s_waitcnt lgkmcnt(6)\n"
        "v_mfma_f32_16x16x32_f16 a[204:207], v[240:243], v[124:127], a[204:207]\n"
        "ds_read_b128 v[120:123], %[bbase] offset:1024\n"
        "v_mfma_f32_16x16x32_f16 a[236:239], v[244:247], v[124:127], a[236:239]\n"
        "buffer_load_dwordx4 v[228:231], v64, %[rsA], s77 offen offset:1024 sc0 sc1\n"
        "s_waitcnt lgkmcnt(6)\n"
        "v_mfma_f32_16x16x32_f16 a[208:211], v[240:243], v[72:75], a[208:211]\n"
        "ds_read_b128 v[124:127], %[bbase] offset:1536\n"
        "v_mfma_f32_16x16x32_f16 a[240:243], v[244:247], v[72:75], a[240:243]\n"
        "s_waitcnt lgkmcnt(6)\n"
        "v_mfma_f32_16x16x32_f16 a[212:215], v[240:243], v[76:79], a[212:215]\n"
        "ds_read_b128 v[72:75], %[bbase] offset:256\n"
        "v_mfma_f32_16x16x32_f16 a[244:247], v[244:247], v[76:79], a[244:247]\n"
        "buffer_load_dwordx4 v[232:235], v64, %[rsA], s77 offen offset:2048 sc0 sc1\n"
        "s_waitcnt lgkmcnt(6)\n"
        "v_mfma_f32_16x16x32_f16 a[216:219], v[240:243], v[104:107], a[216:219]\n"
        "ds_read_b128 v[76:79], %[bbase] offset:768\n"
        "v_mfma_f32_16x16x32_f16 a[248:251], v[244:247], v[104:107], a[248:251]\n"
        "s_waitcnt lgkmcnt(6)\n"
        "v_mfma_f32_16x16x32_f16 a[220:223], v[240:243], v[108:111], a[220:223]\n"
        "ds_read_b128 v[104:107], %[bbase] offset:1280\n"
        "v_mfma_f32_16x16x32_f16 a[252:255], v[244:247], v[108:111], a[252:255]\n"
        "buffer_load_dwordx4 v[236:239], v64, %[rsA], s77 offen offset:3072 sc0 sc1\n"
        "s_waitcnt vmcnt(63) lgkmcnt(6)\n"
        "v_mfma_f32_16x16x32_f16 a[192:195], v[248:251], v[112:115], a[192:195]\n"
        "ds_read_b128 v[108:111], %[bbase] offset:1792\n"
        "v_mfma_f32_16x16x32_f16 a[224:227], v[252:255], v[112:115], a[224:227]\n"
        "s_waitcnt lgkmcnt(6)\n"
        "v_mfma_f32_16x16x32_f16 a[196:199], v[248:251], v[116:119], a[196:199]\n"
        "ds_read_b128 v[112:115], %[bbase] offset:8192\n"
        "v_mfma_f32_16x16x32_f16 a[228:231], v[252:255], v[116:119], a[228:231]\n"
        "s_waitcnt lgkmcnt(6)\n"
        "v_mfma_f32_16x16x32_f16 a[200:203], v[248:251], v[120:123], a[200:203]\n"
        "ds_read_b128 v[116:119], %[bbase] offset:8704\n"
        "v_mfma_f32_16x16x32_f16 a[232:235], v[252:255], v[120:123], a[232:235]\n"
        "s_waitcnt lgkmcnt(6)\n"
        "v_mfma_f32_16x16x32_f16 a[204:207], v[248:251], v[124:127], a[204:207]\n"
        "ds_read_b128 v[120:123], %[bbase] offset:9216\n"
        "v_mfma_f32_16x16x32_f16 a[236:239], v[252:255], v[124:127], a[236:239]\n"
        "s_waitcnt lgkmcnt(6)\n"
        "v_mfma_f32_16x16x32_f16 a[208:211], v[248:251], v[72:75], a[208:211]\n"
        "ds_read_b128 v[124:127], %[bbase] offset:9728\n"
        "v_mfma_f32_16x16x32_f16 a[240:243], v[252:255], v[72:75], a[240:243]\n"
        "s_waitcnt lgkmcnt(6)\n"
        "v_mfma_f32_16x16x32_f16 a[212:215], v[248:251], v[76:79], a[212:215]\n"
        "ds_read_b128 v[72:75], %[bbase] offset:8448\n"
        "v_mfma_f32_16x16x32_f16 a[244:247], v[252:255], v[76:79], a[244:247]\n"
        "s_waitcnt lgkmcnt(6)\n"
        "v_mfma_f32_16x16x32_f16 a[216:219], v[248:251], v[104:107], a[216:219]\n"
        "ds_read_b128 v[76:79], %[bbase] offset:8960\n"
        "v_mfma_f32_16x16x32_f16 a[248:251], v[252:255], v[104:107], a[248:251]\n"
        "s_waitcnt lgkmcnt(6)\n"
        "v_mfma_f32_16x16x32_f16 a[220:223], v[248:251], v[108:111], a[220:223]\n"
        "ds_read_b128 v[104:107], %[bbase] offset:9472\n"
        "v_mfma_f32_16x16x32_f16 a[252:255], v[252:255], v[108:111], a[252:255]\n"
        "s_waitcnt vmcnt(62) lgkmcnt(6)\n"
        "v_mfma_f32_16x16x32_f16 a[192:195], v[12:15], v[112:115], a[192:195]\n"
        "ds_read_b128 v[108:111], %[bbase] offset:9984\n"
        "v_mfma_f32_16x16x32_f16 a[224:227], v[16:19], v[112:115], a[224:227]\n"
        "v_readlane_b32 s41, %[tabh], 10\n"
        "s_xor_b32 s41, s41, 0\n"
        "s_waitcnt lgkmcnt(6)\n"
        "v_mfma_f32_16x16x32_f16 a[196:199], v[12:15], v[116:119], a[196:199]\n"
        "ds_read_b128 v[112:115], %[bbase] offset:16384\n"
        "v_mfma_f32_16x16x32_f16 a[228:231], v[16:19], v[116:119], a[228:231]\n"
        "buffer_load_dwordx4 v[240:243], v64, %[rsA], s41 offen offset:0 sc0 sc1\n"
        "s_waitcnt lgkmcnt(6)\n"
        "v_mfma_f32_16x16x32_f16 a[200:203], v[12:15], v[120:123], a[200:203]\n"
        "ds_read_b128 v[116:119], %[bbase] offset:16896\n"
        "v_mfma_f32_16x16x32_f16 a[232:235], v[16:19], v[120:123], a[232:235]\n"
        "s_waitcnt lgkmcnt(6)\n"
        "v_mfma_f32_16x16x32_f16 a[204:207], v[12:15], v[124:127], a[204:207]\n"
        "ds_read_b128 v[120:123], %[bbase] offset:17408\n"
        "v_mfma_f32_16x16x32_f16 a[236:239], v[16:19], v[124:127], a[236:239]\n"
        "buffer_load_dwordx4 v[244:247], v64, %[rsA], s41 offen offset:1024 sc0 sc1\n"
        "s_waitcnt lgkmcnt(6)\n"
        "v_mfma_f32_16x16x32_f16 a[208:211], v[12:15], v[72:75], a[208:211]\n"
        "ds_read_b128 v[124:127], %[bbase] offset:17920\n"
        "v_mfma_f32_16x16x32_f16 a[240:243], v[16:19], v[72:75], a[240:243]\n"
        "s_waitcnt lgkmcnt(6)\n"
        "v_mfma_f32_16x16x32_f16 a[212:215], v[12:15], v[76:79], a[212:215]\n"
        "ds_read_b128 v[72:75], %[bbase] offset:16640\n"
        "v_mfma_f32_16x16x32_f16 a[244:247], v[16:19], v[76:79], a[244:247]\n"
        "buffer_load_dwordx4 v[248:251], v64, %[rsA], s41 offen offset:2048 sc0 sc1\n"
        "s_waitcnt lgkmcnt(6)\n"
        "v_mfma_f32_16x16x32_f16 a[216:219], v[12:15], v[104:107], a[216:219]\n"
        "ds_read_b128 v[76:79], %[bbase] offset:17152\n"
        "v_mfma_f32_16x16x32_f16 a[248:251], v[16:19], v[104:107], a[248:251]\n"
        "s_waitcnt lgkmcnt(6)\n"
        "v_mfma_f32_16x16x32_f16 a[220:223], v[12:15], v[108:111], a[220:223]\n"
        "ds_read_b128 v[104:107], %[bbase] offset:17664\n"
        "v_mfma_f32_16x16x32_f16 a[252:255], v[16:19], v[108:111], a[252:255]\n"
        "buffer_load_dwordx4 v[252:255], v64, %[rsA], s41 offen offset:3072 sc0 sc1\n"
        "s_waitcnt vmcnt(63) lgkmcnt(6)\n"
        "v_mfma_f32_16x16x32_f16 a[192:195], v[20:23], v[112:115], a[192:195]\n"
        "ds_read_b128 v[108:111], %[bbase] offset:18176\n"
        "v_mfma_f32_16x16x32_f16 a[224:227], v[24:27], v[112:115], a[224:227]\n"
        "s_waitcnt lgkmcnt(6)\n"
        "v_mfma_f32_16x16x32_f16 a[196:199], v[20:23], v[116:119], a[196:199]\n"
        "ds_read_b128 v[112:115], %[bbase] offset:24576\n"
        "v_mfma_f32_16x16x32_f16 a[228:231], v[24:27], v[116:119], a[228:231]\n"
        "s_waitcnt lgkmcnt(6)\n"
        "v_mfma_f32_16x16x32_f16 a[200:203], v[20:23], v[120:123], a[200:203]\n"
        "ds_read_b128 v[116:119], %[bbase] offset:25088\n"
        "v_mfma_f32_16x16x32_f16 a[232:235], v[24:27], v[120:123], a[232:235]\n"
        "s_waitcnt lgkmcnt(6)\n"
        "v_mfma_f32_16x16x32_f16 a[204:207], v[20:23], v[124:127], a[204:207]\n"
        "ds_read_b128 v[120:123], %[bbase] offset:25600\n"
        "v_mfma_f32_16x16x32_f16 a[236:239], v[24:27], v[124:127], a[236:239]\n"
        "s_waitcnt lgkmcnt(6)\n"
        "v_mfma_f32_16x16x32_f16 a[208:211], v[20:23], v[72:75], a[208:211]\n"
        "ds_read_b128 v[124:127], %[bbase] offset:26112\n"
        "v_mfma_f32_16x16x32_f16 a[240:243], v[24:27], v[72:75], a[240:243]\n"
        "s_waitcnt lgkmcnt(6)\n"
        "v_mfma_f32_16x16x32_f16 a[212:215], v[20:23], v[76:79], a[212:215]\n"
        "ds_read_b128 v[72:75], %[bbase] offset:24832\n"
        "v_mfma_f32_16x16x32_f16 a[244:247], v[24:27], v[76:79], a[244:247]\n"
        "s_waitcnt lgkmcnt(6)\n"
        "v_mfma_f32_16x16x32_f16 a[216:219], v[20:23], v[104:107], a[216:219]\n"
        "ds_read_b128 v[76:79], %[bbase] offset:25344\n"
        "v_mfma_f32_16x16x32_f16 a[248:251], v[24:27], v[104:107], a[248:251]\n"
        "s_waitcnt lgkmcnt(6)\n"
        "v_mfma_f32_16x16x32_f16 a[220:223], v[20:23], v[108:111], a[220:223]\n"
        "ds_read_b128 v[104:107], %[bbase] offset:25856\n"
        "v_mfma_f32_16x16x32_f16 a[252:255], v[24:27], v[108:111], a[252:255]\n"
        "s_waitcnt vmcnt(62) lgkmcnt(6)\n"
        "v_mfma_f32_16x16x32_f16 a[192:195], v[28:31], v[112:115], a[192:195]\n"
        "ds_read_b128 v[108:111], %[bbase] offset:26368\n"
        "v_mfma_f32_16x16x32_f16 a[224:227], v[32:35], v[112:115], a[224:227]\n"
        "v_readlane_b32 s77, %[tabh], 11\n"
        "s_xor_b32 s77, s77, 0\n"
        "s_waitcnt lgkmcnt(6)\n"
        "v_mfma_f32_16x16x32_f16 a[196:199], v[28:31], v[116:119], a[196:199]\n"
        "ds_read_b128 v[112:115], %[bbase] offset:32768\n"
        "v_mfma_f32_16x16x32_f16 a[228:231], v[32:35], v[116:119], a[228:231]\n"
        "buffer_load_dwordx4 v[12:15], v64, %[rsA], s77 offen offset:0 sc0 sc1\n"
        "s_waitcnt lgkmcnt(6)\n"
        "v_mfma_f32_16x16x32_f16 a[200:203], v[28:31], v[120:123], a[200:203]\n"
        "ds_read_b128 v[116:119], %[bbase] offset:33280\n"
        "v_mfma_f32_16x16x32_f16 a[232:235], v[32:35], v[120:123], a[232:235]\n"
        "s_waitcnt lgkmcnt(6)\n"
        "v_mfma_f32_16x16x32_f16 a[204:207], v[28:31], v[124:127], a[204:207]\n"
        "ds_read_b128 v[120:123], %[bbase] offset:33792\n"
        "v_mfma_f32_16x16x32_f16 a[236:239], v[32:35], v[124:127], a[236:239]\n"
        "buffer_load_dwordx4 v[16:19], v64, %[rsA], s77 offen offset:1024 sc0 sc1\n"
        "s_waitcnt lgkmcnt(6)\n"
        "v_mfma_f32_16x16x32_f16 a[208:211], v[28:31], v[72:75], a[208:211]\n"
        "ds_read_b128 v[124:127], %[bbase] offset:34304\n"
        "v_mfma_f32_16x16x32_f16 a[240:243], v[32:35], v[72:75], a[240:243]\n"
        "s_waitcnt lgkmcnt(6)\n"
        "v_mfma_f32_16x16x32_f16 a[212:215], v[28:31], v[76:79], a[212:215]\n"
        "ds_read_b128 v[72:75], %[bbase] offset:33024\n"
        "v_mfma_f32_16x16x32_f16 a[244:247], v[32:35], v[76:79], a[244:247]\n"
        "buffer_load_dwordx4 v[20:23], v64, %[rsA], s77 offen offset:2048 sc0 sc1\n"
        "s_waitcnt lgkmcnt(6)\n"
        "v_mfma_f32_16x16x32_f16 a[216:219], v[28:31], v[104:107], a[216:219]\n"
        "ds_read_b128 v[76:79], %[bbase] offset:33536\n"
        "v_mfma_f32_16x16x32_f16 a[248:251], v[32:35], v[104:107], a[248:251]\n"
        "s_waitcnt lgkmcnt(6)\n"
        "v_mfma_f32_16x16x32_f16 a[220:223], v[28:31], v[108:111], a[220:223]\n"
        "ds_read_b128 v[104:107], %[bbase] offset:34048\n"
        "v_mfma_f32_16x16x32_f16 a[252:255], v[32:35], v[108:111], a[252:255]\n"
        "buffer_load_dwordx4 v[24:27], v64, %[rsA], s77 offen offset:3072 sc0 sc1\n"
        "s_waitcnt vmcnt(60) lgkmcnt(6)\n"
        "v_mfma_f32_16x16x32_f16 a[0:3], v[36:39], v[112:115], a[0:3]\n"
        "ds_read_b128 v[108:111], %[bbase] offset:34560\n"
        "v_mfma_f32_16x16x32_f16 a[32:35], v[40:43], v[112:115], a[32:35]\n"
        "v_mfma_f32_16x16x32_f16 a[64:67], v[44:47], v[112:115], a[64:67]\n"
        "v_mfma_f32_16x16x32_f16 a[96:99], v[48:51], v[112:115], a[96:99]\n"
        "s_waitcnt lgkmcnt(6)\n"
        "v_mfma_f32_16x16x32_f16 a[4:7], v[36:39], v[116:119], a[4:7]\n"
        "ds_read_b128 v[112:115], %[bbase] offset:40960\n"
        "v_mfma_f32_16x16x32_f16 a[36:39], v[40:43], v[116:119], a[36:39]\n"
        "v_mfma_f32_16x16x32_f16 a[68:71], v[44:47], v[116:119], a[68:71]\n"
        "v_mfma_f32_16x16x32_f16 a[100:103], v[48:51], v[116:119], a[100:103]\n"
        "s_waitcnt lgkmcnt(6)\n"
        "v_mfma_f32_16x16x32_f16 a[8:11], v[36:39], v[120:123], a[8:11]\n"
        "ds_read_b128 v[116:119], %[bbase] offset:41472\n"
        "v_mfma_f32_16x16x32_f16 a[40:43], v[40:43], v[120:123], a[40:43]\n"
        "v_mfma_f32_16x16x32_f16 a[72:75], v[44:47], v[120:123], a[72:75]\n"
        "s_add_u32 s43, %[xsend], 16384\n"
        "v_add_u32 v60, s43, v61\n"
        "global_store_dwordx4 v60, a[192:195], %[my] offset:0 sc0 sc1\n"
        "v_mfma_f32_16x16x32_f16 a[104:107], v[48:51], v[120:123], a[104:107]\n"
        "s_waitcnt lgkmcnt(6)\n"
        "v_mfma_f32_16x16x32_f16 a[12:15], v[36:39], v[124:127], a[12:15]\n"
        "ds_read_b128 v[120:123], %[bbase] offset:41984\n"
        "v_mfma_f32_16x16x32_f16 a[44:47], v[40:43], v[124:127], a[44:47]\n"
        "v_mfma_f32_16x16x32_f16 a[76:79], v[44:47], v[124:127], a[76:79]\n"
        "v_mfma_f32_16x16x32_f16 a[108:111], v[48:51], v[124:127], a[108:111]\n"
        "s_waitcnt lgkmcnt(6)\n"
        "v_mfma_f32_16x16x32_f16 a[16:19], v[36:39], v[72:75], a[16:19]\n"
        "ds_read_b128 v[124:127], %[bbase] offset:42496\n"
        "v_mfma_f32_16x16x32_f16 a[48:51], v[40:43], v[72:75], a[48:51]\n"
        "v_mfma_f32_16x16x32_f16 a[80:83], v[44:47], v[72:75], a[80:83]\n"
        "global_store_dwordx4 v60, a[196:199], %[my] offset:1024 sc0 sc1\n"
        "v_mfma_f32_16x16x32_f16 a[112:115], v[48:51], v[72:75], a[112:115]\n"
        "s_waitcnt lgkmcnt(6)\n"
        "v_mfma_f32_16x16x32_f16 a[20:23], v[36:39], v[76:79], a[20:23]\n"
        "ds_read_b128 v[72:75], %[bbase] offset:41216\n"
        "v_mfma_f32_16x16x32_f16 a[52:55], v[40:43], v[76:79], a[52:55]\n"
        "v_mfma_f32_16x16x32_f16 a[84:87], v[44:47], v[76:79], a[84:87]\n"
        "v_mfma_f32_16x16x32_f16 a[116:119], v[48:51], v[76:79], a[116:119]\n"
        "s_waitcnt lgkmcnt(6)\n"
        "v_mfma_f32_16x16x32_f16 a[24:27], v[36:39], v[104:107], a[24:27]\n"
        "ds_read_b128 v[76:79], %[bbase] offset:41728\n"
        "v_mfma_f32_16x16x32_f16 a[56:59], v[40:43], v[104:107], a[56:59]\n"
        "v_mfma_f32_16x16x32_f16 a[88:91], v[44:47], v[104:107], a[88:91]\n"
        "global_store_dwordx4 v60, a[200:203], %[my] offset:2048 sc0 sc1\n"
        "v_mfma_f32_16x16x32_f16 a[120:123], v[48:51], v[104:107], a[120:123]\n"
        "s_waitcnt lgkmcnt(6)\n"
        "v_mfma_f32_16x16x32_f16 a[28:31], v[36:39], v[108:111], a[28:31]\n"
        "ds_read_b128 v[104:107], %[bbase] offset:42240\n"
        "v_mfma_f32_16x16x32_f16 a[60:63], v[40:43], v[108:111], a[60:63]\n"
        "v_mfma_f32_16x16x32_f16 a[92:95], v[44:47], v[108:111], a[92:95]\n"
        "v_mfma_f32_16x16x32_f16 a[124:127], v[48:51], v[108:111], a[124:127]\n"
        "s_waitcnt vmcnt(57) lgkmcnt(6)\n"
        "v_mfma_f32_16x16x32_f16 a[0:3], v[52:55], v[112:115], a[0:3]\n"
        "ds_read_b128 v[108:111], %[bbase] offset:42752\n"
        "v_mfma_f32_16x16x32_f16 a[32:35], v[56:59], v[112:115], a[32:35]\n"
        "v_readlane_b32 s41, %[tabh], 12\n"
        "s_xor_b32 s41, s41, 0\n"
        "v_mfma_f32_16x16x32_f16 a[64:67], v[80:83], v[112:115], a[64:67]\n"
        "buffer_load_dwordx4 v[28:31], v64, %[rsA], s41 offen offset:0 sc0 sc1\n"
        "v_mfma_f32_16x16x32_f16 a[96:99], v[84:87], v[112:115], a[96:99]\n"
        "global_store_dwordx4 v60, a[204:207], %[my] offset:3072 sc0 sc1\n"
        "s_waitcnt lgkmcnt(6)\n"
        "v_mfma_f32_16x16x32_f16 a[4:7], v[52:55], v[116:119], a[4:7]\n"
        "ds_read_b128 v[112:115], %[bbase] offset:49152\n"
        "v_mfma_f32_16x16x32_f16 a[36:39], v[56:59], v[116:119], a[36:39]\n"
        "v_mfma_f32_16x16x32_f16 a[68:71], v[80:83], v[116:119], a[68:71]\n"
        "buffer_load_dwordx4 v[32:35], v64, %[rsA], s41 offen offset:1024 sc0 sc1\n"
        "v_mfma_f32_16x16x32_f16 a[100:103], v[84:87], v[116:119], a[100:103]\n"
        "s_waitcnt lgkmcnt(6)\n"
        "v_mfma_f32_16x16x32_f16 a[8:11], v[52:55], v[120:123], a[8:11]\n"
        "ds_read_b128 v[116:119], %[bbase] offset:49664\n"
        "v_mfma_f32_16x16x32_f16 a[40:43], v[56:59], v[120:123], a[40:43]\n"
        "v_mfma_f32_16x16x32_f16 a[72:75], v[80:83], v[120:123], a[72:75]\n"
        "buffer_load_dwordx4 v[36:39], v64, %[rsA], s41 offen offset:2048 sc0 sc1\n"
        "v_mfma_f32_16x16x32_f16 a[104:107], v[84:87], v[120:123], a[104:107]\n"
        "s_add_u32 s43, %[xsend], 20480\n"
        "v_add_u32 v60, s43, v61\n"
        "global_store_dwordx4 v60, a[208:211], %[my] offset:0 sc0 sc1\n"
        "s_waitcnt lgkmcnt(6)\n"
        "v_mfma_f32_16x16x32_f16 a[12:15], v[52:55], v[124:127], a[12:15]\n"
        "ds_read_b128 v[120:123], %[bbase] offset:50176\n"
        "v_mfma_f32_16x16x32_f16 a[44:47], v[56:59], v[124:127], a[44:47]\n"
        "v_mfma_f32_16x16x32_f16 a[76:79], v[80:83], v[124:127], a[76:79]\n"
        "buffer_load_dwordx4 v[40:43], v64, %[rsA], s41 offen offset:3072 sc0 sc1\n"
        "v_mfma_f32_16x16x32_f16 a[108:111], v[84:87], v[124:127], a[108:111]\n"
        "s_waitcnt lgkmcnt(6)\n"
        "v_mfma_f32_16x16x32_f16 a[16:19], v[52:55], v[72:75], a[16:19]\n"
        "ds_read_b128 v[124:127], %[bbase] offset:50688\n"
        "v_mfma_f32_16x16x32_f16 a[48:51], v[56:59], v[72:75], a[48:51]\n"
        "v_mfma_f32_16x16x32_f16 a[80:83], v[80:83], v[72:75], a[80:83]\n"
        "v_mfma_f32_16x16x32_f16 a[112:115], v[84:87], v[72:75], a[112:115]\n"
        "global_store_dwordx4 v60, a[212:215], %[my] offset:1024 sc0 sc1\n"
        "s_waitcnt lgkmcnt(6)\n"
        "v_mfma_f32_16x16x32_f16 a[20:23], v[52:55], v[76:79], a[20:23]\n"
        "ds_read_b128 v[72:75], %[bbase] offset:49408\n"
        "v_mfma_f32_16x16x32_f16 a[52:55], v[56:59], v[76:79], a[52:55]\n"
        "v_mfma_f32_16x16x32_f16 a[84:87], v[80:83], v[76:79], a[84:87]\n"
        "v_mfma_f32_16x16x32_f16 a[116:119], v[84:87], v[76:79], a[116:119]\n"
        "s_waitcnt lgkmcnt(6)\n"
        "v_mfma_f32_16x16x32_f16 a[24:27], v[52:55], v[104:107], a[24:27]\n"
        "ds_read_b128 v[76:79], %[bbase] offset:49920\n"
        "v_mfma_f32_16x16x32_f16 a[56:59], v[56:59], v[104:107], a[56:59]\n"
        "v_mfma_f32_16x16x32_f16 a[88:91], v[80:83], v[104:107], a[88:91]\n"
        "v_mfma_f32_16x16x32_f16 a[120:123], v[84:87], v[104:107], a[120:123]\n"
        "global_store_dwordx4 v60, a[216:219], %[my] offset:2048 sc0 sc1\n"
        "s_waitcnt lgkmcnt(6)\n"
        "v_mfma_f32_16x16x32_f16 a[28:31], v[52:55], v[108:111], a[28:31]\n"
        "ds_read_b128 v[104:107], %[bbase] offset:50432\n"
        "v_mfma_f32_16x16x32_f16 a[60:63], v[56:59], v[108:111], a[60:63]\n"
        "v_mfma_f32_16x16x32_f16 a[92:95], v[80:83], v[108:111], a[92:95]\n"
        "v_mfma_f32_16x16x32_f16 a[124:127], v[84:87], v[108:111], a[124:127]\n"
        "s_waitcnt vmcnt(59) lgkmcnt(6)\n"
        "v_mfma_f32_16x16x32_f16 a[0:3], v[88:91], v[112:115], a[0:3]\n"
        "ds_read_b128 v[108:111], %[bbase] offset:50944\n"
        "v_mfma_f32_16x16x32_f16 a[32:35], v[92:95], v[112:115], a[32:35]\n"
        "v_readlane_b32 s77, %[tabh], 13\n"
        "s_xor_b32 s77, s77, 0\n"
        "v_mfma_f32_16x16x32_f16 a[64:67], v[136:139], v[112:115], a[64:67]\n"
        "buffer_load_dwordx4 v[44:47], v64, %[rsA], s77 offen offset:0 sc0 sc1\n"
        "v_mfma_f32_16x16x32_f16 a[96:99], v[140:143], v[112:115], a[96:99]\n"
        "global_store_dwordx4 v60, a[220:223], %[my] offset:3072 sc0 sc1\n"
        "s_waitcnt lgkmcnt(6)\n"
        "v_mfma_f32_16x16x32_f16 a[4:7], v[88:91], v[116:119], a[4:7]\n"
        "ds_read_b128 v[112:115], %[bbase] offset:57344\n"
        "v_mfma_f32_16x16x32_f16 a[36:39], v[92:95], v[116:119], a[36:39]\n"
        "v_mfma_f32_16x16x32_f16 a[68:71], v[136:139], v[116:119], a[68:71]\n"
        "buffer_load_dwordx4 v[48:51], v64, %[rsA], s77 offen offset:1024 sc0 sc1\n"
        "v_mfma_f32_16x16x32_f16 a[100:103], v[140:143], v[116:119], a[100:103]\n"
        "s_waitcnt lgkmcnt(6)\n"
        "v_mfma_f32_16x16x32_f16 a[8:11], v[88:91], v[120:123], a[8:11]\n"
        "ds_read_b128 v[116:119], %[bbase] offset:57856\n"
        "v_mfma_f32_16x16x32_f16 a[40:43], v[92:95], v[120:123], a[40:43]\n"
        "v_mfma_f32_16x16x32_f16 a[72:75], v[136:139], v[120:123], a[72:75]\n"
        "buffer_load_dwordx4 v[52:55], v64, %[rsA], s77 offen offset:2048 sc0 sc1\n"
        "v_mfma_f32_16x16x32_f16 a[104:107], v[140:143], v[120:123], a[104:107]\n"
        "s_add_u32 s43, %[xsend], 24576\n"
        "v_add_u32 v60, s43, v61\n"
        "global_store_dwordx4 v60, a[224:227], %[my] offset:0 sc0 sc1\n"
        "s_waitcnt lgkmcnt(6)\n"
        "v_mfma_f32_16x16x32_f16 a[12:15], v[88:91], v[124:127], a[12:15]\n"
        "ds_read_b128 v[120:123], %[bbase] offset:58368\n"
        "v_mfma_f32_16x16x32_f16 a[44:47], v[92:95], v[124:127], a[44:47]\n"
        "v_mfma_f32_16x16x32_f16 a[76:79], v[136:139], v[124:127], a[76:79]\n"
        "buffer_load_dwordx4 v[56:59], v64, %[rsA], s77 offen offset:3072 sc0 sc1\n"
        "v_mfma_f32_16x16x32_f16 a[108:111], v[140:143], v[124:127], a[108:111]\n"
        "s_waitcnt lgkmcnt(6)\n"
        "v_mfma_f32_16x16x32_f16 a[16:19], v[88:91], v[72:75], a[16:19]\n"
        "ds_read_b128 v[124:127], %[bbase] offset:58880\n"
        "v_mfma_f32_16x16x32_f16 a[48:51], v[92:95], v[72:75], a[48:51]\n"
        "v_mfma_f32_16x16x32_f16 a[80:83], v[136:139], v[72:75], a[80:83]\n"
        "v_mfma_f32_16x16x32_f16 a[112:115], v[140:143], v[72:75], a[112:115]\n"
        "global_store_dwordx4 v60, a[228:231], %[my] offset:1024 sc0 sc1\n"
        "s_waitcnt lgkmcnt(6)\n"
        "v_mfma_f32_16x16x32_f16 a[20:23], v[88:91], v[76:79], a[20:23]\n"
        "ds_read_b128 v[72:75], %[bbase] offset:57600\n"
        "v_mfma_f32_16x16x32_f16 a[52:55], v[92:95], v[76:79], a[52:55]\n"
        "v_mfma_f32_16x16x32_f16 a[84:87], v[136:139], v[76:79], a[84:87]\n"
        "v_mfma_f32_16x16x32_f16 a[116:119], v[140:143], v[76:79], a[116:119]\n"
        "s_waitcnt lgkmcnt(6)\n"
        "v_mfma_f32_16x16x32_f16 a[24:27], v[88:91], v[104:107], a[24:27]\n"
        "ds_read_b128 v[76:79], %[bbase] offset:58112\n"
        "v_mfma_f32_16x16x32_f16 a[56:59], v[92:95], v[104:107], a[56:59]\n"
        "v_mfma_f32_16x16x32_f16 a[88:91], v[136:139], v[104:107], a[88:91]\n"
        "v_mfma_f32_16x16x32_f16 a[120:123], v[140:143], v[104:107], a[120:123]\n"
        "global_store_dwordx4 v60, a[232:235], %[my] offset:2048 sc0 sc1\n"
        "s_waitcnt lgkmcnt(6)\n"
        "v_mfma_f32_16x16x32_f16 a[28:31], v[88:91], v[108:111], a[28:31]\n"
        "ds_read_b128 v[104:107], %[bbase] offset:58624\n"
        "v_mfma_f32_16x16x32_f16 a[60:63], v[92:95], v[108:111], a[60:63]\n"
        "v_mfma_f32_16x16x32_f16 a[92:95], v[136:139], v[108:111], a[92:95]\n"
        "v_mfma_f32_16x16x32_f16 a[124:127], v[140:143], v[108:111], a[124:127]\n"
        "s_waitcnt vmcnt(61) lgkmcnt(6)\n"
        "v_mfma_f32_16x16x32_f16 a[0:3], v[152:155], v[112:115], a[0:3]\n"
        "ds_read_b128 v[108:111], %[bbase] offset:59136\n"
        "v_mfma_f32_16x16x32_f16 a[32:35], v[156:159], v[112:115], a[32:35]\n"
        "v_readlane_b32 s41, %[tabh], 14\n"
        "s_xor_b32 s41, s41, 0\n"
        "v_mfma_f32_16x16x32_f16 a[64:67], v[168:171], v[112:115], a[64:67]\n"
        "buffer_load_dwordx4 v[80:83], v64, %[rsA], s41 offen offset:0 sc0 sc1\n"
        "v_mfma_f32_16x16x32_f16 a[96:99], v[172:175], v[112:115], a[96:99]\n"
        "global_store_dwordx4 v60, a[236:239], %[my] offset:3072 sc0 sc1\n"
        "s_waitcnt lgkmcnt(6)\n"
        "v_mfma_f32_16x16x32_f16 a[4:7], v[152:155], v[116:119], a[4:7]\n"
        "ds_read_b128 v[112:115], v70 offset:0\n"
        "v_mfma_f32_16x16x32_f16 a[36:39], v[156:159], v[116:119], a[36:39]\n"
        "v_mfma_f32_16x16x32_f16 a[68:71], v[168:171], v[116:119], a[68:71]\n"
        "buffer_load_dwordx4 v[84:87], v64, %[rsA], s41 offen offset:1024 sc0 sc1\n"
        "v_mfma_f32_16x16x32_f16 a[100:103], v[172:175], v[116:119], a[100:103]\n"
        "s_waitcnt lgkmcnt(6)\n"
        "v_mfma_f32_16x16x32_f16 a[8:11], v[152:155], v[120:123], a[8:11]\n"
        "ds_read_b128 v[116:119], v70 offset:512\n"
        "v_mfma_f32_16x16x32_f16 a[40:43], v[156:159], v[120:123], a[40:43]\n"
        "v_mfma_f32_16x16x32_f16 a[72:75], v[168:171], v[120:123], a[72:75]\n"
        "buffer_load_dwordx4 v[88:91], v64, %[rsA], s41 offen offset:2048 sc0 sc1\n"
        "v_mfma_f32_16x16x32_f16 a[104:107], v[172:175], v[120:123], a[104:107]\n"
        "s_add_u32 s43, %[xsend], 28672\n"
        "v_add_u32 v60, s43, v61\n"
        "global_store_dwordx4 v60, a[240:243], %[my] offset:0 sc0 sc1\n"
        "s_waitcnt lgkmcnt(6)\n"
        "v_mfma_f32_16x16x32_f16 a[12:15], v[152:155], v[124:127], a[12:15]\n"
        "ds_read_b128 v[120:123], v70 offset:1024\n"
        "v_mfma_f32_16x16x32_f16 a[44:47], v[156:159], v[124:127], a[44:47]\n"
        "v_mfma_f32_16x16x32_f16 a[76:79], v[168:171], v[124:127], a[76:79]\n"
        "buffer_load_dwordx4 v[92:95], v64, %[rsA], s41 offen offset:3072 sc0 sc1\n"
        "v_mfma_f32_16x16x32_f16 a[108:111], v[172:175], v[124:127], a[108:111]\n"
        "s_waitcnt lgkmcnt(6)\n"
        "v_mfma_f32_16x16x32_f16 a[16:19], v[152:155], v[72:75], a[16:19]\n"
        "ds_read_b128 v[124:127], v70 offset:1536\n"
        "v_mfma_f32_16x16x32_f16 a[48:51], v[156:159], v[72:75], a[48:51]\n"
        "v_mfma_f32_16x16x32_f16 a[80:83], v[168:171], v[72:75], a[80:83]\n"
        "v_mfma_f32_16x16x32_f16 a[112:115], v[172:175], v[72:75], a[112:115]\n"
        "global_store_dwordx4 v60, a[244:247], %[my] offset:1024 sc0 sc1\n"
        "s_waitcnt lgkmcnt(6)\n"
        "v_mfma_f32_16x16x32_f16 a[20:23], v[152:155], v[76:79], a[20:23]\n"
        "ds_read_b128 v[72:75], v70 offset:256\n"
        "v_mfma_f32_16x16x32_f16 a[52:55], v[156:159], v[76:79], a[52:55]\n"
        "v_mfma_f32_16x16x32_f16 a[84:87], v[168:171], v[76:79], a[84:87]\n"
        "v_mfma_f32_16x16x32_f16 a[116:119], v[172:175], v[76:79], a[116:119]\n"
        "s_waitcnt lgkmcnt(6)\n"
        "v_mfma_f32_16x16x32_f16 a[24:27], v[152:155], v[104:107], a[24:27]\n"
        "ds_read_b128 v[76:79], v70 offset:768\n"
        "v_mfma_f32_16x16x32_f16 a[56:59], v[156:159], v[104:107], a[56:59]\n"
        "v_mfma_f32_16x16x32_f16 a[88:91], v[168:171], v[104:107], a[88:91]\n"
        "v_mfma_f32_16x16x32_f16 a[120:123], v[172:175], v[104:107], a[120:123]\n"
        "global_store_dwordx4 v60, a[248:251], %[my] offset:2048 sc0 sc1\n"
        "s_waitcnt lgkmcnt(6)\n"
        "v_mfma_f32_16x16x32_f16 a[28:31], v[152:155], v[108:111], a[28:31]\n"
        "ds_read_b128 v[104:107], v70 offset:1280\n"
        "v_mfma_f32_16x16x32_f16 a[60:63], v[156:159], v[108:111], a[60:63]\n"
        "v_mfma_f32_16x16x32_f16 a[92:95], v[168:171], v[108:111], a[92:95]\n"
        "v_mfma_f32_16x16x32_f16 a[124:127], v[172:175], v[108:111], a[124:127]\n"
        "v_mov_b32 v65, %[mflag]\n"
        "v_mov_b32 v66, 0x20000\n"
        "buffer_load_dword v65, v65, %[rsA], 0 offen sc0 sc1\n"
        "buffer_load_dword v66, v66, %[rsA], 0 offen sc1\n"
        "s_waitcnt vmcnt(63) lgkmcnt(6)\n"
        "v_mfma_f32_16x16x32_f16 a[0:3], v[184:187], v[112:115], a[0:3]\n"
        "ds_read_b128 v[108:111], v70 offset:1792\n"
        "v_mfma_f32_16x16x32_f16 a[32:35], v[188:191], v[112:115], a[32:35]\n"
        "v_readlane_b32 s77, %[tabh], 15\n"
        "s_xor_b32 s77, s77, 0\n"
        "v_mfma_f32_16x16x32_f16 a[64:67], v[96:99], v[112:115], a[64:67]\n"
        "buffer_load_dwordx4 v[136:139], v64, %[rsA], s77 offen offset:0 sc0 sc1\n"
        "v_mfma_f32_16x16x32_f16 a[96:99], v[100:103], v[112:115], a[96:99]\n"
        "global_store_dwordx4 v60, a[252:255], %[my] offset:3072 sc0 sc1\n"
        "s_waitcnt lgkmcnt(6)\n"
        "v_mfma_f32_16x16x32_f16 a[4:7], v[184:187], v[116:119], a[4:7]\n"
        "ds_read_b128 v[112:115], v70 offset:8192\n"
        "v_mfma_f32_16x16x32_f16 a[36:39], v[188:191], v[116:119], a[36:39]\n"
        "v_mfma_f32_16x16x32_f16 a[68:71], v[96:99], v[116:119], a[68:71]\n"
        "buffer_load_dwordx4 v[140:143], v64, %[rsA], s77 offen offset:1024 sc0 sc1\n"
        "v_mfma_f32_16x16x32_f16 a[100:103], v[100:103], v[116:119], a[100:103]\n"
        "s_waitcnt lgkmcnt(6)\n"
        "v_mfma_f32_16x16x32_f16 a[8:11], v[184:187], v[120:123], a[8:11]\n"
        "ds_read_b128 v[116:119], v70 offset:8704\n"
        "v_mfma_f32_16x16x32_f16 a[40:43], v[188:191], v[120:123], a[40:43]\n"
        "v_mfma_f32_16x16x32_f16 a[72:75], v[96:99], v[120:123], a[72:75]\n"
        "buffer_load_dwordx4 v[152:155], v64, %[rsA], s77 offen offset:2048 sc0 sc1\n"
        "v_mfma_f32_16x16x32_f16 a[104:107], v[100:103], v[120:123], a[104:107]\n"
        "s_waitcnt lgkmcnt(6)\n"
        "v_mfma_f32_16x16x32_f16 a[12:15], v[184:187], v[124:127], a[12:15]\n"
        "ds_read_b128 v[120:123], v70 offset:9216\n"
        "v_mfma_f32_16x16x32_f16 a[44:47], v[188:191], v[124:127], a[44:47]\n"
        "v_mfma_f32_16x16x32_f16 a[76:79], v[96:99], v[124:127], a[76:79]\n"
        "buffer_load_dwordx4 v[156:159], v64, %[rsA], s77 offen offset:3072 sc0 sc1\n"
        "v_mfma_f32_16x16x32_f16 a[108:111], v[100:103], v[124:127], a[108:111]\n"
        "s_waitcnt lgkmcnt(6)\n"
        "v_mfma_f32_16x16x32_f16 a[16:19], v[184:187], v[72:75], a[16:19]\n"
        "ds_read_b128 v[124:127], v70 offset:9728\n"
        "v_mfma_f32_16x16x32_f16 a[48:51], v[188:191], v[72:75], a[48:51]\n"
        "v_mfma_f32_16x16x32_f16 a[80:83], v[96:99], v[72:75], a[80:83]\n"
        "v_mfma_f32_16x16x32_f16 a[112:115], v[100:103], v[72:75], a[112:115]\n"
        "s_waitcnt lgkmcnt(6)\n"
        "v_mfma_f32_16x16x32_f16 a[20:23], v[184:187], v[76:79], a[20:23]\n"
        "ds_read_b128 v[72:75], v70 offset:8448\n"
        "v_mfma_f32_16x16x32_f16 a[52:55], v[188:191], v[76:79], a[52:55]\n"
        "v_mfma_f32_16x16x32_f16 a[84:87], v[96:99], v[76:79], a[84:87]\n"
        "v_mfma_f32_16x16x32_f16 a[116:119], v[100:103], v[76:79], a[116:119]\n"
        "s_waitcnt lgkmcnt(6)\n"
        "v_mfma_f32_16x16x32_f16 a[24:27], v[184:187], v[104:107], a[24:27]\n"
        "ds_read_b128 v[76:79], v70 offset:8960\n"
        "v_mfma_f32_16x16x32_f16 a[56:59], v[188:191], v[104:107], a[56:59]\n"
        "v_mfma_f32_16x16x32_f16 a[88:91], v[96:99], v[104:107], a[88:91]\n"
        "v_mfma_f32_16x16x32_f16 a[120:123], v[100:103], v[104:107], a[120:123]\n"
        "s_waitcnt lgkmcnt(6)\n"
        "v_mfma_f32_16x16x32_f16 a[28:31], v[184:187], v[108:111], a[28:31]\n"
        "ds_read_b128 v[104:107], v70 offset:9472\n"
        "v_mfma_f32_16x16x32_f16 a[60:63], v[188:191], v[108:111], a[60:63]\n"
        "v_mfma_f32_16x16x32_f16 a[92:95], v[96:99], v[108:111], a[92:95]\n"
        "v_mfma_f32_16x16x32_f16 a[124:127], v[100:103], v[108:111], a[124:127]\n"
        "s_add_u32 s43, s66, 3\n"
        "v_mov_b32 v68, %[pflag]\n"
        "v_mov_b32 v67, s43\n"
        "s_waitcnt vmcnt(21)\n"
        "s_mov_b64 exec, 1\n"
        "buffer_store_dword v67, v68, %[rsA], 0 offen sc0 sc1\n"
        "s_mov_b64 exec, -1\n"
        "s_waitcnt vmcnt(63) lgkmcnt(6)\n"
        "v_mfma_f32_16x16x32_f16 a[0:3], v[128:131], v[112:115], a[0:3]\n"
        "ds_read_b128 v[108:111], v70 offset:9984\n"
        "v_mfma_f32_16x16x32_f16 a[32:35], v[132:135], v[112:115], a[32:35]\n"
        "v_readlane_b32 s41, %[tabh], 16\n"
        "s_xor_b32 s41, s41, 0\n"
        "v_mfma_f32_16x16x32_f16 a[64:67], v[144:147], v[112:115], a[64:67]\n"
        "buffer_load_dwordx4 v[168:171], v64, %[rsA], s41 offen offset:0 sc0 sc1\n"
        "v_mfma_f32_16x16x32_f16 a[96:99], v[148:151], v[112:115], a[96:99]\n"
        "s_waitcnt lgkmcnt(6)\n"
        "v_mfma_f32_16x16x32_f16 a[4:7], v[128:131], v[116:119], a[4:7]\n"
        "ds_read_b128 v[112:115], v70 offset:16384\n"
        "v_mfma_f32_16x16x32_f16 a[36:39], v[132:135], v[116:119], a[36:39]\n"
        "v_mfma_f32_16x16x32_f16 a[68:71], v[144:147], v[116:119], a[68:71]\n"
        "buffer_load_dwordx4 v[172:175], v64, %[rsA], s41 offen offset:1024 sc0 sc1\n"
        "v_mfma_f32_16x16x32_f16 a[100:103], v[148:151], v[116:119], a[100:103]\n"
        "s_waitcnt lgkmcnt(6)\n"
        "v_mfma_f32_16x16x32_f16 a[8:11], v[128:131], v[120:123], a[8:11]\n"
        "ds_read_b128 v[116:119], v70 offset:16896\n"
        "v_mfma_f32_16x16x32_f16 a[40:43], v[132:135], v[120:123], a[40:43]\n"
        "v_mfma_f32_16x16x32_f16 a[72:75], v[144:147], v[120:123], a[72:75]\n"
        "buffer_load_dwordx4 v[184:187], v64, %[rsA], s41 offen offset:2048 sc0 sc1\n"
        "v_mfma_f32_16x16x32_f16 a[104:107], v[148:151], v[120:123], a[104:107]\n"
        "s_waitcnt lgkmcnt(6)\n"
        "v_mfma_f32_16x16x32_f16 a[12:15], v[128:131], v[124:127], a[12:15]\n"
        "ds_read_b128 v[120:123], v70 offset:17408\n"
        "v_mfma_f32_16x16x32_f16 a[44:47], v[132:135], v[124:127], a[44:47]\n"
        "v_mfma_f32_16x16x32_f16 a[76:79], v[144:147], v[124:127], a[76:79]\n"
        "buffer_load_dwordx4 v[188:191], v64, %[rsA], s41 offen offset:3072 sc0 sc1\n"
        "v_mfma_f32_16x16x32_f16 a[108:111], v[148:151], v[124:127], a[108:111]\n"
        "s_waitcnt lgkmcnt(6)\n"
        "v_mfma_f32_16x16x32_f16 a[16:19], v[128:131], v[72:75], a[16:19]\n"
        "ds_read_b128 v[124:127], v70 offset:17920\n"
        "v_mfma_f32_16x16x32_f16 a[48:51], v[132:135], v[72:75], a[48:51]\n"
        "v_mfma_f32_16x16x32_f16 a[80:83], v[144:147], v[72:75], a[80:83]\n"
        "v_mfma_f32_16x16x32_f16 a[112:115], v[148:151], v[72:75], a[112:115]\n"
        "s_waitcnt lgkmcnt(6)\n"
        "v_mfma_f32_16x16x32_f16 a[20:23], v[128:131], v[76:79], a[20:23]\n"
        "ds_read_b128 v[72:75], v70 offset:16640\n"
        "v_mfma_f32_16x16x32_f16 a[52:55], v[132:135], v[76:79], a[52:55]\n"
        "v_mfma_f32_16x16x32_f16 a[84:87], v[144:147], v[76:79], a[84:87]\n"
        "v_mfma_f32_16x16x32_f16 a[116:119], v[148:151], v[76:79], a[116:119]\n"
        "s_waitcnt lgkmcnt(6)\n"
        "v_mfma_f32_16x16x32_f16 a[24:27], v[128:131], v[104:107], a[24:27]\n"
        "ds_read_b128 v[76:79], v70 offset:17152\n"
        "v_mfma_f32_16x16x32_f16 a[56:59], v[132:135], v[104:107], a[56:59]\n"
        "v_mfma_f32_16x16x32_f16 a[88:91], v[144:147], v[104:107], a[88:91]\n"
        "v_mfma_f32_16x16x32_f16 a[120:123], v[148:151], v[104:107], a[120:123]\n"
        "s_waitcnt lgkmcnt(6)\n"
        "v_mfma_f32_16x16x32_f16 a[28:31], v[128:131], v[108:111], a[28:31]\n"
        "ds_read_b128 v[104:107], v70 offset:17664\n"
        "v_mfma_f32_16x16x32_f16 a[60:63], v[132:135], v[108:111], a[60:63]\n"
        "v_mfma_f32_16x16x32_f16 a[92:95], v[144:147], v[108:111], a[92:95]\n"
        "v_mfma_f32_16x16x32_f16 a[124:127], v[148:151], v[108:111], a[124:127]\n"
        "s_waitcnt vmcnt(10)\n"
        "v_readfirstlane_b32 s69, v65\n"
        "v_readfirstlane_b32 s45, v66\n"
        "s_nop 1\n"
        "s_cmp_eq_u32 s45, s70\n"
        "s_cselect_b32 s53, 1, s53\n"
        "s_max_u32 s69, s69, s66\n"
        "s_sub_u32 s69, s69, s66\n"
        "s_min_u32 s69, s69, 2\n"
        "s_cmp_eq_u32 s67, 0\n"
        "s_cselect_b32 s67, s69, s67\n"
        "s_waitcnt vmcnt(61) lgkmcnt(6)\n"
        "v_mfma_f32_16x16x32_f16 a[0:3], v[160:163], v[112:115], a[0:3]\n"
        "ds_read_b128 v[108:111], v70 offset:18176\n"
        "v_mfma_f32_16x16x32_f16 a[32:35], v[164:167], v[112:115], a[32:35]\n"
        "v_readlane_b32 s77, %[tabh], 17\n"
        "s_xor_b32 s77, s77, 0\n"
        "v_mfma_f32_16x16x32_f16 a[64:67], v[176:179], v[112:115], a[64:67]\n"
        "buffer_load_dwordx4 v[96:99], v64, %[rsA], s77 offen offset:0 sc0 sc1\n"
        "v_mfma_f32_16x16x32_f16 a[96:99], v[180:183], v[112:115], a[96:99]\n"
        "s_add_u32 s52, %[xrecv], 0\n"
        "s_cmp_gt_u32 s67, 0\n"
        "s_cselect_b32 s52, s52, 0x6000000\n"
        "buffer_load_dwordx4 a[128:131], v61, %[rsA], s52 offen offset:0 sc0 sc1\n"
        "s_waitcnt lgkmcnt(6)\n"
        "v_mfma_f32_16x16x32_f16 a[4:7], v[160:163], v[116:119], a[4:7]\n"
        "ds_read_b128 v[112:115], v70 offset:24576\n"
        "v_mfma_f32_16x16x32_f16 a[36:39], v[164:167], v[116:119], a[36:39]\n"
        "v_mfma_f32_16x16x32_f16 a[68:71], v[176:179], v[116:119], a[68:71]\n"
        "buffer_load_dwordx4 v[100:103], v64, %[rsA], s77 offen offset:1024 sc0 sc1\n"
        "v_mfma_f32_16x16x32_f16 a[100:103], v[180:183], v[116:119], a[100:103]\n"
        "s_waitcnt lgkmcnt(6)\n"
        "v_mfma_f32_16x16x32_f16 a[8:11], v[160:163], v[120:123], a[8:11]\n"
        "ds_read_b128 v[116:119], v70 offset:25088\n"
        "v_mfma_f32_16x16x32_f16 a[40:43], v[164:167], v[120:123], a[40:43]\n"
        "v_mfma_f32_16x16x32_f16 a[72:75], v[176:179], v[120:123], a[72:75]\n"
        "buffer_load_dwordx4 v[128:131], v64, %[rsA], s77 offen offset:2048 sc0 sc1\n"
        "v_mfma_f32_16x16x32_f16 a[104:107], v[180:183], v[120:123], a[104:107]\n"
        "buffer_load_dwordx4 a[132:135], v61, %[rsA], s52 offen offset:1024 sc0 sc1\n"
        "s_waitcnt lgkmcnt(6)\n"
        "v_mfma_f32_16x16x32_f16 a[12:15], v[160:163], v[124:127], a[12:15]\n"
        "ds_read_b128 v[120:123], v70 offset:25600\n"
        "v_mfma_f32_16x16x32_f16 a[44:47], v[164:167], v[124:127], a[44:47]\n"
        "v_mfma_f32_16x16x32_f16 a[76:79], v[176:179], v[124:127], a[76:79]\n"
        "buffer_load_dwordx4 v[132:135], v64, %[rsA], s77 offen offset:3072 sc0 sc1\n"
        "v_mfma_f32_16x16x32_f16 a[108:111], v[180:183], v[124:127], a[108:111]\n"
        "s_waitcnt lgkmcnt(6)\n"
        "v_mfma_f32_16x16x32_f16 a[16:19], v[160:163], v[72:75], a[16:19]\n"
        "ds_read_b128 v[124:127], v70 offset:26112\n"
        "v_mfma_f32_16x16x32_f16 a[48:51], v[164:167], v[72:75], a[48:51]\n"
        "v_mfma_f32_16x16x32_f16 a[80:83], v[176:179], v[72:75], a[80:83]\n"
        "v_mfma_f32_16x16x32_f16 a[112:115], v[180:183], v[72:75], a[112:115]\n"
        "buffer_load_dwordx4 a[136:139], v61, %[rsA], s52 offen offset:2048 sc0 sc1\n"
        "s_waitcnt lgkmcnt(6)\n"
        "v_mfma_f32_16x16x32_f16 a[20:23], v[160:163], v[76:79], a[20:23]\n"
        "ds_read_b128 v[72:75], v70 offset:24832\n"
        "v_mfma_f32_16x16x32_f16 a[52:55], v[164:167], v[76:79], a[52:55]\n"
        "v_mfma_f32_16x16x32_f16 a[84:87], v[176:179], v[76:79], a[84:87]\n"
        "v_mfma_f32_16x16x32_f16 a[116:119], v[180:183], v[76:79], a[116:119]\n"
        "s_waitcnt lgkmcnt(6)\n"
        "v_mfma_f32_16x16x32_f16 a[24:27], v[160:163], v[104:107], a[24:27]\n"
        "ds_read_b128 v[76:79], v70 offset:25344\n"
        "v_mfma_f32_16x16x32_f16 a[56:59], v[164:167], v[104:107], a[56:59]\n"
        "v_mfma_f32_16x16x32_f16 a[88:91], v[176:179], v[104:107], a[88:91]\n"
        "v_mfma_f32_16x16x32_f16 a[120:123], v[180:183], v[104:107], a[120:123]\n"
        "buffer_load_dwordx4 a[140:143], v61, %[rsA], s52 offen offset:3072 sc0 sc1\n"
        "s_waitcnt lgkmcnt(6)\n"
        "v_mfma_f32_16x16x32_f16 a[28:31], v[160:163], v[108:111], a[28:31]\n"
        "ds_read_b128 v[104:107], v70 offset:25856\n"
        "v_mfma_f32_16x16x32_f16 a[60:63], v[164:167], v[108:111], a[60:63]\n"
        "v_mfma_f32_16x16x32_f16 a[92:95], v[176:179], v[108:111], a[92:95]\n"
        "v_mfma_f32_16x16x32_f16 a[124:127], v[180:183], v[108:111], a[124:127]\n"
        "s_add_u32 s43, s66, 4\n"
        "v_mov_b32 v68, %[pflag]\n"
        "v_mov_b32 v67, s43\n"
        "s_waitcnt vmcnt(16)\n"
        "s_mov_b64 exec, 1\n"
        "buffer_store_dword v67, v68, %[rsA], 0 offen sc0 sc1\n"
        "s_mov_b64 exec, -1\n"
        "s_waitcnt vmcnt(63) lgkmcnt(6)\n"
        "v_mfma_f32_16x16x32_f16 a[0:3], v[192:195], v[112:115], a[0:3]\n"
        "ds_read_b128 v[108:111], v70 offset:26368\n"
        "v_mfma_f32_16x16x32_f16 a[32:35], v[196:199], v[112:115], a[32:35]\n"
        "v_readlane_b32 s41, %[tabh], 18\n"
        "s_xor_b32 s41, s41, 0\n"
        "v_mfma_f32_16x16x32_f16 a[64:67], v[200:203], v[112:115], a[64:67]\n"
        "buffer_load_dwordx4 v[144:147], v64, %[rsA], s41 offen offset:0 sc0 sc1\n"
        "v_mfma_f32_16x16x32_f16 a[96:99], v[204:207], v[112:115], a[96:99]\n"
        "s_add_u32 s52, %[xrecv], 4096\n"
        "s_cmp_gt_u32 s67, 0\n"
        "s_cselect_b32 s52, s52, 0x6000000\n"
        "buffer_load_dwordx4 a[144:147], v61, %[rsA], s52 offen offset:0 sc0 sc1\n"
        "s_waitcnt lgkmcnt(6)\n"
        "v_mfma_f32_16x16x32_f16 a[4:7], v[192:195], v[116:119], a[4:7]\n"
        "ds_read_b128 v[112:115], v70 offset:32768\n"
        "v_mfma_f32_16x16x32_f16 a[36:39], v[196:199], v[116:119], a[36:39]\n"
        "v_mfma_f32_16x16x32_f16 a[68:71], v[200:203], v[116:119], a[68:71]\n"
        "buffer_load_dwordx4 v[148:151], v64, %[rsA], s41 offen offset:1024 sc0 sc1\n"
        "v_mfma_f32_16x16x32_f16 a[100:103], v[204:207], v[116:119], a[100:103]\n"
        "s_waitcnt lgkmcnt(6)\n"
        "v_mfma_f32_16x16x32_f16 a[8:11], v[192:195], v[120:123], a[8:11]\n"
        "ds_read_b128 v[116:119], v70 offset:33280\n"
        "v_mfma_f32_16x16x32_f16 a[40:43], v[196:199], v[120:123], a[40:43]\n"
        "v_mfma_f32_16x16x32_f16 a[72:75], v[200:203], v[120:123], a[72:75]\n"
        "buffer_load_dwordx4 v[160:163], v64, %[rsA], s41 offen offset:2048 sc0 sc1\n"
        "v_mfma_f32_16x16x32_f16 a[104:107], v[204:207], v[120:123], a[104:107]\n"
        "buffer_load_dwordx4 a[148:151], v61, %[rsA], s52 offen offset:1024 sc0 sc1\n"
        "s_waitcnt lgkmcnt(6)\n"
        "v_mfma_f32_16x16x32_f16 a[12:15], v[192:195], v[124:127], a[12:15]\n"
        "ds_read_b128 v[120:123], v70 offset:33792\n"
        "v_mfma_f32_16x16x32_f16 a[44:47], v[196:199], v[124:127], a[44:47]\n"
        "v_mfma_f32_16x16x32_f16 a[76:79], v[200:203], v[124:127], a[76:79]\n"
        "buffer_load_dwordx4 v[164:167], v64, %[rsA], s41 offen offset:3072 sc0 sc1\n"
        "v_mfma_f32_16x16x32_f16 a[108:111], v[204:207], v[124:127], a[108:111]\n"
        "s_waitcnt lgkmcnt(6)\n"
        "v_mfma_f32_16x16x32_f16 a[16:19], v[192:195], v[72:75], a[16:19]\n"
        "ds_read_b128 v[124:127], v70 offset:34304\n"
        "v_mfma_f32_16x16x32_f16 a[48:51], v[196:199], v[72:75], a[48:51]\n"
        "v_mfma_f32_16x16x32_f16 a[80:83], v[200:203], v[72:75], a[80:83]\n"
        "v_mfma_f32_16x16x32_f16 a[112:115], v[204:207], v[72:75], a[112:115]\n"
        "buffer_load_dwordx4 a[152:155], v61, %[rsA], s52 offen offset:2048 sc0 sc1\n"
        "s_waitcnt lgkmcnt(6)\n"
        "v_mfma_f32_16x16x32_f16 a[20:23], v[192:195], v[76:79], a[20:23]\n"
        "ds_read_b128 v[72:75], v70 offset:33024\n"
        "v_mfma_f32_16x16x32_f16 a[52:55], v[196:199], v[76:79], a[52:55]\n"
        "v_mfma_f32_16x16x32_f16 a[84:87], v[200:203], v[76:79], a[84:87]\n"
        "v_mfma_f32_16x16x32_f16 a[116:119], v[204:207], v[76:79], a[116:119]\n"
        "s_waitcnt lgkmcnt(6)\n"
        "v_mfma_f32_16x16x32_f16 a[24:27], v[192:195], v[104:107], a[24:27]\n"
        "ds_read_b128 v[76:79], v70 offset:33536\n"
        "v_mfma_f32_16x16x32_f16 a[56:59], v[196:199], v[104:107], a[56:59]\n"
        "v_mfma_f32_16x16x32_f16 a[88:91], v[200:203], v[104:107], a[88:91]\n"
        "v_mfma_f32_16x16x32_f16 a[120:123], v[204:207], v[104:107], a[120:123]\n"
        "buffer_load_dwordx4 a[156:159], v61, %[rsA], s52 offen offset:3072 sc0 sc1\n"
        "s_waitcnt lgkmcnt(6)\n"
        "v_mfma_f32_16x16x32_f16 a[28:31], v[192:195], v[108:111], a[28:31]\n"
        "ds_read_b128 v[104:107], v70 offset:34048\n"
        "v_mfma_f32_16x16x32_f16 a[60:63], v[196:199], v[108:111], a[60:63]\n"
        "v_mfma_f32_16x16x32_f16 a[92:95], v[200:203], v[108:111], a[92:95]\n"
        "v_mfma_f32_16x16x32_f16 a[124:127], v[204:207], v[108:111], a[124:127]\n"
        "s_waitcnt vmcnt(63) lgkmcnt(6)\n"
        "v_mfma_f32_16x16x32_f16 a[0:3], v[208:211], v[112:115], a[0:3]\n"
        "ds_read_b128 v[108:111], v70 offset:34560\n"
        "v_mfma_f32_16x16x32_f16 a[32:35], v[212:215], v[112:115], a[32:35]\n"
        "v_readlane_b32 s77, %[tabh], 19\n"
        "s_xor_b32 s77, s77, 0\n"
        "v_mfma_f32_16x16x32_f16 a[64:67], v[216:219], v[112:115], a[64:67]\n"
        "buffer_load_dwordx4 v[176:179], v64, %[rsA], s77 offen offset:0 sc0 sc1\n"
        "v_mfma_f32_16x16x32_f16 a[96:99], v[220:223], v[112:115], a[96:99]\n"
        "s_add_u32 s52, %[xrecv], 8192\n"
        "s_cmp_gt_u32 s67, 1\n"
        "s_cselect_b32 s52, s52, 0x6000000\n"
        "buffer_load_dwordx4 a[160:163], v61, %[rsA], s52 offen offset:0 sc0 sc1\n"
        "s_waitcnt lgkmcnt(6)\n"
        "v_mfma_f32_16x16x32_f16 a[4:7], v[208:211], v[116:119], a[4:7]\n"
        "ds_read_b128 v[112:115], v70 offset:40960\n"
        "v_mfma_f32_16x16x32_f16 a[36:39], v[212:215], v[116:119], a[36:39]\n"
        "v_mfma_f32_16x16x32_f16 a[68:71], v[216:219], v[116:119], a[68:71]\n"
        "buffer_load_dwordx4 v[180:183], v64, %[rsA], s77 offen offset:1024 sc0 sc1\n"
        "v_mfma_f32_16x16x32_f16 a[100:103], v[220:223], v[116:119], a[100:103]\n"
        "s_waitcnt lgkmcnt(6)\n"
        "v_mfma_f32_16x16x32_f16 a[8:11], v[208:211], v[120:123], a[8:11]\n"
        "ds_read_b128 v[116:119], v70 offset:41472\n"
        "v_mfma_f32_16x16x32_f16 a[40:43], v[212:215], v[120:123], a[40:43]\n"
        "v_mfma_f32_16x16x32_f16 a[72:75], v[216:219], v[120:123], a[72:75]\n"
        "buffer_load_dwordx4 v[192:195], v64, %[rsA], s77 offen offset:2048 sc0 sc1\n"
        "v_mfma_f32_16x16x32_f16 a[104:107], v[220:223], v[120:123], a[104:107]\n"
        "buffer_load_dwordx4 a[164:167], v61, %[rsA], s52 offen offset:1024 sc0 sc1\n"
        "s_waitcnt lgkmcnt(6)\n"
        "v_mfma_f32_16x16x32_f16 a[12:15], v[208:211], v[124:127], a[12:15]\n"
        "ds_read_b128 v[120:123], v70 offset:41984\n"
        "v_mfma_f32_16x16x32_f16 a[44:47], v[212:215], v[124:127], a[44:47]\n"
        "v_mfma_f32_16x16x32_f16 a[76:79], v[216:219], v[124:127], a[76:79]\n"
        "buffer_load_dwordx4 v[196:199], v64, %[rsA], s77 offen offset:3072 sc0 sc1\n"
        "v_mfma_f32_16x16x32_f16 a[108:111], v[220:223], v[124:127], a[108:111]\n"
        "s_waitcnt lgkmcnt(6)\n"
        "v_mfma_f32_16x16x32_f16 a[16:19], v[208:211], v[72:75], a[16:19]\n"
        "ds_read_b128 v[124:127], v70 offset:42496\n"
        "v_mfma_f32_16x16x32_f16 a[48:51], v[212:215], v[72:75], a[48:51]\n"
        "v_mfma_f32_16x16x32_f16 a[80:83], v[216:219], v[72:75], a[80:83]\n"
        "v_mfma_f32_16x16x32_f16 a[112:115], v[220:223], v[72:75], a[112:115]\n"
        "buffer_load_dwordx4 a[168:171], v61, %[rsA], s52 offen offset:2048 sc0 sc1\n"
        "s_waitcnt lgkmcnt(6)\n"
        "v_mfma_f32_16x16x32_f16 a[20:23], v[208:211], v[76:79], a[20:23]\n"
        "ds_read_b128 v[72:75], v70 offset:41216\n"
        "v_mfma_f32_16x16x32_f16 a[52:55], v[212:215], v[76:79], a[52:55]\n"
        "v_mfma_f32_16x16x32_f16 a[84:87], v[216:219], v[76:79], a[84:87]\n"
        "v_mfma_f32_16x16x32_f16 a[116:119], v[220:223], v[76:79], a[116:119]\n"
        "s_waitcnt lgkmcnt(6)\n"
        "v_mfma_f32_16x16x32_f16 a[24:27], v[208:211], v[104:107], a[24:27]\n"
        "ds_read_b128 v[76:79], v70 offset:41728\n"
        "v_mfma_f32_16x16x32_f16 a[56:59], v[212:215], v[104:107], a[56:59]\n"
        "v_mfma_f32_16x16x32_f16 a[88:91], v[216:219], v[104:107], a[88:91]\n"
        "v_mfma_f32_16x16x32_f16 a[120:123], v[220:223], v[104:107], a[120:123]\n"
        "buffer_load_dwordx4 a[172:175], v61, %[rsA], s52 offen offset:3072 sc0 sc1\n"
        "s_waitcnt lgkmcnt(6)\n"
        "v_mfma_f32_16x16x32_f16 a[28:31], v[208:211], v[108:111], a[28:31]\n"
        "ds_read_b128 v[104:107], v70 offset:42240\n"
        "v_mfma_f32_16x16x32_f16 a[60:63], v[212:215], v[108:111], a[60:63]\n"
        "v_mfma_f32_16x16x32_f16 a[92:95], v[216:219], v[108:111], a[92:95]\n"
        "v_mfma_f32_16x16x32_f16 a[124:127], v[220:223], v[108:111], a[124:127]\n"
        "s_waitcnt vmcnt(63) lgkmcnt(6)\n"
        "v_mfma_f32_16x16x32_f16 a[0:3], v[224:227], v[112:115], a[0:3]\n"
        "ds_read_b128 v[108:111], v70 offset:42752\n"
        "v_mfma_f32_16x16x32_f16 a[32:35], v[228:231], v[112:115], a[32:35]\n"
        "v_mfma_f32_16x16x32_f16 a[64:67], v[232:235], v[112:115], a[64:67]\n"
        "v_mfma_f32_16x16x32_f16 a[96:99], v[236:239], v[112:115], a[96:99]\n"
        "s_add_u32 s52, %[xrecv], 12288\n"
        "s_cmp_gt_u32 s67, 1\n"
        "s_cselect_b32 s52, s52, 0x6000000\n"
        "buffer_load_dwordx4 a[176:179], v61, %[rsA], s52 offen offset:0 sc0 sc1\n"
        "s_waitcnt lgkmcnt(6)\n"
        "v_mfma_f32_16x16x32_f16 a[4:7], v[224:227], v[116:119], a[4:7]\n"
        "ds_read_b128 v[112:115], v70 offset:49152\n"
        "v_mfma_f32_16x16x32_f16 a[36:39], v[228:231], v[116:119], a[36:39]\n"
        "v_mfma_f32_16x16x32_f16 a[68:71], v[232:235], v[116:119], a[68:71]\n"
        "v_mfma_f32_16x16x32_f16 a[100:103], v[236:239], v[116:119], a[100:103]\n"
        "s_waitcnt lgkmcnt(6)\n"
        "v_mfma_f32_16x16x32_f16 a[8:11], v[224:227], v[120:123], a[8:11]\n"
        "ds_read_b128 v[116:119], v70 offset:49664\n"
        "v_mfma_f32_16x16x32_f16 a[40:43], v[228:231], v[120:123], a[40:43]\n"
        "v_mfma_f32_16x16x32_f16 a[72:75], v[232:235], v[120:123], a[72:75]\n"
        "v_mfma_f32_16x16x32_f16 a[104:107], v[236:239], v[120:123], a[104:107]\n"
        "buffer_load_dwordx4 a[180:183], v61, %[rsA], s52 offen offset:1024 sc0 sc1\n"
        "s_waitcnt lgkmcnt(6)\n"
        "v_mfma_f32_16x16x32_f16 a[12:15], v[224:227], v[124:127], a[12:15]\n"
        "ds_read_b128 v[120:123], v70 offset:50176\n"
        "v_mfma_f32_16x16x32_f16 a[44:47], v[228:231], v[124:127], a[44:47]\n"
        "v_mfma_f32_16x16x32_f16 a[76:79], v[232:235], v[124:127], a[76:79]\n"
        "v_mfma_f32_16x16x32_f16 a[108:111], v[236:239], v[124:127], a[108:111]\n"
        "s_waitcnt lgkmcnt(6)\n"
        "v_mfma_f32_16x16x32_f16 a[16:19], v[224:227], v[72:75], a[16:19]\n"
        "ds_read_b128 v[124:127], v70 offset:50688\n"
        "v_mfma_f32_16x16x32_f16 a[48:51], v[228:231], v[72:75], a[48:51]\n"
        "v_mfma_f32_16x16x32_f16 a[80:83], v[232:235], v[72:75], a[80:83]\n"
        "v_mfma_f32_16x16x32_f16 a[112:115], v[236:239], v[72:75], a[112:115]\n"
        "buffer_load_dwordx4 a[184:187], v61, %[rsA], s52 offen offset:2048 sc0 sc1\n"
        "s_waitcnt lgkmcnt(6)\n"
        "v_mfma_f32_16x16x32_f16 a[20:23], v[224:227], v[76:79], a[20:23]\n"
        "ds_read_b128 v[72:75], v70 offset:49408\n"
        "v_mfma_f32_16x16x32_f16 a[52:55], v[228:231], v[76:79], a[52:55]\n"
        "v_mfma_f32_16x16x32_f16 a[84:87], v[232:235], v[76:79], a[84:87]\n"
        "v_mfma_f32_16x16x32_f16 a[116:119], v[236:239], v[76:79], a[116:119]\n"
        "s_waitcnt lgkmcnt(6)\n"
        "v_mfma_f32_16x16x32_f16 a[24:27], v[224:227], v[104:107], a[24:27]\n"
        "ds_read_b128 v[76:79], v70 offset:49920\n"
        "v_mfma_f32_16x16x32_f16 a[56:59], v[228:231], v[104:107], a[56:59]\n"
        "v_mfma_f32_16x16x32_f16 a[88:91], v[232:235], v[104:107], a[88:91]\n"
        "v_mfma_f32_16x16x32_f16 a[120:123], v[236:239], v[104:107], a[120:123]\n"
        "buffer_load_dwordx4 a[188:191], v61, %[rsA], s52 offen offset:3072 sc0 sc1\n"
        "s_waitcnt lgkmcnt(6)\n"
        "v_mfma_f32_16x16x32_f16 a[28:31], v[224:227], v[108:111], a[28:31]\n"
        "ds_read_b128 v[104:107], v70 offset:50432\n"
        "v_mfma_f32_16x16x32_f16 a[60:63], v[228:231], v[108:111], a[60:63]\n"
        "v_mfma_f32_16x16x32_f16 a[92:95], v[232:235], v[108:111], a[92:95]\n"
        "v_mfma_f32_16x16x32_f16 a[124:127], v[236:239], v[108:111], a[124:127]\n"
        "s_waitcnt vmcnt(63) lgkmcnt(6)\n"
        "v_mfma_f32_16x16x32_f16 a[0:3], v[240:243], v[112:115], a[0:3]\n"
        "ds_read_b128 v[108:111], v70 offset:50944\n"
        "v_mfma_f32_16x16x32_f16 a[32:35], v[244:247], v[112:115], a[32:35]\n"
        "v_mfma_f32_16x16x32_f16 a[64:67], v[248:251], v[112:115], a[64:67]\n"
        "v_mfma_f32_16x16x32_f16 a[96:99], v[252:255], v[112:115], a[96:99]\n"
        "s_waitcnt lgkmcnt(6)\n"
        "v_mfma_f32_16x16x32_f16 a[4:7], v[240:243], v[116:119], a[4:7]\n"
        "ds_read_b128 v[112:115], v70 offset:57344\n"
        "v_mfma_f32_16x16x32_f16 a[36:39], v[244:247], v[116:119], a[36:39]\n"
        "v_mfma_f32_16x16x32_f16 a[68:71], v[248:251], v[116:119], a[68:71]\n"
        "v_mfma_f32_16x16x32_f16 a[100:103], v[252:255], v[116:119], a[100:103]\n"
        "s_waitcnt lgkmcnt(6)\n"
        "v_mfma_f32_16x16x32_f16 a[8:11], v[240:243], v[120:123], a[8:11]\n"
        "ds_read_b128 v[116:119], v70 offset:57856\n"
        "v_mfma_f32_16x16x32_f16 a[40:43], v[244:247], v[120:123], a[40:43]\n"
        "v_mfma_f32_16x16x32_f16 a[72:75], v[248:251], v[120:123], a[72:75]\n"
        "v_mfma_f32_16x16x32_f16 a[104:107], v[252:255], v[120:123], a[104:107]\n"
        "s_waitcnt lgkmcnt(6)\n"
        "v_mfma_f32_16x16x32_f16 a[12:15], v[240:243], v[124:127], a[12:15]\n"
        "ds_read_b128 v[120:123], v70 offset:58368\n"
        "v_mfma_f32_16x16x32_f16 a[44:47], v[244:247], v[124:127], a[44:47]\n"
        "v_mfma_f32_16x16x32_f16 a[76:79], v[248:251], v[124:127], a[76:79]\n"
        "v_mfma_f32_16x16x32_f16 a[108:111], v[252:255], v[124:127], a[108:111]\n"
        "s_waitcnt lgkmcnt(6)\n"
        "v_mfma_f32_16x16x32_f16 a[16:19], v[240:243], v[72:75], a[16:19]\n"
        "ds_read_b128 v[124:127], v70 offset:58880\n"
        "v_mfma_f32_16x16x32_f16 a[48:51], v[244:247], v[72:75], a[48:51]\n"
        "v_mfma_f32_16x16x32_f16 a[80:83], v[248:251], v[72:75], a[80:83]\n"
        "v_mfma_f32_16x16x32_f16 a[112:115], v[252:255], v[72:75], a[112:115]\n"
        "s_waitcnt lgkmcnt(6)\n"
        "v_mfma_f32_16x16x32_f16 a[20:23], v[240:243], v[76:79], a[20:23]\n"
        "ds_read_b128 v[72:75], v70 offset:57600\n"
        "v_mfma_f32_16x16x32_f16 a[52:55], v[244:247], v[76:79], a[52:55]\n"
        "v_mfma_f32_16x16x32_f16 a[84:87], v[248:251], v[76:79], a[84:87]\n"
        "v_mfma_f32_16x16x32_f16 a[116:119], v[252:255], v[76:79], a[116:119]\n"
        "s_waitcnt lgkmcnt(6)\n"
        "v_mfma_f32_16x16x32_f16 a[24:27], v[240:243], v[104:107], a[24:27]\n"
        "ds_read_b128 v[76:79], v70 offset:58112\n"
        "v_mfma_f32_16x16x32_f16 a[56:59], v[244:247], v[104:107], a[56:59]\n"
        "v_mfma_f32_16x16x32_f16 a[88:91], v[248:251], v[104:107], a[88:91]\n"
        "v_mfma_f32_16x16x32_f16 a[120:123], v[252:255], v[104:107], a[120:123]\n"
        "s_waitcnt lgkmcnt(6)\n"
        "v_mfma_f32_16x16x32_f16 a[28:31], v[240:243], v[108:111], a[28:31]\n"
        "ds_read_b128 v[104:107], v70 offset:58624\n"
        "v_mfma_f32_16x16x32_f16 a[60:63], v[244:247], v[108:111], a[60:63]\n"
        "v_mfma_f32_16x16x32_f16 a[92:95], v[248:251], v[108:111], a[92:95]\n"
        "v_mfma_f32_16x16x32_f16 a[124:127], v[252:255], v[108:111], a[124:127]\n"
        "s_waitcnt vmcnt(63) lgkmcnt(6)\n"
        "v_mfma_f32_16x16x32_f16 a[0:3], v[12:15], v[112:115], a[0:3]\n"
        "ds_read_b128 v[108:111], v70 offset:59136\n"
        "v_mfma_f32_16x16x32_f16 a[32:35], v[16:19], v[112:115], a[32:35]\n"
        "v_mfma_f32_16x16x32_f16 a[64:67], v[20:23], v[112:115], a[64:67]\n"
        "v_mfma_f32_16x16x32_f16 a[96:99], v[24:27], v[112:115], a[96:99]\n"
        "s_waitcnt lgkmcnt(6)\n"
        "v_mfma_f32_16x16x32_f16 a[4:7], v[12:15], v[116:119], a[4:7]\n"
        "ds_read_b128 v[112:115], v71 offset:0\n"
        "v_mfma_f32_16x16x32_f16 a[36:39], v[16:19], v[116:119], a[36:39]\n"
        "v_mfma_f32_16x16x32_f16 a[68:71], v[20:23], v[116:119], a[68:71]\n"
        "v_mfma_f32_16x16x32_f16 a[100:103], v[24:27], v[116:119], a[100:103]\n"
        "s_waitcnt lgkmcnt(6)\n"
        "v_mfma_f32_16x16x32_f16 a[8:11], v[12:15], v[120:123], a[8:11]\n"
        "ds_read_b128 v[116:119], v71 offset:512\n"
        "v_mfma_f32_16x16x32_f16 a[40:43], v[16:19], v[120:123], a[40:43]\n"
        "v_mfma_f32_16x16x32_f16 a[72:75], v[20:23], v[120:123], a[72:75]\n"
        "v_mfma_f32_16x16x32_f16 a[104:107], v[24:27], v[120:123], a[104:107]\n"
        "s_waitcnt lgkmcnt(6)\n"
        "v_mfma_f32_16x16x32_f16 a[12:15], v[12:15], v[124:127], a[12:15]\n"
        "ds_read_b128 v[120:123], v71 offset:1024\n"
        "v_mfma_f32_16x16x32_f16 a[44:47], v[16:19], v[124:127], a[44:47]\n"
        "v_mfma_f32_16x16x32_f16 a[76:79], v[20:23], v[124:127], a[76:79]\n"
        "v_mfma_f32_16x16x32_f16 a[108:111], v[24:27], v[124:127], a[108:111]\n"
        "s_waitcnt lgkmcnt(6)\n"
        "v_mfma_f32_16x16x32_f16 a[16:19], v[12:15], v[72:75], a[16:19]\n"
        "ds_read_b128 v[124:127], v71 offset:1536\n"
        "v_mfma_f32_16x16x32_f16 a[48:51], v[16:19], v[72:75], a[48:51]\n"
        "v_mfma_f32_16x16x32_f16 a[80:83], v[20:23], v[72:75], a[80:83]\n"
        "v_mfma_f32_16x16x32_f16 a[112:115], v[24:27], v[72:75], a[112:115]\n"
        "s_waitcnt lgkmcnt(6)\n"
        "v_mfma_f32_16x16x32_f16 a[20:23], v[12:15], v[76:79], a[20:23]\n"
        "ds_read_b128 v[72:75], v71 offset:256\n"
        "v_mfma_f32_16x16x32_f16 a[52:55], v[16:19], v[76:79], a[52:55]\n"
        "v_mfma_f32_16x16x32_f16 a[84:87], v[20:23], v[76:79], a[84:87]\n"
        "v_mfma_f32_16x16x32_f16 a[116:119], v[24:27], v[76:79], a[116:119]\n"
        "s_waitcnt lgkmcnt(6)\n"
        "v_mfma_f32_16x16x32_f16 a[24:27], v[12:15], v[104:107], a[24:27]\n"
        "ds_read_b128 v[76:79], v71 offset:768\n"
        "v_mfma_f32_16x16x32_f16 a[56:59], v[16:19], v[104:107], a[56:59]\n"
        "v_mfma_f32_16x16x32_f16 a[88:91], v[20:23], v[104:107], a[88:91]\n"
        "v_mfma_f32_16x16x32_f16 a[120:123], v[24:27], v[104:107], a[120:123]\n"
        "s_waitcnt lgkmcnt(6)\n"
        "v_mfma_f32_16x16x32_f16 a[28:31], v[12:15], v[108:111], a[28:31]\n"
        "ds_read_b128 v[104:107], v71 offset:1280\n"
        "v_mfma_f32_16x16x32_f16 a[60:63], v[16:19], v[108:111], a[60:63]\n"
        "v_mfma_f32_16x16x32_f16 a[92:95], v[20:23], v[108:111], a[92:95]\n"
        "v_mfma_f32_16x16x32_f16 a[124:127], v[24:27], v[108:111], a[124:127]\n"
        "v_mov_b32 v65, %[mflag]\n"
        "v_mov_b32 v66, 0x20000\n"
        "buffer_load_dword v65, v65, %[rsA], 0 offen sc0 sc1\n"
        "buffer_load_dword v66, v66, %[rsA], 0 offen sc1\n"
        "s_waitcnt vmcnt(61) lgkmcnt(6)\n"
        "v_mfma_f32_16x16x32_f16 a[0:3], v[28:31], v[112:115], a[0:3]\n"
        "ds_read_b128 v[108:111], v71 offset:1792\n"
        "v_mfma_f32_16x16x32_f16 a[32:35], v[32:35], v[112:115], a[32:35]\n"
        "v_mfma_f32_16x16x32_f16 a[64:67], v[36:39], v[112:115], a[64:67]\n"
        "v_mfma_f32_16x16x32_f16 a[96:99], v[40:43], v[112:115], a[96:99]\n"
        "s_waitcnt lgkmcnt(6)\n"
        "v_mfma_f32_16x16x32_f16 a[4:7], v[28:31], v[116:119], a[4:7]\n"
        "ds_read_b128 v[112:115], v71 offset:8192\n"
        "v_mfma_f32_16x16x32_f16 a[36:39], v[32:35], v[116:119], a[36:39]\n"
        "v_mfma_f32_16x16x32_f16 a[68:71], v[36:39], v[116:119], a[68:71]\n"
        "v_mfma_f32_16x16x32_f16 a[100:103], v[40:43], v[116:119], a[100:103]\n"
        "s_waitcnt lgkmcnt(6)\n"
        "v_mfma_f32_16x16x32_f16 a[8:11], v[28:31], v[120:123], a[8:11]\n"
        "ds_read_b128 v[116:119], v71 offset:8704\n"
        "v_mfma_f32_16x16x32_f16 a[40:43], v[32:35], v[120:123], a[40:43]\n"
        "v_mfma_f32_16x16x32_f16 a[72:75], v[36:39], v[120:123], a[72:75]\n"
        "v_mfma_f32_16x16x32_f16 a[104:107], v[40:43], v[120:123], a[104:107]\n"
        "s_waitcnt lgkmcnt(6)\n"
        "v_mfma_f32_16x16x32_f16 a[12:15], v[28:31], v[124:127], a[12:15]\n"
        "ds_read_b128 v[120:123], v71 offset:9216\n"
        "v_mfma_f32_16x16x32_f16 a[44:47], v[32:35], v[124:127], a[44:47]\n"
        "v_mfma_f32_16x16x32_f16 a[76:79], v[36:39], v[124:127], a[76:79]\n"
        "v_mfma_f32_16x16x32_f16 a[108:111], v[40:43], v[124:127], a[108:111]\n"
        "s_waitcnt lgkmcnt(6)\n"
        "v_mfma_f32_16x16x32_f16 a[16:19], v[28:31], v[72:75], a[16:19]\n"
        "ds_read_b128 v[124:127], v71 offset:9728\n"
        "v_mfma_f32_16x16x32_f16 a[48:51], v[32:35], v[72:75], a[48:51]\n"
        "v_mfma_f32_16x16x32_f16 a[80:83], v[36:39], v[72:75], a[80:83]\n"
        "v_mfma_f32_16x16x32_f16 a[112:115], v[40:43], v[72:75], a[112:115]\n"
        "s_waitcnt lgkmcnt(6)\n"
        "v_mfma_f32_16x16x32_f16 a[20:23], v[28:31], v[76:79], a[20:23]\n"
        "ds_read_b128 v[72:75], v71 offset:8448\n"
        "v_mfma_f32_16x16x32_f16 a[52:55], v[32:35], v[76:79], a[52:55]\n"
        "v_mfma_f32_16x16x32_f16 a[84:87], v[36:39], v[76:79], a[84:87]\n"
        "v_mfma_f32_16x16x32_f16 a[116:119], v[40:43], v[76:79], a[116:119]\n"
        "s_waitcnt lgkmcnt(6)\n"
        "v_mfma_f32_16x16x32_f16 a[24:27], v[28:31], v[104:107], a[24:27]\n"
        "ds_read_b128 v[76:79], v71 offset:8960\n"
        "v_mfma_f32_16x16x32_f16 a[56:59], v[32:35], v[104:107], a[56:59]\n"
        "v_mfma_f32_16x16x32_f16 a[88:91], v[36:39], v[104:107], a[88:91]\n"
        "v_mfma_f32_16x16x32_f16 a[120:123], v[40:43], v[104:107], a[120:123]\n"
        "s_waitcnt lgkmcnt(6)\n"
        "v_mfma_f32_16x16x32_f16 a[28:31], v[28:31], v[108:111], a[28:31]\n"
        "ds_read_b128 v[104:107], v71 offset:9472\n"
        "v_mfma_f32_16x16x32_f16 a[60:63], v[32:35], v[108:111], a[60:63]\n"
        "v_mfma_f32_16x16x32_f16 a[92:95], v[36:39], v[108:111], a[92:95]\n"
        "v_mfma_f32_16x16x32_f16 a[124:127], v[40:43], v[108:111], a[124:127]\n"
        "s_waitcnt vmcnt(53) lgkmcnt(6)\n"
        "v_mfma_f32_16x16x32_f16 a[0:3], v[44:47], v[112:115], a[0:3]\n"
        "ds_read_b128 v[108:111], v71 offset:9984\n"
        "v_mfma_f32_16x16x32_f16 a[32:35], v[48:51], v[112:115], a[32:35]\n"
        "v_mfma_f32_16x16x32_f16 a[64:67], v[52:55], v[112:115], a[64:67]\n"
        "v_mfma_f32_16x16x32_f16 a[96:99], v[56:59], v[112:115], a[96:99]\n"
        "s_waitcnt lgkmcnt(6)\n"
        "v_mfma_f32_16x16x32_f16 a[4:7], v[44:47], v[116:119], a[4:7]\n"
        "ds_read_b128 v[112:115], v71 offset:16384\n"
        "v_mfma_f32_16x16x32_f16 a[36:39], v[48:51], v[116:119], a[36:39]\n"
        "v_mfma_f32_16x16x32_f16 a[68:71], v[52:55], v[116:119], a[68:71]\n"
        "v_mfma_f32_16x16x32_f16 a[100:103], v[56:59], v[116:119], a[100:103]\n"
        "s_waitcnt lgkmcnt(6)\n"
        "v_mfma_f32_16x16x32_f16 a[8:11], v[44:47], v[120:123], a[8:11]\n"
        "ds_read_b128 v[116:119], v71 offset:16896\n"
        "v_mfma_f32_16x16x32_f16 a[40:43], v[48:51], v[120:123], a[40:43]\n"
        "v_mfma_f32_16x16x32_f16 a[72:75], v[52:55], v[120:123], a[72:75]\n"
        "v_mfma_f32_16x16x32_f16 a[104:107], v[56:59], v[120:123], a[104:107]\n"
        "s_waitcnt lgkmcnt(6)\n"
        "v_mfma_f32_16x16x32_f16 a[12:15], v[44:47], v[124:127], a[12:15]\n"
        "ds_read_b128 v[120:123], v71 offset:17408\n"
        "v_mfma_f32_16x16x32_f16 a[44:47], v[48:51], v[124:127], a[44:47]\n"
        "v_mfma_f32_16x16x32_f16 a[76:79], v[52:55], v[124:127], a[76:79]\n"
        "v_mfma_f32_16x16x32_f16 a[108:111], v[56:59], v[124:127], a[108:111]\n"
        "s_waitcnt lgkmcnt(6)\n"
        "v_mfma_f32_16x16x32_f16 a[16:19], v[44:47], v[72:75], a[16:19]\n"
        "ds_read_b128 v[124:127], v71 offset:17920\n"
        "v_mfma_f32_16x16x32_f16 a[48:51], v[48:51], v[72:75], a[48:51]\n"
        "v_mfma_f32_16x16x32_f16 a[80:83], v[52:55], v[72:75], a[80:83]\n"
        "v_mfma_f32_16x16x32_f16 a[112:115], v[56:59], v[72:75], a[112:115]\n"
        "s_waitcnt lgkmcnt(6)\n"
        "v_mfma_f32_16x16x32_f16 a[20:23], v[44:47], v[76:79], a[20:23]\n"
        "ds_read_b128 v[72:75], v71 offset:16640\n"
        "v_mfma_f32_16x16x32_f16 a[52:55], v[48:51], v[76:79], a[52:55]\n"
        "v_mfma_f32_16x16x32_f16 a[84:87], v[52:55], v[76:79], a[84:87]\n"
        "v_mfma_f32_16x16x32_f16 a[116:119], v[56:59], v[76:79], a[116:119]\n"
        "s_waitcnt lgkmcnt(6)\n"
        "v_mfma_f32_16x16x32_f16 a[24:27], v[44:47], v[104:107], a[24:27]\n"
        "ds_read_b128 v[76:79], v71 offset:17152\n"
        "v_mfma_f32_16x16x32_f16 a[56:59], v[48:51], v[104:107], a[56:59]\n"
        "v_mfma_f32_16x16x32_f16 a[88:91], v[52:55], v[104:107], a[88:91]\n"
        "v_mfma_f32_16x16x32_f16 a[120:123], v[56:59], v[104:107], a[120:123]\n"
        "s_waitcnt lgkmcnt(6)\n"
        "v_mfma_f32_16x16x32_f16 a[28:31], v[44:47], v[108:111], a[28:31]\n"
        "ds_read_b128 v[104:107], v71 offset:17664\n"
        "v_mfma_f32_16x16x32_f16 a[60:63], v[48:51], v[108:111], a[60:63]\n"
        "v_mfma_f32_16x16x32_f16 a[92:95], v[52:55], v[108:111], a[92:95]\n"
        "v_mfma_f32_16x16x32_f16 a[124:127], v[56:59], v[108:111], a[124:127]\n"
        "s_waitcnt vmcnt(0)\n"
        "v_readfirstlane_b32 s69, v65\n"
        "v_readfirstlane_b32 s45, v66\n"
        "s_nop 1\n"
        "s_cmp_eq_u32 s45, s70\n"
        "s_cselect_b32 s53, 1, s53\n"
        "s_max_u32 s69, s69, s66\n"
        "s_sub_u32 s69, s69, s66\n"
        "s_min_u32 s69, s69, 4\n"
        "s_cmp_eq_u32 s67, 2\n"
        "s_cselect_b32 s67, s69, s67\n"
        "s_waitcnt vmcnt(45) lgkmcnt(6)\n"
        "v_mfma_f32_16x16x32_f16 a[0:3], v[80:83], v[112:115], a[0:3]\n"
        "ds_read_b128 v[108:111], v71 offset:18176\n"
        "v_mfma_f32_16x16x32_f16 a[32:35], v[84:87], v[112:115], a[32:35]\n"
        "s_add_u32 s52, %[xrecv], 16384\n"
        "s_cmp_gt_u32 s67, 2\n"
        "s_cselect_b32 s52, s52, 0x6000000\n"
        "buffer_load_dwordx4 a[192:195], v61, %[rsA], s52 offen offset:0 sc0 sc1\n"
        "v_mfma_f32_16x16x32_f16 a[64:67], v[88:91], v[112:115], a[64:67]\n"
        "v_mfma_f32_16x16x32_f16 a[96:99], v[92:95], v[112:115], a[96:99]\n"
        "s_waitcnt lgkmcnt(6)\n"
        "v_mfma_f32_16x16x32_f16 a[4:7], v[80:83], v[116:119], a[4:7]\n"
        "ds_read_b128 v[112:115], v71 offset:24576\n"
        "v_mfma_f32_16x16x32_f16 a[36:39], v[84:87], v[116:119], a[36:39]\n"
        "v_mfma_f32_16x16x32_f16 a[68:71], v[88:91], v[116:119], a[68:71]\n"
        "v_mfma_f32_16x16x32_f16 a[100:103], v[92:95], v[116:119], a[100:103]\n"
        "s_waitcnt lgkmcnt(6)\n"
        "v_mfma_f32_16x16x32_f16 a[8:11], v[80:83], v[120:123], a[8:11]\n"
        "ds_read_b128 v[116:119], v71 offset:25088\n"
        "v_mfma_f32_16x16x32_f16 a[40:43], v[84:87], v[120:123], a[40:43]\n"
        "buffer_load_dwordx4 a[196:199], v61, %[rsA], s52 offen offset:1024 sc0 sc1\n"
        "v_mfma_f32_16x16x32_f16 a[72:75], v[88:91], v[120:123], a[72:75]\n"
        "v_mfma_f32_16x16x32_f16 a[104:107], v[92:95], v[120:123], a[104:107]\n"
        "s_waitcnt lgkmcnt(6)\n"
        "v_mfma_f32_16x16x32_f16 a[12:15], v[80:83], v[124:127], a[12:15]\n"
        "ds_read_b128 v[120:123], v71 offset:25600\n"
        "v_mfma_f32_16x16x32_f16 a[44:47], v[84:87], v[124:127], a[44:47]\n"
        "v_mfma_f32_16x16x32_f16 a[76:79], v[88:91], v[124:127], a[76:79]\n"
        "v_mfma_f32_16x16x32_f16 a[108:111], v[92:95], v[124:127], a[108:111]\n"
        "s_waitcnt lgkmcnt(6)\n"
        "v_mfma_f32_16x16x32_f16 a[16:19], v[80:83], v[72:75], a[16:19]\n"
        "ds_read_b128 v[124:127], v71 offset:26112\n"
        "v_mfma_f32_16x16x32_f16 a[48:51], v[84:87], v[72:75], a[48:51]\n"
        "buffer_load_dwordx4 a[200:203], v61, %[rsA], s52 offen offset:2048 sc0 sc1\n"
        "v_mfma_f32_16x16x32_f16 a[80:83], v[88:91], v[72:75], a[80:83]\n"
        "v_mfma_f32_16x16x32_f16 a[112:115], v[92:95], v[72:75], a[112:115]\n"
        "s_waitcnt lgkmcnt(6)\n"
        "v_mfma_f32_16x16x32_f16 a[20:23], v[80:83], v[76:79], a[20:23]\n"
        "ds_read_b128 v[72:75], v71 offset:24832\n"
        "v_mfma_f32_16x16x32_f16 a[52:55], v[84:87], v[76:79], a[52:55]\n"
        "v_mfma_f32_16x16x32_f16 a[84:87], v[88:91], v[76:79], a[84:87]\n"
        "v_mfma_f32_16x16x32_f16 a[116:119], v[92:95], v[76:79], a[116:119]\n"
        "s_waitcnt lgkmcnt(6)\n"
        "v_mfma_f32_16x16x32_f16 a[24:27], v[80:83], v[104:107], a[24:27]\n"
        "ds_read_b128 v[76:79], v71 offset:25344\n"
        "v_mfma_f32_16x16x32_f16 a[56:59], v[84:87], v[104:107], a[56:59]\n"
        "buffer_load_dwordx4 a[204:207], v61, %[rsA], s52 offen offset:3072 sc0 sc1\n"
        "v_mfma_f32_16x16x32_f16 a[88:91], v[88:91], v[104:107], a[88:91]\n"
        "v_mfma_f32_16x16x32_f16 a[120:123], v[92:95], v[104:107], a[120:123]\n"
        "s_waitcnt lgkmcnt(6)\n"
        "v_mfma_f32_16x16x32_f16 a[28:31], v[80:83], v[108:111], a[28:31]\n"
        "ds_read_b128 v[104:107], v71 offset:25856\n"
        "v_mfma_f32_16x16x32_f16 a[60:63], v[84:87], v[108:111], a[60:63]\n"
        "v_mfma_f32_16x16x32_f16 a[92:95], v[88:91], v[108:111], a[92:95]\n"
        "v_mfma_f32_16x16x32_f16 a[124:127], v[92:95], v[108:111], a[124:127]\n"
        "s_waitcnt vmcnt(40) lgkmcnt(6)\n"
        "v_mfma_f32_16x16x32_f16 a[0:3], v[136:139], v[112:115], a[0:3]\n"
        "ds_read_b128 v[108:111], v71 offset:26368\n"
        "v_mfma_f32_16x16x32_f16 a[32:35], v[140:143], v[112:115], a[32:35]\n"
        "s_add_u32 s52, %[xrecv], 20480\n"
        "s_cmp_gt_u32 s67, 2\n"
        "s_cselect_b32 s52, s52, 0x6000000\n"
        "buffer_load_dwordx4 a[208:211], v61, %[rsA], s52 offen offset:0 sc0 sc1\n"
        "v_mfma_f32_16x16x32_f16 a[64:67], v[152:155], v[112:115], a[64:67]\n"
        "v_mfma_f32_16x16x32_f16 a[96:99], v[156:159], v[112:115], a[96:99]\n"
        "s_waitcnt lgkmcnt(6)\n"
        "v_mfma_f32_16x16x32_f16 a[4:7], v[136:139], v[116:119], a[4:7]\n"
        "ds_read_b128 v[112:115], %[bbase] offset:0\n"
        "v_mfma_f32_16x16x32_f16 a[36:39], v[140:143], v[116:119], a[36:39]\n"
        "v_mfma_f32_16x16x32_f16 a[68:71], v[152:155], v[116:119], a[68:71]\n"
        "v_mfma_f32_16x16x32_f16 a[100:103], v[156:159], v[116:119], a[100:103]\n"
        "s_waitcnt lgkmcnt(6)\n"
        "v_mfma_f32_16x16x32_f16 a[8:11], v[136:139], v[120:123], a[8:11]\n"
        "ds_read_b128 v[116:119], %[bbase] offset:512\n"
        "v_mfma_f32_16x16x32_f16 a[40:43], v[140:143], v[120:123], a[40:43]\n"
        "buffer_load_dwordx4 a[212:215], v61, %[rsA], s52 offen offset:1024 sc0 sc1\n"
        "v_mfma_f32_16x16x32_f16 a[72:75], v[152:155], v[120:123], a[72:75]\n"
        "v_mfma_f32_16x16x32_f16 a[104:107], v[156:159], v[120:123], a[104:107]\n"
        "s_waitcnt lgkmcnt(6)\n"
        "v_mfma_f32_16x16x32_f16 a[12:15], v[136:139], v[124:127], a[12:15]\n"
        "ds_read_b128 v[120:123], %[bbase] offset:1024\n"
        "v_mfma_f32_16x16x32_f16 a[44:47], v[140:143], v[124:127], a[44:47]\n"
        "v_mfma_f32_16x16x32_f16 a[76:79], v[152:155], v[124:127], a[76:79]\n"
        "v_mfma_f32_16x16x32_f16 a[108:111], v[156:159], v[124:127], a[108:111]\n"
        "s_waitcnt lgkmcnt(6)\n"
        "v_mfma_f32_16x16x32_f16 a[16:19], v[136:139], v[72:75], a[16:19]\n"
        "ds_read_b128 v[124:127], %[bbase] offset:1536\n"
        "v_mfma_f32_16x16x32_f16 a[48:51], v[140:143], v[72:75], a[48:51]\n"
        "buffer_load_dwordx4 a[216:219], v61, %[rsA], s52 offen offset:2048 sc0 sc1\n"
        "v_mfma_f32_16x16x32_f16 a[80:83], v[152:155], v[72:75], a[80:83]\n"
        "v_mfma_f32_16x16x32_f16 a[112:115], v[156:159], v[72:75], a[112:115]\n"
        "s_waitcnt lgkmcnt(6)\n"
        "v_mfma_f32_16x16x32_f16 a[20:23], v[136:139], v[76:79], a[20:23]\n"
        "ds_read_b128 v[72:75], %[bbase] offset:256\n"
        "v_mfma_f32_16x16x32_f16 a[52:55], v[140:143], v[76:79], a[52:55]\n"
        "v_mfma_f32_16x16x32_f16 a[84:87], v[152:155], v[76:79], a[84:87]\n"
        "v_mfma_f32_16x16x32_f16 a[116:119], v[156:159], v[76:79], a[116:119]\n"
        "s_waitcnt lgkmcnt(6)\n"
        "v_mfma_f32_16x16x32_f16 a[24:27], v[136:139], v[104:107], a[24:27]\n"
        "ds_read_b128 v[76:79], %[bbase] offset:768\n"
        "v_mfma_f32_16x16x32_f16 a[56:59], v[140:143], v[104:107], a[56:59]\n"
        "buffer_load_dwordx4 a[220:223], v61, %[rsA], s52 offen offset:3072 sc0 sc1\n"
        "v_mfma_f32_16x16x32_f16 a[88:91], v[152:155], v[104:107], a[88:91]\n"
        "v_mfma_f32_16x16x32_f16 a[120:123], v[156:159], v[104:107], a[120:123]\n"
        "s_waitcnt lgkmcnt(6)\n"
        "v_mfma_f32_16x16x32_f16 a[28:31], v[136:139], v[108:111], a[28:31]\n"
        "ds_read_b128 v[104:107], %[bbase] offset:1280\n"
        "v_mfma_f32_16x16x32_f16 a[60:63], v[140:143], v[108:111], a[60:63]\n"
        "v_mfma_f32_16x16x32_f16 a[92:95], v[152:155], v[108:111], a[92:95]\n"
        "v_mfma_f32_16x16x32_f16 a[124:127], v[156:159], v[108:111], a[124:127]\n"
        "s_waitcnt vmcnt(39) lgkmcnt(6)\n"
        "v_mfma_f32_16x16x32_f16 a[0:3], v[168:171], v[112:115], a[0:3]\n"
        "ds_read_b128 v[108:111], %[bbase] offset:1792\n"
        "v_mfma_f32_16x16x32_f16 a[32:35], v[172:175], v[112:115], a[32:35]\n"
        "s_add_u32 s52, %[xrecv], 24576\n"
        "s_cmp_gt_u32 s67, 3\n"
        "s_cselect_b32 s52, s52, 0x6000000\n"
        "buffer_load_dwordx4 a[224:227], v61, %[rsA], s52 offen offset:0 sc0 sc1\n"
        "v_mfma_f32_16x16x32_f16 a[64:67], v[184:187], v[112:115], a[64:67]\n"
        "v_mfma_f32_16x16x32_f16 a[96:99], v[188:191], v[112:115], a[96:99]\n"
        "s_waitcnt lgkmcnt(6)\n"
        "v_mfma_f32_16x16x32_f16 a[4:7], v[168:171], v[116:119], a[4:7]\n"
        "ds_read_b128 v[112:115], %[bbase] offset:8192\n"
        "v_mfma_f32_16x16x32_f16 a[36:39], v[172:175], v[116:119], a[36:39]\n"
        "v_mfma_f32_16x16x32_f16 a[68:71], v[184:187], v[116:119], a[68:71]\n"
        "v_mfma_f32_16x16x32_f16 a[100:103], v[188:191], v[116:119], a[100:103]\n"
        "s_waitcnt lgkmcnt(6)\n"
        "v_mfma_f32_16x16x32_f16 a[8:11], v[168:171], v[120:123], a[8:11]\n"
        "ds_read_b128 v[116:119], %[bbase] offset:8704\n"
        "v_mfma_f32_16x16x32_f16 a[40:43], v[172:175], v[120:123], a[40:43]\n"
        "buffer_load_dwordx4 a[228:231], v61, %[rsA], s52 offen offset:1024 sc0 sc1\n"
        "v_mfma_f32_16x16x32_f16 a[72:75], v[184:187], v[120:123], a[72:75]\n"
        "v_mfma_f32_16x16x32_f16 a[104:107], v[188:191], v[120:123], a[104:107]\n"
        "s_waitcnt lgkmcnt(6)\n"
        "v_mfma_f32_16x16x32_f16 a[12:15], v[168:171], v[124:127], a[12:15]\n"
        "ds_read_b128 v[120:123], %[bbase] offset:9216\n"
        "v_mfma_f32_16x16x32_f16 a[44:47], v[172:175], v[124:127], a[44:47]\n"
        "v_mfma_f32_16x16x32_f16 a[76:79], v[184:187], v[124:127], a[76:79]\n"
        "v_mfma_f32_16x16x32_f16 a[108:111], v[188:191], v[124:127], a[108:111]\n"
        "s_waitcnt lgkmcnt(6)\n"
        "v_mfma_f32_16x16x32_f16 a[16:19], v[168:171], v[72:75], a[16:19]\n"
        "ds_read_b128 v[124:127], %[bbase] offset:9728\n"
        "v_mfma_f32_16x16x32_f16 a[48:51], v[172:175], v[72:75], a[48:51]\n"
        "buffer_load_dwordx4 a[232:235], v61, %[rsA], s52 offen offset:2048 sc0 sc1\n"
        "v_mfma_f32_16x16x32_f16 a[80:83], v[184:187], v[72:75], a[80:83]\n"
        "v_mfma_f32_16x16x32_f16 a[112:115], v[188:191], v[72:75], a[112:115]\n"
        "s_waitcnt lgkmcnt(6)\n"
        "v_mfma_f32_16x16x32_f16 a[20:23], v[168:171], v[76:79], a[20:23]\n"
        "ds_read_b128 v[72:75], %[bbase] offset:8448\n"
        "v_mfma_f32_16x16x32_f16 a[52:55], v[172:175], v[76:79], a[52:55]\n"
        "v_mfma_f32_16x16x32_f16 a[84:87], v[184:187], v[76:79], a[84:87]\n"
        "v_mfma_f32_16x16x32_f16 a[116:119], v[188:191], v[76:79], a[116:119]\n"
        "s_waitcnt lgkmcnt(6)\n"
        "v_mfma_f32_16x16x32_f16 a[24:27], v[168:171], v[104:107], a[24:27]\n"
        "ds_read_b128 v[76:79], %[bbase] offset:8960\n"
        "v_mfma_f32_16x16x32_f16 a[56:59], v[172:175], v[104:107], a[56:59]\n"
        "buffer_load_dwordx4 a[236:239], v61, %[rsA], s52 offen offset:3072 sc0 sc1\n"
        "v_mfma_f32_16x16x32_f16 a[88:91], v[184:187], v[104:107], a[88:91]\n"
        "v_mfma_f32_16x16x32_f16 a[120:123], v[188:191], v[104:107], a[120:123]\n"
        "s_waitcnt lgkmcnt(6)\n"
        "v_mfma_f32_16x16x32_f16 a[28:31], v[168:171], v[108:111], a[28:31]\n"
        "ds_read_b128 v[104:107], %[bbase] offset:9472\n"
        "v_mfma_f32_16x16x32_f16 a[60:63], v[172:175], v[108:111], a[60:63]\n"
        "v_mfma_f32_16x16x32_f16 a[92:95], v[184:187], v[108:111], a[92:95]\n"
        "v_mfma_f32_16x16x32_f16 a[124:127], v[188:191], v[108:111], a[124:127]\n"
        "s_waitcnt vmcnt(37) lgkmcnt(6)\n"
        "v_mfma_f32_16x16x32_f16 a[0:3], v[96:99], v[112:115], a[0:3]\n"
        "ds_read_b128 v[108:111], %[bbase] offset:9984\n"
        "v_mfma_f32_16x16x32_f16 a[32:35], v[100:103], v[112:115], a[32:35]\n"
        "s_add_u32 s52, %[xrecv], 28672\n"
        "s_cmp_gt_u32 s67, 3\n"
        "s_cselect_b32 s52, s52, 0x6000000\n"
        "buffer_load_dwordx4 a[240:243], v61, %[rsA], s52 offen offset:0 sc0 sc1\n"
        "v_mfma_f32_16x16x32_f16 a[64:67], v[128:131], v[112:115], a[64:67]\n"
        "v_mfma_f32_16x16x32_f16 a[96:99], v[132:135], v[112:115], a[96:99]\n"
        "s_waitcnt lgkmcnt(6)\n"
        "v_mfma_f32_16x16x32_f16 a[4:7], v[96:99], v[116:119], a[4:7]\n"
        "ds_read_b128 v[112:115], %[bbase] offset:16384\n"
        "v_mfma_f32_16x16x32_f16 a[36:39], v[100:103], v[116:119], a[36:39]\n"
        "v_mfma_f32_16x16x32_f16 a[68:71], v[128:131], v[116:119], a[68:71]\n"
        "v_mfma_f32_16x16x32_f16 a[100:103], v[132:135], v[116:119], a[100:103]\n"
        "s_waitcnt lgkmcnt(6)\n"
        "v_mfma_f32_16x16x32_f16 a[8:11], v[96:99], v[120:123], a[8:11]\n"
        "ds_read_b128 v[116:119], %[bbase] offset:16896\n"
        "v_mfma_f32_16x16x32_f16 a[40:43], v[100:103], v[120:123], a[40:43]\n"
        "buffer_load_dwordx4 a[244:247], v61, %[rsA], s52 offen offset:1024 sc0 sc1\n"
        "v_mfma_f32_16x16x32_f16 a[72:75], v[128:131], v[120:123], a[72:75]\n"
        "v_mfma_f32_16x16x32_f16 a[104:107], v[132:135], v[120:123], a[104:107]\n"
        "s_waitcnt lgkmcnt(6)\n"
        "v_mfma_f32_16x16x32_f16 a[12:15], v[96:99], v[124:127], a[12:15]\n"
        "ds_read_b128 v[120:123], %[bbase] offset:17408\n"
        "v_mfma_f32_16x16x32_f16 a[44:47], v[100:103], v[124:127], a[44:47]\n"
        "v_mfma_f32_16x16x32_f16 a[76:79], v[128:131], v[124:127], a[76:79]\n"
        "v_mfma_f32_16x16x32_f16 a[108:111], v[132:135], v[124:127], a[108:111]\n"
        "s_waitcnt lgkmcnt(6)\n"
        "v_mfma_f32_16x16x32_f16 a[16:19], v[96:99], v[72:75], a[16:19]\n"
        "ds_read_b128 v[124:127], %[bbase] offset:17920\n"
        "v_mfma_f32_16x16x32_f16 a[48:51], v[100:103], v[72:75], a[48:51]\n"
        "buffer_load_dwordx4 a[248:251], v61, %[rsA], s52 offen offset:2048 sc0 sc1\n"
        "v_mfma_f32_16x16x32_f16 a[80:83], v[128:131], v[72:75], a[80:83]\n"
        "v_mfma_f32_16x16x32_f16 a[112:115], v[132:135], v[72:75], a[112:115]\n"
        "s_waitcnt lgkmcnt(6)\n"
        "v_mfma_f32_16x16x32_f16 a[20:23], v[96:99], v[76:79], a[20:23]\n"
        "ds_read_b128 v[72:75], %[bbase] offset:16640\n"
        "v_mfma_f32_16x16x32_f16 a[52:55], v[100:103], v[76:79], a[52:55]\n"
        "v_mfma_f32_16x16x32_f16 a[84:87], v[128:131], v[76:79], a[84:87]\n"
        "v_mfma_f32_16x16x32_f16 a[116:119], v[132:135], v[76:79], a[116:119]\n"
        "s_waitcnt lgkmcnt(6)\n"
        "v_mfma_f32_16x16x32_f16 a[24:27], v[96:99], v[104:107], a[24:27]\n"
        "ds_read_b128 v[76:79], %[bbase] offset:17152\n"
        "v_mfma_f32_16x16x32_f16 a[56:59], v[100:103], v[104:107], a[56:59]\n"
        "buffer_load_dwordx4 a[252:255], v61, %[rsA], s52 offen offset:3072 sc0 sc1\n"
        "v_mfma_f32_16x16x32_f16 a[88:91], v[128:131], v[104:107], a[88:91]\n"
        "v_mfma_f32_16x16x32_f16 a[120:123], v[132:135], v[104:107], a[120:123]\n"
        "s_waitcnt lgkmcnt(6)\n"
        "v_mfma_f32_16x16x32_f16 a[28:31], v[96:99], v[108:111], a[28:31]\n"
        "ds_read_b128 v[104:107], %[bbase] offset:17664\n"
        "v_mfma_f32_16x16x32_f16 a[60:63], v[100:103], v[108:111], a[60:63]\n"
        "v_mfma_f32_16x16x32_f16 a[92:95], v[128:131], v[108:111], a[92:95]\n"
        "v_mfma_f32_16x16x32_f16 a[124:127], v[132:135], v[108:111], a[124:127]\n"
        "s_waitcnt vmcnt(32) lgkmcnt(6)\n"
        "v_mfma_f32_16x16x32_f16 a[0:3], v[144:147], v[112:115], a[0:3]\n"
        "ds_read_b128 v[108:111], %[bbase] offset:18176\n"
        "v_mfma_f32_16x16x32_f16 a[32:35], v[148:151], v[112:115], a[32:35]\n"
        "v_mfma_f32_16x16x32_f16 a[64:67], v[160:163], v[112:115], a[64:67]\n"
        "v_mfma_f32_16x16x32_f16 a[96:99], v[164:167], v[112:115], a[96:99]\n"
        "s_waitcnt lgkmcnt(6)\n"
        "v_mfma_f32_16x16x32_f16 a[4:7], v[144:147], v[116:119], a[4:7]\n"
        "ds_read_b128 v[112:115], %[bbase] offset:24576\n"
        "v_mfma_f32_16x16x32_f16 a[36:39], v[148:151], v[116:119], a[36:39]\n"
        "v_mfma_f32_16x16x32_f16 a[68:71], v[160:163], v[116:119], a[68:71]\n"
        "v_mfma_f32_16x16x32_f16 a[100:103], v[164:167], v[116:119], a[100:103]\n"
        "s_waitcnt lgkmcnt(6)\n"
        "v_mfma_f32_16x16x32_f16 a[8:11], v[144:147], v[120:123], a[8:11]\n"
        "ds_read_b128 v[116:119], %[bbase] offset:25088\n"
        "v_mfma_f32_16x16x32_f16 a[40:43], v[148:151], v[120:123], a[40:43]\n"
        "v_mfma_f32_16x16x32_f16 a[72:75], v[160:163], v[120:123], a[72:75]\n"
        "v_mfma_f32_16x16x32_f16 a[104:107], v[164:167], v[120:123], a[104:107]\n"
        "s_waitcnt lgkmcnt(6)\n"
        "v_mfma_f32_16x16x32_f16 a[12:15], v[144:147], v[124:127], a[12:15]\n"
        "ds_read_b128 v[120:123], %[bbase] offset:25600\n"
        "v_mfma_f32_16x16x32_f16 a[44:47], v[148:151], v[124:127], a[44:47]\n"
        "v_mfma_f32_16x16x32_f16 a[76:79], v[160:163], v[124:127], a[76:79]\n"
        "v_mfma_f32_16x16x32_f16 a[108:111], v[164:167], v[124:127], a[108:111]\n"
        "s_waitcnt lgkmcnt(6)\n"
        "v_mfma_f32_16x16x32_f16 a[16:19], v[144:147], v[72:75], a[16:19]\n"
        "ds_read_b128 v[124:127], %[bbase] offset:26112\n"
        "v_mfma_f32_16x16x32_f16 a[48:51], v[148:151], v[72:75], a[48:51]\n"
        "v_mfma_f32_16x16x32_f16 a[80:83], v[160:163], v[72:75], a[80:83]\n"
        "v_mfma_f32_16x16x32_f16 a[112:115], v[164:167], v[72:75], a[112:115]\n"
        "s_waitcnt lgkmcnt(6)\n"
        "v_mfma_f32_16x16x32_f16 a[20:23], v[144:147], v[76:79], a[20:23]\n"
        "ds_read_b128 v[72:75], %[bbase] offset:24832\n"
        "v_mfma_f32_16x16x32_f16 a[52:55], v[148:151], v[76:79], a[52:55]\n"
        "v_mfma_f32_16x16x32_f16 a[84:87], v[160:163], v[76:79], a[84:87]\n"
        "v_mfma_f32_16x16x32_f16 a[116:119], v[164:167], v[76:79], a[116:119]\n"
        "s_waitcnt lgkmcnt(6)\n"
        "v_mfma_f32_16x16x32_f16 a[24:27], v[144:147], v[104:107], a[24:27]\n"
        "ds_read_b128 v[76:79], %[bbase] offset:25344\n"
        "v_mfma_f32_16x16x32_f16 a[56:59], v[148:151], v[104:107], a[56:59]\n"
        "v_mfma_f32_16x16x32_f16 a[88:91], v[160:163], v[104:107], a[88:91]\n"
        "v_mfma_f32_16x16x32_f16 a[120:123], v[164:167], v[104:107], a[120:123]\n"
        "s_waitcnt lgkmcnt(6)\n"
        "v_mfma_f32_16x16x32_f16 a[28:31], v[144:147], v[108:111], a[28:31]\n"
        "ds_read_b128 v[104:107], %[bbase] offset:25856\n"
        "v_mfma_f32_16x16x32_f16 a[60:63], v[148:151], v[108:111], a[60:63]\n"
        "v_mfma_f32_16x16x32_f16 a[92:95], v[160:163], v[108:111], a[92:95]\n"
        "v_mfma_f32_16x16x32_f16 a[124:127], v[164:167], v[108:111], a[124:127]\n"
        "s_waitcnt vmcnt(24) lgkmcnt(6)\n"
        "v_mfma_f32_16x16x32_f16 a[0:3], v[176:179], v[112:115], a[0:3]\n"
        "ds_read_b128 v[108:111], %[bbase] offset:26368\n"
        "v_mfma_f32_16x16x32_f16 a[32:35], v[180:183], v[112:115], a[32:35]\n"
        "v_mfma_f32_16x16x32_f16 a[64:67], v[192:195], v[112:115], a[64:67]\n"
        "v_mfma_f32_16x16x32_f16 a[96:99], v[196:199], v[112:115], a[96:99]\n"
        "s_waitcnt lgkmcnt(6)\n"
        "v_mfma_f32_16x16x32_f16 a[4:7], v[176:179], v[116:119], a[4:7]\n"
        "v_mfma_f32_16x16x32_f16 a[36:39], v[180:183], v[116:119], a[36:39]\n"
        "v_mfma_f32_16x16x32_f16 a[68:71], v[192:195], v[116:119], a[68:71]\n"
        "v_mfma_f32_16x16x32_f16 a[100:103], v[196:199], v[116:119], a[100:103]\n"
        "s_waitcnt lgkmcnt(5)\n"
        "v_mfma_f32_16x16x32_f16 a[8:11], v[176:179], v[120:123], a[8:11]\n"
        "v_mfma_f32_16x16x32_f16 a[40:43], v[180:183], v[120:123], a[40:43]\n"
        "v_mfma_f32_16x16x32_f16 a[72:75], v[192:195], v[120:123], a[72:75]\n"
        "v_mfma_f32_16x16x32_f16 a[104:107], v[196:199], v[120:123], a[104:107]\n"
        "s_waitcnt lgkmcnt(4)\n"
        "v_mfma_f32_16x16x32_f16 a[12:15], v[176:179], v[124:127], a[12:15]\n"
        "v_mfma_f32_16x16x32_f16 a[44:47], v[180:183], v[124:127], a[44:47]\n"
        "v_mfma_f32_16x16x32_f16 a[76:79], v[192:195], v[124:127], a[76:79]\n"
        "v_mfma_f32_16x16x32_f16 a[108:111], v[196:199], v[124:127], a[108:111]\n"
        "s_waitcnt lgkmcnt(3)\n"
        "v_mfma_f32_16x16x32_f16 a[16:19], v[176:179], v[72:75], a[16:19]\n"
        "v_mfma_f32_16x16x32_f16 a[48:51], v[180:183], v[72:75], a[48:51]\n"
        "v_mfma_f32_16x16x32_f16 a[80:83], v[192:195], v[72:75], a[80:83]\n"
        "v_mfma_f32_16x16x32_f16 a[112:115], v[196:199], v[72:75], a[112:115]\n"
        "s_waitcnt lgkmcnt(2)\n"
        "v_mfma_f32_16x16x32_f16 a[20:23], v[176:179], v[76:79], a[20:23]\n"
        "v_mfma_f32_16x16x32_f16 a[52:55], v[180:183], v[76:79], a[52:55]\n"
        "v_mfma_f32_16x16x32_f16 a[84:87], v[192:195], v[76:79], a[84:87]\n"
        "v_mfma_f32_16x16x32_f16 a[116:119], v[196:199], v[76:79], a[116:119]\n"
        "s_waitcnt lgkmcnt(1)\n"
        "v_mfma_f32_16x16x32_f16 a[24:27], v[176:179], v[104:107], a[24:27]\n"
        "v_mfma_f32_16x16x32_f16 a[56:59], v[180:183], v[104:107], a[56:59]\n"
        "v_mfma_f32_16x16x32_f16 a[88:91], v[192:195], v[104:107], a[88:91]\n"
        "v_mfma_f32_16x16x32_f16 a[120:123], v[196:199], v[104:107], a[120:123]\n"
        "s_waitcnt lgkmcnt(0)\n"
        "v_mfma_f32_16x16x32_f16 a[28:31], v[176:179], v[108:111], a[28:31]\n"
        "v_mfma_f32_16x16x32_f16 a[60:63], v[180:183], v[108:111], a[60:63]\n"
        "v_mfma_f32_16x16x32_f16 a[92:95], v[192:195], v[108:111], a[92:95]\n"
        "v_mfma_f32_16x16x32_f16 a[124:127], v[196:199], v[108:111], a[124:127]\n"
        "s_nop 7\n"
        "s_nop 7\n"
        "s_branch LEND_%=\n"
        "LOOPH_%=:\n"
        "s_cmp_lg_u32 s85, 0\n"
        "s_cbranch_scc1 LOOPG_%=\n"
        "s_sub_u32 s45, s40, 8\n"
        "s_cmp_lt_u32 s45, 40\n"
        "s_cbranch_scc1 LTPH0_%=\n"
        "LTRH0_%=:\n"
        "s_barrier\n"
        "s_waitcnt vmcnt(8) lgkmcnt(2)\n"
        "v_mfma_f32_16x16x32_f16 a[0:3], v[128:131], v[112:115], a[0:3]\n"
        "buffer_load_dwordx4 v[80:83], %[wv0], %[rsW], s42 offen nt\n"
        "v_readlane_b32 s77, %[tablo], s40\n"
        "v_mfma_f32_16x16x32_f16 a[32:35], v[132:135], v[112:115], a[32:35]\n"
        "ds_read_b128 v[124:127], v71 offset:1536\n"
        "v_readlane_b32 s78, %[tabhi], s40\n"
        "s_add_u32 s40, s40, 1\n"
        "v_mfma_f32_16x16x32_f16 a[64:67], v[136:139], v[112:115], a[64:67]\n"
        "buffer_load_dwordx4 v[84:87], v62, %[rsW], s42 offen nt\n"
        "v_mfma_f32_16x16x32_f16 a[96:99], v[140:143], v[112:115], a[96:99]\n"
        "buffer_load_dwordx4 v[88:91], v63, %[rsW], s42 offen nt\n"
        "v_mfma_f32_16x16x32_f16 a[128:131], v[144:147], v[112:115], a[128:131]\n"
        "buffer_load_dwordx4 v[92:95], v66, %[rsW], s42 offen nt\n"
        "v_mfma_f32_16x16x32_f16 a[160:163], v[148:151], v[112:115], a[160:163]\n"
        "buffer_load_dwordx4 v[96:99], %[wv0], %[rsW], s68 offen nt\n"
        "v_mfma_f32_16x16x32_f16 a[192:195], v[152:155], v[112:115], a[192:195]\n"
        "buffer_load_dwordx4 v[100:103], v62, %[rsW], s68 offen nt\n"
        "v_mfma_f32_16x16x32_f16 a[224:227], v[156:159], v[112:115], a[224:227]\n"
        "s_waitcnt lgkmcnt(2)\n"
        "v_mfma_f32_16x16x32_f16 a[4:7], v[128:131], v[116:119], a[4:7]\n"
        "buffer_load_dwordx4 v[104:107], v63, %[rsW], s68 offen nt\n"
        "v_mfma_f32_16x16x32_f16 a[36:39], v[132:135], v[116:119], a[36:39]\n"
        "ds_read_b128 v[112:115], v71 offset:256\n"
        "v_mfma_f32_16x16x32_f16 a[68:71], v[136:139], v[116:119], a[68:71]\n"
        "buffer_load_dwordx4 v[108:111], v66, %[rsW], s68 offen nt\n"
        "s_mov_b64 exec, s[88:89]\n"
        "buffer_load_dword v61, %[cnt], %[rsA], s60 offen sc0 sc1\n"
        "s_mov_b64 exec, -1\n"
        "buffer_load_dword v65, v64, %[rsA], s58 offen sc0 sc1\n"
        "v_mfma_f32_16x16x32_f16 a[100:103], v[140:143], v[116:119], a[100:103]\n"
        "v_readlane_b32 s42, %[tabw], s66\n"
        "s_cmp_lt_u32 s66, 64\n"
        "s_cselect_b32 s42, s42, 0x80000000\n"
        "s_add_u32 s66, s66, 1\n"
        "v_mfma_f32_16x16x32_f16 a[132:135], v[144:147], v[116:119], a[132:135]\n"
        "v_readlane_b32 s68, %[tabw], s66\n"
        "s_cmp_lt_u32 s66, 64\n"
        "s_cselect_b32 s68, s68, 0x80000000\n"
        "s_add_u32 s66, s66, 1\n"
        "v_mfma_f32_16x16x32_f16 a[164:167], v[148:151], v[116:119], a[164:167]\n"
        "s_cmp_eq_u32 s40, s61\n"
        "s_cbranch_scc1 LSENDH0_%=\n"
        "LSENTH0_%=:\n"
        "v_mfma_f32_16x16x32_f16 a[196:199], v[152:155], v[116:119], a[196:199]\n"
        "v_mfma_f32_16x16x32_f16 a[228:231], v[156:159], v[116:119], a[228:231]\n"
        "s_waitcnt lgkmcnt(2)\n"
        "v_mfma_f32_16x16x32_f16 a[8:11], v[128:131], v[120:123], a[8:11]\n"
        "v_add_u32 v69, s67, %[wdst]\n"
        "s_add_u32 s67, s67, 0x2000\n"
        "v_mfma_f32_16x16x32_f16 a[40:43], v[132:135], v[120:123], a[40:43]\n"
        "ds_read_b128 v[116:119], v71 offset:768\n"
        "v_mfma_f32_16x16x32_f16 a[72:75], v[136:139], v[120:123], a[72:75]\n"
        "s_waitcnt vmcnt(22)\n"
        "s_cmp_lt_u32 s67, 163840\n"
        "s_cselect_b32 s67, s67, 0\n"
        "v_mfma_f32_16x16x32_f16 a[104:107], v[140:143], v[120:123], a[104:107]\n"
        "v_cvt_pk_f16_f32 v72, v12, v16\n"
        "v_cvt_pk_f16_f32 v73, v20, v24\n"
        "v_mfma_f32_16x16x32_f16 a[136:139], v[144:147], v[120:123], a[136:139]\n"
        "ds_write_b64 v69, v[72:73] offset:0\n"
        "v_mfma_f32_16x16x32_f16 a[168:171], v[148:151], v[120:123], a[168:171]\n"
        "v_cvt_pk_f16_f32 v74, v13, v17\n"
        "v_cvt_pk_f16_f32 v75, v21, v25\n"
        "v_mfma_f32_16x16x32_f16 a[200:203], v[152:155], v[120:123], a[200:203]\n"
        "ds_write_b64 v69, v[74:75] offset:512\n"
        "v_mfma_f32_16x16x32_f16 a[232:235], v[156:159], v[120:123], a[232:235]\n"
        "s_waitcnt lgkmcnt(4)\n"
        "v_mfma_f32_16x16x32_f16 a[12:15], v[128:131], v[124:127], a[12:15]\n"
        "v_cvt_pk_f16_f32 v76, v14, v18\n"
        "v_cvt_pk_f16_f32 v77, v22, v26\n"
        "v_mfma_f32_16x16x32_f16 a[44:47], v[132:135], v[124:127], a[44:47]\n"
        "ds_read_b128 v[120:123], v71 offset:1280\n"
        "v_mfma_f32_16x16x32_f16 a[76:79], v[136:139], v[124:127], a[76:79]\n"
        "ds_write_b64 v69, v[76:77] offset:1024\n"
        "v_mfma_f32_16x16x32_f16 a[108:111], v[140:143], v[124:127], a[108:111]\n"
        "v_cvt_pk_f16_f32 v78, v15, v19\n"
        "v_cvt_pk_f16_f32 v79, v23, v27\n"
        "v_mfma_f32_16x16x32_f16 a[140:143], v[144:147], v[124:127], a[140:143]\n"
        "ds_write_b64 v69, v[78:79] offset:1536\n"
        "v_mfma_f32_16x16x32_f16 a[172:175], v[148:151], v[124:127], a[172:175]\n"
        "v_add_u32 v69, s67, %[wdst]\n"
        "s_add_u32 s67, s67, 0x2000\n"
        "v_mfma_f32_16x16x32_f16 a[204:207], v[152:155], v[124:127], a[204:207]\n"
        "s_waitcnt vmcnt(18)\n"
        "s_cmp_lt_u32 s67, 163840\n"
        "s_cselect_b32 s67, s67, 0\n"
        "v_mfma_f32_16x16x32_f16 a[236:239], v[156:159], v[124:127], a[236:239]\n"
        "s_waitcnt lgkmcnt(6)\n"
        "v_mfma_f32_16x16x32_f16 a[16:19], v[128:131], v[112:115], a[16:19]\n"
        "v_cvt_pk_f16_f32 v72, v44, v48\n"
        "v_cvt_pk_f16_f32 v73, v52, v56\n"
        "v_mfma_f32_16x16x32_f16 a[48:51], v[132:135], v[112:115], a[48:51]\n"
        "ds_read_b128 v[124:127], v71 offset:1792\n"
        "v_mfma_f32_16x16x32_f16 a[80:83], v[136:139], v[112:115], a[80:83]\n"
        "ds_write_b64 v69, v[72:73] offset:0\n"
        "v_mfma_f32_16x16x32_f16 a[112:115], v[140:143], v[112:115], a[112:115]\n"
        "v_cvt_pk_f16_f32 v74, v45, v49\n"
        "v_cvt_pk_f16_f32 v75, v53, v57\n"
        "v_mfma_f32_16x16x32_f16 a[144:147], v[144:147], v[112:115], a[144:147]\n"
        "ds_write_b64 v69, v[74:75] offset:512\n"
        "v_mfma_f32_16x16x32_f16 a[176:179], v[148:151], v[112:115], a[176:179]\n"
        "v_cvt_pk_f16_f32 v76, v46, v50\n"
        "v_cvt_pk_f16_f32 v77, v54, v58\n"
        "v_mfma_f32_16x16x32_f16 a[208:211], v[152:155], v[112:115], a[208:211]\n"
        "ds_write_b64 v69, v[76:77] offset:1024\n"
        "v_mfma_f32_16x16x32_f16 a[240:243], v[156:159], v[112:115], a[240:243]\n"
        "s_waitcnt lgkmcnt(9)\n"
        "v_mfma_f32_16x16x32_f16 a[20:23], v[128:131], v[116:119], a[20:23]\n"
        "buffer_load_dwordx4 v[192:195], v64, %[rsA], s77 offen offset:0 sc0 sc1\n"
        "v_mfma_f32_16x16x32_f16 a[52:55], v[132:135], v[116:119], a[52:55]\n"
        "ds_read_b128 v[112:115], v71 offset:8192\n"
        "v_mfma_f32_16x16x32_f16 a[84:87], v[136:139], v[116:119], a[84:87]\n"
        "v_cvt_pk_f16_f32 v78, v47, v51\n"
        "v_cvt_pk_f16_f32 v79, v55, v59\n"
        "v_mfma_f32_16x16x32_f16 a[116:119], v[140:143], v[116:119], a[116:119]\n"
        "buffer_load_dwordx4 v[196:199], v64, %[rsA], s77 offen offset:1024 sc0 sc1\n"
        "v_mfma_f32_16x16x32_f16 a[148:151], v[144:147], v[116:119], a[148:151]\n"
        "ds_write_b64 v69, v[78:79] offset:1536\n"
        "v_mfma_f32_16x16x32_f16 a[180:183], v[148:151], v[116:119], a[180:183]\n"
        "v_mfma_f32_16x16x32_f16 a[212:215], v[152:155], v[116:119], a[212:215]\n"
        "buffer_load_dwordx4 v[200:203], v64, %[rsA], s77 offen offset:2048 sc0 sc1\n"
        "v_mfma_f32_16x16x32_f16 a[244:247], v[156:159], v[116:119], a[244:247]\n"
        "s_waitcnt lgkmcnt(8)\n"
        "v_mfma_f32_16x16x32_f16 a[24:27], v[128:131], v[120:123], a[24:27]\n"
        "v_mfma_f32_16x16x32_f16 a[56:59], v[132:135], v[120:123], a[56:59]\n"
        "buffer_load_dwordx4 v[204:207], v64, %[rsA], s77 offen offset:3072 sc0 sc1\n"
        "v_mfma_f32_16x16x32_f16 a[88:91], v[136:139], v[120:123], a[88:91]\n"
        "ds_read_b128 v[116:119], v71 offset:8704\n"
        "v_mfma_f32_16x16x32_f16 a[120:123], v[140:143], v[120:123], a[120:123]\n"
        "v_mfma_f32_16x16x32_f16 a[152:155], v[144:147], v[120:123], a[152:155]\n"
        "buffer_load_dwordx4 v[208:211], v64, %[rsA], s78 offen offset:0 sc0 sc1\n"
        "v_mfma_f32_16x16x32_f16 a[184:187], v[148:151], v[120:123], a[184:187]\n"
        "v_mfma_f32_16x16x32_f16 a[216:219], v[152:155], v[120:123], a[216:219]\n"
        "buffer_load_dwordx4 v[212:215], v64, %[rsA], s78 offen offset:1024 sc0 sc1\n"
        "v_mfma_f32_16x16x32_f16 a[248:251], v[156:159], v[120:123], a[248:251]\n"
        "s_waitcnt lgkmcnt(6)\n"
        "v_mfma_f32_16x16x32_f16 a[28:31], v[128:131], v[124:127], a[28:31]\n"
        "v_mfma_f32_16x16x32_f16 a[60:63], v[132:135], v[124:127], a[60:63]\n"
        "ds_read_b128 v[120:123], v71 offset:9216\n"
        "v_mfma_f32_16x16x32_f16 a[92:95], v[136:139], v[124:127], a[92:95]\n"
        "buffer_load_dwordx4 v[216:219], v64, %[rsA], s78 offen offset:2048 sc0 sc1\n"
        "v_mfma_f32_16x16x32_f16 a[124:127], v[140:143], v[124:127], a[124:127]\n"
        "v_mfma_f32_16x16x32_f16 a[156:159], v[144:147], v[124:127], a[156:159]\n"
        "buffer_load_dwordx4 v[220:223], v64, %[rsA], s78 offen offset:3072 sc0 sc1\n"
        "v_mfma_f32_16x16x32_f16 a[188:191], v[148:151], v[124:127], a[188:191]\n"
        "v_mfma_f32_16x16x32_f16 a[220:223], v[152:155], v[124:127], a[220:223]\n"
        "v_mfma_f32_16x16x32_f16 a[252:255], v[156:159], v[124:127], a[252:255]\n"
        "s_sub_u32 s45, s40, 8\n"
        "s_cmp_lt_u32 s45, 40\n"
        "s_cbranch_scc1 LTPH1_%=\n"
        "LTRH1_%=:\n"
        "s_barrier\n"
        "s_waitcnt vmcnt(18) lgkmcnt(3)\n"
        "v_mfma_f32_16x16x32_f16 a[0:3], v[160:163], v[112:115], a[0:3]\n"
        "buffer_load_dwordx4 v[12:15], %[wv0], %[rsW], s42 offen nt\n"
        "v_readlane_b32 s77, %[tablo], s40\n"
        "v_mfma_f32_16x16x32_f16 a[32:35], v[164:167], v[112:115], a[32:35]\n"
        "ds_read_b128 v[124:127], v71 offset:9728\n"
        "v_readlane_b32 s78, %[tabhi], s40\n"
        "s_add_u32 s40, s40, 1\n"
        "v_mfma_f32_16x16x32_f16 a[64:67], v[168:171], v[112:115], a[64:67]\n"
        "buffer_load_dwordx4 v[16:19], v62, %[rsW], s42 offen nt\n"
        "s_lshl_b64 s[54:55], s[54:55], 2\n"
        "v_mfma_f32_16x16x32_f16 a[96:99], v[172:175], v[112:115], a[96:99]\n"
        "buffer_load_dwordx4 v[20:23], v63, %[rsW], s42 offen nt\n"
        "s_or_b32 s54, s54, 3\n"
        "v_mfma_f32_16x16x32_f16 a[128:131], v[176:179], v[112:115], a[128:131]\n"
        "buffer_load_dwordx4 v[24:27], v66, %[rsW], s42 offen nt\n"
        "v_mfma_f32_16x16x32_f16 a[160:163], v[180:183], v[112:115], a[160:163]\n"
        "buffer_load_dwordx4 v[44:47], %[wv0], %[rsW], s68 offen nt\n"
        "v_mfma_f32_16x16x32_f16 a[192:195], v[184:187], v[112:115], a[192:195]\n"
        "buffer_load_dwordx4 v[48:51], v62, %[rsW], s68 offen nt\n"
        "v_mfma_f32_16x16x32_f16 a[224:227], v[188:191], v[112:115], a[224:227]\n"
        "s_waitcnt lgkmcnt(2)\n"
        "v_mfma_f32_16x16x32_f16 a[4:7], v[160:163], v[116:119], a[4:7]\n"
        "buffer_load_dwordx4 v[52:55], v63, %[rsW], s68 offen nt\n"
        "v_mfma_f32_16x16x32_f16 a[36:39], v[164:167], v[116:119], a[36:39]\n"
        "ds_read_b128 v[112:115], v71 offset:8448\n"
        "v_mfma_f32_16x16x32_f16 a[68:71], v[168:171], v[116:119], a[68:71]\n"
        "buffer_load_dwordx4 v[56:59], v66, %[rsW], s68 offen nt\n"
        "v_mfma_f32_16x16x32_f16 a[100:103], v[172:175], v[116:119], a[100:103]\n"
        "v_readlane_b32 s42, %[tabw], s66\n"
        "s_cmp_lt_u32 s66, 64\n"
        "s_cselect_b32 s42, s42, 0x80000000\n"
        "s_add_u32 s66, s66, 1\n"
        "v_mfma_f32_16x16x32_f16 a[132:135], v[176:179], v[116:119], a[132:135]\n"
        "v_readlane_b32 s68, %[tabw], s66\n"
        "s_cmp_lt_u32 s66, 64\n"
        "s_cselect_b32 s68, s68, 0x80000000\n"
        "s_add_u32 s66, s66, 1\n"
        "v_mfma_f32_16x16x32_f16 a[164:167], v[180:183], v[116:119], a[164:167]\n"
        "s_cmp_eq_u32 s40, s61\n"
        "s_cbranch_scc1 LSENDH1_%=\n"
        "LSENTH1_%=:\n"
        "v_mfma_f32_16x16x32_f16 a[196:199], v[184:187], v[116:119], a[196:199]\n"
        "s_cmp_ge_u32 s40, 41\n"
        "s_cselect_b64 s[54:55], -1, s[54:55]\n"
        "v_mfma_f32_16x16x32_f16 a[228:231], v[188:191], v[116:119], a[228:231]\n"
        "s_waitcnt lgkmcnt(2)\n"
        "v_mfma_f32_16x16x32_f16 a[8:11], v[160:163], v[120:123], a[8:11]\n"
        "v_add_u32 v69, s67, %[wdst]\n"
        "s_add_u32 s67, s67, 0x2000\n"
        "v_mfma_f32_16x16x32_f16 a[40:43], v[164:167], v[120:123], a[40:43]\n"
        "ds_read_b128 v[116:119], v71 offset:8960\n"
        "v_mfma_f32_16x16x32_f16 a[72:75], v[168:171], v[120:123], a[72:75]\n"
        "s_waitcnt vmcnt(22)\n"
        "s_cmp_lt_u32 s67, 163840\n"
        "s_cselect_b32 s67, s67, 0\n"
        "v_mfma_f32_16x16x32_f16 a[104:107], v[172:175], v[120:123], a[104:107]\n"
        "v_cvt_pk_f16_f32 v72, v80, v84\n"
        "v_cvt_pk_f16_f32 v73, v88, v92\n"
        "v_mfma_f32_16x16x32_f16 a[136:139], v[176:179], v[120:123], a[136:139]\n"
        "ds_write_b64 v69, v[72:73] offset:0\n"
        "v_mfma_f32_16x16x32_f16 a[168:171], v[180:183], v[120:123], a[168:171]\n"
        "v_cvt_pk_f16_f32 v74, v81, v85\n"
        "v_cvt_pk_f16_f32 v75, v89, v93\n"
        "v_mfma_f32_16x16x32_f16 a[200:203], v[184:187], v[120:123], a[200:203]\n"
        "ds_write_b64 v69, v[74:75] offset:512\n"
        "v_mfma_f32_16x16x32_f16 a[232:235], v[188:191], v[120:123], a[232:235]\n"
        "s_waitcnt lgkmcnt(4)\n"
        "v_mfma_f32_16x16x32_f16 a[12:15], v[160:163], v[124:127], a[12:15]\n"
        "v_cvt_pk_f16_f32 v76, v82, v86\n"
        "v_cvt_pk_f16_f32 v77, v90, v94\n"
        "v_mfma_f32_16x16x32_f16 a[44:47], v[164:167], v[124:127], a[44:47]\n"
        "ds_read_b128 v[120:123], v71 offset:9472\n"
        "v_mfma_f32_16x16x32_f16 a[76:79], v[168:171], v[124:127], a[76:79]\n"
        "ds_write_b64 v69, v[76:77] offset:1024\n"
        "v_mfma_f32_16x16x32_f16 a[108:111], v[172:175], v[124:127], a[108:111]\n"
        "v_cvt_pk_f16_f32 v78, v83, v87\n"
        "v_cvt_pk_f16_f32 v79, v91, v95\n"
        "v_mfma_f32_16x16x32_f16 a[140:143], v[176:179], v[124:127], a[140:143]\n"
        "ds_write_b64 v69, v[78:79] offset:1536\n"
        "v_mfma_f32_16x16x32_f16 a[172:175], v[180:183], v[124:127], a[172:175]\n"
        "v_add_u32 v69, s67, %[wdst]\n"
        "s_add_u32 s67, s67, 0x2000\n"
        "v_mfma_f32_16x16x32_f16 a[204:207], v[184:187], v[124:127], a[204:207]\n"
        "s_waitcnt vmcnt(18)\n"
        "s_cmp_lt_u32 s67, 163840\n"
        "s_cselect_b32 s67, s67, 0\n"
        "v_mfma_f32_16x16x32_f16 a[236:239], v[188:191], v[124:127], a[236:239]\n"
        "s_waitcnt lgkmcnt(6)\n"
        "v_mfma_f32_16x16x32_f16 a[16:19], v[160:163], v[112:115], a[16:19]\n"
        "v_cvt_pk_f16_f32 v72, v96, v100\n"
        "v_cvt_pk_f16_f32 v73, v104, v108\n"
        "v_mfma_f32_16x16x32_f16 a[48:51], v[164:167], v[112:115], a[48:51]\n"
        "ds_read_b128 v[124:127], v71 offset:9984\n"
        "v_mfma_f32_16x16x32_f16 a[80:83], v[168:171], v[112:115], a[80:83]\n"
        "ds_write_b64 v69, v[72:73] offset:0\n"
        "v_mfma_f32_16x16x32_f16 a[112:115], v[172:175], v[112:115], a[112:115]\n"
        "v_cvt_pk_f16_f32 v74, v97, v101\n"
        "v_cvt_pk_f16_f32 v75, v105, v109\n"
        "v_mfma_f32_16x16x32_f16 a[144:147], v[176:179], v[112:115], a[144:147]\n"
        "ds_write_b64 v69, v[74:75] offset:512\n"
        "v_mfma_f32_16x16x32_f16 a[176:179], v[180:183], v[112:115], a[176:179]\n"
        "v_cvt_pk_f16_f32 v76, v98, v102\n"
        "v_cvt_pk_f16_f32 v77, v106, v110\n"
        "v_mfma_f32_16x16x32_f16 a[208:211], v[184:187], v[112:115], a[208:211]\n"
        "ds_write_b64 v69, v[76:77] offset:1024\n"
        "v_mfma_f32_16x16x32_f16 a[240:243], v[188:191], v[112:115], a[240:243]\n"
        "s_waitcnt lgkmcnt(9)\n"
        "v_mfma_f32_16x16x32_f16 a[20:23], v[160:163], v[116:119], a[20:23]\n"
        "buffer_load_dwordx4 v[224:227], v64, %[rsA], s77 offen offset:0 sc0 sc1\n"
        "v_mfma_f32_16x16x32_f16 a[52:55], v[164:167], v[116:119], a[52:55]\n"
        "ds_read_b128 v[112:115], v71 offset:16384\n"
        "v_mfma_f32_16x16x32_f16 a[84:87], v[168:171], v[116:119], a[84:87]\n"
        "v_cvt_pk_f16_f32 v78, v99, v103\n"
        "v_cvt_pk_f16_f32 v79, v107, v111\n"
        "v_mfma_f32_16x16x32_f16 a[116:119], v[172:175], v[116:119], a[116:119]\n"
        "buffer_load_dwordx4 v[228:231], v64, %[rsA], s77 offen offset:1024 sc0 sc1\n"
        "v_mfma_f32_16x16x32_f16 a[148:151], v[176:179], v[116:119], a[148:151]\n"
        "ds_write_b64 v69, v[78:79] offset:1536\n"
        "v_mfma_f32_16x16x32_f16 a[180:183], v[180:183], v[116:119], a[180:183]\n"
        "s_waitcnt vmcnt(19)\n"
        "v_cmp_le_u32 vcc, s71, v61\n"
        "v_mfma_f32_16x16x32_f16 a[212:215], v[184:187], v[116:119], a[212:215]\n"
        "buffer_load_dwordx4 v[232:235], v64, %[rsA], s77 offen offset:2048 sc0 sc1\n"
        "v_mfma_f32_16x16x32_f16 a[244:247], v[188:191], v[116:119], a[244:247]\n"
        "s_waitcnt lgkmcnt(8)\n"
        "v_mfma_f32_16x16x32_f16 a[24:27], v[160:163], v[120:123], a[24:27]\n"
        "s_or_b64 s[46:47], s[46:47], vcc\n"
        "v_mfma_f32_16x16x32_f16 a[56:59], v[164:167], v[120:123], a[56:59]\n"
        "buffer_load_dwordx4 v[236:239], v64, %[rsA], s77 offen offset:3072 sc0 sc1\n"
        "s_lshl_b64 s[62:63], s[54:55], 8\n"
        "v_mfma_f32_16x16x32_f16 a[88:91], v[168:171], v[120:123], a[88:91]\n"
        "ds_read_b128 v[116:119], v71 offset:16896\n"
        "v_mfma_f32_16x16x32_f16 a[120:123], v[172:175], v[120:123], a[120:123]\n"
        "s_andn2_b64 s[62:63], s[62:63], s[46:47]\n"
        "s_cselect_b32 s60, 0, 0x6000000\n"
        "v_mfma_f32_16x16x32_f16 a[152:155], v[176:179], v[120:123], a[152:155]\n"
        "buffer_load_dwordx4 v[240:243], v64, %[rsA], s78 offen offset:0 sc0 sc1\n"
        "s_ff0_i32_b64 s52, s[46:47]\n"
        "v_mfma_f32_16x16x32_f16 a[184:187], v[180:183], v[120:123], a[184:187]\n"
        "s_bfm_b64 s[88:89], 44, s52\n"
        "v_mfma_f32_16x16x32_f16 a[216:219], v[184:187], v[120:123], a[216:219]\n"
        "buffer_load_dwordx4 v[244:247], v64, %[rsA], s78 offen offset:1024 sc0 sc1\n"
        "s_andn2_b64 s[88:89], s[88:89], s[46:47]\n"
        "v_mfma_f32_16x16x32_f16 a[248:251], v[188:191], v[120:123], a[248:251]\n"
        "s_waitcnt lgkmcnt(6)\n"
        "v_mfma_f32_16x16x32_f16 a[28:31], v[160:163], v[124:127], a[28:31]\n"
        "s_cmp_lt_u32 s40, 14\n"
        "s_cselect_b64 s[62:63], 0x3fffff, -1\n"
        "v_mfma_f32_16x16x32_f16 a[60:63], v[164:167], v[124:127], a[60:63]\n"
        "ds_read_b128 v[120:123], v71 offset:17408\n"
        "s_and_b64 s[88:89], s[88:89], s[62:63]\n"
        "v_mfma_f32_16x16x32_f16 a[92:95], v[168:171], v[124:127], a[92:95]\n"
        "buffer_load_dwordx4 v[248:251], v64, %[rsA], s78 offen offset:2048 sc0 sc1\n"
        "s_or_b32 s88, s88, 1\n"
        "v_mfma_f32_16x16x32_f16 a[124:127], v[172:175], v[124:127], a[124:127]\n"
        "s_cmp_eq_u32 s40, 14\n"
        "s_cbranch_scc1 LCWH_%=\n"
        "LCXH_%=:\n"
        "v_mfma_f32_16x16x32_f16 a[156:159], v[176:179], v[124:127], a[156:159]\n"
        "buffer_load_dwordx4 v[252:255], v64, %[rsA], s78 offen offset:3072 sc0 sc1\n"
        "v_mfma_f32_16x16x32_f16 a[188:191], v[180:183], v[124:127], a[188:191]\n"
        "s_andn2_b64 s[56:57], s[54:55], s[46:47]\n"
        "s_cbranch_scc1 LS_H1_%=\n"
        "LR_H1_%=:\n"
        "v_mfma_f32_16x16x32_f16 a[220:223], v[184:187], v[124:127], a[220:223]\n"
        "v_readlane_b32 s58, %[tabpf], s40\n"
        "v_mfma_f32_16x16x32_f16 a[252:255], v[188:191], v[124:127], a[252:255]\n"
        "s_bitcmp1_b64 s[46:47], s58\n"
        "s_cselect_b32 s58, s58, 0x6000000\n"
        "s_andn2_b32 s58, s58, 63\n"
        "s_sub_u32 s45, s40, 8\n"
        "s_cmp_lt_u32 s45, 40\n"
        "s_cbranch_scc1 LTPH2_%=\n"
        "LTRH2_%=:\n"
        "s_barrier\n"
        "s_waitcnt vmcnt(16) lgkmcnt(3)\n"
        "v_mfma_f32_16x16x32_f16 a[0:3], v[192:195], v[112:115], a[0:3]\n"
        "buffer_load_dwordx4 v[80:83], %[wv0], %[rsW], s42 offen nt\n"
        "v_readlane_b32 s77, %[tablo], s40\n"
        "v_mfma_f32_16x16x32_f16 a[32:35], v[196:199], v[112:115], a[32:35]\n"
        "ds_read_b128 v[124:127], v71 offset:17920\n"
        "v_readlane_b32 s78, %[tabhi], s40\n"
        "s_add_u32 s40, s40, 1\n"
        "v_mfma_f32_16x16x32_f16 a[64:67], v[200:203], v[112:115], a[64:67]\n"
        "buffer_load_dwordx4 v[84:87], v62, %[rsW], s42 offen nt\n"
        "v_mfma_f32_16x16x32_f16 a[96:99], v[204:207], v[112:115], a[96:99]\n"
        "buffer_load_dwordx4 v[88:91], v63, %[rsW], s42 offen nt\n"
        "v_mfma_f32_16x16x32_f16 a[128:131], v[208:211], v[112:115], a[128:131]\n"
        "buffer_load_dwordx4 v[92:95], v66, %[rsW], s42 offen nt\n"
        "v_mfma_f32_16x16x32_f16 a[160:163], v[212:215], v[112:115], a[160:163]\n"
        "buffer_load_dwordx4 v[96:99], %[wv0], %[rsW], s68 offen nt\n"
        "v_mfma_f32_16x16x32_f16 a[192:195], v[216:219], v[112:115], a[192:195]\n"
        "buffer_load_dwordx4 v[100:103], v62, %[rsW], s68 offen nt\n"
        "v_mfma_f32_16x16x32_f16 a[224:227], v[220:223], v[112:115], a[224:227]\n"
        "s_waitcnt lgkmcnt(2)\n"
        "v_mfma_f32_16x16x32_f16 a[4:7], v[192:195], v[116:119], a[4:7]\n"
        "buffer_load_dwordx4 v[104:107], v63, %[rsW], s68 offen nt\n"
        "v_mfma_f32_16x16x32_f16 a[36:39], v[196:199], v[116:119], a[36:39]\n"
        "ds_read_b128 v[112:115], v71 offset:16640\n"
        "v_mfma_f32_16x16x32_f16 a[68:71], v[200:203], v[116:119], a[68:71]\n"
        "buffer_load_dwordx4 v[108:111], v66, %[rsW], s68 offen nt\n"
        "s_mov_b64 exec, s[88:89]\n"
        "buffer_load_dword v61, %[cnt], %[rsA], s60 offen sc0 sc1\n"
        "s_mov_b64 exec, -1\n"
        "buffer_load_dword v65, v64, %[rsA], s58 offen sc0 sc1\n"
        "v_mfma_f32_16x16x32_f16 a[100:103], v[204:207], v[116:119], a[100:103]\n"
        "v_readlane_b32 s42, %[tabw], s66\n"
        "s_cmp_lt_u32 s66, 64\n"
        "s_cselect_b32 s42, s42, 0x80000000\n"
        "s_add_u32 s66, s66, 1\n"
        "v_mfma_f32_16x16x32_f16 a[132:135], v[208:211], v[116:119], a[132:135]\n"
        "v_readlane_b32 s68, %[tabw], s66\n"
        "s_cmp_lt_u32 s66, 64\n"
        "s_cselect_b32 s68, s68, 0x80000000\n"
        "s_add_u32 s66, s66, 1\n"
        "v_mfma_f32_16x16x32_f16 a[164:167], v[212:215], v[116:119], a[164:167]\n"
        "s_cmp_eq_u32 s40, s61\n"
        "s_cbranch_scc1 LSENDH2_%=\n"
        "LSENTH2_%=:\n"
        "v_mfma_f32_16x16x32_f16 a[196:199], v[216:219], v[116:119], a[196:199]\n"
        "v_mfma_f32_16x16x32_f16 a[228:231], v[220:223], v[116:119], a[228:231]\n"
        "s_waitcnt lgkmcnt(2)\n"
        "v_mfma_f32_16x16x32_f16 a[8:11], v[192:195], v[120:123], a[8:11]\n"
        "v_add_u32 v69, s67, %[wdst]\n"
        "s_add_u32 s67, s67, 0x2000\n"
        "v_mfma_f32_16x16x32_f16 a[40:43], v[196:199], v[120:123], a[40:43]\n"
        "ds_read_b128 v[116:119], v71 offset:17152\n"
        "v_mfma_f32_16x16x32_f16 a[72:75], v[200:203], v[120:123], a[72:75]\n"
        "s_waitcnt vmcnt(22)\n"
        "s_cmp_lt_u32 s67, 163840\n"
        "s_cselect_b32 s67, s67, 0\n"
        "v_mfma_f32_16x16x32_f16 a[104:107], v[204:207], v[120:123], a[104:107]\n"
        "v_cvt_pk_f16_f32 v72, v12, v16\n"
        "v_cvt_pk_f16_f32 v73, v20, v24\n"
        "v_mfma_f32_16x16x32_f16 a[136:139], v[208:211], v[120:123], a[136:139]\n"
        "ds_write_b64 v69, v[72:73] offset:0\n"
        "v_mfma_f32_16x16x32_f16 a[168:171], v[212:215], v[120:123], a[168:171]\n"
        "v_cvt_pk_f16_f32 v74, v13, v17\n"
        "v_cvt_pk_f16_f32 v75, v21, v25\n"
        "v_mfma_f32_16x16x32_f16 a[200:203], v[216:219], v[120:123], a[200:203]\n"
        "ds_write_b64 v69, v[74:75] offset:512\n"
        "v_mfma_f32_16x16x32_f16 a[232:235], v[220:223], v[120:123], a[232:235]\n"
        "s_waitcnt lgkmcnt(4)\n"
        "v_mfma_f32_16x16x32_f16 a[12:15], v[192:195], v[124:127], a[12:15]\n"
        "v_cvt_pk_f16_f32 v76, v14, v18\n"
        "v_cvt_pk_f16_f32 v77, v22, v26\n"
        "v_mfma_f32_16x16x32_f16 a[44:47], v[196:199], v[124:127], a[44:47]\n"
        "ds_read_b128 v[120:123], v71 offset:17664\n"
        "v_mfma_f32_16x16x32_f16 a[76:79], v[200:203], v[124:127], a[76:79]\n"
        "ds_write_b64 v69, v[76:77] offset:1024\n"
        "v_mfma_f32_16x16x32_f16 a[108:111], v[204:207], v[124:127], a[108:111]\n"
        "v_cvt_pk_f16_f32 v78, v15, v19\n"
        "v_cvt_pk_f16_f32 v79, v23, v27\n"
        "v_mfma_f32_16x16x32_f16 a[140:143], v[208:211], v[124:127], a[140:143]\n"
        "ds_write_b64 v69, v[78:79] offset:1536\n"
        "v_mfma_f32_16x16x32_f16 a[172:175], v[212:215], v[124:127], a[172:175]\n"
        "v_add_u32 v69, s67, %[wdst]\n"
        "s_add_u32 s67, s67, 0x2000\n"
        "v_mfma_f32_16x16x32_f16 a[204:207], v[216:219], v[124:127], a[204:207]\n"
        "s_waitcnt vmcnt(18)\n"
        "s_cmp_lt_u32 s67, 163840\n"
        "s_cselect_b32 s67, s67, 0\n"
        "v_mfma_f32_16x16x32_f16 a[236:239], v[220:223], v[124:127], a[236:239]\n"
        "s_waitcnt lgkmcnt(6)\n"
        "v_mfma_f32_16x16x32_f16 a[16:19], v[192:195], v[112:115], a[16:19]\n"
        "v_cvt_pk_f16_f32 v72, v44, v48\n"
        "v_cvt_pk_f16_f32 v73, v52, v56\n"
        "v_mfma_f32_16x16x32_f16 a[48:51], v[196:199], v[112:115], a[48:51]\n"
        "ds_read_b128 v[124:127], v71 offset:18176\n"
        "v_mfma_f32_16x16x32_f16 a[80:83], v[200:203], v[112:115], a[80:83]\n"
        "ds_write_b64 v69, v[72:73] offset:0\n"
        "v_mfma_f32_16x16x32_f16 a[112:115], v[204:207], v[112:115], a[112:115]\n"
        "v_cvt_pk_f16_f32 v74, v45, v49\n"
        "v_cvt_pk_f16_f32 v75, v53, v57\n"
        "v_mfma_f32_16x16x32_f16 a[144:147], v[208:211], v[112:115], a[144:147]\n"
        "ds_write_b64 v69, v[74:75] offset:512\n"
        "v_mfma_f32_16x16x32_f16 a[176:179], v[212:215], v[112:115], a[176:179]\n"
        "v_cvt_pk_f16_f32 v76, v46, v50\n"
        "v_cvt_pk_f16_f32 v77, v54, v58\n"
        "v_mfma_f32_16x16x32_f16 a[208:211], v[216:219], v[112:115], a[208:211]\n"
        "ds_write_b64 v69, v[76:77] offset:1024\n"
        "v_mfma_f32_16x16x32_f16 a[240:243], v[220:223], v[112:115], a[240:243]\n"
        "s_waitcnt lgkmcnt(9)\n"
        "v_mfma_f32_16x16x32_f16 a[20:23], v[192:195], v[116:119], a[20:23]\n"
        "buffer_load_dwordx4 v[128:131], v64, %[rsA], s77 offen offset:0 sc0 sc1\n"
        "v_mfma_f32_16x16x32_f16 a[52:55], v[196:199], v[116:119], a[52:55]\n"
        "ds_read_b128 v[112:115], v71 offset:24576\n"
        "v_mfma_f32_16x16x32_f16 a[84:87], v[200:203], v[116:119], a[84:87]\n"
        "v_cvt_pk_f16_f32 v78, v47, v51\n"
        "v_cvt_pk_f16_f32 v79, v55, v59\n"
        "v_mfma_f32_16x16x32_f16 a[116:119], v[204:207], v[116:119], a[116:119]\n"
        "buffer_load_dwordx4 v[132:135], v64, %[rsA], s77 offen offset:1024 sc0 sc1\n"
        "v_mfma_f32_16x16x32_f16 a[148:151], v[208:211], v[116:119], a[148:151]\n"
        "ds_write_b64 v69, v[78:79] offset:1536\n"
        "v_mfma_f32_16x16x32_f16 a[180:183], v[212:215], v[116:119], a[180:183]\n"
        "v_mfma_f32_16x16x32_f16 a[212:215], v[216:219], v[116:119], a[212:215]\n"
        "buffer_load_dwordx4 v[136:139], v64, %[rsA], s77 offen offset:2048 sc0 sc1\n"
        "v_mfma_f32_16x16x32_f16 a[244:247], v[220:223], v[116:119], a[244:247]\n"
        "s_waitcnt lgkmcnt(8)\n"
        "v_mfma_f32_16x16x32_f16 a[24:27], v[192:195], v[120:123], a[24:27]\n"
        "v_mfma_f32_16x16x32_f16 a[56:59], v[196:199], v[120:123], a[56:59]\n"
        "buffer_load_dwordx4 v[140:143], v64, %[rsA], s77 offen offset:3072 sc0 sc1\n"
        "v_mfma_f32_16x16x32_f16 a[88:91], v[200:203], v[120:123], a[88:91]\n"
        "ds_read_b128 v[116:119], v71 offset:25088\n"
        "v_mfma_f32_16x16x32_f16 a[120:123], v[204:207], v[120:123], a[120:123]\n"
        "v_mfma_f32_16x16x32_f16 a[152:155], v[208:211], v[120:123], a[152:155]\n"
        "buffer_load_dwordx4 v[144:147], v64, %[rsA], s78 offen offset:0 sc0 sc1\n"
        "v_mfma_f32_16x16x32_f16 a[184:187], v[212:215], v[120:123], a[184:187]\n"
        "v_mfma_f32_16x16x32_f16 a[216:219], v[216:219], v[120:123], a[216:219]\n"
        "buffer_load_dwordx4 v[148:151], v64, %[rsA], s78 offen offset:1024 sc0 sc1\n"
        "v_mfma_f32_16x16x32_f16 a[248:251], v[220:223], v[120:123], a[248:251]\n"
        "s_waitcnt lgkmcnt(6)\n"
        "v_mfma_f32_16x16x32_f16 a[28:31], v[192:195], v[124:127], a[28:31]\n"
        "v_mfma_f32_16x16x32_f16 a[60:63], v[196:199], v[124:127], a[60:63]\n"
        "ds_read_b128 v[120:123], v71 offset:25600\n"
        "v_mfma_f32_16x16x32_f16 a[92:95], v[200:203], v[124:127], a[92:95]\n"
        "buffer_load_dwordx4 v[152:155], v64, %[rsA], s78 offen offset:2048 sc0 sc1\n"
        "v_mfma_f32_16x16x32_f16 a[124:127], v[204:207], v[124:127], a[124:127]\n"
        "v_mfma_f32_16x16x32_f16 a[156:159], v[208:211], v[124:127], a[156:159]\n"
        "buffer_load_dwordx4 v[156:159], v64, %[rsA], s78 offen offset:3072 sc0 sc1\n"
        "v_mfma_f32_16x16x32_f16 a[188:191], v[212:215], v[124:127], a[188:191]\n"
        "v_mfma_f32_16x16x32_f16 a[220:223], v[216:219], v[124:127], a[220:223]\n"
        "v_mfma_f32_16x16x32_f16 a[252:255], v[220:223], v[124:127], a[252:255]\n"
        "s_sub_u32 s45, s40, 8\n"
        "s_cmp_lt_u32 s45, 40\n"
        "s_cbranch_scc1 LTPH3_%=\n"
        "LTRH3_%=:\n"
        "s_barrier\n"
        "s_waitcnt vmcnt(18) lgkmcnt(3)\n"
        "v_mfma_f32_16x16x32_f16 a[0:3], v[224:227], v[112:115], a[0:3]\n"
        "buffer_load_dwordx4 v[12:15], %[wv0], %[rsW], s42 offen nt\n"
        "v_readlane_b32 s77, %[tablo], s40\n"
        "v_mfma_f32_16x16x32_f16 a[32:35], v[228:231], v[112:115], a[32:35]\n"
        "ds_read_b128 v[124:127], v71 offset:26112\n"
        "v_readlane_b32 s78, %[tabhi], s40\n"
        "s_add_u32 s40, s40, 1\n"
        "v_mfma_f32_16x16x32_f16 a[64:67], v[232:235], v[112:115], a[64:67]\n"
        "buffer_load_dwordx4 v[16:19], v62, %[rsW], s42 offen nt\n"
        "s_lshl_b64 s[54:55], s[54:55], 2\n"
        "v_mfma_f32_16x16x32_f16 a[96:99], v[236:239], v[112:115], a[96:99]\n"
        "buffer_load_dwordx4 v[20:23], v63, %[rsW], s42 offen nt\n"
        "s_or_b32 s54, s54, 3\n"
        "v_mfma_f32_16x16x32_f16 a[128:131], v[240:243], v[112:115], a[128:131]\n"
        "buffer_load_dwordx4 v[24:27], v66, %[rsW], s42 offen nt\n"
        "s_add_u32 s69, s69, 0x8000\n"
        "v_mfma_f32_16x16x32_f16 a[160:163], v[244:247], v[112:115], a[160:163]\n"
        "buffer_load_dwordx4 v[44:47], %[wv0], %[rsW], s68 offen nt\n"
        "v_mfma_f32_16x16x32_f16 a[192:195], v[248:251], v[112:115], a[192:195]\n"
        "buffer_load_dwordx4 v[48:51], v62, %[rsW], s68 offen nt\n"
        "v_mfma_f32_16x16x32_f16 a[224:227], v[252:255], v[112:115], a[224:227]\n"
        "s_waitcnt lgkmcnt(2)\n"
        "v_mfma_f32_16x16x32_f16 a[4:7], v[224:227], v[116:119], a[4:7]\n"
        "buffer_load_dwordx4 v[52:55], v63, %[rsW], s68 offen nt\n"
        "v_mfma_f32_16x16x32_f16 a[36:39], v[228:231], v[116:119], a[36:39]\n"
        "ds_read_b128 v[112:115], v71 offset:24832\n"
        "v_mfma_f32_16x16x32_f16 a[68:71], v[232:235], v[116:119], a[68:71]\n"
        "buffer_load_dwordx4 v[56:59], v66, %[rsW], s68 offen nt\n"
        "v_mfma_f32_16x16x32_f16 a[100:103], v[236:239], v[116:119], a[100:103]\n"
        "v_readlane_b32 s42, %[tabw], s66\n"
        "s_cmp_lt_u32 s66, 64\n"
        "s_cselect_b32 s42, s42, 0x80000000\n"
        "s_add_u32 s66, s66, 1\n"
        "v_mfma_f32_16x16x32_f16 a[132:135], v[240:243], v[116:119], a[132:135]\n"
        "v_readlane_b32 s68, %[tabw], s66\n"
        "s_cmp_lt_u32 s66, 64\n"
        "s_cselect_b32 s68, s68, 0x80000000\n"
        "s_add_u32 s66, s66, 1\n"
        "v_mfma_f32_16x16x32_f16 a[164:167], v[244:247], v[116:119], a[164:167]\n"
        "s_cmp_eq_u32 s40, s61\n"
        "s_cbranch_scc1 LSENDH3_%=\n"
        "LSENTH3_%=:\n"
        "v_mfma_f32_16x16x32_f16 a[196:199], v[248:251], v[116:119], a[196:199]\n"
        "s_cmp_ge_u32 s40, 41\n"
        "s_cselect_b64 s[54:55], -1, s[54:55]\n"
        "v_mfma_f32_16x16x32_f16 a[228:231], v[252:255], v[116:119], a[228:231]\n"
        "s_waitcnt lgkmcnt(2)\n"
        "v_mfma_f32_16x16x32_f16 a[8:11], v[224:227], v[120:123], a[8:11]\n"
        "v_add_u32 v69, s67, %[wdst]\n"
        "s_add_u32 s67, s67, 0x2000\n"
        "v_mfma_f32_16x16x32_f16 a[40:43], v[228:231], v[120:123], a[40:43]\n"
        "ds_read_b128 v[116:119], v71 offset:25344\n"
        "v_mfma_f32_16x16x32_f16 a[72:75], v[232:235], v[120:123], a[72:75]\n"
        "s_waitcnt vmcnt(22)\n"
        "s_cmp_lt_u32 s67, 163840\n"
        "s_cselect_b32 s67, s67, 0\n"
        "v_mfma_f32_16x16x32_f16 a[104:107], v[236:239], v[120:123], a[104:107]\n"
        "v_cvt_pk_f16_f32 v72, v80, v84\n"
        "v_cvt_pk_f16_f32 v73, v88, v92\n"
        "v_mfma_f32_16x16x32_f16 a[136:139], v[240:243], v[120:123], a[136:139]\n"
        "ds_write_b64 v69, v[72:73] offset:0\n"
        "v_mfma_f32_16x16x32_f16 a[168:171], v[244:247], v[120:123], a[168:171]\n"
        "v_cvt_pk_f16_f32 v74, v81, v85\n"
        "v_cvt_pk_f16_f32 v75, v89, v93\n"
        "v_mfma_f32_16x16x32_f16 a[200:203], v[248:251], v[120:123], a[200:203]\n"
        "ds_write_b64 v69, v[74:75] offset:512\n"
        "v_mfma_f32_16x16x32_f16 a[232:235], v[252:255], v[120:123], a[232:235]\n"
        "s_waitcnt lgkmcnt(4)\n"
        "v_mfma_f32_16x16x32_f16 a[12:15], v[224:227], v[124:127], a[12:15]\n"
        "v_cvt_pk_f16_f32 v76, v82, v86\n"
        "v_cvt_pk_f16_f32 v77, v90, v94\n"
        "v_mfma_f32_16x16x32_f16 a[44:47], v[228:231], v[124:127], a[44:47]\n"
        "ds_read_b128 v[120:123], v71 offset:25856\n"
        "v_mfma_f32_16x16x32_f16 a[76:79], v[232:235], v[124:127], a[76:79]\n"
        "ds_write_b64 v69, v[76:77] offset:1024\n"
        "v_mfma_f32_16x16x32_f16 a[108:111], v[236:239], v[124:127], a[108:111]\n"
        "v_cvt_pk_f16_f32 v78, v83, v87\n"
        "v_cvt_pk_f16_f32 v79, v91, v95\n"
        "v_mfma_f32_16x16x32_f16 a[140:143], v[240:243], v[124:127], a[140:143]\n"
        "ds_write_b64 v69, v[78:79] offset:1536\n"
        "v_mfma_f32_16x16x32_f16 a[172:175], v[244:247], v[124:127], a[172:175]\n"
        "v_add_u32 v69, s67, %[wdst]\n"
        "s_add_u32 s67, s67, 0x2000\n"
        "v_mfma_f32_16x16x32_f16 a[204:207], v[248:251], v[124:127], a[204:207]\n"
        "s_waitcnt vmcnt(18)\n"
        "s_cmp_lt_u32 s67, 163840\n"
        "s_cselect_b32 s67, s67, 0\n"
        "v_mfma_f32_16x16x32_f16 a[236:239], v[252:255], v[124:127], a[236:239]\n"
        "s_waitcnt lgkmcnt(6)\n"
        "v_mfma_f32_16x16x32_f16 a[16:19], v[224:227], v[112:115], a[16:19]\n"
        "v_cvt_pk_f16_f32 v72, v96, v100\n"
        "v_cvt_pk_f16_f32 v73, v104, v108\n"
        "v_mfma_f32_16x16x32_f16 a[48:51], v[228:231], v[112:115], a[48:51]\n"
        "ds_read_b128 v[124:127], v71 offset:26368\n"
        "v_mfma_f32_16x16x32_f16 a[80:83], v[232:235], v[112:115], a[80:83]\n"
        "ds_write_b64 v69, v[72:73] offset:0\n"
        "v_mfma_f32_16x16x32_f16 a[112:115], v[236:239], v[112:115], a[112:115]\n"
        "v_cvt_pk_f16_f32 v74, v97, v101\n"
        "v_cvt_pk_f16_f32 v75, v105, v109\n"
        "v_mfma_f32_16x16x32_f16 a[144:147], v[240:243], v[112:115], a[144:147]\n"
        "ds_write_b64 v69, v[74:75] offset:512\n"
        "v_mfma_f32_16x16x32_f16 a[176:179], v[244:247], v[112:115], a[176:179]\n"
        "v_cvt_pk_f16_f32 v76, v98, v102\n"
        "v_cvt_pk_f16_f32 v77, v106, v110\n"
        "v_mfma_f32_16x16x32_f16 a[208:211], v[248:251], v[112:115], a[208:211]\n"
        "ds_write_b64 v69, v[76:77] offset:1024\n"
        "v_mfma_f32_16x16x32_f16 a[240:243], v[252:255], v[112:115], a[240:243]\n"
        "s_waitcnt lgkmcnt(9)\n"
        "v_mfma_f32_16x16x32_f16 a[20:23], v[224:227], v[116:119], a[20:23]\n"
        "buffer_load_dwordx4 v[160:163], v64, %[rsA], s77 offen offset:0 sc0 sc1\n"
        "v_mfma_f32_16x16x32_f16 a[52:55], v[228:231], v[116:119], a[52:55]\n"
        "ds_read_b128 v[112:115], v70 offset:0\n"
        "v_mfma_f32_16x16x32_f16 a[84:87], v[232:235], v[116:119], a[84:87]\n"
        "v_cvt_pk_f16_f32 v78, v99, v103\n"
        "v_cvt_pk_f16_f32 v79, v107, v111\n"
        "v_mfma_f32_16x16x32_f16 a[116:119], v[236:239], v[116:119], a[116:119]\n"
        "buffer_load_dwordx4 v[164:167], v64, %[rsA], s77 offen offset:1024 sc0 sc1\n"
        "v_mfma_f32_16x16x32_f16 a[148:151], v[240:243], v[116:119], a[148:151]\n"
        "ds_write_b64 v69, v[78:79] offset:1536\n"
        "v_mfma_f32_16x16x32_f16 a[180:183], v[244:247], v[116:119], a[180:183]\n"
        "s_waitcnt vmcnt(19)\n"
        "v_cmp_le_u32 vcc, s71, v61\n"
        "v_mfma_f32_16x16x32_f16 a[212:215], v[248:251], v[116:119], a[212:215]\n"
        "buffer_load_dwordx4 v[168:171], v64, %[rsA], s77 offen offset:2048 sc0 sc1\n"
        "v_mfma_f32_16x16x32_f16 a[244:247], v[252:255], v[116:119], a[244:247]\n"
        "s_waitcnt lgkmcnt(8)\n"
        "v_mfma_f32_16x16x32_f16 a[24:27], v[224:227], v[120:123], a[24:27]\n"
        "s_or_b64 s[46:47], s[46:47], vcc\n"
        "v_mfma_f32_16x16x32_f16 a[56:59], v[228:231], v[120:123], a[56:59]\n"
        "buffer_load_dwordx4 v[172:175], v64, %[rsA], s77 offen offset:3072 sc0 sc1\n"
        "s_lshl_b64 s[62:63], s[54:55], 8\n"
        "v_mfma_f32_16x16x32_f16 a[88:91], v[232:235], v[120:123], a[88:91]\n"
        "ds_read_b128 v[116:119], v70 offset:512\n"
        "v_mfma_f32_16x16x32_f16 a[120:123], v[236:239], v[120:123], a[120:123]\n"
        "s_andn2_b64 s[62:63], s[62:63], s[46:47]\n"
        "s_cselect_b32 s60, 0, 0x6000000\n"
        "v_mfma_f32_16x16x32_f16 a[152:155], v[240:243], v[120:123], a[152:155]\n"
        "buffer_load_dwordx4 v[176:179], v64, %[rsA], s78 offen offset:0 sc0 sc1\n"
        "s_ff0_i32_b64 s52, s[46:47]\n"
        "v_mfma_f32_16x16x32_f16 a[184:187], v[244:247], v[120:123], a[184:187]\n"
        "s_bfm_b64 s[88:89], 44, s52\n"
        "v_mfma_f32_16x16x32_f16 a[216:219], v[248:251], v[120:123], a[216:219]\n"
        "buffer_load_dwordx4 v[180:183], v64, %[rsA], s78 offen offset:1024 sc0 sc1\n"
        "s_andn2_b64 s[88:89], s[88:89], s[46:47]\n"
        "v_mfma_f32_16x16x32_f16 a[248:251], v[252:255], v[120:123], a[248:251]\n"
        "s_waitcnt lgkmcnt(6)\n"
        "v_mfma_f32_16x16x32_f16 a[28:31], v[224:227], v[124:127], a[28:31]\n"
        "s_cmp_lt_u32 s40, 14\n"
        "s_cselect_b64 s[62:63], 0x3fffff, -1\n"
        "v_mfma_f32_16x16x32_f16 a[60:63], v[228:231], v[124:127], a[60:63]\n"
        "ds_read_b128 v[120:123], v70 offset:1024\n"
        "s_and_b64 s[88:89], s[88:89], s[62:63]\n"
        "v_mfma_f32_16x16x32_f16 a[92:95], v[232:235], v[124:127], a[92:95]\n"
        "buffer_load_dwordx4 v[184:187], v64, %[rsA], s78 offen offset:2048 sc0 sc1\n"
        "s_or_b32 s88, s88, 1\n"
        "v_mfma_f32_16x16x32_f16 a[124:127], v[236:239], v[124:127], a[124:127]\n"
        "s_andn2_b64 s[56:57], s[54:55], s[46:47]\n"
        "s_cbranch_scc1 LS_H3_%=\n"
        "LR_H3_%=:\n"
        "v_mfma_f32_16x16x32_f16 a[156:159], v[240:243], v[124:127], a[156:159]\n"
        "buffer_load_dwordx4 v[188:191], v64, %[rsA], s78 offen offset:3072 sc0 sc1\n"
        "v_mfma_f32_16x16x32_f16 a[188:191], v[244:247], v[124:127], a[188:191]\n"
        "v_readlane_b32 s58, %[tabpf], s40\n"
        "v_mfma_f32_16x16x32_f16 a[220:223], v[248:251], v[124:127], a[220:223]\n"
        "s_bitcmp1_b64 s[46:47], s58\n"
        "s_cselect_b32 s58, s58, 0x6000000\n"
        "v_mfma_f32_16x16x32_f16 a[252:255], v[252:255], v[124:127], a[252:255]\n"
        "s_andn2_b32 s58, s58, 63\n"
        "s_cmp_lt_u32 s69, 163840\n"
        "s_cselect_b32 s69, s69, 0\n"
        "v_mov_b32 v71, v70\n"
        "v_add_u32 v70, s69, %[bbase]\n"
        "s_sub_u32 s82, s82, 4\n"
        "s_cmp_eq_u32 s40, 16\n"
        "s_cbranch_scc1 LCLS_%=\n"
        "s_branch LBACK_%=\n"
        "LS_p_%=:\n"
        "s_getpc_b64 s[64:65]\n"
        "s_branch LPOLL_%=\n"
        "s_branch LR_p_%=\n"
        "LSR0_%=:\n"
        "s_getpc_b64 s[64:65]\n"
        "s_branch LSWAP_%=\n"
        "s_branch LSB0_%=\n"
        "LSENDN0_%=:\n"
        "s_getpc_b64 s[64:65]\n"
        "s_branch LSEND_%=\n"
        "s_branch LSENTN0_%=\n"
        "LTQN0_%=:\n"
        "s_sleep 1\n"
        "LTPN0_%=:\n"
        "s_memrealtime s[50:51]\n"
        "s_waitcnt lgkmcnt(0)\n"
        "s_sub_u32 s45, s50, s73\n"
        "s_cmp_lt_i32 s45, 0\n"
        "s_cbranch_scc1 LTQN0_%=\n"
        "s_sub_i32 s52, s40, 16\n"
        "s_max_i32 s52, s52, 0\n"
        "s_mul_i32 s52, s52, s75\n"
        "s_sub_i32 s52, s74, s52\n"
        "s_max_i32 s52, s52, s79\n"
        "s_cmp_lt_u32 s40, 12\n"
        "s_cselect_b32 s52, s92, s52\n"
        "s_mul_i32 s52, s52, 196\n"
        "s_lshr_b32 s52, s52, 8\n"
        "s_add_u32 s72, s72, s52\n"
        "s_add_u32 s73, s50, s52\n"
        "s_sub_u32 s73, s73, s76\n"
        "s_sub_u32 s45, s72, s73\n"
        "s_max_i32 s45, s45, 0\n"
        "s_min_i32 s45, s45, 22\n"
        "s_add_u32 s73, s73, s45\n"
        "s_branch LTRN0_%=\n"
        "LSENDN1_%=:\n"
        "s_getpc_b64 s[64:65]\n"
        "s_branch LSEND_%=\n"
        "s_branch LSENTN1_%=\n"
        "LSW1_%=:\n"
        "v_mov_b32 v68, %[xl]\n"
        "s_lshl_b32 s45, s85, 31\n"
        "v_mov_b32 v67, s46\n"
        "v_and_b32 v67, 0x3fffff, v67\n"
        "v_or_b32 v67, s45, v67\n"
        "ds_write_b32 v68, v67\n"
        "s_waitcnt lgkmcnt(0)\n"
        "s_branch LSX1_%=\n"
        "LCWN_%=:\n"
        "s_cmp_eq_u32 s60, 0\n"
        "s_cselect_b64 s[88:89], s[88:89], 0\n"
        "s_or_b32 s88, s88, 3\n"
        "s_mov_b32 s60, 0\n"
        "s_add_u32 s45, s86, 4\n"
        "v_writelane_b32 %[cnt], s86, 0\n"
        "v_writelane_b32 %[cnt], s45, 1\n"
        "s_branch LCXN_%=\n"
        "LS_N1_%=:\n"
        "s_getpc_b64 s[64:65]\n"
        "s_branch LPOLL_%=\n"
        "s_branch LR_N1_%=\n"
        "LSR2_%=:\n"
        "s_getpc_b64 s[64:65]\n"
        "s_branch LSWAP_%=\n"
        "s_branch LSB2_%=\n"
        "LSENDN2_%=:\n"
        "s_getpc_b64 s[64:65]\n"
        "s_branch LSEND_%=\n"
        "s_branch LSENTN2_%=\n"
        "LTQN2_%=:\n"
        "s_sleep 1\n"
        "LTPN2_%=:\n"
        "s_memrealtime s[50:51]\n"
        "s_waitcnt lgkmcnt(0)\n"
        "s_sub_u32 s45, s50, s73\n"
        "s_cmp_lt_i32 s45, 0\n"
        "s_cbranch_scc1 LTQN2_%=\n"
        "s_sub_i32 s52, s40, 16\n"
        "s_max_i32 s52, s52, 0\n"
        "s_mul_i32 s52, s52, s75\n"
        "s_sub_i32 s52, s74, s52\n"
        "s_max_i32 s52, s52, s79\n"
        "s_cmp_lt_u32 s40, 12\n"
        "s_cselect_b32 s52, s92, s52\n"
        "s_mul_i32 s52, s52, 196\n"
        "s_lshr_b32 s52, s52, 8\n"
        "s_add_u32 s72, s72, s52\n"
        "s_add_u32 s73, s50, s52\n"
        "s_sub_u32 s73, s73, s76\n"
        "s_sub_u32 s45, s72, s73\n"
        "s_max_i32 s45, s45, 0\n"
        "s_min_i32 s45, s45, 22\n"
        "s_add_u32 s73, s73, s45\n"
        "s_branch LTRN2_%=\n"
        "LSENDN3_%=:\n"
        "s_getpc_b64 s[64:65]\n"
        "s_branch LSEND_%=\n"
        "s_branch LSENTN3_%=\n"
        "LSW3_%=:\n"
        "v_mov_b32 v68, %[xl]\n"
        "s_lshl_b32 s45, s85, 31\n"
        "v_mov_b32 v67, s46\n"
        "v_and_b32 v67, 0x3fffff, v67\n"
        "v_or_b32 v67, s45, v67\n"
        "ds_write_b32 v68, v67\n"
        "s_waitcnt lgkmcnt(0)\n"
        "s_branch LSX3_%=\n"
        "LS_N3_%=:\n"
        "s_getpc_b64 s[64:65]\n"
        "s_branch LPOLL_%=\n"
        "s_branch LR_N3_%=\n"
        "LSENDH0_%=:\n"
        "s_getpc_b64 s[64:65]\n"
        "s_branch LSEND_%=\n"
        "s_branch LSENTH0_%=\n"
        "LTQH0_%=:\n"
        "s_sleep 1\n"
        "LTPH0_%=:\n"
        "s_memrealtime s[50:51]\n"
        "s_waitcnt lgkmcnt(0)\n"
        "s_sub_u32 s45, s50, s73\n"
        "s_cmp_lt_i32 s45, 0\n"
        "s_cbranch_scc1 LTQH0_%=\n"
        "s_sub_i32 s52, s40, 16\n"
        "s_max_i32 s52, s52, 0\n"
        "s_mul_i32 s52, s52, s75\n"
        "s_sub_i32 s52, s74, s52\n"
        "s_max_i32 s52, s52, s79\n"
        "s_cmp_lt_u32 s40, 12\n"
        "s_cselect_b32 s52, s92, s52\n"
        "s_mul_i32 s52, s52, 137\n"
        "s_lshr_b32 s52, s52, 8\n"
        "s_add_u32 s72, s72, s52\n"
        "s_add_u32 s73, s50, s52\n"
        "s_sub_u32 s73, s73, s76\n"
        "s_sub_u32 s45, s72, s73\n"
        "s_max_i32 s45, s45, 0\n"
        "s_min_i32 s45, s45, 22\n"
        "s_add_u32 s73, s73, s45\n"
        "s_branch LTRH0_%=\n"
        "LSENDH1_%=:\n"
        "s_getpc_b64 s[64:65]\n"
        "s_branch LSEND_%=\n"
        "s_branch LSENTH1_%=\n"
        "LCWH_%=:\n"
        "s_cmp_eq_u32 s60, 0\n"
        "s_cselect_b64 s[88:89], s[88:89], 0\n"
        "s_or_b32 s88, s88, 3\n"
        "s_mov_b32 s60, 0\n"
        "s_add_u32 s45, s86, 4\n"
        "v_writelane_b32 %[cnt], s86, 0\n"
        "v_writelane_b32 %[cnt], s45, 1\n"
        "s_branch LCXH_%=\n"
        "LS_H1_%=:\n"
        "s_getpc_b64 s[64:65]\n"
        "s_branch LPOLL_%=\n"
        "s_branch LR_H1_%=\n"
        "LTQH1_%=:\n"
        "s_sleep 1\n"
        "LTPH1_%=:\n"
        "s_memrealtime s[50:51]\n"
        "s_waitcnt lgkmcnt(0)\n"
        "s_sub_u32 s45, s50, s73\n"
        "s_cmp_lt_i32 s45, 0\n"
        "s_cbranch_scc1 LTQH1_%=\n"
        "s_sub_i32 s52, s40, 16\n"
        "s_max_i32 s52, s52, 0\n"
        "s_mul_i32 s52, s52, s75\n"
        "s_sub_i32 s52, s74, s52\n"
        "s_max_i32 s52, s52, s79\n"
        "s_cmp_lt_u32 s40, 12\n"
        "s_cselect_b32 s52, s92, s52\n"
        "s_mul_i32 s52, s52, 137\n"
        "s_lshr_b32 s52, s52, 8\n"
        "s_add_u32 s72, s72, s52\n"
        "s_add_u32 s73, s50, s52\n"
        "s_sub_u32 s73, s73, s76\n"
        "s_sub_u32 s45, s72, s73\n"
        "s_max_i32 s45, s45, 0\n"
        "s_min_i32 s45, s45, 22\n"
        "s_add_u32 s73, s73, s45\n"
        "s_branch LTRH1_%=\n"
        "LSENDH2_%=:\n"
        "s_getpc_b64 s[64:65]\n"
        "s_branch LSEND_%=\n"
        "s_branch LSENTH2_%=\n"
        "LTQH2_%=:\n"
        "s_sleep 1\n"
        "LTPH2_%=:\n"
        "s_memrealtime s[50:51]\n"
        "s_waitcnt lgkmcnt(0)\n"
        "s_sub_u32 s45, s50, s73\n"
        "s_cmp_lt_i32 s45, 0\n"
        "s_cbranch_scc1 LTQH2_%=\n"
        "s_sub_i32 s52, s40, 16\n"
        "s_max_i32 s52, s52, 0\n"
        "s_mul_i32 s52, s52, s75\n"
        "s_sub_i32 s52, s74, s52\n"
        "s_max_i32 s52, s52, s79\n"
        "s_cmp_lt_u32 s40, 12\n"
        "s_cselect_b32 s52, s92, s52\n"
        "s_mul_i32 s52, s52, 137\n"
        "s_lshr_b32 s52, s52, 8\n"
        "s_add_u32 s72, s72, s52\n"
        "s_add_u32 s73, s50, s52\n"
        "s_sub_u32 s73, s73, s76\n"
        "s_sub_u32 s45, s72, s73\n"
        "s_max_i32 s45, s45, 0\n"
        "s_min_i32 s45, s45, 22\n"
        "s_add_u32 s73, s73, s45\n"
        "s_branch LTRH2_%=\n"
        "LSENDH3_%=:\n"
        "s_getpc_b64 s[64:65]\n"
        "s_branch LSEND_%=\n"
        "s_branch LSENTH3_%=\n"
        "LS_H3_%=:\n"
        "s_getpc_b64 s[64:65]\n"
        "s_branch LPOLL_%=\n"
        "s_branch LR_H3_%=\n"
        "LTQH3_%=:\n"
        "s_sleep 1\n"
        "LTPH3_%=:\n"
        "s_memrealtime s[50:51]\n"
        "s_waitcnt lgkmcnt(0)\n"
        "s_sub_u32 s45, s50, s73\n"
        "s_cmp_lt_i32 s45, 0\n"
        "s_cbranch_scc1 LTQH3_%=\n"
        "s_sub_i32 s52, s40, 16\n"
        "s_max_i32 s52, s52, 0\n"
        "s_mul_i32 s52, s52, s75\n"
        "s_sub_i32 s52, s74, s52\n"
        "s_max_i32 s52, s52, s79\n"
        "s_cmp_lt_u32 s40, 12\n"
        "s_cselect_b32 s52, s92, s52\n"
        "s_mul_i32 s52, s52, 137\n"
        "s_lshr_b32 s52, s52, 8\n"
        "s_add_u32 s72, s72, s52\n"
        "s_add_u32 s73, s50, s52\n"
        "s_sub_u32 s73, s73, s76\n"
        "s_sub_u32 s45, s72, s73\n"
        "s_max_i32 s45, s45, 0\n"
        "s_min_i32 s45, s45, 22\n"
        "s_add_u32 s73, s73, s45\n"
        "s_branch LTRH3_%=\n"
        "LCLP_%=:\n"
        "s_bitcmp1_b32 %[role], 4\n"
        "s_cbranch_scc0 LBACK_%=\n"
        "s_memtime s[56:57]\n"
        "s_memrealtime s[50:51]\n"
        "s_waitcnt lgkmcnt(0)\n"
        "s_sub_u32 s56, s56, s83\n"
        "s_sub_u32 s50, s50, s84\n"
        "v_mov_b32 v68, s50\n"
        "v_mov_b32 v69, s56\n"
        "v_mov_b32 v67, s86\n"
        "s_mov_b64 exec, 1\n"
        "buffer_atomic_add_x2 v[68:69], v67, %[rsA], 0 offen sc1\n"
        "s_bitcmp1_b32 %[role], 3\n"
        "s_cbranch_scc0 LCLQ_%=\n"
        "v_xor_b32 v67, 0x100, v67\n"
        "v_mov_b32 v68, 0\n"
        "v_mov_b32 v69, 0\n"
        "buffer_store_dwordx2 v[68:69], v67, %[rsA], 0 offen sc1\n"
        "LCLQ_%=:\n"
        "s_mov_b64 exec, -1\n"
        "s_branch LBACK_%=\n"
        "LCLS_%=:\n"
        "s_cmp_lg_u32 s85, 0\n"
        "s_cbranch_scc1 LBACK_%=\n"
        "v_readlane_b32 s50, v61, 0\n"
        "v_readlane_b32 s56, v61, 1\n"
        "s_nop 1\n"
        "s_cmp_lt_u32 s50, 2500\n"
        "s_cbranch_scc1 LBACK_%=\n"
        "s_mul_i32 s56, s56, 100\n"
        "s_mul_i32 s45, s50, 1795\n"
        "s_cmp_lt_u32 s56, s45\n"
        "s_cbranch_scc0 LCL0_%=\n"
        "s_mov_b32 s74, 254\n"
        "s_mov_b32 s75, 4\n"
        "s_branch LBACK_%=\n"
        "LCL0_%=:\n"
        "s_branch LBACK_%=\n"
        "LPOLL_%=:\n"
        "s_add_u32 s64, s64, 4\n"
        "s_addc_u32 s65, s65, 0\n"
        "LPOLL1_%=:\n"
        "v_mov_b32 v68, 0\n"
        "s_andn2_b64 exec, s[54:55], s[46:47]\n"
        "buffer_load_dword v68, %[cnt], %[rsA], 0 offen sc0 sc1\n"
        "s_mov_b64 exec, -1\n"
        "v_mov_b32 v67, 0x20000\n"
        "buffer_load_dword v67, v67, %[rsA], 0 offen sc1\n"
        "s_memrealtime s[50:51]\n"
        "s_waitcnt vmcnt(0) lgkmcnt(0)\n"
        "v_cmp_le_u32 s[56:57], s71, v68\n"
        "v_readfirstlane_b32 s45, v67\n"
        "s_nop 1\n"
        "s_or_b64 s[46:47], s[46:47], s[56:57]\n"
        "s_andn2_b64 s[56:57], s[54:55], s[46:47]\n"
        "s_cbranch_scc0 LPOLL3_%=\n"
        "s_cmp_eq_u32 s45, s70\n"
        "s_cbranch_scc1 LPOLL2_%=\n"
        "s_sub_u32 s50, s50, s48\n"
        "s_subb_u32 s51, s51, s49\n"
        "s_cmp_lg_u32 s51, 0\n"
        "s_cbranch_scc1 LPOLL2_%=\n"
        "s_cmp_gt_u32 s50, 50000000\n"
        "s_cbranch_scc1 LPOLL2_%=\n"
        "s_sleep 1\n"
        "s_branch LPOLL1_%=\n"
        "LPOLL2_%=:\n"
        "v_mov_b32 v68, 0x20000\n"
        "v_mov_b32 v67, s70\n"
        "buffer_store_dword v67, v68, %[rsA], 0 offen sc1\n"
        "s_mov_b64 s[46:47], -1\n"
        "s_mov_b32 s53, 1\n"
        "LPOLL3_%=:\n"
        "s_setpc_b64 s[64:65]\n"
        "LSEND_%=:\n"
        "s_add_u32 s64, s64, 4\n"
        "s_addc_u32 s65, s65, 0\n"
        "s_sub_u32 s45, s61, %[slim]\n"
        "s_add_u32 s61, s61, 3\n"
        "s_cmp_eq_u32 s45, 0\n"
        "s_cbranch_scc0 LEV1_%=\n"
        "v_add_u32 v60, %[soff], v64\n"
        "global_store_dwordx4 v60, v[28:31], %[dws] offset:0 sc0 sc1\n"
        "global_store_dwordx4 v60, v[32:35], %[dws] offset:1024 sc0 sc1\n"
        "v_add_u32 v60, 0x8000, v60\n"
        "global_store_dwordx4 v60, v[36:39], %[dws] offset:0 sc0 sc1\n"
        "global_store_dwordx4 v60, v[40:43], %[dws] offset:1024 sc0 sc1\n"
        "s_nop 1\n"
        "s_setpc_b64 s[64:65]\n"
        "LEV1_%=:\n"
        "s_waitcnt vmcnt(24)\n"
        "v_mov_b32 v68, %[scnt]\n"
        "v_mov_b32 v67, 2\n"
        "s_mov_b64 exec, 1\n"
        "global_atomic_add v68, v67, %[dws] sc1\n"
        "s_mov_b64 exec, -1\n"
        "s_mov_b32 s61, -1\n"
        "s_setpc_b64 s[64:65]\n"
        "LSWAP_%=:\n"
        "s_add_u32 s64, s64, 4\n"
        "s_addc_u32 s65, s65, 0\n"
        "s_cmp_eq_u32 s40, 1\n"
        "s_cbranch_scc1 LSWE_%=\n"
        "v_mov_b32 v68, 147456\n"
        "ds_read_b32 v67, v68\n"
        "s_waitcnt lgkmcnt(0)\n"
        "v_readfirstlane_b32 s56, v67\n"
        "s_nop 1\n"
        "s_bitcmp1_b32 s56, 31\n"
        "s_cselect_b32 s87, 0, s87\n"
        "s_cbranch_scc1 LSWE_%=\n"
        "s_or_b32 s46, s46, s56\n"
        "s_cmp_eq_u32 s56, 0x3fffff\n"
        "s_cselect_b32 s87, 0, s87\n"
        "s_bitcmp1_b32 s56, s66\n"
        "s_cbranch_scc1 LSWE_%=\n"
        "s_lshr_b32 s62, s56, s66\n"
        "s_cmp_eq_u32 s62, 0\n"
        "s_cbranch_scc1 LSWE_%=\n"
        "s_ff1_i32_b32 s62, s62\n"
        "s_add_u32 s62, s62, s66\n"
        "s_add_u32 s57, s66, 0\n"
        "s_add_u32 s45, s62, 0\n"
        "v_readlane_b32 s63, %[tabw], s57\n"
        "v_readlane_b32 s50, %[tabw], s45\n"
        "s_mov_b32 m0, s57\n"
        "s_nop 3\n"
        "v_writelane_b32 %[tabw], s50, m0\n"
        "s_mov_b32 m0, s45\n"
        "s_nop 0\n"
        "v_writelane_b32 %[tabw], s63, m0\n"
        "s_nop 0\n"
        "s_add_u32 s57, s66, 1\n"
        "s_add_u32 s45, s62, 1\n"
        "v_readlane_b32 s63, %[tabw], s57\n"
        "v_readlane_b32 s50, %[tabw], s45\n"
        "s_mov_b32 m0, s57\n"
        "s_nop 3\n"
        "v_writelane_b32 %[tabw], s50, m0\n"
        "s_mov_b32 m0, s45\n"
        "s_nop 0\n"
        "v_writelane_b32 %[tabw], s63, m0\n"
        "s_nop 0\n"
        "s_add_u32 s57, s66, 0\n"
        "s_add_u32 s45, s62, 0\n"
        "v_readlane_b32 s63, %[cnt], s57\n"
        "v_readlane_b32 s50, %[cnt], s45\n"
        "s_mov_b32 m0, s57\n"
        "s_nop 3\n"
        "v_writelane_b32 %[cnt], s50, m0\n"
        "s_mov_b32 m0, s45\n"
        "s_nop 0\n"
        "v_writelane_b32 %[cnt], s63, m0\n"
        "s_nop 0\n"
        "s_add_u32 s57, s66, 1\n"
        "s_add_u32 s45, s62, 1\n"
        "v_readlane_b32 s63, %[cnt], s57\n"
        "v_readlane_b32 s50, %[cnt], s45\n"
        "s_mov_b32 m0, s57\n"
        "s_nop 3\n"
        "v_writelane_b32 %[cnt], s50, m0\n"
        "s_mov_b32 m0, s45\n"
        "s_nop 0\n"
        "v_writelane_b32 %[cnt], s63, m0\n"
        "s_nop 0\n"
        "s_add_u32 s57, s66, -2\n"
        "s_add_u32 s45, s62, -2\n"
        "v_readlane_b32 s63, %[tablo], s57\n"
        "v_readlane_b32 s50, %[tablo], s45\n"
        "s_mov_b32 m0, s57\n"
        "s_nop 3\n"
        "v_writelane_b32 %[tablo], s50, m0\n"
        "s_mov_b32 m0, s45\n"
        "s_nop 0\n"
        "v_writelane_b32 %[tablo], s63, m0\n"
        "s_nop 0\n"
        "s_add_u32 s57, s66, -1\n"
        "s_add_u32 s45, s62, -1\n"
        "v_readlane_b32 s63, %[tablo], s57\n"
        "v_readlane_b32 s50, %[tablo], s45\n"
        "s_mov_b32 m0, s57\n"
        "s_nop 3\n"
        "v_writelane_b32 %[tablo], s50, m0\n"
        "s_mov_b32 m0, s45\n"
        "s_nop 0\n"
        "v_writelane_b32 %[tablo], s63, m0\n"
        "s_nop 0\n"
        "s_add_u32 s57, s66, -2\n"
        "s_add_u32 s45, s62, -2\n"
        "v_readlane_b32 s63, %[tabhi], s57\n"
        "v_readlane_b32 s50, %[tabhi], s45\n"
        "s_mov_b32 m0, s57\n"
        "s_nop 3\n"
        "v_writelane_b32 %[tabhi], s50, m0\n"
        "s_mov_b32 m0, s45\n"
        "s_nop 0\n"
        "v_writelane_b32 %[tabhi], s63, m0\n"
        "s_nop 0\n"
        "s_add_u32 s57, s66, -1\n"
        "s_add_u32 s45, s62, -1\n"
        "v_readlane_b32 s63, %[tabhi], s57\n"
        "v_readlane_b32 s50, %[tabhi], s45\n"
        "s_mov_b32 m0, s57\n"
        "s_nop 3\n"
        "v_writelane_b32 %[tabhi], s50, m0\n"
        "s_mov_b32 m0, s45\n"
        "s_nop 0\n"
        "v_writelane_b32 %[tabhi], s63, m0\n"
        "s_nop 0\n"
        "s_lshl_b32 s63, 3, s66\n"
        "s_lshl_b32 s50, 3, s62\n"
        "s_or_b32 s46, s46, s63\n"
        "s_andn2_b32 s46, s46, s50\n"
        "s_or_b32 s88, s88, s50\n"
        "s_mov_b32 s60, 0\n"
        "s_mov_b32 s58, 0x6000000\n"
        "s_mov_b64 exec, 16383\n"
        "v_mov_b32 %[tabpf], 0x6000000\n"
        "s_mov_b64 exec, -1\n"
        "LSWE_%=:\n"
        "s_setpc_b64 s[64:65]\n"
        "LSENDG0_%=:\n"
        "s_getpc_b64 s[64:65]\n"
        "s_branch LSEND_%=\n"
        "s_branch LSENTG0_%=\n"
        "LTQG0_%=:\n"
        "s_sleep 1\n"
        "LTPG0_%=:\n"
        "s_memrealtime s[50:51]\n"
        "s_waitcnt lgkmcnt(0)\n"
        "s_sub_u32 s45, s50, s73\n"
        "s_cmp_lt_i32 s45, 0\n"
        "s_cbranch_scc1 LTQG0_%=\n"
        "s_sub_i32 s52, s40, 16\n"
        "s_max_i32 s52, s52, 0\n"
        "s_mul_i32 s52, s52, s75\n"
        "s_sub_i32 s52, s74, s52\n"
        "s_max_i32 s52, s52, s79\n"
        "s_cmp_lt_u32 s40, 12\n"
        "s_cselect_b32 s52, s92, s52\n"
        "s_mul_i32 s52, s52, 137\n"
        "s_lshr_b32 s52, s52, 8\n"
        "s_add_u32 s72, s72, s52\n"
        "s_add_u32 s73, s50, s52\n"
        "s_sub_u32 s73, s73, s76\n"
        "s_sub_u32 s45, s72, s73\n"
        "s_max_i32 s45, s45, 0\n"
        "s_min_i32 s45, s45, 22\n"
        "s_add_u32 s73, s73, s45\n"
        "s_branch LTRG0_%=\n"
        "LSENDG1_%=:\n"
        "s_getpc_b64 s[64:65]\n"
        "s_branch LSEND_%=\n"
        "s_branch LSENTG1_%=\n"
        "LCWG_%=:\n"
        "s_cmp_eq_u32 s60, 0\n"
        "s_cselect_b64 s[88:89], s[88:89], 0\n"
        "s_or_b32 s88, s88, 3\n"
        "s_mov_b32 s60, 0\n"
        "s_add_u32 s45, s86, 4\n"
        "v_writelane_b32 %[cnt], s86, 0\n"
        "v_writelane_b32 %[cnt], s45, 1\n"
        "s_branch LCXG_%=\n"
        "LS_G1_%=:\n"
        "s_getpc_b64 s[64:65]\n"
        "s_branch LPOLL_%=\n"
        "s_branch LR_G1_%=\n"
        "LTQG1_%=:\n"
        "s_sleep 1\n"
        "LTPG1_%=:\n"
        "s_memrealtime s[50:51]\n"
        "s_waitcnt lgkmcnt(0)\n"
        "s_sub_u32 s45, s50, s73\n"
        "s_cmp_lt_i32 s45, 0\n"
        "s_cbranch_scc1 LTQG1_%=\n"
        "s_sub_i32 s52, s40, 16\n"
        "s_max_i32 s52, s52, 0\n"
        "s_mul_i32 s52, s52, s75\n"
        "s_sub_i32 s52, s74, s52\n"
        "s_max_i32 s52, s52, s79\n"
        "s_cmp_lt_u32 s40, 12\n"
        "s_cselect_b32 s52, s92, s52\n"
        "s_mul_i32 s52, s52, 137\n"
        "s_lshr_b32 s52, s52, 8\n"
        "s_add_u32 s72, s72, s52\n"
        "s_add_u32 s73, s50, s52\n"
        "s_sub_u32 s73, s73, s76\n"
        "s_sub_u32 s45, s72, s73\n"
        "s_max_i32 s45, s45, 0\n"
        "s_min_i32 s45, s45, 22\n"
        "s_add_u32 s73, s73, s45\n"
        "s_branch LTRG1_%=\n"
        "LSENDG2_%=:\n"
        "s_getpc_b64 s[64:65]\n"
        "s_branch LSEND_%=\n"
        "s_branch LSENTG2_%=\n"
        "LTQG2_%=:\n"
        "s_sleep 1\n"
        "LTPG2_%=:\n"
        "s_memrealtime s[50:51]\n"
        "s_waitcnt lgkmcnt(0)\n"
        "s_sub_u32 s45, s50, s73\n"
        "s_cmp_lt_i32 s45, 0\n"
        "s_cbranch_scc1 LTQG2_%=\n"
        "s_sub_i32 s52, s40, 16\n"
        "s_max_i32 s52, s52, 0\n"
        "s_mul_i32 s52, s52, s75\n"
        "s_sub_i32 s52, s74, s52\n"
        "s_max_i32 s52, s52, s79\n"
        "s_cmp_lt_u32 s40, 12\n"
        "s_cselect_b32 s52, s92, s52\n"
        "s_mul_i32 s52, s52, 137\n"
        "s_lshr_b32 s52, s52, 8\n"
        "s_add_u32 s72, s72, s52\n"
        "s_add_u32 s73, s50, s52\n"
        "s_sub_u32 s73, s73, s76\n"
        "s_sub_u32 s45, s72, s73\n"
        "s_max_i32 s45, s45, 0\n"
        "s_min_i32 s45, s45, 22\n"
        "s_add_u32 s73, s73, s45\n"
        "s_branch LTRG2_%=\n"
        "LSENDG3_%=:\n"
        "s_getpc_b64 s[64:65]\n"
        "s_branch LSEND_%=\n"
        "s_branch LSENTG3_%=\n"
        "LS_G3_%=:\n"
        "s_getpc_b64 s[64:65]\n"
        "s_branch LPOLL_%=\n"
        "s_branch LR_G3_%=\n"
        "LTQG3_%=:\n"
        "s_sleep 1\n"
        "LTPG3_%=:\n"
        "s_memrealtime s[50:51]\n"
        "s_waitcnt lgkmcnt(0)\n"
        "s_sub_u32 s45, s50, s73\n"
        "s_cmp_lt_i32 s45, 0\n"
        "s_cbranch_scc1 LTQG3_%=\n"
        "s_sub_i32 s52, s40, 16\n"
        "s_max_i32 s52, s52, 0\n"
        "s_mul_i32 s52, s52, s75\n"
        "s_sub_i32 s52, s74, s52\n"
        "s_max_i32 s52, s52, s79\n"
        "s_cmp_lt_u32 s40, 12\n"
        "s_cselect_b32 s52, s92, s52\n"
        "s_mul_i32 s52, s52, 137\n"
        "s_lshr_b32 s52, s52, 8\n"
        "s_add_u32 s72, s72, s52\n"
        "s_add_u32 s73, s50, s52\n"
        "s_sub_u32 s73, s73, s76\n"
        "s_sub_u32 s45, s72, s73\n"
        "s_max_i32 s45, s45, 0\n"
        "s_min_i32 s45, s45, 22\n"
        "s_add_u32 s73, s73, s45\n"
        "s_branch LTRG3_%=\n"
        "LEND_%=:\n"
        "v_add_u32 v60, 256, %[okeep]\n"
        "s_mov_b32 s46, 0xbdd2d3e8\n"
        "s_mov_b32 s47, s46\n"
        "v_mov_b32 v62, 0xc0135761\n"
        "v_mov_b32 v63, v62\n"
        "s_memrealtime s[48:49]\n"
        "s_mov_b32 s68, 3\n"
        "s_cmp_lg_u32 s53, 0\n"
        "s_cbranch_scc1 LXBAD_%=\n"
        "s_waitcnt vmcnt(0) lgkmcnt(0)\n"
        "v_accvgpr_read_b32 v192, a0\n"
        "v_accvgpr_read_b32 v193, a1\n"
        "v_accvgpr_read_b32 v194, a2\n"
        "v_accvgpr_read_b32 v195, a3\n"
        "v_accvgpr_read_b32 v196, a4\n"
        "v_accvgpr_read_b32 v197, a5\n"
        "v_accvgpr_read_b32 v198, a6\n"
        "v_accvgpr_read_b32 v199, a7\n"
        "v_accvgpr_read_b32 v200, a8\n"
        "v_accvgpr_read_b32 v201, a9\n"
        "v_accvgpr_read_b32 v202, a10\n"
        "v_accvgpr_read_b32 v203, a11\n"
        "v_accvgpr_read_b32 v204, a12\n"
        "v_accvgpr_read_b32 v205, a13\n"
        "v_accvgpr_read_b32 v206, a14\n"
        "v_accvgpr_read_b32 v207, a15\n"
        "s_cmp_gt_u32 s67, 0\n"
        "s_cbranch_scc1 LHAVE0_%=\n"
        "s_mov_b32 s52, 0\n"
        "s_getpc_b64 s[64:65]\n"
        "s_branch LFETCH_%=\n"
        "LHAVE0_%=:\n"
        "s_waitcnt vmcnt(24)\n"
        "s_cmp_gt_u32 s68, 2\n"
        "s_cbranch_scc1 LGO0_%=\n"
        "s_waitcnt vmcnt(16)\n"
        "s_cmp_gt_u32 s68, 1\n"
        "s_cbranch_scc1 LGO0_%=\n"
        "s_waitcnt vmcnt(8)\n"
        "s_cmp_gt_u32 s68, 0\n"
        "s_cbranch_scc1 LGO0_%=\n"
        "s_waitcnt vmcnt(0)\n"
        "LGO0_%=:\n"
        "v_accvgpr_read_b32 v208, a128\n"
        "v_accvgpr_read_b32 v209, a129\n"
        "v_accvgpr_read_b32 v210, a130\n"
        "v_accvgpr_read_b32 v211, a131\n"
        "v_accvgpr_read_b32 v212, a132\n"
        "v_accvgpr_read_b32 v213, a133\n"
        "v_accvgpr_read_b32 v214, a134\n"
        "v_accvgpr_read_b32 v215, a135\n"
        "v_accvgpr_read_b32 v216, a136\n"
        "v_accvgpr_read_b32 v217, a137\n"
        "v_accvgpr_read_b32 v218, a138\n"
        "v_accvgpr_read_b32 v219, a139\n"
        "v_accvgpr_read_b32 v220, a140\n"
        "v_accvgpr_read_b32 v221, a141\n"
        "v_accvgpr_read_b32 v222, a142\n"
        "v_accvgpr_read_b32 v223, a143\n"
        "v_pk_add_f32 v[192:193], v[192:193], v[208:209]\n"
        "v_pk_add_f32 v[194:195], v[194:195], v[210:211]\n"
        "v_pk_add_f32 v[196:197], v[196:197], v[212:213]\n"
        "v_pk_add_f32 v[198:199], v[198:199], v[214:215]\n"
        "v_pk_add_f32 v[200:201], v[200:201], v[216:217]\n"
        "v_pk_add_f32 v[202:203], v[202:203], v[218:219]\n"
        "v_pk_add_f32 v[204:205], v[204:205], v[220:221]\n"
        "v_pk_add_f32 v[206:207], v[206:207], v[222:223]\n"
        "v_pk_mul_f32 v[208:209], v[192:193], v[192:193]\n"
        "v_pk_mul_f32 v[210:211], v[194:195], v[194:195]\n"
        "v_pk_mul_f32 v[212:213], v[196:197], v[196:197]\n"
        "v_pk_mul_f32 v[214:215], v[198:199], v[198:199]\n"
        "v_pk_mul_f32 v[216:217], v[200:201], v[200:201]\n"
        "v_pk_mul_f32 v[218:219], v[202:203], v[202:203]\n"
        "v_pk_mul_f32 v[220:221], v[204:205], v[204:205]\n"
        "v_pk_mul_f32 v[222:223], v[206:207], v[206:207]\n"
        "v_pk_fma_f32 v[208:209], v[208:209], s[46:47], v[62:63]\n"
        "v_pk_fma_f32 v[210:211], v[210:211], s[46:47], v[62:63]\n"
        "v_pk_fma_f32 v[212:213], v[212:213], s[46:47], v[62:63]\n"
        "v_pk_fma_f32 v[214:215], v[214:215], s[46:47], v[62:63]\n"
        "v_pk_fma_f32 v[216:217], v[216:217], s[46:47], v[62:63]\n"
        "v_pk_fma_f32 v[218:219], v[218:219], s[46:47], v[62:63]\n"
        "v_pk_fma_f32 v[220:221], v[220:221], s[46:47], v[62:63]\n"
        "v_pk_fma_f32 v[222:223], v[222:223], s[46:47], v[62:63]\n"
        "v_pk_mul_f32 v[208:209], v[208:209], v[192:193]\n"
        "v_pk_mul_f32 v[210:211], v[210:211], v[194:195]\n"
        "v_pk_mul_f32 v[212:213], v[212:213], v[196:197]\n"
        "v_pk_mul_f32 v[214:215], v[214:215], v[198:199]\n"
        "v_pk_mul_f32 v[216:217], v[216:217], v[200:201]\n"
        "v_pk_mul_f32 v[218:219], v[218:219], v[202:203]\n"
        "v_pk_mul_f32 v[220:221], v[220:221], v[204:205]\n"
        "v_pk_mul_f32 v[222:223], v[222:223], v[206:207]\n"
        "v_exp_f32 v208, v208\n"
        "v_exp_f32 v209, v209\n"
        "v_exp_f32 v210, v210\n"
        "v_exp_f32 v211, v211\n"
        "v_exp_f32 v212, v212\n"
        "v_exp_f32 v213, v213\n"
        "v_exp_f32 v214, v214\n"
        "v_exp_f32 v215, v215\n"
        "v_exp_f32 v216, v216\n"
        "v_exp_f32 v217, v217\n"
        "v_exp_f32 v218, v218\n"
        "v_exp_f32 v219, v219\n"
        "v_exp_f32 v220, v220\n"
        "v_exp_f32 v221, v221\n"
        "v_exp_f32 v222, v222\n"
        "v_exp_f32 v223, v223\n"
        "s_nop 0\n"
        "v_pk_add_f32 v[208:209], v[208:209], 1.0 op_sel_hi:[1,0]\n"
        "v_pk_add_f32 v[210:211], v[210:211], 1.0 op_sel_hi:[1,0]\n"
        "v_pk_add_f32 v[212:213], v[212:213], 1.0 op_sel_hi:[1,0]\n"
        "v_pk_add_f32 v[214:215], v[214:215], 1.0 op_sel_hi:[1,0]\n"
        "v_pk_add_f32 v[216:217], v[216:217], 1.0 op_sel_hi:[1,0]\n"
        "v_pk_add_f32 v[218:219], v[218:219], 1.0 op_sel_hi:[1,0]\n"
        "v_pk_add_f32 v[220:221], v[220:221], 1.0 op_sel_hi:[1,0]\n"
        "v_pk_add_f32 v[222:223], v[222:223], 1.0 op_sel_hi:[1,0]\n"
        "v_rcp_f32 v208, v208\n"
        "v_rcp_f32 v209, v209\n"
        "v_rcp_f32 v210, v210\n"
        "v_rcp_f32 v211, v211\n"
        "v_rcp_f32 v212, v212\n"
        "v_rcp_f32 v213, v213\n"
        "v_rcp_f32 v214, v214\n"
        "v_rcp_f32 v215, v215\n"
        "v_rcp_f32 v216, v216\n"
        "v_rcp_f32 v217, v217\n"
        "v_rcp_f32 v218, v218\n"
        "v_rcp_f32 v219, v219\n"
        "v_rcp_f32 v220, v220\n"
        "v_rcp_f32 v221, v221\n"
        "v_rcp_f32 v222, v222\n"
        "v_rcp_f32 v223, v223\n"
        "s_nop 0\n"
        "v_mul_f32 v224, v192, v208\n"
        "v_mul_f32 v225, v196, v212\n"
        "v_mul_f32 v226, v200, v216\n"
        "v_mul_f32 v227, v204, v220\n"
        "s_mov_b32 s43, 0\n"
        "buffer_store_dwordx4 v[224:227], %[okeep], %[rsO], s43 offen sc1 nt\n"
        "v_mul_f32 v228, v193, v209\n"
        "v_mul_f32 v229, v197, v213\n"
        "v_mul_f32 v230, v201, v217\n"
        "v_mul_f32 v231, v205, v221\n"
        "s_mov_b32 s43, 65536\n"
        "buffer_store_dwordx4 v[228:231], %[okeep], %[rsO], s43 offen sc1 nt\n"
        "v_mul_f32 v232, v194, v210\n"
        "v_mul_f32 v233, v198, v214\n"
        "v_mul_f32 v234, v202, v218\n"
        "v_mul_f32 v235, v206, v222\n"
        "s_mov_b32 s43, 131072\n"
        "buffer_store_dwordx4 v[232:235], %[okeep], %[rsO], s43 offen sc1 nt\n"
        "v_mul_f32 v236, v195, v211\n"
        "v_mul_f32 v237, v199, v215\n"
        "v_mul_f32 v238, v203, v219\n"
        "v_mul_f32 v239, v207, v223\n"
        "s_mov_b32 s43, 196608\n"
        "buffer_store_dwordx4 v[236:239], %[okeep], %[rsO], s43 offen sc1 nt\n"
        "v_accvgpr_read_b32 v192, a16\n"
        "v_accvgpr_read_b32 v193, a17\n"
        "v_accvgpr_read_b32 v194, a18\n"
        "v_accvgpr_read_b32 v195, a19\n"
        "v_accvgpr_read_b32 v196, a20\n"
        "v_accvgpr_read_b32 v197, a21\n"
        "v_accvgpr_read_b32 v198, a22\n"
        "v_accvgpr_read_b32 v199, a23\n"
        "v_accvgpr_read_b32 v200, a24\n"
        "v_accvgpr_read_b32 v201, a25\n"
        "v_accvgpr_read_b32 v202, a26\n"
        "v_accvgpr_read_b32 v203, a27\n"
        "v_accvgpr_read_b32 v204, a28\n"
        "v_accvgpr_read_b32 v205, a29\n"
        "v_accvgpr_read_b32 v206, a30\n"
        "v_accvgpr_read_b32 v207, a31\n"
        "v_accvgpr_read_b32 v208, a144\n"
        "v_accvgpr_read_b32 v209, a145\n"
        "v_accvgpr_read_b32 v210, a146\n"
        "v_accvgpr_read_b32 v211, a147\n"
        "v_accvgpr_read_b32 v212, a148\n"
        "v_accvgpr_read_b32 v213, a149\n"
        "v_accvgpr_read_b32 v214, a150\n"
        "v_accvgpr_read_b32 v215, a151\n"
        "v_accvgpr_read_b32 v216, a152\n"
        "v_accvgpr_read_b32 v217, a153\n"
        "v_accvgpr_read_b32 v218, a154\n"
        "v_accvgpr_read_b32 v219, a155\n"
        "v_accvgpr_read_b32 v220, a156\n"
        "v_accvgpr_read_b32 v221, a157\n"
        "v_accvgpr_read_b32 v222, a158\n"
        "v_accvgpr_read_b32 v223, a159\n"
        "v_pk_add_f32 v[192:193], v[192:193], v[208:209]\n"
        "v_pk_add_f32 v[194:195], v[194:195], v[210:211]\n"
        "v_pk_add_f32 v[196:197], v[196:197], v[212:213]\n"
        "v_pk_add_f32 v[198:199], v[198:199], v[214:215]\n"
        "v_pk_add_f32 v[200:201], v[200:201], v[216:217]\n"
        "v_pk_add_f32 v[202:203], v[202:203], v[218:219]\n"
        "v_pk_add_f32 v[204:205], v[204:205], v[220:221]\n"
        "v_pk_add_f32 v[206:207], v[206:207], v[222:223]\n"
        "v_pk_mul_f32 v[208:209], v[192:193], v[192:193]\n"
        "v_pk_mul_f32 v[210:211], v[194:195], v[194:195]\n"
        "v_pk_mul_f32 v[212:213], v[196:197], v[196:197]\n"
        "v_pk_mul_f32 v[214:215], v[198:199], v[198:199]\n"
        "v_pk_mul_f32 v[216:217], v[200:201], v[200:201]\n"
        "v_pk_mul_f32 v[218:219], v[202:203], v[202:203]\n"
        "v_pk_mul_f32 v[220:221], v[204:205], v[204:205]\n"
        "v_pk_mul_f32 v[222:223], v[206:207], v[206:207]\n"
        "v_pk_fma_f32 v[208:209], v[208:209], s[46:47], v[62:63]\n"
        "v_pk_fma_f32 v[210:211], v[210:211], s[46:47], v[62:63]\n"
        "v_pk_fma_f32 v[212:213], v[212:213], s[46:47], v[62:63]\n"
        "v_pk_fma_f32 v[214:215], v[214:215], s[46:47], v[62:63]\n"
        "v_pk_fma_f32 v[216:217], v[216:217], s[46:47], v[62:63]\n"
        "v_pk_fma_f32 v[218:219], v[218:219], s[46:47], v[62:63]\n"
        "v_pk_fma_f32 v[220:221], v[220:221], s[46:47], v[62:63]\n"
        "v_pk_fma_f32 v[222:223], v[222:223], s[46:47], v[62:63]\n"
        "v_pk_mul_f32 v[208:209], v[208:209], v[192:193]\n"
        "v_pk_mul_f32 v[210:211], v[210:211], v[194:195]\n"
        "v_pk_mul_f32 v[212:213], v[212:213], v[196:197]\n"
        "v_pk_mul_f32 v[214:215], v[214:215], v[198:199]\n"
        "v_pk_mul_f32 v[216:217], v[216:217], v[200:201]\n"
        "v_pk_mul_f32 v[218:219], v[218:219], v[202:203]\n"
        "v_pk_mul_f32 v[220:221], v[220:221], v[204:205]\n"
        "v_pk_mul_f32 v[222:223], v[222:223], v[206:207]\n"
        "v_exp_f32 v208, v208\n"
        "v_exp_f32 v209, v209\n"
        "v_exp_f32 v210, v210\n"
        "v_exp_f32 v211, v211\n"
        "v_exp_f32 v212, v212\n"
        "v_exp_f32 v213, v213\n"
        "v_exp_f32 v214, v214\n"
        "v_exp_f32 v215, v215\n"
        "v_exp_f32 v216, v216\n"
        "v_exp_f32 v217, v217\n"
        "v_exp_f32 v218, v218\n"
        "v_exp_f32 v219, v219\n"
        "v_exp_f32 v220, v220\n"
        "v_exp_f32 v221, v221\n"
        "v_exp_f32 v222, v222\n"
        "v_exp_f32 v223, v223\n"
        "s_nop 0\n"
        "v_pk_add_f32 v[208:209], v[208:209], 1.0 op_sel_hi:[1,0]\n"
        "v_pk_add_f32 v[210:211], v[210:211], 1.0 op_sel_hi:[1,0]\n"
        "v_pk_add_f32 v[212:213], v[212:213], 1.0 op_sel_hi:[1,0]\n"
        "v_pk_add_f32 v[214:215], v[214:215], 1.0 op_sel_hi:[1,0]\n"
        "v_pk_add_f32 v[216:217], v[216:217], 1.0 op_sel_hi:[1,0]\n"
        "v_pk_add_f32 v[218:219], v[218:219], 1.0 op_sel_hi:[1,0]\n"
        "v_pk_add_f32 v[220:221], v[220:221], 1.0 op_sel_hi:[1,0]\n"
        "v_pk_add_f32 v[222:223], v[222:223], 1.0 op_sel_hi:[1,0]\n"
        "v_rcp_f32 v208, v208\n"
        "v_rcp_f32 v209, v209\n"
        "v_rcp_f32 v210, v210\n"
        "v_rcp_f32 v211, v211\n"
        "v_rcp_f32 v212, v212\n"
        "v_rcp_f32 v213, v213\n"
        "v_rcp_f32 v214, v214\n"
        "v_rcp_f32 v215, v215\n"
        "v_rcp_f32 v216, v216\n"
        "v_rcp_f32 v217, v217\n"
        "v_rcp_f32 v218, v218\n"
        "v_rcp_f32 v219, v219\n"
        "v_rcp_f32 v220, v220\n"
        "v_rcp_f32 v221, v221\n"
        "v_rcp_f32 v222, v222\n"
        "v_rcp_f32 v223, v223\n"
        "s_nop 0\n"
        "v_mul_f32 v240, v192, v208\n"
        "v_mul_f32 v241, v196, v212\n"
        "v_mul_f32 v242, v200, v216\n"
        "v_mul_f32 v243, v204, v220\n"
        "s_mov_b32 s43, 0\n"
        "buffer_store_dwordx4 v[240:243], v60, %[rsO], s43 offen sc1 nt\n"
        "v_mul_f32 v244, v193, v209\n"
        "v_mul_f32 v245, v197, v213\n"
        "v_mul_f32 v246, v201, v217\n"
        "v_mul_f32 v247, v205, v221\n"
        "s_mov_b32 s43, 65536\n"
        "buffer_store_dwordx4 v[244:247], v60, %[rsO], s43 offen sc1 nt\n"
        "v_mul_f32 v248, v194, v210\n"
        "v_mul_f32 v249, v198, v214\n"
        "v_mul_f32 v250, v202, v218\n"
        "v_mul_f32 v251, v206, v222\n"
        "s_mov_b32 s43, 131072\n"
        "buffer_store_dwordx4 v[248:251], v60, %[rsO], s43 offen sc1 nt\n"
        "v_mul_f32 v252, v195, v211\n"
        "v_mul_f32 v253, v199, v215\n"
        "v_mul_f32 v254, v203, v219\n"
        "v_mul_f32 v255, v207, v223\n"
        "s_mov_b32 s43, 196608\n"
        "buffer_store_dwordx4 v[252:255], v60, %[rsO], s43 offen sc1 nt\n"
        "v_accvgpr_read_b32 v192, a32\n"
        "v_accvgpr_read_b32 v193, a33\n"
        "v_accvgpr_read_b32 v194, a34\n"
        "v_accvgpr_read_b32 v195, a35\n"
        "v_accvgpr_read_b32 v196, a36\n"
        "v_accvgpr_read_b32 v197, a37\n"
        "v_accvgpr_read_b32 v198, a38\n"
        "v_accvgpr_read_b32 v199, a39\n"
        "v_accvgpr_read_b32 v200, a40\n"
        "v_accvgpr_read_b32 v201, a41\n"
        "v_accvgpr_read_b32 v202, a42\n"
        "v_accvgpr_read_b32 v203, a43\n"
        "v_accvgpr_read_b32 v204, a44\n"
        "v_accvgpr_read_b32 v205, a45\n"
        "v_accvgpr_read_b32 v206, a46\n"
        "v_accvgpr_read_b32 v207, a47\n"
        "s_cmp_gt_u32 s67, 1\n"
        "s_cbranch_scc1 LHAVE1_%=\n"
        "s_mov_b32 s52, 1\n"
        "s_getpc_b64 s[64:65]\n"
        "s_branch LFETCH_%=\n"
        "LHAVE1_%=:\n"
        "s_waitcnt vmcnt(24)\n"
        "s_cmp_gt_u32 s68, 2\n"
        "s_cbranch_scc1 LGO1_%=\n"
        "s_waitcnt vmcnt(16)\n"
        "s_cmp_gt_u32 s68, 1\n"
        "s_cbranch_scc1 LGO1_%=\n"
        "s_waitcnt vmcnt(8)\n"
        "s_cmp_gt_u32 s68, 0\n"
        "s_cbranch_scc1 LGO1_%=\n"
        "s_waitcnt vmcnt(0)\n"
        "LGO1_%=:\n"
        "v_accvgpr_read_b32 v208, a160\n"
        "v_accvgpr_read_b32 v209, a161\n"
        "v_accvgpr_read_b32 v210, a162\n"
        "v_accvgpr_read_b32 v211, a163\n"
        "v_accvgpr_read_b32 v212, a164\n"
        "v_accvgpr_read_b32 v213, a165\n"
        "v_accvgpr_read_b32 v214, a166\n"
        "v_accvgpr_read_b32 v215, a167\n"
        "v_accvgpr_read_b32 v216, a168\n"
        "v_accvgpr_read_b32 v217, a169\n"
        "v_accvgpr_read_b32 v218, a170\n"
        "v_accvgpr_read_b32 v219, a171\n"
        "v_accvgpr_read_b32 v220, a172\n"
        "v_accvgpr_read_b32 v221, a173\n"
        "v_accvgpr_read_b32 v222, a174\n"
        "v_accvgpr_read_b32 v223, a175\n"
        "v_pk_add_f32 v[192:193], v[192:193], v[208:209]\n"
        "v_pk_add_f32 v[194:195], v[194:195], v[210:211]\n"
        "v_pk_add_f32 v[196:197], v[196:197], v[212:213]\n"
        "v_pk_add_f32 v[198:199], v[198:199], v[214:215]\n"
        "v_pk_add_f32 v[200:201], v[200:201], v[216:217]\n"
        "v_pk_add_f32 v[202:203], v[202:203], v[218:219]\n"
        "v_pk_add_f32 v[204:205], v[204:205], v[220:221]\n"
        "v_pk_add_f32 v[206:207], v[206:207], v[222:223]\n"
        "v_pk_mul_f32 v[208:209], v[192:193], v[192:193]\n"
        "v_pk_mul_f32 v[210:211], v[194:195], v[194:195]\n"
        "v_pk_mul_f32 v[212:213], v[196:197], v[196:197]\n"
        "v_pk_mul_f32 v[214:215], v[198:199], v[198:199]\n"
        "v_pk_mul_f32 v[216:217], v[200:201], v[200:201]\n"
        "v_pk_mul_f32 v[218:219], v[202:203], v[202:203]\n"
        "v_pk_mul_f32 v[220:221], v[204:205], v[204:205]\n"
        "v_pk_mul_f32 v[222:223], v[206:207], v[206:207]\n"
        "v_pk_fma_f32 v[208:209], v[208:209], s[46:47], v[62:63]\n"
        "v_pk_fma_f32 v[210:211], v[210:211], s[46:47], v[62:63]\n"
        "v_pk_fma_f32 v[212:213], v[212:213], s[46:47], v[62:63]\n"
        "v_pk_fma_f32 v[214:215], v[214:215], s[46:47], v[62:63]\n"
        "v_pk_fma_f32 v[216:217], v[216:217], s[46:47], v[62:63]\n"
        "v_pk_fma_f32 v[218:219], v[218:219], s[46:47], v[62:63]\n"
        "v_pk_fma_f32 v[220:221], v[220:221], s[46:47], v[62:63]\n"
        "v_pk_fma_f32 v[222:223], v[222:223], s[46:47], v[62:63]\n"
        "v_pk_mul_f32 v[208:209], v[208:209], v[192:193]\n"
        "v_pk_mul_f32 v[210:211], v[210:211], v[194:195]\n"
        "v_pk_mul_f32 v[212:213], v[212:213], v[196:197]\n"
        "v_pk_mul_f32 v[214:215], v[214:215], v[198:199]\n"
        "v_pk_mul_f32 v[216:217], v[216:217], v[200:201]\n"
        "v_pk_mul_f32 v[218:219], v[218:219], v[202:203]\n"
        "v_pk_mul_f32 v[220:221], v[220:221], v[204:205]\n"
        "v_pk_mul_f32 v[222:223], v[222:223], v[206:207]\n"
        "v_exp_f32 v208, v208\n"
        "v_exp_f32 v209, v209\n"
        "v_exp_f32 v210, v210\n"
        "v_exp_f32 v211, v211\n"
        "v_exp_f32 v212, v212\n"
        "v_exp_f32 v213, v213\n"
        "v_exp_f32 v214, v214\n"
        "v_exp_f32 v215, v215\n"
        "v_exp_f32 v216, v216\n"
        "v_exp_f32 v217, v217\n"
        "v_exp_f32 v218, v218\n"
        "v_exp_f32 v219, v219\n"
        "v_exp_f32 v220, v220\n"
        "v_exp_f32 v221, v221\n"
        "v_exp_f32 v222, v222\n"
        "v_exp_f32 v223, v223\n"
        "s_nop 0\n"
        "v_pk_add_f32 v[208:209], v[208:209], 1.0 op_sel_hi:[1,0]\n"
        "v_pk_add_f32 v[210:211], v[210:211], 1.0 op_sel_hi:[1,0]\n"
        "v_pk_add_f32 v[212:213], v[212:213], 1.0 op_sel_hi:[1,0]\n"
        "v_pk_add_f32 v[214:215], v[214:215], 1.0 op_sel_hi:[1,0]\n"
        "v_pk_add_f32 v[216:217], v[216:217], 1.0 op_sel_hi:[1,0]\n"
        "v_pk_add_f32 v[218:219], v[218:219], 1.0 op_sel_hi:[1,0]\n"
        "v_pk_add_f32 v[220:221], v[220:221], 1.0 op_sel_hi:[1,0]\n"
        "v_pk_add_f32 v[222:223], v[222:223], 1.0 op_sel_hi:[1,0]\n"
        "v_rcp_f32 v208, v208\n"
        "v_rcp_f32 v209, v209\n"
        "v_rcp_f32 v210, v210\n"
        "v_rcp_f32 v211, v211\n"
        "v_rcp_f32 v212, v212\n"
        "v_rcp_f32 v213, v213\n"
        "v_rcp_f32 v214, v214\n"
        "v_rcp_f32 v215, v215\n"
        "v_rcp_f32 v216, v216\n"
        "v_rcp_f32 v217, v217\n"
        "v_rcp_f32 v218, v218\n"
        "v_rcp_f32 v219, v219\n"
        "v_rcp_f32 v220, v220\n"
        "v_rcp_f32 v221, v221\n"
        "v_rcp_f32 v222, v222\n"
        "v_rcp_f32 v223, v223\n"
        "s_nop 0\n"
        "v_mul_f32 v224, v192, v208\n"
        "v_mul_f32 v225, v196, v212\n"
        "v_mul_f32 v226, v200, v216\n"
        "v_mul_f32 v227, v204, v220\n"
        "s_mov_b32 s43, 1048576\n"
        "buffer_store_dwordx4 v[224:227], %[okeep], %[rsO], s43 offen sc1 nt\n"
        "v_mul_f32 v228, v193, v209\n"
        "v_mul_f32 v229, v197, v213\n"
        "v_mul_f32 v230, v201, v217\n"
        "v_mul_f32 v231, v205, v221\n"
        "s_mov_b32 s43, 1114112\n"
        "buffer_store_dwordx4 v[228:231], %[okeep], %[rsO], s43 offen sc1 nt\n"
        "v_mul_f32 v232, v194, v210\n"
        "v_mul_f32 v233, v198, v214\n"
        "v_mul_f32 v234, v202, v218\n"
        "v_mul_f32 v235, v206, v222\n"
        "s_mov_b32 s43, 1179648\n"
        "buffer_store_dwordx4 v[232:235], %[okeep], %[rsO], s43 offen sc1 nt\n"
        "v_mul_f32 v236, v195, v211\n"
        "v_mul_f32 v237, v199, v215\n"
        "v_mul_f32 v238, v203, v219\n"
        "v_mul_f32 v239, v207, v223\n"
        "s_mov_b32 s43, 1245184\n"
        "buffer_store_dwordx4 v[236:239], %[okeep], %[rsO], s43 offen sc1 nt\n"
        "v_accvgpr_read_b32 v192, a48\n"
        "v_accvgpr_read_b32 v193, a49\n"
        "v_accvgpr_read_b32 v194, a50\n"
        "v_accvgpr_read_b32 v195, a51\n"
        "v_accvgpr_read_b32 v196, a52\n"
        "v_accvgpr_read_b32 v197, a53\n"
        "v_accvgpr_read_b32 v198, a54\n"
        "v_accvgpr_read_b32 v199, a55\n"
        "v_accvgpr_read_b32 v200, a56\n"
        "v_accvgpr_read_b32 v201, a57\n"
        "v_accvgpr_read_b32 v202, a58\n"
        "v_accvgpr_read_b32 v203, a59\n"
        "v_accvgpr_read_b32 v204, a60\n"
        "v_accvgpr_read_b32 v205, a61\n"
        "v_accvgpr_read_b32 v206, a62\n"
        "v_accvgpr_read_b32 v207, a63\n"
        "v_accvgpr_read_b32 v208, a176\n"
        "v_accvgpr_read_b32 v209, a177\n"
        "v_accvgpr_read_b32 v210, a178\n"
        "v_accvgpr_read_b32 v211, a179\n"
        "v_accvgpr_read_b32 v212, a180\n"
        "v_accvgpr_read_b32 v213, a181\n"
        "v_accvgpr_read_b32 v214, a182\n"
        "v_accvgpr_read_b32 v215, a183\n"
        "v_accvgpr_read_b32 v216, a184\n"
        "v_accvgpr_read_b32 v217, a185\n"
        "v_accvgpr_read_b32 v218, a186\n"
        "v_accvgpr_read_b32 v219, a187\n"
        "v_accvgpr_read_b32 v220, a188\n"
        "v_accvgpr_read_b32 v221, a189\n"
        "v_accvgpr_read_b32 v222, a190\n"
        "v_accvgpr_read_b32 v223, a191\n"
        "v_pk_add_f32 v[192:193], v[192:193], v[208:209]\n"
        "v_pk_add_f32 v[194:195], v[194:195], v[210:211]\n"
        "v_pk_add_f32 v[196:197], v[196:197], v[212:213]\n"
        "v_pk_add_f32 v[198:199], v[198:199], v[214:215]\n"
        "v_pk_add_f32 v[200:201], v[200:201], v[216:217]\n"
        "v_pk_add_f32 v[202:203], v[202:203], v[218:219]\n"
        "v_pk_add_f32 v[204:205], v[204:205], v[220:221]\n"
        "v_pk_add_f32 v[206:207], v[206:207], v[222:223]\n"
        "v_pk_mul_f32 v[208:209], v[192:193], v[192:193]\n"
        "v_pk_mul_f32 v[210:211], v[194:195], v[194:195]\n"
        "v_pk_mul_f32 v[212:213], v[196:197], v[196:197]\n"
        "v_pk_mul_f32 v[214:215], v[198:199], v[198:199]\n"
        "v_pk_mul_f32 v[216:217], v[200:201], v[200:201]\n"
        "v_pk_mul_f32 v[218:219], v[202:203], v[202:203]\n"
        "v_pk_mul_f32 v[220:221], v[204:205], v[204:205]\n"
        "v_pk_mul_f32 v[222:223], v[206:207], v[206:207]\n"
        "v_pk_fma_f32 v[208:209], v[208:209], s[46:47], v[62:63]\n"
        "v_pk_fma_f32 v[210:211], v[210:211], s[46:47], v[62:63]\n"
        "v_pk_fma_f32 v[212:213], v[212:213], s[46:47], v[62:63]\n"
        "v_pk_fma_f32 v[214:215], v[214:215], s[46:47], v[62:63]\n"
        "v_pk_fma_f32 v[216:217], v[216:217], s[46:47], v[62:63]\n"
        "v_pk_fma_f32 v[218:219], v[218:219], s[46:47], v[62:63]\n"
        "v_pk_fma_f32 v[220:221], v[220:221], s[46:47], v[62:63]\n"
        "v_pk_fma_f32 v[222:223], v[222:223], s[46:47], v[62:63]\n"
        "v_pk_mul_f32 v[208:209], v[208:209], v[192:193]\n"
        "v_pk_mul_f32 v[210:211], v[210:211], v[194:195]\n"
        "v_pk_mul_f32 v[212:213], v[212:213], v[196:197]\n"
        "v_pk_mul_f32 v[214:215], v[214:215], v[198:199]\n"
        "v_pk_mul_f32 v[216:217], v[216:217], v[200:201]\n"
        "v_pk_mul_f32 v[218:219], v[218:219], v[202:203]\n"
        "v_pk_mul_f32 v[220:221], v[220:221], v[204:205]\n"
        "v_pk_mul_f32 v[222:223], v[222:223], v[206:207]\n"
        "v_exp_f32 v208, v208\n"
        "v_exp_f32 v209, v209\n"
        "v_exp_f32 v210, v210\n"
        "v_exp_f32 v211, v211\n"
        "v_exp_f32 v212, v212\n"
        "v_exp_f32 v213, v213\n"
        "v_exp_f32 v214, v214\n"
        "v_exp_f32 v215, v215\n"
        "v_exp_f32 v216, v216\n"
        "v_exp_f32 v217, v217\n"
        "v_exp_f32 v218, v218\n"
        "v_exp_f32 v219, v219\n"
        "v_exp_f32 v220, v220\n"
        "v_exp_f32 v221, v221\n"
        "v_exp_f32 v222, v222\n"
        "v_exp_f32 v223, v223\n"
        "s_nop 0\n"
        "v_pk_add_f32 v[208:209], v[208:209], 1.0 op_sel_hi:[1,0]\n"
        "v_pk_add_f32 v[210:211], v[210:211], 1.0 op_sel_hi:[1,0]\n"
        "v_pk_add_f32 v[212:213], v[212:213], 1.0 op_sel_hi:[1,0]\n"
        "v_pk_add_f32 v[214:215], v[214:215], 1.0 op_sel_hi:[1,0]\n"
        "v_pk_add_f32 v[216:217], v[216:217], 1.0 op_sel_hi:[1,0]\n"
        "v_pk_add_f32 v[218:219], v[218:219], 1.0 op_sel_hi:[1,0]\n"
        "v_pk_add_f32 v[220:221], v[220:221], 1.0 op_sel_hi:[1,0]\n"
        "v_pk_add_f32 v[222:223], v[222:223], 1.0 op_sel_hi:[1,0]\n"
        "v_rcp_f32 v208, v208\n"
        "v_rcp_f32 v209, v209\n"
        "v_rcp_f32 v210, v210\n"
        "v_rcp_f32 v211, v211\n"
        "v_rcp_f32 v212, v212\n"
        "v_rcp_f32 v213, v213\n"
        "v_rcp_f32 v214, v214\n"
        "v_rcp_f32 v215, v215\n"
        "v_rcp_f32 v216, v216\n"
        "v_rcp_f32 v217, v217\n"
        "v_rcp_f32 v218, v218\n"
        "v_rcp_f32 v219, v219\n"
        "v_rcp_f32 v220, v220\n"
        "v_rcp_f32 v221, v221\n"
        "v_rcp_f32 v222, v222\n"
        "v_rcp_f32 v223, v223\n"
        "s_nop 0\n"
        "v_mul_f32 v240, v192, v208\n"
        "v_mul_f32 v241, v196, v212\n"
        "v_mul_f32 v242, v200, v216\n"
        "v_mul_f32 v243, v204, v220\n"
        "s_mov_b32 s43, 1048576\n"
        "buffer_store_dwordx4 v[240:243], v60, %[rsO], s43 offen sc1 nt\n"
        "v_mul_f32 v244, v193, v209\n"
        "v_mul_f32 v245, v197, v213\n"
        "v_mul_f32 v246, v201, v217\n"
        "v_mul_f32 v247, v205, v221\n"
        "s_mov_b32 s43, 1114112\n"
        "buffer_store_dwordx4 v[244:247], v60, %[rsO], s43 offen sc1 nt\n"
        "v_mul_f32 v248, v194, v210\n"
        "v_mul_f32 v249, v198, v214\n"
        "v_mul_f32 v250, v202, v218\n"
        "v_mul_f32 v251, v206, v222\n"
        "s_mov_b32 s43, 1179648\n"
        "buffer_store_dwordx4 v[248:251], v60, %[rsO], s43 offen sc1 nt\n"
        "v_mul_f32 v252, v195, v211\n"
        "v_mul_f32 v253, v199, v215\n"
        "v_mul_f32 v254, v203, v219\n"
        "v_mul_f32 v255, v207, v223\n"
        "s_mov_b32 s43, 1245184\n"
        "buffer_store_dwordx4 v[252:255], v60, %[rsO], s43 offen sc1 nt\n"
        "v_accvgpr_read_b32 v192, a64\n"
        "v_accvgpr_read_b32 v193, a65\n"
        "v_accvgpr_read_b32 v194, a66\n"
        "v_accvgpr_read_b32 v195, a67\n"
        "v_accvgpr_read_b32 v196, a68\n"
        "v_accvgpr_read_b32 v197, a69\n"
        "v_accvgpr_read_b32 v198, a70\n"
        "v_accvgpr_read_b32 v199, a71\n"
        "v_accvgpr_read_b32 v200, a72\n"
        "v_accvgpr_read_b32 v201, a73\n"
        "v_accvgpr_read_b32 v202, a74\n"
        "v_accvgpr_read_b32 v203, a75\n"
        "v_accvgpr_read_b32 v204, a76\n"
        "v_accvgpr_read_b32 v205, a77\n"
        "v_accvgpr_read_b32 v206, a78\n"
        "v_accvgpr_read_b32 v207, a79\n"
        "s_cmp_gt_u32 s67, 2\n"
        "s_cbranch_scc1 LHAVE2_%=\n"
        "s_mov_b32 s52, 2\n"
        "s_getpc_b64 s[64:65]\n"
        "s_branch LFETCH_%=\n"
        "LHAVE2_%=:\n"
        "s_waitcnt vmcnt(24)\n"
        "s_cmp_gt_u32 s68, 2\n"
        "s_cbranch_scc1 LGO2_%=\n"
        "s_waitcnt vmcnt(16)\n"
        "s_cmp_gt_u32 s68, 1\n"
        "s_cbranch_scc1 LGO2_%=\n"
        "s_waitcnt vmcnt(8)\n"
        "s_cmp_gt_u32 s68, 0\n"
        "s_cbranch_scc1 LGO2_%=\n"
        "s_waitcnt vmcnt(0)\n"
        "LGO2_%=:\n"
        "v_accvgpr_read_b32 v208, a192\n"
        "v_accvgpr_read_b32 v209, a193\n"
        "v_accvgpr_read_b32 v210, a194\n"
        "v_accvgpr_read_b32 v211, a195\n"
        "v_accvgpr_read_b32 v212, a196\n"
        "v_accvgpr_read_b32 v213, a197\n"
        "v_accvgpr_read_b32 v214, a198\n"
        "v_accvgpr_read_b32 v215, a199\n"
        "v_accvgpr_read_b32 v216, a200\n"
        "v_accvgpr_read_b32 v217, a201\n"
        "v_accvgpr_read_b32 v218, a202\n"
        "v_accvgpr_read_b32 v219, a203\n"
        "v_accvgpr_read_b32 v220, a204\n"
        "v_accvgpr_read_b32 v221, a205\n"
        "v_accvgpr_read_b32 v222, a206\n"
        "v_accvgpr_read_b32 v223, a207\n"
        "v_pk_add_f32 v[192:193], v[192:193], v[208:209]\n"
        "v_pk_add_f32 v[194:195], v[194:195], v[210:211]\n"
        "v_pk_add_f32 v[196:197], v[196:197], v[212:213]\n"
        "v_pk_add_f32 v[198:199], v[198:199], v[214:215]\n"
        "v_pk_add_f32 v[200:201], v[200:201], v[216:217]\n"
        "v_pk_add_f32 v[202:203], v[202:203], v[218:219]\n"
        "v_pk_add_f32 v[204:205], v[204:205], v[220:221]\n"
        "v_pk_add_f32 v[206:207], v[206:207], v[222:223]\n"
        "v_pk_mul_f32 v[208:209], v[192:193], v[192:193]\n"
        "v_pk_mul_f32 v[210:211], v[194:195], v[194:195]\n"
        "v_pk_mul_f32 v[212:213], v[196:197], v[196:197]\n"
        "v_pk_mul_f32 v[214:215], v[198:199], v[198:199]\n"
        "v_pk_mul_f32 v[216:217], v[200:201], v[200:201]\n"
        "v_pk_mul_f32 v[218:219], v[202:203], v[202:203]\n"
        "v_pk_mul_f32 v[220:221], v[204:205], v[204:205]\n"
        "v_pk_mul_f32 v[222:223], v[206:207], v[206:207]\n"
        "v_pk_fma_f32 v[208:209], v[208:209], s[46:47], v[62:63]\n"
        "v_pk_fma_f32 v[210:211], v[210:211], s[46:47], v[62:63]\n"
        "v_pk_fma_f32 v[212:213], v[212:213], s[46:47], v[62:63]\n"
        "v_pk_fma_f32 v[214:215], v[214:215], s[46:47], v[62:63]\n"
        "v_pk_fma_f32 v[216:217], v[216:217], s[46:47], v[62:63]\n"
        "v_pk_fma_f32 v[218:219], v[218:219], s[46:47], v[62:63]\n"
        "v_pk_fma_f32 v[220:221], v[220:221], s[46:47], v[62:63]\n"
        "v_pk_fma_f32 v[222:223], v[222:223], s[46:47], v[62:63]\n"
        "v_pk_mul_f32 v[208:209], v[208:209], v[192:193]\n"
        "v_pk_mul_f32 v[210:211], v[210:211], v[194:195]\n"
        "v_pk_mul_f32 v[212:213], v[212:213], v[196:197]\n"
        "v_pk_mul_f32 v[214:215], v[214:215], v[198:199]\n"
        "v_pk_mul_f32 v[216:217], v[216:217], v[200:201]\n"
        "v_pk_mul_f32 v[218:219], v[218:219], v[202:203]\n"
        "v_pk_mul_f32 v[220:221], v[220:221], v[204:205]\n"
        "v_pk_mul_f32 v[222:223], v[222:223], v[206:207]\n"
        "v_exp_f32 v208, v208\n"
        "v_exp_f32 v209, v209\n"
        "v_exp_f32 v210, v210\n"
        "v_exp_f32 v211, v211\n"
        "v_exp_f32 v212, v212\n"
        "v_exp_f32 v213, v213\n"
        "v_exp_f32 v214, v214\n"
        "v_exp_f32 v215, v215\n"
        "v_exp_f32 v216, v216\n"
        "v_exp_f32 v217, v217\n"
        "v_exp_f32 v218, v218\n"
        "v_exp_f32 v219, v219\n"
        "v_exp_f32 v220, v220\n"
        "v_exp_f32 v221, v221\n"
        "v_exp_f32 v222, v222\n"
        "v_exp_f32 v223, v223\n"
        "s_nop 0\n"
        "v_pk_add_f32 v[208:209], v[208:209], 1.0 op_sel_hi:[1,0]\n"
        "v_pk_add_f32 v[210:211], v[210:211], 1.0 op_sel_hi:[1,0]\n"
        "v_pk_add_f32 v[212:213], v[212:213], 1.0 op_sel_hi:[1,0]\n"
        "v_pk_add_f32 v[214:215], v[214:215], 1.0 op_sel_hi:[1,0]\n"
        "v_pk_add_f32 v[216:217], v[216:217], 1.0 op_sel_hi:[1,0]\n"
        "v_pk_add_f32 v[218:219], v[218:219], 1.0 op_sel_hi:[1,0]\n"
        "v_pk_add_f32 v[220:221], v[220:221], 1.0 op_sel_hi:[1,0]\n"
        "v_pk_add_f32 v[222:223], v[222:223], 1.0 op_sel_hi:[1,0]\n"
        "v_rcp_f32 v208, v208\n"
        "v_rcp_f32 v209, v209\n"
        "v_rcp_f32 v210, v210\n"
        "v_rcp_f32 v211, v211\n"
        "v_rcp_f32 v212, v212\n"
        "v_rcp_f32 v213, v213\n"
        "v_rcp_f32 v214, v214\n"
        "v_rcp_f32 v215, v215\n"
        "v_rcp_f32 v216, v216\n"
        "v_rcp_f32 v217, v217\n"
        "v_rcp_f32 v218, v218\n"
        "v_rcp_f32 v219, v219\n"
        "v_rcp_f32 v220, v220\n"
        "v_rcp_f32 v221, v221\n"
        "v_rcp_f32 v222, v222\n"
        "v_rcp_f32 v223, v223\n"
        "s_nop 0\n"
        "v_mul_f32 v224, v192, v208\n"
        "v_mul_f32 v225, v196, v212\n"
        "v_mul_f32 v226, v200, v216\n"
        "v_mul_f32 v227, v204, v220\n"
        "s_mov_b32 s43, 2097152\n"
        "buffer_store_dwordx4 v[224:227], %[okeep], %[rsO], s43 offen sc1 nt\n"
        "v_mul_f32 v228, v193, v209\n"
        "v_mul_f32 v229, v197, v213\n"
        "v_mul_f32 v230, v201, v217\n"
        "v_mul_f32 v231, v205, v221\n"
        "s_mov_b32 s43, 2162688\n"
        "buffer_store_dwordx4 v[228:231], %[okeep], %[rsO], s43 offen sc1 nt\n"
        "v_mul_f32 v232, v194, v210\n"
        "v_mul_f32 v233, v198, v214\n"
        "v_mul_f32 v234, v202, v218\n"
        "v_mul_f32 v235, v206, v222\n"
        "s_mov_b32 s43, 2228224\n"
        "buffer_store_dwordx4 v[232:235], %[okeep], %[rsO], s43 offen sc1 nt\n"
        "v_mul_f32 v236, v195, v211\n"
        "v_mul_f32 v237, v199, v215\n"
        "v_mul_f32 v238, v203, v219\n"
        "v_mul_f32 v239, v207, v223\n"
        "s_mov_b32 s43, 2293760\n"
        "buffer_store_dwordx4 v[236:239], %[okeep], %[rsO], s43 offen sc1 nt\n"
        "v_accvgpr_read_b32 v192, a80\n"
        "v_accvgpr_read_b32 v193, a81\n"
        "v_accvgpr_read_b32 v194, a82\n"
        "v_accvgpr_read_b32 v195, a83\n"
        "v_accvgpr_read_b32 v196, a84\n"
        "v_accvgpr_read_b32 v197, a85\n"
        "v_accvgpr_read_b32 v198, a86\n"
        "v_accvgpr_read_b32 v199, a87\n"
        "v_accvgpr_read_b32 v200, a88\n"
        "v_accvgpr_read_b32 v201, a89\n"
        "v_accvgpr_read_b32 v202, a90\n"
        "v_accvgpr_read_b32 v203, a91\n"
        "v_accvgpr_read_b32 v204, a92\n"
        "v_accvgpr_read_b32 v205, a93\n"
        "v_accvgpr_read_b32 v206, a94\n"
        "v_accvgpr_read_b32 v207, a95\n"
        "v_accvgpr_read_b32 v208, a208\n"
        "v_accvgpr_read_b32 v209, a209\n"
        "v_accvgpr_read_b32 v210, a210\n"
        "v_accvgpr_read_b32 v211, a211\n"
        "v_accvgpr_read_b32 v212, a212\n"
        "v_accvgpr_read_b32 v213, a213\n"
        "v_accvgpr_read_b32 v214, a214\n"
        "v_accvgpr_read_b32 v215, a215\n"
        "v_accvgpr_read_b32 v216, a216\n"
        "v_accvgpr_read_b32 v217, a217\n"
        "v_accvgpr_read_b32 v218, a218\n"
        "v_accvgpr_read_b32 v219, a219\n"
        "v_accvgpr_read_b32 v220, a220\n"
        "v_accvgpr_read_b32 v221, a221\n"
        "v_accvgpr_read_b32 v222, a222\n"
        "v_accvgpr_read_b32 v223, a223\n"
        "v_pk_add_f32 v[192:193], v[192:193], v[208:209]\n"
        "v_pk_add_f32 v[194:195], v[194:195], v[210:211]\n"
        "v_pk_add_f32 v[196:197], v[196:197], v[212:213]\n"
        "v_pk_add_f32 v[198:199], v[198:199], v[214:215]\n"
        "v_pk_add_f32 v[200:201], v[200:201], v[216:217]\n"
        "v_pk_add_f32 v[202:203], v[202:203], v[218:219]\n"
        "v_pk_add_f32 v[204:205], v[204:205], v[220:221]\n"
        "v_pk_add_f32 v[206:207], v[206:207], v[222:223]\n"
        "v_pk_mul_f32 v[208:209], v[192:193], v[192:193]\n"
        "v_pk_mul_f32 v[210:211], v[194:195], v[194:195]\n"
        "v_pk_mul_f32 v[212:213], v[196:197], v[196:197]\n"
        "v_pk_mul_f32 v[214:215], v[198:199], v[198:199]\n"
        "v_pk_mul_f32 v[216:217], v[200:201], v[200:201]\n"
        "v_pk_mul_f32 v[218:219], v[202:203], v[202:203]\n"
        "v_pk_mul_f32 v[220:221], v[204:205], v[204:205]\n"
        "v_pk_mul_f32 v[222:223], v[206:207], v[206:207]\n"
        "v_pk_fma_f32 v[208:209], v[208:209], s[46:47], v[62:63]\n"
        "v_pk_fma_f32 v[210:211], v[210:211], s[46:47], v[62:63]\n"
        "v_pk_fma_f32 v[212:213], v[212:213], s[46:47], v[62:63]\n"
        "v_pk_fma_f32 v[214:215], v[214:215], s[46:47], v[62:63]\n"
        "v_pk_fma_f32 v[216:217], v[216:217], s[46:47], v[62:63]\n"
        "v_pk_fma_f32 v[218:219], v[218:219], s[46:47], v[62:63]\n"
        "v_pk_fma_f32 v[220:221], v[220:221], s[46:47], v[62:63]\n"
        "v_pk_fma_f32 v[222:223], v[222:223], s[46:47], v[62:63]\n"
        "v_pk_mul_f32 v[208:209], v[208:209], v[192:193]\n"
        "v_pk_mul_f32 v[210:211], v[210:211], v[194:195]\n"
        "v_pk_mul_f32 v[212:213], v[212:213], v[196:197]\n"
        "v_pk_mul_f32 v[214:215], v[214:215], v[198:199]\n"
        "v_pk_mul_f32 v[216:217], v[216:217], v[200:201]\n"
        "v_pk_mul_f32 v[218:219], v[218:219], v[202:203]\n"
        "v_pk_mul_f32 v[220:221], v[220:221], v[204:205]\n"
        "v_pk_mul_f32 v[222:223], v[222:223], v[206:207]\n"
        "v_exp_f32 v208, v208\n"
        "v_exp_f32 v209, v209\n"
        "v_exp_f32 v210, v210\n"
        "v_exp_f32 v211, v211\n"
        "v_exp_f32 v212, v212\n"
        "v_exp_f32 v213, v213\n"
        "v_exp_f32 v214, v214\n"
        "v_exp_f32 v215, v215\n"
        "v_exp_f32 v216, v216\n"
        "v_exp_f32 v217, v217\n"
        "v_exp_f32 v218, v218\n"
        "v_exp_f32 v219, v219\n"
        "v_exp_f32 v220, v220\n"
        "v_exp_f32 v221, v221\n"
        "v_exp_f32 v222, v222\n"
        "v_exp_f32 v223, v223\n"
        "s_nop 0\n"
        "v_pk_add_f32 v[208:209], v[208:209], 1.0 op_sel_hi:[1,0]\n"
        "v_pk_add_f32 v[210:211], v[210:211], 1.0 op_sel_hi:[1,0]\n"
        "v_pk_add_f32 v[212:213], v[212:213], 1.0 op_sel_hi:[1,0]\n"
        "v_pk_add_f32 v[214:215], v[214:215], 1.0 op_sel_hi:[1,0]\n"
        "v_pk_add_f32 v[216:217], v[216:217], 1.0 op_sel_hi:[1,0]\n"
        "v_pk_add_f32 v[218:219], v[218:219], 1.0 op_sel_hi:[1,0]\n"
        "v_pk_add_f32 v[220:221], v[220:221], 1.0 op_sel_hi:[1,0]\n"
        "v_pk_add_f32 v[222:223], v[222:223], 1.0 op_sel_hi:[1,0]\n"
        "v_rcp_f32 v208, v208\n"
        "v_rcp_f32 v209, v209\n"
        "v_rcp_f32 v210, v210\n"
        "v_rcp_f32 v211, v211\n"
        "v_rcp_f32 v212, v212\n"
        "v_rcp_f32 v213, v213\n"
        "v_rcp_f32 v214, v214\n"
        "v_rcp_f32 v215, v215\n"
        "v_rcp_f32 v216, v216\n"
        "v_rcp_f32 v217, v217\n"
        "v_rcp_f32 v218, v218\n"
        "v_rcp_f32 v219, v219\n"
        "v_rcp_f32 v220, v220\n"
        "v_rcp_f32 v221, v221\n"
        "v_rcp_f32 v222, v222\n"
        "v_rcp_f32 v223, v223\n"
        "s_nop 0\n"
        "v_mul_f32 v240, v192, v208\n"
        "v_mul_f32 v241, v196, v212\n"
        "v_mul_f32 v242, v200, v216\n"
        "v_mul_f32 v243, v204, v220\n"
        "s_mov_b32 s43, 2097152\n"
        "buffer_store_dwordx4 v[240:243], v60, %[rsO], s43 offen sc1 nt\n"
        "v_mul_f32 v244, v193, v209\n"
        "v_mul_f32 v245, v197, v213\n"
        "v_mul_f32 v246, v201, v217\n"
        "v_mul_f32 v247, v205, v221\n"
        "s_mov_b32 s43, 2162688\n"
        "buffer_store_dwordx4 v[244:247], v60, %[rsO], s43 offen sc1 nt\n"
        "v_mul_f32 v248, v194, v210\n"
        "v_mul_f32 v249, v198, v214\n"
        "v_mul_f32 v250, v202, v218\n"
        "v_mul_f32 v251, v206, v222\n"
        "s_mov_b32 s43, 2228224\n"
        "buffer_store_dwordx4 v[248:251], v60, %[rsO], s43 offen sc1 nt\n"
        "v_mul_f32 v252, v195, v211\n"
        "v_mul_f32 v253, v199, v215\n"
        "v_mul_f32 v254, v203, v219\n"
        "v_mul_f32 v255, v207, v223\n"
        "s_mov_b32 s43, 2293760\n"
        "buffer_store_dwordx4 v[252:255], v60, %[rsO], s43 offen sc1 nt\n"
        "v_accvgpr_read_b32 v192, a96\n"
        "v_accvgpr_read_b32 v193, a97\n"
        "v_accvgpr_read_b32 v194, a98\n"
        "v_accvgpr_read_b32 v195, a99\n"
        "v_accvgpr_read_b32 v196, a100\n"
        "v_accvgpr_read_b32 v197, a101\n"
        "v_accvgpr_read_b32 v198, a102\n"
        "v_accvgpr_read_b32 v199, a103\n"
        "v_accvgpr_read_b32 v200, a104\n"
        "v_accvgpr_read_b32 v201, a105\n"
        "v_accvgpr_read_b32 v202, a106\n"
        "v_accvgpr_read_b32 v203, a107\n"
        "v_accvgpr_read_b32 v204, a108\n"
        "v_accvgpr_read_b32 v205, a109\n"
        "v_accvgpr_read_b32 v206, a110\n"
        "v_accvgpr_read_b32 v207, a111\n"
        "s_cmp_gt_u32 s67, 3\n"
        "s_cbranch_scc1 LHAVE3_%=\n"
        "s_mov_b32 s52, 3\n"
        "s_getpc_b64 s[64:65]\n"
        "s_branch LFETCH_%=\n"
        "LHAVE3_%=:\n"
        "s_waitcnt vmcnt(24)\n"
        "s_cmp_gt_u32 s68, 2\n"
        "s_cbranch_scc1 LGO3_%=\n"
        "s_waitcnt vmcnt(16)\n"
        "s_cmp_gt_u32 s68, 1\n"
        "s_cbranch_scc1 LGO3_%=\n"
        "s_waitcnt vmcnt(8)\n"
        "s_cmp_gt_u32 s68, 0\n"
        "s_cbranch_scc1 LGO3_%=\n"
        "s_waitcnt vmcnt(0)\n"
        "LGO3_%=:\n"
        "v_accvgpr_read_b32 v208, a224\n"
        "v_accvgpr_read_b32 v209, a225\n"
        "v_accvgpr_read_b32 v210, a226\n"
        "v_accvgpr_read_b32 v211, a227\n"
        "v_accvgpr_read_b32 v212, a228\n"
        "v_accvgpr_read_b32 v213, a229\n"
        "v_accvgpr_read_b32 v214, a230\n"
        "v_accvgpr_read_b32 v215, a231\n"
        "v_accvgpr_read_b32 v216, a232\n"
        "v_accvgpr_read_b32 v217, a233\n"
        "v_accvgpr_read_b32 v218, a234\n"
        "v_accvgpr_read_b32 v219, a235\n"
        "v_accvgpr_read_b32 v220, a236\n"
        "v_accvgpr_read_b32 v221, a237\n"
        "v_accvgpr_read_b32 v222, a238\n"
        "v_accvgpr_read_b32 v223, a239\n"
        "v_pk_add_f32 v[192:193], v[192:193], v[208:209]\n"
        "v_pk_add_f32 v[194:195], v[194:195], v[210:211]\n"
        "v_pk_add_f32 v[196:197], v[196:197], v[212:213]\n"
        "v_pk_add_f32 v[198:199], v[198:199], v[214:215]\n"
        "v_pk_add_f32 v[200:201], v[200:201], v[216:217]\n"
        "v_pk_add_f32 v[202:203], v[202:203], v[218:219]\n"
        "v_pk_add_f32 v[204:205], v[204:205], v[220:221]\n"
        "v_pk_add_f32 v[206:207], v[206:207], v[222:223]\n"
        "v_pk_mul_f32 v[208:209], v[192:193], v[192:193]\n"
        "v_pk_mul_f32 v[210:211], v[194:195], v[194:195]\n"
        "v_pk_mul_f32 v[212:213], v[196:197], v[196:197]\n"
        "v_pk_mul_f32 v[214:215], v[198:199], v[198:199]\n"
        "v_pk_mul_f32 v[216:217], v[200:201], v[200:201]\n"
        "v_pk_mul_f32 v[218:219], v[202:203], v[202:203]\n"
        "v_pk_mul_f32 v[220:221], v[204:205], v[204:205]\n"
        "v_pk_mul_f32 v[222:223], v[206:207], v[206:207]\n"
        "v_pk_fma_f32 v[208:209], v[208:209], s[46:47], v[62:63]\n"
        "v_pk_fma_f32 v[210:211], v[210:211], s[46:47], v[62:63]\n"
        "v_pk_fma_f32 v[212:213], v[212:213], s[46:47], v[62:63]\n"
        "v_pk_fma_f32 v[214:215], v[214:215], s[46:47], v[62:63]\n"
        "v_pk_fma_f32 v[216:217], v[216:217], s[46:47], v[62:63]\n"
        "v_pk_fma_f32 v[218:219], v[218:219], s[46:47], v[62:63]\n"
        "v_pk_fma_f32 v[220:221], v[220:221], s[46:47], v[62:63]\n"
        "v_pk_fma_f32 v[222:223], v[222:223], s[46:47], v[62:63]\n"
        "v_pk_mul_f32 v[208:209], v[208:209], v[192:193]\n"
        "v_pk_mul_f32 v[210:211], v[210:211], v[194:195]\n"
        "v_pk_mul_f32 v[212:213], v[212:213], v[196:197]\n"
        "v_pk_mul_f32 v[214:215], v[214:215], v[198:199]\n"
        "v_pk_mul_f32 v[216:217], v[216:217], v[200:201]\n"
        "v_pk_mul_f32 v[218:219], v[218:219], v[202:203]\n"
        "v_pk_mul_f32 v[220:221], v[220:221], v[204:205]\n"
        "v_pk_mul_f32 v[222:223], v[222:223], v[206:207]\n"
        "v_exp_f32 v208, v208\n"
        "v_exp_f32 v209, v209\n"
        "v_exp_f32 v210, v210\n"
        "v_exp_f32 v211, v211\n"
        "v_exp_f32 v212, v212\n"
        "v_exp_f32 v213, v213\n"
        "v_exp_f32 v214, v214\n"
        "v_exp_f32 v215, v215\n"
        "v_exp_f32 v216, v216\n"
        "v_exp_f32 v217, v217\n"
        "v_exp_f32 v218, v218\n"
        "v_exp_f32 v219, v219\n"
        "v_exp_f32 v220, v220\n"
        "v_exp_f32 v221, v221\n"
        "v_exp_f32 v222, v222\n"
        "v_exp_f32 v223, v223\n"
        "s_nop 0\n"
        "v_pk_add_f32 v[208:209], v[208:209], 1.0 op_sel_hi:[1,0]\n"
        "v_pk_add_f32 v[210:211], v[210:211], 1.0 op_sel_hi:[1,0]\n"
        "v_pk_add_f32 v[212:213], v[212:213], 1.0 op_sel_hi:[1,0]\n"
        "v_pk_add_f32 v[214:215], v[214:215], 1.0 op_sel_hi:[1,0]\n"
        "v_pk_add_f32 v[216:217], v[216:217], 1.0 op_sel_hi:[1,0]\n"
        "v_pk_add_f32 v[218:219], v[218:219], 1.0 op_sel_hi:[1,0]\n"
        "v_pk_add_f32 v[220:221], v[220:221], 1.0 op_sel_hi:[1,0]\n"
        "v_pk_add_f32 v[222:223], v[222:223], 1.0 op_sel_hi:[1,0]\n"
        "v_rcp_f32 v208, v208\n"
        "v_rcp_f32 v209, v209\n"
        "v_rcp_f32 v210, v210\n"
        "v_rcp_f32 v211, v211\n"
        "v_rcp_f32 v212, v212\n"
        "v_rcp_f32 v213, v213\n"
        "v_rcp_f32 v214, v214\n"
        "v_rcp_f32 v215, v215\n"
        "v_rcp_f32 v216, v216\n"
        "v_rcp_f32 v217, v217\n"
        "v_rcp_f32 v218, v218\n"
        "v_rcp_f32 v219, v219\n"
        "v_rcp_f32 v220, v220\n"
        "v_rcp_f32 v221, v221\n"
        "v_rcp_f32 v222, v222\n"
        "v_rcp_f32 v223, v223\n"
        "s_nop 0\n"
        "v_mul_f32 v224, v192, v208\n"
        "v_mul_f32 v225, v196, v212\n"
        "v_mul_f32 v226, v200, v216\n"
        "v_mul_f32 v227, v204, v220\n"
        "s_mov_b32 s43, 3145728\n"
        "buffer_store_dwordx4 v[224:227], %[okeep], %[rsO], s43 offen sc1 nt\n"
        "v_mul_f32 v228, v193, v209\n"
        "v_mul_f32 v229, v197, v213\n"
        "v_mul_f32 v230, v201, v217\n"
        "v_mul_f32 v231, v205, v221\n"
        "s_mov_b32 s43, 3211264\n"
        "buffer_store_dwordx4 v[228:231], %[okeep], %[rsO], s43 offen sc1 nt\n"
        "v_mul_f32 v232, v194, v210\n"
        "v_mul_f32 v233, v198, v214\n"
        "v_mul_f32 v234, v202, v218\n"
        "v_mul_f32 v235, v206, v222\n"
        "s_mov_b32 s43, 3276800\n"
        "buffer_store_dwordx4 v[232:235], %[okeep], %[rsO], s43 offen sc1 nt\n"
        "v_mul_f32 v236, v195, v211\n"
        "v_mul_f32 v237, v199, v215\n"
        "v_mul_f32 v238, v203, v219\n"
        "v_mul_f32 v239, v207, v223\n"
        "s_mov_b32 s43, 3342336\n"
        "buffer_store_dwordx4 v[236:239], %[okeep], %[rsO], s43 offen sc1 nt\n"
        "v_accvgpr_read_b32 v192, a112\n"
        "v_accvgpr_read_b32 v193, a113\n"
        "v_accvgpr_read_b32 v194, a114\n"
        "v_accvgpr_read_b32 v195, a115\n"
        "v_accvgpr_read_b32 v196, a116\n"
        "v_accvgpr_read_b32 v197, a117\n"
        "v_accvgpr_read_b32 v198, a118\n"
        "v_accvgpr_read_b32 v199, a119\n"
        "v_accvgpr_read_b32 v200, a120\n"
        "v_accvgpr_read_b32 v201, a121\n"
        "v_accvgpr_read_b32 v202, a122\n"
        "v_accvgpr_read_b32 v203, a123\n"
        "v_accvgpr_read_b32 v204, a124\n"
        "v_accvgpr_read_b32 v205, a125\n"
        "v_accvgpr_read_b32 v206, a126\n"
        "v_accvgpr_read_b32 v207, a127\n"
        "v_accvgpr_read_b32 v208, a240\n"
        "v_accvgpr_read_b32 v209, a241\n"
        "v_accvgpr_read_b32 v210, a242\n"
        "v_accvgpr_read_b32 v211, a243\n"
        "v_accvgpr_read_b32 v212, a244\n"
        "v_accvgpr_read_b32 v213, a245\n"
        "v_accvgpr_read_b32 v214, a246\n"
        "v_accvgpr_read_b32 v215, a247\n"
        "v_accvgpr_read_b32 v216, a248\n"
        "v_accvgpr_read_b32 v217, a249\n"
        "v_accvgpr_read_b32 v218, a250\n"
        "v_accvgpr_read_b32 v219, a251\n"
        "v_accvgpr_read_b32 v220, a252\n"
        "v_accvgpr_read_b32 v221, a253\n"
        "v_accvgpr_read_b32 v222, a254\n"
        "v_accvgpr_read_b32 v223, a255\n"
        "v_pk_add_f32 v[192:193], v[192:193], v[208:209]\n"
        "v_pk_add_f32 v[194:195], v[194:195], v[210:211]\n"
        "v_pk_add_f32 v[196:197], v[196:197], v[212:213]\n"
        "v_pk_add_f32 v[198:199], v[198:199], v[214:215]\n"
        "v_pk_add_f32 v[200:201], v[200:201], v[216:217]\n"
        "v_pk_add_f32 v[202:203], v[202:203], v[218:219]\n"
        "v_pk_add_f32 v[204:205], v[204:205], v[220:221]\n"
        "v_pk_add_f32 v[206:207], v[206:207], v[222:223]\n"
        "v_pk_mul_f32 v[208:209], v[192:193], v[192:193]\n"
        "v_pk_mul_f32 v[210:211], v[194:195], v[194:195]\n"
        "v_pk_mul_f32 v[212:213], v[196:197], v[196:197]\n"
        "v_pk_mul_f32 v[214:215], v[198:199], v[198:199]\n"
        "v_pk_mul_f32 v[216:217], v[200:201], v[200:201]\n"
        "v_pk_mul_f32 v[218:219], v[202:203], v[202:203]\n"
        "v_pk_mul_f32 v[220:221], v[204:205], v[204:205]\n"
        "v_pk_mul_f32 v[222:223], v[206:207], v[206:207]\n"
        "v_pk_fma_f32 v[208:209], v[208:209], s[46:47], v[62:63]\n"
        "v_pk_fma_f32 v[210:211], v[210:211], s[46:47], v[62:63]\n"
        "v_pk_fma_f32 v[212:213], v[212:213], s[46:47], v[62:63]\n"
        "v_pk_fma_f32 v[214:215], v[214:215], s[46:47], v[62:63]\n"
        "v_pk_fma_f32 v[216:217], v[216:217], s[46:47], v[62:63]\n"
        "v_pk_fma_f32 v[218:219], v[218:219], s[46:47], v[62:63]\n"
        "v_pk_fma_f32 v[220:221], v[220:221], s[46:47], v[62:63]\n"
        "v_pk_fma_f32 v[222:223], v[222:223], s[46:47], v[62:63]\n"
        "v_pk_mul_f32 v[208:209], v[208:209], v[192:193]\n"
        "v_pk_mul_f32 v[210:211], v[210:211], v[194:195]\n"
        "v_pk_mul_f32 v[212:213], v[212:213], v[196:197]\n"
        "v_pk_mul_f32 v[214:215], v[214:215], v[198:199]\n"
        "v_pk_mul_f32 v[216:217], v[216:217], v[200:201]\n"
        "v_pk_mul_f32 v[218:219], v[218:219], v[202:203]\n"
        "v_pk_mul_f32 v[220:221], v[220:221], v[204:205]\n"
        "v_pk_mul_f32 v[222:223], v[222:223], v[206:207]\n"
        "v_exp_f32 v208, v208\n"
        "v_exp_f32 v209, v209\n"
        "v_exp_f32 v210, v210\n"
        "v_exp_f32 v211, v211\n"
        "v_exp_f32 v212, v212\n"
        "v_exp_f32 v213, v213\n"
        "v_exp_f32 v214, v214\n"
        "v_exp_f32 v215, v215\n"
        "v_exp_f32 v216, v216\n"
        "v_exp_f32 v217, v217\n"
        "v_exp_f32 v218, v218\n"
        "v_exp_f32 v219, v219\n"
        "v_exp_f32 v220, v220\n"
        "v_exp_f32 v221, v221\n"
        "v_exp_f32 v222, v222\n"
        "v_exp_f32 v223, v223\n"
        "s_nop 0\n"
        "v_pk_add_f32 v[208:209], v[208:209], 1.0 op_sel_hi:[1,0]\n"
        "v_pk_add_f32 v[210:211], v[210:211], 1.0 op_sel_hi:[1,0]\n"
        "v_pk_add_f32 v[212:213], v[212:213], 1.0 op_sel_hi:[1,0]\n"
        "v_pk_add_f32 v[214:215], v[214:215], 1.0 op_sel_hi:[1,0]\n"
        "v_pk_add_f32 v[216:217], v[216:217], 1.0 op_sel_hi:[1,0]\n"
        "v_pk_add_f32 v[218:219], v[218:219], 1.0 op_sel_hi:[1,0]\n"
        "v_pk_add_f32 v[220:221], v[220:221], 1.0 op_sel_hi:[1,0]\n"
        "v_pk_add_f32 v[222:223], v[222:223], 1.0 op_sel_hi:[1,0]\n"
        "v_rcp_f32 v208, v208\n"
        "v_rcp_f32 v209, v209\n"
        "v_rcp_f32 v210, v210\n"
        "v_rcp_f32 v211, v211\n"
        "v_rcp_f32 v212, v212\n"
        "v_rcp_f32 v213, v213\n"
        "v_rcp_f32 v214, v214\n"
        "v_rcp_f32 v215, v215\n"
        "v_rcp_f32 v216, v216\n"
        "v_rcp_f32 v217, v217\n"
        "v_rcp_f32 v218, v218\n"
        "v_rcp_f32 v219, v219\n"
        "v_rcp_f32 v220, v220\n"
        "v_rcp_f32 v221, v221\n"
        "v_rcp_f32 v222, v222\n"
        "v_rcp_f32 v223, v223\n"
        "s_nop 0\n"
        "v_mul_f32 v240, v192, v208\n"
        "v_mul_f32 v241, v196, v212\n"
        "v_mul_f32 v242, v200, v216\n"
        "v_mul_f32 v243, v204, v220\n"
        "s_mov_b32 s43, 3145728\n"
        "buffer_store_dwordx4 v[240:243], v60, %[rsO], s43 offen sc1 nt\n"
        "v_mul_f32 v244, v193, v209\n"
        "v_mul_f32 v245, v197, v213\n"
        "v_mul_f32 v246, v201, v217\n"
        "v_mul_f32 v247, v205, v221\n"
        "s_mov_b32 s43, 3211264\n"
        "buffer_store_dwordx4 v[244:247], v60, %[rsO], s43 offen sc1 nt\n"
        "v_mul_f32 v248, v194, v210\n"
        "v_mul_f32 v249, v198, v214\n"
        "v_mul_f32 v250, v202, v218\n"
        "v_mul_f32 v251, v206, v222\n"
        "s_mov_b32 s43, 3276800\n"
        "buffer_store_dwordx4 v[248:251], v60, %[rsO], s43 offen sc1 nt\n"
        "v_mul_f32 v252, v195, v211\n"
        "v_mul_f32 v253, v199, v215\n"
        "v_mul_f32 v254, v203, v219\n"
        "v_mul_f32 v255, v207, v223\n"
        "s_mov_b32 s43, 3342336\n"
        "buffer_store_dwordx4 v[252:255], v60, %[rsO], s43 offen sc1 nt\n"
        "s_branch LXEND_%=\n"
        "LFETCH_%=:\n"
        "s_add_u32 s64, s64, 4\n"
        "s_addc_u32 s65, s65, 0\n"
        "LXP_%=:\n"
        "v_mov_b32 v68, %[mflag]\n"
        "v_mov_b32 v67, 0x20000\n"
        "buffer_load_dword v68, v68, %[rsA], 0 offen sc0 sc1\n"
        "buffer_load_dword v67, v67, %[rsA], 0 offen sc1\n"
        "s_memrealtime s[50:51]\n"
        "s_waitcnt vmcnt(0) lgkmcnt(0)\n"
        "v_readfirstlane_b32 s69, v68\n"
        "v_readfirstlane_b32 s45, v67\n"
        "s_nop 1\n"
        "s_cmp_eq_u32 s45, s70\n"
        "s_cbranch_scc1 LXBAD_%=\n"
        "s_max_u32 s69, s69, s66\n"
        "s_sub_u32 s69, s69, s66\n"
        "s_min_u32 s69, s69, 4\n"
        "s_cmp_gt_u32 s69, s52\n"
        "s_cbranch_scc1 LXOK_%=\n"
        "s_cmp_eq_u32 s45, s70\n"
        "s_cbranch_scc1 LXBAD_%=\n"
        "s_sub_u32 s50, s50, s48\n"
        "s_subb_u32 s51, s51, s49\n"
        "s_cmp_lg_u32 s51, 0\n"
        "s_cbranch_scc1 LXBAD_%=\n"
        "s_cmp_gt_u32 s50, 50000000\n"
        "s_cbranch_scc1 LXBAD_%=\n"
        "s_sleep 1\n"
        "s_branch LXP_%=\n"
        "LXOK_%=:\n"
        "s_cmp_gt_u32 s67, 0\n"
        "s_cbranch_scc1 LSK0_%=\n"
        "s_cmp_le_u32 s69, 0\n"
        "s_cbranch_scc1 LSK0_%=\n"
        "s_add_u32 s43, %[xrecv], 0\n"
        "buffer_load_dwordx4 a[128:131], v61, %[rsA], s43 offen offset:0 sc0 sc1\n"
        "buffer_load_dwordx4 a[132:135], v61, %[rsA], s43 offen offset:1024 sc0 sc1\n"
        "buffer_load_dwordx4 a[136:139], v61, %[rsA], s43 offen offset:2048 sc0 sc1\n"
        "buffer_load_dwordx4 a[140:143], v61, %[rsA], s43 offen offset:3072 sc0 sc1\n"
        "s_add_u32 s43, %[xrecv], 4096\n"
        "buffer_load_dwordx4 a[144:147], v61, %[rsA], s43 offen offset:0 sc0 sc1\n"
        "buffer_load_dwordx4 a[148:151], v61, %[rsA], s43 offen offset:1024 sc0 sc1\n"
        "buffer_load_dwordx4 a[152:155], v61, %[rsA], s43 offen offset:2048 sc0 sc1\n"
        "buffer_load_dwordx4 a[156:159], v61, %[rsA], s43 offen offset:3072 sc0 sc1\n"
        "LSK0_%=:\n"
        "s_cmp_gt_u32 s67, 1\n"
        "s_cbranch_scc1 LSK1_%=\n"
        "s_cmp_le_u32 s69, 1\n"
        "s_cbranch_scc1 LSK1_%=\n"
        "s_add_u32 s43, %[xrecv], 8192\n"
        "buffer_load_dwordx4 a[160:163], v61, %[rsA], s43 offen offset:0 sc0 sc1\n"
        "buffer_load_dwordx4 a[164:167], v61, %[rsA], s43 offen offset:1024 sc0 sc1\n"
        "buffer_load_dwordx4 a[168:171], v61, %[rsA], s43 offen offset:2048 sc0 sc1\n"
        "buffer_load_dwordx4 a[172:175], v61, %[rsA], s43 offen offset:3072 sc0 sc1\n"
        "s_add_u32 s43, %[xrecv], 12288\n"
        "buffer_load_dwordx4 a[176:179], v61, %[rsA], s43 offen offset:0 sc0 sc1\n"
        "buffer_load_dwordx4 a[180:183], v61, %[rsA], s43 offen offset:1024 sc0 sc1\n"
        "buffer_load_dwordx4 a[184:187], v61, %[rsA], s43 offen offset:2048 sc0 sc1\n"
        "buffer_load_dwordx4 a[188:191], v61, %[rsA], s43 offen offset:3072 sc0 sc1\n"
        "LSK1_%=:\n"
        "s_cmp_gt_u32 s67, 2\n"
        "s_cbranch_scc1 LSK2_%=\n"
        "s_cmp_le_u32 s69, 2\n"
        "s_cbranch_scc1 LSK2_%=\n"
        "s_add_u32 s43, %[xrecv], 16384\n"
        "buffer_load_dwordx4 a[192:195], v61, %[rsA], s43 offen offset:0 sc0 sc1\n"
        "buffer_load_dwordx4 a[196:199], v61, %[rsA], s43 offen offset:1024 sc0 sc1\n"
        "buffer_load_dwordx4 a[200:203], v61, %[rsA], s43 offen offset:2048 sc0 sc1\n"
        "buffer_load_dwordx4 a[204:207], v61, %[rsA], s43 offen offset:3072 sc0 sc1\n"
        "s_add_u32 s43, %[xrecv], 20480\n"
        "buffer_load_dwordx4 a[208:211], v61, %[rsA], s43 offen offset:0 sc0 sc1\n"
        "buffer_load_dwordx4 a[212:215], v61, %[rsA], s43 offen offset:1024 sc0 sc1\n"
        "buffer_load_dwordx4 a[216:219], v61, %[rsA], s43 offen offset:2048 sc0 sc1\n"
        "buffer_load_dwordx4 a[220:223], v61, %[rsA], s43 offen offset:3072 sc0 sc1\n"
        "LSK2_%=:\n"
        "s_cmp_gt_u32 s67, 3\n"
        "s_cbranch_scc1 LSK3_%=\n"
        "s_cmp_le_u32 s69, 3\n"
        "s_cbranch_scc1 LSK3_%=\n"
        "s_add_u32 s43, %[xrecv], 24576\n"
        "buffer_load_dwordx4 a[224:227], v61, %[rsA], s43 offen offset:0 sc0 sc1\n"
        "buffer_load_dwordx4 a[228:231], v61, %[rsA], s43 offen offset:1024 sc0 sc1\n"
        "buffer_load_dwordx4 a[232:235], v61, %[rsA], s43 offen offset:2048 sc0 sc1\n"
        "buffer_load_dwordx4 a[236:239], v61, %[rsA], s43 offen offset:3072 sc0 sc1\n"
        "s_add_u32 s43, %[xrecv], 28672\n"
        "buffer_load_dwordx4 a[240:243], v61, %[rsA], s43 offen offset:0 sc0 sc1\n"
        "buffer_load_dwordx4 a[244:247], v61, %[rsA], s43 offen offset:1024 sc0 sc1\n"
        "buffer_load_dwordx4 a[248:251], v61, %[rsA], s43 offen offset:2048 sc0 sc1\n"
        "buffer_load_dwordx4 a[252:255], v61, %[rsA], s43 offen offset:3072 sc0 sc1\n"
        "LSK3_%=:\n"
        "s_sub_u32 s68, s69, s52\n"
        "s_sub_u32 s68, s68, 1\n"
        "s_mov_b32 s67, s69\n"
        "s_setpc_b64 s[64:65]\n"
        "LXBAD_%=:\n"
        "v_mov_b32 v68, 0x20000\n"
        "v_mov_b32 v67, s70\n"
        "buffer_store_dword v67, v68, %[rsA], 0 offen sc1\n"
        "s_waitcnt vmcnt(0)\n"
        "v_mov_b32 v192, 0x7fc00000\n"
        "v_mov_b32 v193, 0x7fc00000\n"
        "v_mov_b32 v194, 0x7fc00000\n"
        "v_mov_b32 v195, 0x7fc00000\n"
        "s_mov_b32 s43, 0\n"
        "buffer_store_dwordx4 v[192:195], %[okeep], %[rsO], s43 offen\n"
        "buffer_store_dwordx4 v[192:195], v60, %[rsO], s43 offen\n"
        "s_mov_b32 s43, 65536\n"
        "buffer_store_dwordx4 v[192:195], %[okeep], %[rsO], s43 offen\n"
        "buffer_store_dwordx4 v[192:195], v60, %[rsO], s43 offen\n"
        "s_mov_b32 s43, 131072\n"
        "buffer_store_dwordx4 v[192:195], %[okeep], %[rsO], s43 offen\n"
        "buffer_store_dwordx4 v[192:195], v60, %[rsO], s43 offen\n"
        "s_mov_b32 s43, 196608\n"
        "buffer_store_dwordx4 v[192:195], %[okeep], %[rsO], s43 offen\n"
        "buffer_store_dwordx4 v[192:195], v60, %[rsO], s43 offen\n"
        "s_mov_b32 s43, 1048576\n"
        "buffer_store_dwordx4 v[192:195], %[okeep], %[rsO], s43 offen\n"
        "buffer_store_dwordx4 v[192:195], v60, %[rsO], s43 offen\n"
        "s_mov_b32 s43, 1114112\n"
        "buffer_store_dwordx4 v[192:195], %[okeep], %[rsO], s43 offen\n"
        "buffer_store_dwordx4 v[192:195], v60, %[rsO], s43 offen\n"
        "s_mov_b32 s43, 1179648\n"
        "buffer_store_dwordx4 v[192:195], %[okeep], %[rsO], s43 offen\n"
        "buffer_store_dwordx4 v[192:195], v60, %[rsO], s43 offen\n"
        "s_mov_b32 s43, 1245184\n"
        "buffer_store_dwordx4 v[192:195], %[okeep], %[rsO], s43 offen\n"
        "buffer_store_dwordx4 v[192:195], v60, %[rsO], s43 offen\n"
        "s_mov_b32 s43, 2097152\n"
        "buffer_store_dwordx4 v[192:195], %[okeep], %[rsO], s43 offen\n"
        "buffer_store_dwordx4 v[192:195], v60, %[rsO], s43 offen\n"
        "s_mov_b32 s43, 2162688\n"
        "buffer_store_dwordx4 v[192:195], %[okeep], %[rsO], s43 offen\n"
        "buffer_store_dwordx4 v[192:195], v60, %[rsO], s43 offen\n"
        "s_mov_b32 s43, 2228224\n"
        "buffer_store_dwordx4 v[192:195], %[okeep], %[rsO], s43 offen\n"
        "buffer_store_dwordx4 v[192:195], v60, %[rsO], s43 offen\n"
        "s_mov_b32 s43, 2293760\n"
        "buffer_store_dwordx4 v[192:195], %[okeep], %[rsO], s43 offen\n"
        "buffer_store_dwordx4 v[192:195], v60, %[rsO], s43 offen\n"
        "s_mov_b32 s43, 3145728\n"
        "buffer_store_dwordx4 v[192:195], %[okeep], %[rsO], s43 offen\n"
        "buffer_store_dwordx4 v[192:195], v60, %[rsO], s43 offen\n"
        "s_mov_b32 s43, 3211264\n"
        "buffer_store_dwordx4 v[192:195], %[okeep], %[rsO], s43 offen\n"
        "buffer_store_dwordx4 v[192:195], v60, %[rsO], s43 offen\n"
        "s_mov_b32 s43, 3276800\n"
        "buffer_store_dwordx4 v[192:195], %[okeep], %[rsO], s43 offen\n"
        "buffer_store_dwordx4 v[192:195], v60, %[rsO], s43 offen\n"
        "s_mov_b32 s43, 3342336\n"
        "buffer_store_dwordx4 v[192:195], %[okeep], %[rsO], s43 offen\n"
        "buffer_store_dwordx4 v[192:195], v60, %[rsO], s43 offen\n"
        "LXEND_%=:\n"
        "s_memrealtime s[50:51]\n"
        "v_mov_b32 v67, %[seq]\n"
        "v_mov_b32 v68, s70\n"
        "s_waitcnt lgkmcnt(0)\n"
        "v_mov_b32 v69, s50\n"
        "s_mov_b64 exec, 1\n"
        "buffer_store_dwordx2 v[68:69], v67, %[rsA], 0 offen sc0 sc1\n"
        "s_mov_b64 exec, -1\n"
        "LXFIN_%=:\n"
        "s_branch LZZ_%=\n"
        "LOOPG_%=:\n"
        "s_sub_u32 s45, s40, 8\n"
        "s_cmp_lt_u32 s45, 40\n"
        "s_cbranch_scc1 LTPG0_%=\n"
        "LTRG0_%=:\n"
        "s_barrier\n"
        "s_waitcnt vmcnt(8) lgkmcnt(2)\n"
        "v_mfma_f32_16x16x32_f16 a[0:3], v[128:131], v[112:115], a[0:3]\n"
        "buffer_load_dwordx4 v[80:83], %[wv0], %[rsW], s42 offen\n"
        "v_readlane_b32 s77, %[tablo], s40\n"
        "v_mfma_f32_16x16x32_f16 a[32:35], v[132:135], v[112:115], a[32:35]\n"
        "ds_read_b128 v[124:127], v71 offset:1536\n"
        "v_readlane_b32 s78, %[tabhi], s40\n"
        "s_add_u32 s40, s40, 1\n"
        "v_mfma_f32_16x16x32_f16 a[64:67], v[136:139], v[112:115], a[64:67]\n"
        "buffer_load_dwordx4 v[84:87], v62, %[rsW], s42 offen\n"
        "v_mfma_f32_16x16x32_f16 a[96:99], v[140:143], v[112:115], a[96:99]\n"
        "buffer_load_dwordx4 v[88:91], v63, %[rsW], s42 offen\n"
        "v_mfma_f32_16x16x32_f16 a[128:131], v[144:147], v[112:115], a[128:131]\n"
        "buffer_load_dwordx4 v[92:95], v66, %[rsW], s42 offen\n"
        "v_mfma_f32_16x16x32_f16 a[160:163], v[148:151], v[112:115], a[160:163]\n"
        "buffer_load_dwordx4 v[96:99], %[wv0], %[rsW], s68 offen\n"
        "v_mfma_f32_16x16x32_f16 a[192:195], v[152:155], v[112:115], a[192:195]\n"
        "buffer_load_dwordx4 v[100:103], v62, %[rsW], s68 offen\n"
        "v_mfma_f32_16x16x32_f16 a[224:227], v[156:159], v[112:115], a[224:227]\n"
        "s_waitcnt lgkmcnt(2)\n"
        "v_mfma_f32_16x16x32_f16 a[4:7], v[128:131], v[116:119], a[4:7]\n"
        "buffer_load_dwordx4 v[104:107], v63, %[rsW], s68 offen\n"
        "v_mfma_f32_16x16x32_f16 a[36:39], v[132:135], v[116:119], a[36:39]\n"
        "ds_read_b128 v[112:115], v71 offset:256\n"
        "v_mfma_f32_16x16x32_f16 a[68:71], v[136:139], v[116:119], a[68:71]\n"
        "buffer_load_dwordx4 v[108:111], v66, %[rsW], s68 offen\n"
        "s_mov_b64 exec, s[88:89]\n"
        "buffer_load_dword v61, %[cnt], %[rsA], s60 offen sc0 sc1\n"
        "s_mov_b64 exec, -1\n"
        "buffer_load_dword v65, v64, %[rsA], s58 offen sc0 sc1\n"
        "v_mfma_f32_16x16x32_f16 a[100:103], v[140:143], v[116:119], a[100:103]\n"
        "v_readlane_b32 s42, %[tabw], s66\n"
        "s_cmp_lt_u32 s66, 64\n"
        "s_cselect_b32 s42, s42, 0x80000000\n"
        "s_add_u32 s66, s66, 1\n"
        "v_mfma_f32_16x16x32_f16 a[132:135], v[144:147], v[116:119], a[132:135]\n"
        "v_readlane_b32 s68, %[tabw], s66\n"
        "s_cmp_lt_u32 s66, 64\n"
        "s_cselect_b32 s68, s68, 0x80000000\n"
        "s_add_u32 s66, s66, 1\n"
        "v_mfma_f32_16x16x32_f16 a[164:167], v[148:151], v[116:119], a[164:167]\n"
        "s_cmp_eq_u32 s40, s61\n"
        "s_cbranch_scc1 LSENDG0_%=\n"
        "LSENTG0_%=:\n"
        "v_mfma_f32_16x16x32_f16 a[196:199], v[152:155], v[116:119], a[196:199]\n"
        "v_mfma_f32_16x16x32_f16 a[228:231], v[156:159], v[116:119], a[228:231]\n"
        "s_waitcnt lgkmcnt(2)\n"
        "v_mfma_f32_16x16x32_f16 a[8:11], v[128:131], v[120:123], a[8:11]\n"
        "v_add_u32 v69, s67, %[wdst]\n"
        "s_add_u32 s67, s67, 0x2000\n"
        "v_mfma_f32_16x16x32_f16 a[40:43], v[132:135], v[120:123], a[40:43]\n"
        "ds_read_b128 v[116:119], v71 offset:768\n"
        "v_mfma_f32_16x16x32_f16 a[72:75], v[136:139], v[120:123], a[72:75]\n"
        "s_waitcnt vmcnt(22)\n"
        "s_cmp_lt_u32 s67, 163840\n"
        "s_cselect_b32 s67, s67, 0\n"
        "v_mfma_f32_16x16x32_f16 a[104:107], v[140:143], v[120:123], a[104:107]\n"
        "v_cvt_pk_f16_f32 v72, v12, v16\n"
        "v_cvt_pk_f16_f32 v73, v20, v24\n"
        "v_mfma_f32_16x16x32_f16 a[136:139], v[144:147], v[120:123], a[136:139]\n"
        "ds_write_b64 v69, v[72:73] offset:0\n"
        "v_mfma_f32_16x16x32_f16 a[168:171], v[148:151], v[120:123], a[168:171]\n"
        "v_cvt_pk_f16_f32 v74, v13, v17\n"
        "v_cvt_pk_f16_f32 v75, v21, v25\n"
        "v_mfma_f32_16x16x32_f16 a[200:203], v[152:155], v[120:123], a[200:203]\n"
        "ds_write_b64 v69, v[74:75] offset:512\n"
        "v_mfma_f32_16x16x32_f16 a[232:235], v[156:159], v[120:123], a[232:235]\n"
        "s_waitcnt lgkmcnt(4)\n"
        "v_mfma_f32_16x16x32_f16 a[12:15], v[128:131], v[124:127], a[12:15]\n"
        "v_cvt_pk_f16_f32 v76, v14, v18\n"
        "v_cvt_pk_f16_f32 v77, v22, v26\n"
        "v_mfma_f32_16x16x32_f16 a[44:47], v[132:135], v[124:127], a[44:47]\n"
        "ds_read_b128 v[120:123], v71 offset:1280\n"
        "v_mfma_f32_16x16x32_f16 a[76:79], v[136:139], v[124:127], a[76:79]\n"
        "ds_write_b64 v69, v[76:77] offset:1024\n"
        "v_mfma_f32_16x16x32_f16 a[108:111], v[140:143], v[124:127], a[108:111]\n"
        "v_cvt_pk_f16_f32 v78, v15, v19\n"
        "v_cvt_pk_f16_f32 v79, v23, v27\n"
        "v_mfma_f32_16x16x32_f16 a[140:143], v[144:147], v[124:127], a[140:143]\n"
        "ds_write_b64 v69, v[78:79] offset:1536\n"
        "v_mfma_f32_16x16x32_f16 a[172:175], v[148:151], v[124:127], a[172:175]\n"
        "v_add_u32 v69, s67, %[wdst]\n"
        "s_add_u32 s67, s67, 0x2000\n"
        "v_mfma_f32_16x16x32_f16 a[204:207], v[152:155], v[124:127], a[204:207]\n"
        "s_waitcnt vmcnt(18)\n"
        "s_cmp_lt_u32 s67, 163840\n"
        "s_cselect_b32 s67, s67, 0\n"
        "v_mfma_f32_16x16x32_f16 a[236:239], v[156:159], v[124:127], a[236:239]\n"
        "s_waitcnt lgkmcnt(6)\n"
        "v_mfma_f32_16x16x32_f16 a[16:19], v[128:131], v[112:115], a[16:19]\n"
        "v_cvt_pk_f16_f32 v72, v44, v48\n"
        "v_cvt_pk_f16_f32 v73, v52, v56\n"
        "v_mfma_f32_16x16x32_f16 a[48:51], v[132:135], v[112:115], a[48:51]\n"
        "ds_read_b128 v[124:127], v71 offset:1792\n"
        "v_mfma_f32_16x16x32_f16 a[80:83], v[136:139], v[112:115], a[80:83]\n"
        "ds_write_b64 v69, v[72:73] offset:0\n"
        "v_mfma_f32_16x16x32_f16 a[112:115], v[140:143], v[112:115], a[112:115]\n"
        "v_cvt_pk_f16_f32 v74, v45, v49\n"
        "v_cvt_pk_f16_f32 v75, v53, v57\n"
        "v_mfma_f32_16x16x32_f16 a[144:147], v[144:147], v[112:115], a[144:147]\n"
        "ds_write_b64 v69, v[74:75] offset:512\n"
        "v_mfma_f32_16x16x32_f16 a[176:179], v[148:151], v[112:115], a[176:179]\n"
        "v_cvt_pk_f16_f32 v76, v46, v50\n"
        "v_cvt_pk_f16_f32 v77, v54, v58\n"
        "v_mfma_f32_16x16x32_f16 a[208:211], v[152:155], v[112:115], a[208:211]\n"
        "ds_write_b64 v69, v[76:77] offset:1024\n"
        "v_mfma_f32_16x16x32_f16 a[240:243], v[156:159], v[112:115], a[240:243]\n"
        "s_waitcnt lgkmcnt(9)\n"
        "v_mfma_f32_16x16x32_f16 a[20:23], v[128:131], v[116:119], a[20:23]\n"
        "buffer_load_dwordx4 v[192:195], v64, %[rsA], s77 offen offset:0 sc0 sc1\n"
        "v_mfma_f32_16x16x32_f16 a[52:55], v[132:135], v[116:119], a[52:55]\n"
        "ds_read_b128 v[112:115], v71 offset:8192\n"
        "v_mfma_f32_16x16x32_f16 a[84:87], v[136:139], v[116:119], a[84:87]\n"
        "v_cvt_pk_f16_f32 v78, v47, v51\n"
        "v_cvt_pk_f16_f32 v79, v55, v59\n"
        "v_mfma_f32_16x16x32_f16 a[116:119], v[140:143], v[116:119], a[116:119]\n"
        "buffer_load_dwordx4 v[196:199], v64, %[rsA], s77 offen offset:1024 sc0 sc1\n"
        "v_mfma_f32_16x16x32_f16 a[148:151], v[144:147], v[116:119], a[148:151]\n"
        "ds_write_b64 v69, v[78:79] offset:1536\n"
        "v_mfma_f32_16x16x32_f16 a[180:183], v[148:151], v[116:119], a[180:183]\n"
        "v_mfma_f32_16x16x32_f16 a[212:215], v[152:155], v[116:119], a[212:215]\n"
        "buffer_load_dwordx4 v[200:203], v64, %[rsA], s77 offen offset:2048 sc0 sc1\n"
        "v_mfma_f32_16x16x32_f16 a[244:247], v[156:159], v[116:119], a[244:247]\n"
        "s_waitcnt lgkmcnt(8)\n"
        "v_mfma_f32_16x16x32_f16 a[24:27], v[128:131], v[120:123], a[24:27]\n"
        "v_mfma_f32_16x16x32_f16 a[56:59], v[132:135], v[120:123], a[56:59]\n"
        "buffer_load_dwordx4 v[204:207], v64, %[rsA], s77 offen offset:3072 sc0 sc1\n"
        "v_mfma_f32_16x16x32_f16 a[88:91], v[136:139], v[120:123], a[88:91]\n"
        "ds_read_b128 v[116:119], v71 offset:8704\n"
        "v_mfma_f32_16x16x32_f16 a[120:123], v[140:143], v[120:123], a[120:123]\n"
        "v_mfma_f32_16x16x32_f16 a[152:155], v[144:147], v[120:123], a[152:155]\n"
        "buffer_load_dwordx4 v[208:211], v64, %[rsA], s78 offen offset:0 sc0 sc1\n"
        "v_mfma_f32_16x16x32_f16 a[184:187], v[148:151], v[120:123], a[184:187]\n"
        "v_mfma_f32_16x16x32_f16 a[216:219], v[152:155], v[120:123], a[216:219]\n"
        "buffer_load_dwordx4 v[212:215], v64, %[rsA], s78 offen offset:1024 sc0 sc1\n"
        "v_mfma_f32_16x16x32_f16 a[248:251], v[156:159], v[120:123], a[248:251]\n"
        "s_waitcnt lgkmcnt(6)\n"
        "v_mfma_f32_16x16x32_f16 a[28:31], v[128:131], v[124:127], a[28:31]\n"
        "v_mfma_f32_16x16x32_f16 a[60:63], v[132:135], v[124:127], a[60:63]\n"
        "ds_read_b128 v[120:123], v71 offset:9216\n"
        "v_mfma_f32_16x16x32_f16 a[92:95], v[136:139], v[124:127], a[92:95]\n"
        "buffer_load_dwordx4 v[216:219], v64, %[rsA], s78 offen offset:2048 sc0 sc1\n"
        "v_mfma_f32_16x16x32_f16 a[124:127], v[140:143], v[124:127], a[124:127]\n"
        "v_mfma_f32_16x16x32_f16 a[156:159], v[144:147], v[124:127], a[156:159]\n"
        "buffer_load_dwordx4 v[220:223], v64, %[rsA], s78 offen offset:3072 sc0 sc1\n"
        "v_mfma_f32_16x16x32_f16 a[188:191], v[148:151], v[124:127], a[188:191]\n"
        "v_mfma_f32_16x16x32_f16 a[220:223], v[152:155], v[124:127], a[220:223]\n"
        "v_mfma_f32_16x16x32_f16 a[252:255], v[156:159], v[124:127], a[252:255]\n"
        "s_sub_u32 s45, s40, 8\n"
        "s_cmp_lt_u32 s45, 40\n"
        "s_cbranch_scc1 LTPG1_%=\n"
        "LTRG1_%=:\n"
        "s_barrier\n"
        "s_waitcnt vmcnt(18) lgkmcnt(3)\n"
        "v_mfma_f32_16x16x32_f16 a[0:3], v[160:163], v[112:115], a[0:3]\n"
        "buffer_load_dwordx4 v[12:15], %[wv0], %[rsW], s42 offen\n"
        "v_readlane_b32 s77, %[tablo], s40\n"
        "v_mfma_f32_16x16x32_f16 a[32:35], v[164:167], v[112:115], a[32:35]\n"
        "ds_read_b128 v[124:127], v71 offset:9728\n"
        "v_readlane_b32 s78, %[tabhi], s40\n"
        "s_add_u32 s40, s40, 1\n"
        "v_mfma_f32_16x16x32_f16 a[64:67], v[168:171], v[112:115], a[64:67]\n"
        "buffer_load_dwordx4 v[16:19], v62, %[rsW], s42 offen\n"
        "s_lshl_b64 s[54:55], s[54:55], 2\n"
        "v_mfma_f32_16x16x32_f16 a[96:99], v[172:175], v[112:115], a[96:99]\n"
        "buffer_load_dwordx4 v[20:23], v63, %[rsW], s42 offen\n"
        "s_or_b32 s54, s54, 3\n"
        "v_mfma_f32_16x16x32_f16 a[128:131], v[176:179], v[112:115], a[128:131]\n"
        "buffer_load_dwordx4 v[24:27], v66, %[rsW], s42 offen\n"
        "v_mfma_f32_16x16x32_f16 a[160:163], v[180:183], v[112:115], a[160:163]\n"
        "buffer_load_dwordx4 v[44:47], %[wv0], %[rsW], s68 offen\n"
        "v_mfma_f32_16x16x32_f16 a[192:195], v[184:187], v[112:115], a[192:195]\n"
        "buffer_load_dwordx4 v[48:51], v62, %[rsW], s68 offen\n"
        "v_mfma_f32_16x16x32_f16 a[224:227], v[188:191], v[112:115], a[224:227]\n"
        "s_waitcnt lgkmcnt(2)\n"
        "v_mfma_f32_16x16x32_f16 a[4:7], v[160:163], v[116:119], a[4:7]\n"
        "buffer_load_dwordx4 v[52:55], v63, %[rsW], s68 offen\n"
        "v_mfma_f32_16x16x32_f16 a[36:39], v[164:167], v[116:119], a[36:39]\n"
        "ds_read_b128 v[112:115], v71 offset:8448\n"
        "v_mfma_f32_16x16x32_f16 a[68:71], v[168:171], v[116:119], a[68:71]\n"
        "buffer_load_dwordx4 v[56:59], v66, %[rsW], s68 offen\n"
        "v_mfma_f32_16x16x32_f16 a[100:103], v[172:175], v[116:119], a[100:103]\n"
        "v_readlane_b32 s42, %[tabw], s66\n"
        "s_cmp_lt_u32 s66, 64\n"
        "s_cselect_b32 s42, s42, 0x80000000\n"
        "s_add_u32 s66, s66, 1\n"
        "v_mfma_f32_16x16x32_f16 a[132:135], v[176:179], v[116:119], a[132:135]\n"
        "v_readlane_b32 s68, %[tabw], s66\n"
        "s_cmp_lt_u32 s66, 64\n"
        "s_cselect_b32 s68, s68, 0x80000000\n"
        "s_add_u32 s66, s66, 1\n"
        "v_mfma_f32_16x16x32_f16 a[164:167], v[180:183], v[116:119], a[164:167]\n"
        "s_cmp_eq_u32 s40, s61\n"
        "s_cbranch_scc1 LSENDG1_%=\n"
        "LSENTG1_%=:\n"
        "v_mfma_f32_16x16x32_f16 a[196:199], v[184:187], v[116:119], a[196:199]\n"
        "s_cmp_ge_u32 s40, 41\n"
        "s_cselect_b64 s[54:55], -1, s[54:55]\n"
        "v_mfma_f32_16x16x32_f16 a[228:231], v[188:191], v[116:119], a[228:231]\n"
        "s_waitcnt lgkmcnt(2)\n"
        "v_mfma_f32_16x16x32_f16 a[8:11], v[160:163], v[120:123], a[8:11]\n"
        "v_add_u32 v69, s67, %[wdst]\n"
        "s_add_u32 s67, s67, 0x2000\n"
        "v_mfma_f32_16x16x32_f16 a[40:43], v[164:167], v[120:123], a[40:43]\n"
        "ds_read_b128 v[116:119], v71 offset:8960\n"
        "v_mfma_f32_16x16x32_f16 a[72:75], v[168:171], v[120:123], a[72:75]\n"
        "s_waitcnt vmcnt(22)\n"
        "s_cmp_lt_u32 s67, 163840\n"
        "s_cselect_b32 s67, s67, 0\n"
        "v_mfma_f32_16x16x32_f16 a[104:107], v[172:175], v[120:123], a[104:107]\n"
        "v_cvt_pk_f16_f32 v72, v80, v84\n"
        "v_cvt_pk_f16_f32 v73, v88, v92\n"
        "v_mfma_f32_16x16x32_f16 a[136:139], v[176:179], v[120:123], a[136:139]\n"
        "ds_write_b64 v69, v[72:73] offset:0\n"
        "v_mfma_f32_16x16x32_f16 a[168:171], v[180:183], v[120:123], a[168:171]\n"
        "v_cvt_pk_f16_f32 v74, v81, v85\n"
        "v_cvt_pk_f16_f32 v75, v89, v93\n"
        "v_mfma_f32_16x16x32_f16 a[200:203], v[184:187], v[120:123], a[200:203]\n"
        "ds_write_b64 v69, v[74:75] offset:512\n"
        "v_mfma_f32_16x16x32_f16 a[232:235], v[188:191], v[120:123], a[232:235]\n"
        "s_waitcnt lgkmcnt(4)\n"
        "v_mfma_f32_16x16x32_f16 a[12:15], v[160:163], v[124:127], a[12:15]\n"
        "v_cvt_pk_f16_f32 v76, v82, v86\n"
        "v_cvt_pk_f16_f32 v77, v90, v94\n"
        "v_mfma_f32_16x16x32_f16 a[44:47], v[164:167], v[124:127], a[44:47]\n"
        "ds_read_b128 v[120:123], v71 offset:9472\n"
        "v_mfma_f32_16x16x32_f16 a[76:79], v[168:171], v[124:127], a[76:79]\n"
        "ds_write_b64 v69, v[76:77] offset:1024\n"
        "v_mfma_f32_16x16x32_f16 a[108:111], v[172:175], v[124:127], a[108:111]\n"
        "v_cvt_pk_f16_f32 v78, v83, v87\n"
        "v_cvt_pk_f16_f32 v79, v91, v95\n"
        "v_mfma_f32_16x16x32_f16 a[140:143], v[176:179], v[124:127], a[140:143]\n"
        "ds_write_b64 v69, v[78:79] offset:1536\n"
        "v_mfma_f32_16x16x32_f16 a[172:175], v[180:183], v[124:127], a[172:175]\n"
        "v_add_u32 v69, s67, %[wdst]\n"
        "s_add_u32 s67, s67, 0x2000\n"
        "v_mfma_f32_16x16x32_f16 a[204:207], v[184:187], v[124:127], a[204:207]\n"
        "s_waitcnt vmcnt(18)\n"
        "s_cmp_lt_u32 s67, 163840\n"
        "s_cselect_b32 s67, s67, 0\n"
        "v_mfma_f32_16x16x32_f16 a[236:239], v[188:191], v[124:127], a[236:239]\n"
        "s_waitcnt lgkmcnt(6)\n"
        "v_mfma_f32_16x16x32_f16 a[16:19], v[160:163], v[112:115], a[16:19]\n"
        "v_cvt_pk_f16_f32 v72, v96, v100\n"
        "v_cvt_pk_f16_f32 v73, v104, v108\n"
        "v_mfma_f32_16x16x32_f16 a[48:51], v[164:167], v[112:115], a[48:51]\n"
        "ds_read_b128 v[124:127], v71 offset:9984\n"
        "v_mfma_f32_16x16x32_f16 a[80:83], v[168:171], v[112:115], a[80:83]\n"
        "ds_write_b64 v69, v[72:73] offset:0\n"
        "v_mfma_f32_16x16x32_f16 a[112:115], v[172:175], v[112:115], a[112:115]\n"
        "v_cvt_pk_f16_f32 v74, v97, v101\n"
        "v_cvt_pk_f16_f32 v75, v105, v109\n"
        "v_mfma_f32_16x16x32_f16 a[144:147], v[176:179], v[112:115], a[144:147]\n"
        "ds_write_b64 v69, v[74:75] offset:512\n"
        "v_mfma_f32_16x16x32_f16 a[176:179], v[180:183], v[112:115], a[176:179]\n"
        "v_cvt_pk_f16_f32 v76, v98, v102\n"
        "v_cvt_pk_f16_f32 v77, v106, v110\n"
        "v_mfma_f32_16x16x32_f16 a[208:211], v[184:187], v[112:115], a[208:211]\n"
        "ds_write_b64 v69, v[76:77] offset:1024\n"
        "v_mfma_f32_16x16x32_f16 a[240:243], v[188:191], v[112:115], a[240:243]\n"
        "s_waitcnt lgkmcnt(9)\n"
        "v_mfma_f32_16x16x32_f16 a[20:23], v[160:163], v[116:119], a[20:23]\n"
        "buffer_load_dwordx4 v[224:227], v64, %[rsA], s77 offen offset:0 sc0 sc1\n"
        "v_mfma_f32_16x16x32_f16 a[52:55], v[164:167], v[116:119], a[52:55]\n"
        "ds_read_b128 v[112:115], v71 offset:16384\n"
        "v_mfma_f32_16x16x32_f16 a[84:87], v[168:171], v[116:119], a[84:87]\n"
        "v_cvt_pk_f16_f32 v78, v99, v103\n"
        "v_cvt_pk_f16_f32 v79, v107, v111\n"
        "v_mfma_f32_16x16x32_f16 a[116:119], v[172:175], v[116:119], a[116:119]\n"
        "buffer_load_dwordx4 v[228:231], v64, %[rsA], s77 offen offset:1024 sc0 sc1\n"
        "v_mfma_f32_16x16x32_f16 a[148:151], v[176:179], v[116:119], a[148:151]\n"
        "ds_write_b64 v69, v[78:79] offset:1536\n"
        "v_mfma_f32_16x16x32_f16 a[180:183], v[180:183], v[116:119], a[180:183]\n"
        "s_waitcnt vmcnt(19)\n"
        "v_cmp_le_u32 vcc, s71, v61\n"
        "v_mfma_f32_16x16x32_f16 a[212:215], v[184:187], v[116:119], a[212:215]\n"
        "buffer_load_dwordx4 v[232:235], v64, %[rsA], s77 offen offset:2048 sc0 sc1\n"
        "v_mfma_f32_16x16x32_f16 a[244:247], v[188:191], v[116:119], a[244:247]\n"
        "s_waitcnt lgkmcnt(8)\n"
        "v_mfma_f32_16x16x32_f16 a[24:27], v[160:163], v[120:123], a[24:27]\n"
        "s_or_b64 s[46:47], s[46:47], vcc\n"
        "v_mfma_f32_16x16x32_f16 a[56:59], v[164:167], v[120:123], a[56:59]\n"
        "buffer_load_dwordx4 v[236:239], v64, %[rsA], s77 offen offset:3072 sc0 sc1\n"
        "s_lshl_b64 s[62:63], s[54:55], 8\n"
        "v_mfma_f32_16x16x32_f16 a[88:91], v[168:171], v[120:123], a[88:91]\n"
        "ds_read_b128 v[116:119], v71 offset:16896\n"
        "v_mfma_f32_16x16x32_f16 a[120:123], v[172:175], v[120:123], a[120:123]\n"
        "s_andn2_b64 s[62:63], s[62:63], s[46:47]\n"
        "s_cselect_b32 s60, 0, 0x6000000\n"
        "v_mfma_f32_16x16x32_f16 a[152:155], v[176:179], v[120:123], a[152:155]\n"
        "buffer_load_dwordx4 v[240:243], v64, %[rsA], s78 offen offset:0 sc0 sc1\n"
        "s_ff0_i32_b64 s52, s[46:47]\n"
        "v_mfma_f32_16x16x32_f16 a[184:187], v[180:183], v[120:123], a[184:187]\n"
        "s_bfm_b64 s[88:89], 44, s52\n"
        "v_mfma_f32_16x16x32_f16 a[216:219], v[184:187], v[120:123], a[216:219]\n"
        "buffer_load_dwordx4 v[244:247], v64, %[rsA], s78 offen offset:1024 sc0 sc1\n"
        "s_andn2_b64 s[88:89], s[88:89], s[46:47]\n"
        "v_mfma_f32_16x16x32_f16 a[248:251], v[188:191], v[120:123], a[248:251]\n"
        "s_waitcnt lgkmcnt(6)\n"
        "v_mfma_f32_16x16x32_f16 a[28:31], v[160:163], v[124:127], a[28:31]\n"
        "s_cmp_lt_u32 s40, 14\n"
        "s_cselect_b64 s[62:63], 0x3fffff, -1\n"
        "v_mfma_f32_16x16x32_f16 a[60:63], v[164:167], v[124:127], a[60:63]\n"
        "ds_read_b128 v[120:123], v71 offset:17408\n"
        "s_and_b64 s[88:89], s[88:89], s[62:63]\n"
        "v_mfma_f32_16x16x32_f16 a[92:95], v[168:171], v[124:127], a[92:95]\n"
        "buffer_load_dwordx4 v[248:251], v64, %[rsA], s78 offen offset:2048 sc0 sc1\n"
        "s_or_b32 s88, s88, 1\n"
        "v_mfma_f32_16x16x32_f16 a[124:127], v[172:175], v[124:127], a[124:127]\n"
        "s_cmp_eq_u32 s40, 14\n"
        "s_cbranch_scc1 LCWG_%=\n"
        "LCXG_%=:\n"
        "v_mfma_f32_16x16x32_f16 a[156:159], v[176:179], v[124:127], a[156:159]\n"
        "buffer_load_dwordx4 v[252:255], v64, %[rsA], s78 offen offset:3072 sc0 sc1\n"
        "v_mfma_f32_16x16x32_f16 a[188:191], v[180:183], v[124:127], a[188:191]\n"
        "s_andn2_b64 s[56:57], s[54:55], s[46:47]\n"
        "s_cbranch_scc1 LS_G1_%=\n"
        "LR_G1_%=:\n"
        "v_mfma_f32_16x16x32_f16 a[220:223], v[184:187], v[124:127], a[220:223]\n"
        "v_readlane_b32 s58, %[tabpf], s40\n"
        "v_mfma_f32_16x16x32_f16 a[252:255], v[188:191], v[124:127], a[252:255]\n"
        "s_bitcmp1_b64 s[46:47], s58\n"
        "s_cselect_b32 s58, s58, 0x6000000\n"
        "s_andn2_b32 s58, s58, 63\n"
        "s_sub_u32 s45, s40, 8\n"
        "s_cmp_lt_u32 s45, 40\n"
        "s_cbranch_scc1 LTPG2_%=\n"
        "LTRG2_%=:\n"
        "s_barrier\n"
        "s_waitcnt vmcnt(16) lgkmcnt(3)\n"
        "v_mfma_f32_16x16x32_f16 a[0:3], v[192:195], v[112:115], a[0:3]\n"
        "buffer_load_dwordx4 v[80:83], %[wv0], %[rsW], s42 offen\n"
        "v_readlane_b32 s77, %[tablo], s40\n"
        "v_mfma_f32_16x16x32_f16 a[32:35], v[196:199], v[112:115], a[32:35]\n"
        "ds_read_b128 v[124:127], v71 offset:17920\n"
        "v_readlane_b32 s78, %[tabhi], s40\n"
        "s_add_u32 s40, s40, 1\n"
        "v_mfma_f32_16x16x32_f16 a[64:67], v[200:203], v[112:115], a[64:67]\n"
        "buffer_load_dwordx4 v[84:87], v62, %[rsW], s42 offen\n"
        "v_mfma_f32_16x16x32_f16 a[96:99], v[204:207], v[112:115], a[96:99]\n"
        "buffer_load_dwordx4 v[88:91], v63, %[rsW], s42 offen\n"
        "v_mfma_f32_16x16x32_f16 a[128:131], v[208:211], v[112:115], a[128:131]\n"
        "buffer_load_dwordx4 v[92:95], v66, %[rsW], s42 offen\n"
        "v_mfma_f32_16x16x32_f16 a[160:163], v[212:215], v[112:115], a[160:163]\n"
        "buffer_load_dwordx4 v[96:99], %[wv0], %[rsW], s68 offen\n"
        "v_mfma_f32_16x16x32_f16 a[192:195], v[216:219], v[112:115], a[192:195]\n"
        "buffer_load_dwordx4 v[100:103], v62, %[rsW], s68 offen\n"
        "v_mfma_f32_16x16x32_f16 a[224:227], v[220:223], v[112:115], a[224:227]\n"
        "s_waitcnt lgkmcnt(2)\n"
        "v_mfma_f32_16x16x32_f16 a[4:7], v[192:195], v[116:119], a[4:7]\n"
        "buffer_load_dwordx4 v[104:107], v63, %[rsW], s68 offen\n"
        "v_mfma_f32_16x16x32_f16 a[36:39], v[196:199], v[116:119], a[36:39]\n"
        "ds_read_b128 v[112:115], v71 offset:16640\n"
        "v_mfma_f32_16x16x32_f16 a[68:71], v[200:203], v[116:119], a[68:71]\n"
        "buffer_load_dwordx4 v[108:111], v66, %[rsW], s68 offen\n"
        "s_mov_b64 exec, s[88:89]\n"
        "buffer_load_dword v61, %[cnt], %[rsA], s60 offen sc0 sc1\n"
        "s_mov_b64 exec, -1\n"
        "buffer_load_dword v65, v64, %[rsA], s58 offen sc0 sc1\n"
        "v_mfma_f32_16x16x32_f16 a[100:103], v[204:207], v[116:119], a[100:103]\n"
        "v_readlane_b32 s42, %[tabw], s66\n"
        "s_cmp_lt_u32 s66, 64\n"
        "s_cselect_b32 s42, s42, 0x80000000\n"
        "s_add_u32 s66, s66, 1\n"
        "v_mfma_f32_16x16x32_f16 a[132:135], v[208:211], v[116:119], a[132:135]\n"
        "v_readlane_b32 s68, %[tabw], s66\n"
        "s_cmp_lt_u32 s66, 64\n"
        "s_cselect_b32 s68, s68, 0x80000000\n"
        "s_add_u32 s66, s66, 1\n"
        "v_mfma_f32_16x16x32_f16 a[164:167], v[212:215], v[116:119], a[164:167]\n"
        "s_cmp_eq_u32 s40, s61\n"
        "s_cbranch_scc1 LSENDG2_%=\n"
        "LSENTG2_%=:\n"
        "v_mfma_f32_16x16x32_f16 a[196:199], v[216:219], v[116:119], a[196:199]\n"
        "v_mfma_f32_16x16x32_f16 a[228:231], v[220:223], v[116:119], a[228:231]\n"
        "s_waitcnt lgkmcnt(2)\n"
        "v_mfma_f32_16x16x32_f16 a[8:11], v[192:195], v[120:123], a[8:11]\n"
        "v_add_u32 v69, s67, %[wdst]\n"
        "s_add_u32 s67, s67, 0x2000\n"
        "v_mfma_f32_16x16x32_f16 a[40:43], v[196:199], v[120:123], a[40:43]\n"
        "ds_read_b128 v[116:119], v71 offset:17152\n"
        "v_mfma_f32_16x16x32_f16 a[72:75], v[200:203], v[120:123], a[72:75]\n"
        "s_waitcnt vmcnt(22)\n"
        "s_cmp_lt_u32 s67, 163840\n"
        "s_cselect_b32 s67, s67, 0\n"
        "v_mfma_f32_16x16x32_f16 a[104:107], v[204:207], v[120:123], a[104:107]\n"
        "v_cvt_pk_f16_f32 v72, v12, v16\n"
        "v_cvt_pk_f16_f32 v73, v20, v24\n"
        "v_mfma_f32_16x16x32_f16 a[136:139], v[208:211], v[120:123], a[136:139]\n"
        "ds_write_b64 v69, v[72:73] offset:0\n"
        "v_mfma_f32_16x16x32_f16 a[168:171], v[212:215], v[120:123], a[168:171]\n"
        "v_cvt_pk_f16_f32 v74, v13, v17\n"
        "v_cvt_pk_f16_f32 v75, v21, v25\n"
        "v_mfma_f32_16x16x32_f16 a[200:203], v[216:219], v[120:123], a[200:203]\n"
        "ds_write_b64 v69, v[74:75] offset:512\n"
        "v_mfma_f32_16x16x32_f16 a[232:235], v[220:223], v[120:123], a[232:235]\n"
        "s_waitcnt lgkmcnt(4)\n"
        "v_mfma_f32_16x16x32_f16 a[12:15], v[192:195], v[124:127], a[12:15]\n"
        "v_cvt_pk_f16_f32 v76, v14, v18\n"
        "v_cvt_pk_f16_f32 v77, v22, v26\n"
        "v_mfma_f32_16x16x32_f16 a[44:47], v[196:199], v[124:127], a[44:47]\n"
        "ds_read_b128 v[120:123], v71 offset:17664\n"
        "v_mfma_f32_16x16x32_f16 a[76:79], v[200:203], v[124:127], a[76:79]\n"
        "ds_write_b64 v69, v[76:77] offset:1024\n"
        "v_mfma_f32_16x16x32_f16 a[108:111], v[204:207], v[124:127], a[108:111]\n"
        "v_cvt_pk_f16_f32 v78, v15, v19\n"
        "v_cvt_pk_f16_f32 v79, v23, v27\n"
        "v_mfma_f32_16x16x32_f16 a[140:143], v[208:211], v[124:127], a[140:143]\n"
        "ds_write_b64 v69, v[78:79] offset:1536\n"
        "v_mfma_f32_16x16x32_f16 a[172:175], v[212:215], v[124:127], a[172:175]\n"
        "v_add_u32 v69, s67, %[wdst]\n"
        "s_add_u32 s67, s67, 0x2000\n"
        "v_mfma_f32_16x16x32_f16 a[204:207], v[216:219], v[124:127], a[204:207]\n"
        "s_waitcnt vmcnt(18)\n"
        "s_cmp_lt_u32 s67, 163840\n"
        "s_cselect_b32 s67, s67, 0\n"
        "v_mfma_f32_16x16x32_f16 a[236:239], v[220:223], v[124:127], a[236:239]\n"
        "s_waitcnt lgkmcnt(6)\n"
        "v_mfma_f32_16x16x32_f16 a[16:19], v[192:195], v[112:115], a[16:19]\n"
        "v_cvt_pk_f16_f32 v72, v44, v48\n"
        "v_cvt_pk_f16_f32 v73, v52, v56\n"
        "v_mfma_f32_16x16x32_f16 a[48:51], v[196:199], v[112:115], a[48:51]\n"
        "ds_read_b128 v[124:127], v71 offset:18176\n"
        "v_mfma_f32_16x16x32_f16 a[80:83], v[200:203], v[112:115], a[80:83]\n"
        "ds_write_b64 v69, v[72:73] offset:0\n"
        "v_mfma_f32_16x16x32_f16 a[112:115], v[204:207], v[112:115], a[112:115]\n"
        "v_cvt_pk_f16_f32 v74, v45, v49\n"
        "v_cvt_pk_f16_f32 v75, v53, v57\n"
        "v_mfma_f32_16x16x32_f16 a[144:147], v[208:211], v[112:115], a[144:147]\n"
        "ds_write_b64 v69, v[74:75] offset:512\n"
        "v_mfma_f32_16x16x32_f16 a[176:179], v[212:215], v[112:115], a[176:179]\n"
        "v_cvt_pk_f16_f32 v76, v46, v50\n"
        "v_cvt_pk_f16_f32 v77, v54, v58\n"
        "v_mfma_f32_16x16x32_f16 a[208:211], v[216:219], v[112:115], a[208:211]\n"
        "ds_write_b64 v69, v[76:77] offset:1024\n"
        "v_mfma_f32_16x16x32_f16 a[240:243], v[220:223], v[112:115], a[240:243]\n"
        "s_waitcnt lgkmcnt(9)\n"
        "v_mfma_f32_16x16x32_f16 a[20:23], v[192:195], v[116:119], a[20:23]\n"
        "buffer_load_dwordx4 v[128:131], v64, %[rsA], s77 offen offset:0 sc0 sc1\n"
        "v_mfma_f32_16x16x32_f16 a[52:55], v[196:199], v[116:119], a[52:55]\n"
        "ds_read_b128 v[112:115], v71 offset:24576\n"
        "v_mfma_f32_16x16x32_f16 a[84:87], v[200:203], v[116:119], a[84:87]\n"
        "v_cvt_pk_f16_f32 v78, v47, v51\n"
        "v_cvt_pk_f16_f32 v79, v55, v59\n"
        "v_mfma_f32_16x16x32_f16 a[116:119], v[204:207], v[116:119], a[116:119]\n"
        "buffer_load_dwordx4 v[132:135], v64, %[rsA], s77 offen offset:1024 sc0 sc1\n"
        "v_mfma_f32_16x16x32_f16 a[148:151], v[208:211], v[116:119], a[148:151]\n"
        "ds_write_b64 v69, v[78:79] offset:1536\n"
        "v_mfma_f32_16x16x32_f16 a[180:183], v[212:215], v[116:119], a[180:183]\n"
        "v_mfma_f32_16x16x32_f16 a[212:215], v[216:219], v[116:119], a[212:215]\n"
        "buffer_load_dwordx4 v[136:139], v64, %[rsA], s77 offen offset:2048 sc0 sc1\n"
        "v_mfma_f32_16x16x32_f16 a[244:247], v[220:223], v[116:119], a[244:247]\n"
        "s_waitcnt lgkmcnt(8)\n"
        "v_mfma_f32_16x16x32_f16 a[24:27], v[192:195], v[120:123], a[24:27]\n"
        "v_mfma_f32_16x16x32_f16 a[56:59], v[196:199], v[120:123], a[56:59]\n"
        "buffer_load_dwordx4 v[140:143], v64, %[rsA], s77 offen offset:3072 sc0 sc1\n"
        "v_mfma_f32_16x16x32_f16 a[88:91], v[200:203], v[120:123], a[88:91]\n"
        "ds_read_b128 v[116:119], v71 offset:25088\n"
        "v_mfma_f32_16x16x32_f16 a[120:123], v[204:207], v[120:123], a[120:123]\n"
        "v_mfma_f32_16x16x32_f16 a[152:155], v[208:211], v[120:123], a[152:155]\n"
        "buffer_load_dwordx4 v[144:147], v64, %[rsA], s78 offen offset:0 sc0 sc1\n"
        "v_mfma_f32_16x16x32_f16 a[184:187], v[212:215], v[120:123], a[184:187]\n"
        "v_mfma_f32_16x16x32_f16 a[216:219], v[216:219], v[120:123], a[216:219]\n"
        "buffer_load_dwordx4 v[148:151], v64, %[rsA], s78 offen offset:1024 sc0 sc1\n"
        "v_mfma_f32_16x16x32_f16 a[248:251], v[220:223], v[120:123], a[248:251]\n"
        "s_waitcnt lgkmcnt(6)\n"
        "v_mfma_f32_16x16x32_f16 a[28:31], v[192:195], v[124:127], a[28:31]\n"
        "v_mfma_f32_16x16x32_f16 a[60:63], v[196:199], v[124:127], a[60:63]\n"
        "ds_read_b128 v[120:123], v71 offset:25600\n"
        "v_mfma_f32_16x16x32_f16 a[92:95], v[200:203], v[124:127], a[92:95]\n"
        "buffer_load_dwordx4 v[152:155], v64, %[rsA], s78 offen offset:2048 sc0 sc1\n"
        "v_mfma_f32_16x16x32_f16 a[124:127], v[204:207], v[124:127], a[124:127]\n"
        "v_mfma_f32_16x16x32_f16 a[156:159], v[208:211], v[124:127], a[156:159]\n"
        "buffer_load_dwordx4 v[156:159], v64, %[rsA], s78 offen offset:3072 sc0 sc1\n"
        "v_mfma_f32_16x16x32_f16 a[188:191], v[212:215], v[124:127], a[188:191]\n"
        "v_mfma_f32_16x16x32_f16 a[220:223], v[216:219], v[124:127], a[220:223]\n"
        "v_mfma_f32_16x16x32_f16 a[252:255], v[220:223], v[124:127], a[252:255]\n"
        "s_sub_u32 s45, s40, 8\n"
        "s_cmp_lt_u32 s45, 40\n"
        "s_cbranch_scc1 LTPG3_%=\n"
        "LTRG3_%=:\n"
        "s_barrier\n"
        "s_waitcnt vmcnt(18) lgkmcnt(3)\n"
        "v_mfma_f32_16x16x32_f16 a[0:3], v[224:227], v[112:115], a[0:3]\n"
        "buffer_load_dwordx4 v[12:15], %[wv0], %[rsW], s42 offen\n"
        "v_readlane_b32 s77, %[tablo], s40\n"
        "v_mfma_f32_16x16x32_f16 a[32:35], v[228:231], v[112:115], a[32:35]\n"
        "ds_read_b128 v[124:127], v71 offset:26112\n"
        "v_readlane_b32 s78, %[tabhi], s40\n"
        "s_add_u32 s40, s40, 1\n"
        "v_mfma_f32_16x16x32_f16 a[64:67], v[232:235], v[112:115], a[64:67]\n"
        "buffer_load_dwordx4 v[16:19], v62, %[rsW], s42 offen\n"
        "s_lshl_b64 s[54:55], s[54:55], 2\n"
        "v_mfma_f32_16x16x32_f16 a[96:99], v[236:239], v[112:115], a[96:99]\n"
        "buffer_load_dwordx4 v[20:23], v63, %[rsW], s42 offen\n"
        "s_or_b32 s54, s54, 3\n"
        "v_mfma_f32_16x16x32_f16 a[128:131], v[240:243], v[112:115], a[128:131]\n"
        "buffer_load_dwordx4 v[24:27], v66, %[rsW], s42 offen\n"
        "s_add_u32 s69, s69, 0x8000\n"
        "v_mfma_f32_16x16x32_f16 a[160:163], v[244:247], v[112:115], a[160:163]\n"
        "buffer_load_dwordx4 v[44:47], %[wv0], %[rsW], s68 offen\n"
        "v_mfma_f32_16x16x32_f16 a[192:195], v[248:251], v[112:115], a[192:195]\n"
        "buffer_load_dwordx4 v[48:51], v62, %[rsW], s68 offen\n"
        "v_mfma_f32_16x16x32_f16 a[224:227], v[252:255], v[112:115], a[224:227]\n"
        "s_waitcnt lgkmcnt(2)\n"
        "v_mfma_f32_16x16x32_f16 a[4:7], v[224:227], v[116:119], a[4:7]\n"
        "buffer_load_dwordx4 v[52:55], v63, %[rsW], s68 offen\n"
        "v_mfma_f32_16x16x32_f16 a[36:39], v[228:231], v[116:119], a[36:39]\n"
        "ds_read_b128 v[112:115], v71 offset:24832\n"
        "v_mfma_f32_16x16x32_f16 a[68:71], v[232:235], v[116:119], a[68:71]\n"
        "buffer_load_dwordx4 v[56:59], v66, %[rsW], s68 offen\n"
        "v_mfma_f32_16x16x32_f16 a[100:103], v[236:239], v[116:119], a[100:103]\n"
        "v_readlane_b32 s42, %[tabw], s66\n"
        "s_cmp_lt_u32 s66, 64\n"
        "s_cselect_b32 s42, s42, 0x80000000\n"
        "s_add_u32 s66, s66, 1\n"
        "v_mfma_f32_16x16x32_f16 a[132:135], v[240:243], v[116:119], a[132:135]\n"
        "v_readlane_b32 s68, %[tabw], s66\n"
        "s_cmp_lt_u32 s66, 64\n"
        "s_cselect_b32 s68, s68, 0x80000000\n"
        "s_add_u32 s66, s66, 1\n"
        "v_mfma_f32_16x16x32_f16 a[164:167], v[244:247], v[116:119], a[164:167]\n"
        "s_cmp_eq_u32 s40, s61\n"
        "s_cbranch_scc1 LSENDG3_%=\n"
        "LSENTG3_%=:\n"
        "v_mfma_f32_16x16x32_f16 a[196:199], v[248:251], v[116:119], a[196:199]\n"
        "s_cmp_ge_u32 s40, 41\n"
        "s_cselect_b64 s[54:55], -1, s[54:55]\n"
        "v_mfma_f32_16x16x32_f16 a[228:231], v[252:255], v[116:119], a[228:231]\n"
        "s_waitcnt lgkmcnt(2)\n"
        "v_mfma_f32_16x16x32_f16 a[8:11], v[224:227], v[120:123], a[8:11]\n"
        "v_add_u32 v69, s67, %[wdst]\n"
        "s_add_u32 s67, s67, 0x2000\n"
        "v_mfma_f32_16x16x32_f16 a[40:43], v[228:231], v[120:123], a[40:43]\n"
        "ds_read_b128 v[116:119], v71 offset:25344\n"
        "v_mfma_f32_16x16x32_f16 a[72:75], v[232:235], v[120:123], a[72:75]\n"
        "s_waitcnt vmcnt(22)\n"
        "s_cmp_lt_u32 s67, 163840\n"
        "s_cselect_b32 s67, s67, 0\n"
        "v_mfma_f32_16x16x32_f16 a[104:107], v[236:239], v[120:123], a[104:107]\n"
        "v_cvt_pk_f16_f32 v72, v80, v84\n"
        "v_cvt_pk_f16_f32 v73, v88, v92\n"
        "v_mfma_f32_16x16x32_f16 a[136:139], v[240:243], v[120:123], a[136:139]\n"
        "ds_write_b64 v69, v[72:73] offset:0\n"
        "v_mfma_f32_16x16x32_f16 a[168:171], v[244:247], v[120:123], a[168:171]\n"
        "v_cvt_pk_f16_f32 v74, v81, v85\n"
        "v_cvt_pk_f16_f32 v75, v89, v93\n"
        "v_mfma_f32_16x16x32_f16 a[200:203], v[248:251], v[120:123], a[200:203]\n"
        "ds_write_b64 v69, v[74:75] offset:512\n"
        "v_mfma_f32_16x16x32_f16 a[232:235], v[252:255], v[120:123], a[232:235]\n"
        "s_waitcnt lgkmcnt(4)\n"
        "v_mfma_f32_16x16x32_f16 a[12:15], v[224:227], v[124:127], a[12:15]\n"
        "v_cvt_pk_f16_f32 v76, v82, v86\n"
        "v_cvt_pk_f16_f32 v77, v90, v94\n"
        "v_mfma_f32_16x16x32_f16 a[44:47], v[228:231], v[124:127], a[44:47]\n"
        "ds_read_b128 v[120:123], v71 offset:25856\n"
        "v_mfma_f32_16x16x32_f16 a[76:79], v[232:235], v[124:127], a[76:79]\n"
        "ds_write_b64 v69, v[76:77] offset:1024\n"
        "v_mfma_f32_16x16x32_f16 a[108:111], v[236:239], v[124:127], a[108:111]\n"
        "v_cvt_pk_f16_f32 v78, v83, v87\n"
        "v_cvt_pk_f16_f32 v79, v91, v95\n"
        "v_mfma_f32_16x16x32_f16 a[140:143], v[240:243], v[124:127], a[140:143]\n"
        "ds_write_b64 v69, v[78:79] offset:1536\n"
        "v_mfma_f32_16x16x32_f16 a[172:175], v[244:247], v[124:127], a[172:175]\n"
        "v_add_u32 v69, s67, %[wdst]\n"
        "s_add_u32 s67, s67, 0x2000\n"
        "v_mfma_f32_16x16x32_f16 a[204:207], v[248:251], v[124:127], a[204:207]\n"
        "s_waitcnt vmcnt(18)\n"
        "s_cmp_lt_u32 s67, 163840\n"
        "s_cselect_b32 s67, s67, 0\n"
        "v_mfma_f32_16x16x32_f16 a[236:239], v[252:255], v[124:127], a[236:239]\n"
        "s_waitcnt lgkmcnt(6)\n"
        "v_mfma_f32_16x16x32_f16 a[16:19], v[224:227], v[112:115], a[16:19]\n"
        "v_cvt_pk_f16_f32 v72, v96, v100\n"
        "v_cvt_pk_f16_f32 v73, v104, v108\n"
        "v_mfma_f32_16x16x32_f16 a[48:51], v[228:231], v[112:115], a[48:51]\n"
        "ds_read_b128 v[124:127], v71 offset:26368\n"
        "v_mfma_f32_16x16x32_f16 a[80:83], v[232:235], v[112:115], a[80:83]\n"
        "ds_write_b64 v69, v[72:73] offset:0\n"
        "v_mfma_f32_16x16x32_f16 a[112:115], v[236:239], v[112:115], a[112:115]\n"
        "v_cvt_pk_f16_f32 v74, v97, v101\n"
        "v_cvt_pk_f16_f32 v75, v105, v109\n"
        "v_mfma_f32_16x16x32_f16 a[144:147], v[240:243], v[112:115], a[144:147]\n"
        "ds_write_b64 v69, v[74:75] offset:512\n"
        "v_mfma_f32_16x16x32_f16 a[176:179], v[244:247], v[112:115], a[176:179]\n"
        "v_cvt_pk_f16_f32 v76, v98, v102\n"
        "v_cvt_pk_f16_f32 v77, v106, v110\n"
        "v_mfma_f32_16x16x32_f16 a[208:211], v[248:251], v[112:115], a[208:211]\n"
        "ds_write_b64 v69, v[76:77] offset:1024\n"
        "v_mfma_f32_16x16x32_f16 a[240:243], v[252:255], v[112:115], a[240:243]\n"
        "s_waitcnt lgkmcnt(9)\n"
        "v_mfma_f32_16x16x32_f16 a[20:23], v[224:227], v[116:119], a[20:23]\n"
        "buffer_load_dwordx4 v[160:163], v64, %[rsA], s77 offen offset:0 sc0 sc1\n"
        "v_mfma_f32_16x16x32_f16 a[52:55], v[228:231], v[116:119], a[52:55]\n"
        "ds_read_b128 v[112:115], v70 offset:0\n"
        "v_mfma_f32_16x16x32_f16 a[84:87], v[232:235], v[116:119], a[84:87]\n"
        "v_cvt_pk_f16_f32 v78, v99, v103\n"
        "v_cvt_pk_f16_f32 v79, v107, v111\n"
        "v_mfma_f32_16x16x32_f16 a[116:119], v[236:239], v[116:119], a[116:119]\n"
        "buffer_load_dwordx4 v[164:167], v64, %[rsA], s77 offen offset:1024 sc0 sc1\n"
        "v_mfma_f32_16x16x32_f16 a[148:151], v[240:243], v[116:119], a[148:151]\n"
        "ds_write_b64 v69, v[78:79] offset:1536\n"
        "v_mfma_f32_16x16x32_f16 a[180:183], v[244:247], v[116:119], a[180:183]\n"
        "s_waitcnt vmcnt(19)\n"
        "v_cmp_le_u32 vcc, s71, v61\n"
        "v_mfma_f32_16x16x32_f16 a[212:215], v[248:251], v[116:119], a[212:215]\n"
        "buffer_load_dwordx4 v[168:171], v64, %[rsA], s77 offen offset:2048 sc0 sc1\n"
        "v_mfma_f32_16x16x32_f16 a[244:247], v[252:255], v[116:119], a[244:247]\n"
        "s_waitcnt lgkmcnt(8)\n"
        "v_mfma_f32_16x16x32_f16 a[24:27], v[224:227], v[120:123], a[24:27]\n"
        "s_or_b64 s[46:47], s[46:47], vcc\n"
        "v_mfma_f32_16x16x32_f16 a[56:59], v[228:231], v[120:123], a[56:59]\n"
        "buffer_load_dwordx4 v[172:175], v64, %[rsA], s77 offen offset:3072 sc0 sc1\n"
        "s_lshl_b64 s[62:63], s[54:55], 8\n"
        "v_mfma_f32_16x16x32_f16 a[88:91], v[232:235], v[120:123], a[88:91]\n"
        "ds_read_b128 v[116:119], v70 offset:512\n"
        "v_mfma_f32_16x16x32_f16 a[120:123], v[236:239], v[120:123], a[120:123]\n"
        "s_andn2_b64 s[62:63], s[62:63], s[46:47]\n"
        "s_cselect_b32 s60, 0, 0x6000000\n"
        "v_mfma_f32_16x16x32_f16 a[152:155], v[240:243], v[120:123], a[152:155]\n"
        "buffer_load_dwordx4 v[176:179], v64, %[rsA], s78 offen offset:0 sc0 sc1\n"
        "s_ff0_i32_b64 s52, s[46:47]\n"
        "v_mfma_f32_16x16x32_f16 a[184:187], v[244:247], v[120:123], a[184:187]\n"
        "s_bfm_b64 s[88:89], 44, s52\n"
        "v_mfma_f32_16x16x32_f16 a[216:219], v[248:251], v[120:123], a[216:219]\n"
        "buffer_load_dwordx4 v[180:183], v64, %[rsA], s78 offen offset:1024 sc0 sc1\n"
        "s_andn2_b64 s[88:89], s[88:89], s[46:47]\n"
        "v_mfma_f32_16x16x32_f16 a[248:251], v[252:255], v[120:123], a[248:251]\n"
        "s_waitcnt lgkmcnt(6)\n"
        "v_mfma_f32_16x16x32_f16 a[28:31], v[224:227], v[124:127], a[28:31]\n"
        "s_cmp_lt_u32 s40, 14\n"
        "s_cselect_b64 s[62:63], 0x3fffff, -1\n"
        "v_mfma_f32_16x16x32_f16 a[60:63], v[228:231], v[124:127], a[60:63]\n"
        "ds_read_b128 v[120:123], v70 offset:1024\n"
        "s_and_b64 s[88:89], s[88:89], s[62:63]\n"
        "v_mfma_f32_16x16x32_f16 a[92:95], v[232:235], v[124:127], a[92:95]\n"
        "buffer_load_dwordx4 v[184:187], v64, %[rsA], s78 offen offset:2048 sc0 sc1\n"
        "s_or_b32 s88, s88, 1\n"
        "v_mfma_f32_16x16x32_f16 a[124:127], v[236:239], v[124:127], a[124:127]\n"
        "s_andn2_b64 s[56:57], s[54:55], s[46:47]\n"
        "s_cbranch_scc1 LS_G3_%=\n"
        "LR_G3_%=:\n"
        "v_mfma_f32_16x16x32_f16 a[156:159], v[240:243], v[124:127], a[156:159]\n"
        "buffer_load_dwordx4 v[188:191], v64, %[rsA], s78 offen offset:3072 sc0 sc1\n"
        "v_mfma_f32_16x16x32_f16 a[188:191], v[244:247], v[124:127], a[188:191]\n"
        "v_readlane_b32 s58, %[tabpf], s40\n"
        "v_mfma_f32_16x16x32_f16 a[220:223], v[248:251], v[124:127], a[220:223]\n"
        "s_bitcmp1_b64 s[46:47], s58\n"
        "s_cselect_b32 s58, s58, 0x6000000\n"
        "v_mfma_f32_16x16x32_f16 a[252:255], v[252:255], v[124:127], a[252:255]\n"
        "s_andn2_b32 s58, s58, 63\n"
        "s_cmp_lt_u32 s69, 163840\n"
        "s_cselect_b32 s69, s69, 0\n"
        "v_mov_b32 v71, v70\n"
        "v_add_u32 v70, s69, %[bbase]\n"
        "s_sub_u32 s82, s82, 4\n"
        "s_branch LBACK_%=\n"
        "LZZ_%=:\n"
        : "+{v64}"(l16), [cnt] "+v"(cntoff), [tablo] "+v"(tablo), [tabhi] "+v"(tabhi), [tabw] "+v"(tabw), [tabpf] "+v"(tabpf),
          "+{v[28:59]}"(remx), "+{v[68:69]}"(slot)
        : [bbase] "v"(bbase), [wdst] "v"(wdst), [wv0] "v"(wv0), [xv] "v"(xv),
          [tabh] "v"(tabh), [okeep] "v"(okeep),
          [rsX] "s"(rsX), [rsA] "s"(rsA), [rsW] "s"(rsW), [rsO] "s"(rsO), [dws] "s"(dws), [my] "s"(myws), [role] "s"(role),
          [xrem] "s"(xrem), [soff] "s"(soff),
          [scnt] "s"(scnt), [slim] "s"(slim), [a0] "s"(a0), [pf0] "s"(pf0), [cpo] "s"(cpo), [xl] "s"(xl), [seq] "s"(seq), [xsend] "s"(xsend), [xrecv] "s"(xrecv),
          [mflag] "s"(mflag), [pflag] "s"(pflag)
        :
        "memory", "v12", "v13", "v14", "v15", "v16", "v17", "v18", "v19", "v20", "v21", "v22", "v23", "v24",
        "v25", "v26", "v27", "v60", "v61", "v62", "v63", "v65", "v66", "v67", "v70", "v71", "v72", "v73",
        "v74", "v75", "v76", "v77", "v78", "v79", "v80", "v81", "v82", "v83", "v84", "v85", "v86", "v87",
        "v88", "v89", "v90", "v91", "v92", "v93", "v94", "v95", "v96", "v97", "v98", "v99", "v100", "v101",
        "v102", "v103", "v104", "v105", "v106", "v107", "v108", "v109", "v110", "v111", "v112", "v113",
        "v114", "v115", "v116", "v117", "v118", "v119", "v120", "v121", "v122", "v123", "v124", "v125",
        "v126", "v127", "v128", "v129", "v130", "v131", "v132", "v133", "v134", "v135", "v136", "v137",
        "v138", "v139", "v140", "v141", "v142", "v143", "v144", "v145", "v146", "v147", "v148", "v149",
        "v150", "v151", "v152", "v153", "v154", "v155", "v156", "v157", "v158", "v159", "v160", "v161",
        "v162", "v163", "v164", "v165", "v166", "v167", "v168", "v169", "v170", "v171", "v172", "v173",
        "v174", "v175", "v176", "v177", "v178", "v179", "v180", "v181", "v182", "v183", "v184", "v185",
        "v186", "v187", "v188", "v189", "v190", "v191", "v192", "v193", "v194", "v195", "v196", "v197",
        "v198", "v199", "v200", "v201", "v202", "v203", "v204", "v205", "v206", "v207", "v208", "v209",
        "v210", "v211", "v212", "v213", "v214", "v215", "v216", "v217", "v218", "v219", "v220", "v221",
        "v222", "v223", "v224", "v225", "v226", "v227", "v228", "v229", "v230", "v231", "v232", "v233",
        "v234", "v235", "v236", "v237", "v238", "v239", "v240", "v241", "v242", "v243", "v244", "v245",
        "v246", "v247", "v248", "v249", "v250", "v251", "v252", "v253", "v254", "v255", "a0", "a1", "a2",
        "a3", "a4", "a5", "a6", "a7", "a8", "a9", "a10", "a11", "a12", "a13", "a14", "a15", "a16", "a17",
        "a18", "a19", "a20", "a21", "a22", "a23", "a24", "a25", "a26", "a27", "a28", "a29", "a30", "a31",
        "a32", "a33", "a34", "a35", "a36", "a37", "a38", "a39", "a40", "a41", "a42", "a43", "a44", "a45",
        "a46", "a47", "a48", "a49", "a50", "a51", "a52", "a53", "a54", "a55", "a56", "a57", "a58", "a59",
        "a60", "a61", "a62", "a63", "a64", "a65", "a66", "a67", "a68", "a69", "a70", "a71", "a72", "a73",
        "a74", "a75", "a76", "a77", "a78", "a79", "a80", "a81", "a82", "a83", "a84", "a85", "a86", "a87",
        "a88", "a89", "a90", "a91", "a92", "a93", "a94", "a95", "a96", "a97", "a98", "a99", "a100", "a101",
        "a102", "a103", "a104", "a105", "a106", "a107", "a108", "a109", "a110", "a111", "a112", "a113",
        "a114", "a115", "a116", "a117", "a118", "a119", "a120", "a121", "a122", "a123", "a124", "a125",
        "a126", "a127", "a128", "a129", "a130", "a131", "a132", "a133", "a134", "a135", "a136", "a137",
        "a138", "a139", "a140", "a141", "a142", "a143", "a144", "a145", "a146", "a147", "a148", "a149",
        "a150", "a151", "a152", "a153", "a154", "a155", "a156", "a157", "a158", "a159", "a160", "a161",
        "a162", "a163", "a164", "a165", "a166", "a167", "a168", "a169", "a170", "a171", "a172", "a173",
        "a174", "a175", "a176", "a177", "a178", "a179", "a180", "a181", "a182", "a183", "a184", "a185",
        "a186", "a187", "a188", "a189", "a190", "a191", "a192", "a193", "a194", "a195", "a196", "a197",
        "a198", "a199", "a200", "a201", "a202", "a203", "a204", "a205", "a206", "a207", "a208", "a209",
        "a210", "a211", "a212", "a213", "a214", "a215", "a216", "a217", "a218", "a219", "a220", "a221",
        "a222", "a223", "a224", "a225", "a226", "a227", "a228", "a229", "a230", "a231", "a232", "a233",
        "a234", "a235", "a236", "a237", "a238", "a239", "a240", "a241", "a242", "a243", "a244", "a245",
        "a246", "a247", "a248", "a249", "a250", "a251", "a252", "a253", "a254", "a255", "s40", "s41", "s42",
        "s43", "s44", "s45", "s46", "s47", "s48", "s49", "s50", "s51", "s52", "s53", "s54", "s55", "s56",
        "s57", "s58", "s59", "s60", "s61", "s62", "s63", "s64", "s65", "s66", "s67", "s68", "s69", "s70",
        "s71", "s72", "s73", "s74", "s75", "s76", "s77", "s78", "s79", "s80", "s81", "s82", "s83", "s84",
        "s85", "s86", "s87", "s88", "s89", "s90", "s91", "s92", "scc", "vcc");
}

}

extern "C" void kernel_launch(int rank, int nranks, ncclComm_t comm,
                              hipStream_t stream,
                              void* const* d_in, const int* in_sizes, int n_in,
                              void* d_out, int out_size,
                              const int* in_dtypes, int out_dtype,
                              void* const* d_out_all,
                              void* const* d_ws_all, size_t ws_bytes) {
    if (nranks != NR || (unsigned)rank >= (unsigned)NR || ws_bytes < (size_t)WS_REC || n_in < 2 ||
        in_sizes[0] != KTOT * KSH || in_sizes[1] != KTOT * NTOT || out_size != MLOC * NTOT) {
        fprintf(stderr, "kernel_launch: unexpected problem shape\n");
        return;
    }
    Params p;
    p.x = (const float*)d_in[0];
    p.w = (const float*)d_in[1];
    p.out = (float*)d_out;
    for (int r = 0; r < NR; ++r) p.ws[r] = (char*)d_ws_all[r];
    p.my = (char*)d_ws_all[rank];
    p.rank = rank;
    p.pad = 0;
    hipLaunchKernelGGL(fused_a2a_gemm_gelu, dim3(NWG), dim3(NTHR), 0, stream, p);
    HIP_CHECK(hipGetLastError());
}
```
